```python
import numpy as np

kernel_name = "gpu_a2a_gemm_m4096_k4096_n16384_e5m2e5m2_dqsilu"


ABSMAX_RTOL = 0.05

COMPUTE_DTYPE = "float32"
EPILOGUE = "dequant_silu"
IN_QUANT = None
OUT_QUANT = None
IN_FP8 = "e5m2e5m2"
IN_DTYPES = {"x": "e5m2", "w_mat": "e5m2",
             "scale_x": "float32", "scale_w": "float32"}
WORLD_SIZE = 8

def _snap_fp8(x, mant_bits, max_abs, min_normal):
    a = np.abs(x.astype(np.float64))
    _m, _e = np.frexp(a)
    q = np.where(a >= min_normal, np.ldexp(1.0, _e - 1 - mant_bits),
                 min_normal * 2.0 ** -mant_bits)
    snapped = np.minimum(np.round(a / q) * q, max_abs)
    return (np.sign(x) * snapped).astype(np.float32)

def setup_inputs():

    x = _snap_fp8(np.random.randn(4096, 4096).astype(np.float32), 2, 57344.0, 2.0 ** -14)
    w_mat = _snap_fp8(np.random.randn(4096, 16384).astype(np.float32), 2, 57344.0, 2.0 ** -14)
    scale_x = (0.02 * (1.0 + np.random.rand(1))).astype(np.float32)
    scale_w = (0.02 * (1.0 + np.random.rand(1))).astype(np.float32)
    return {"x": x, "w_mat": w_mat, "scale_x": scale_x, "scale_w": scale_w}

def reference(x, w_mat, scale_x, scale_w):
    acc = (x @ w_mat).astype(np.float32)
    y = (acc * (scale_x * scale_w)).astype(np.float32)
    return (y / (1.0 + np.exp(-np.clip(y, -60.0, 60.0)))).astype(np.float32)
```

```cpp
#include <hip/hip_runtime.h>
#include <rccl/rccl.h>
#include <cstdio>

#define HIP_CHECK(expr) do { hipError_t _e = (expr); if (_e != hipSuccess) { \
    fprintf(stderr, "HIP error %s at %s:%d\n", hipGetErrorString(_e), __FILE__, __LINE__); \
    return; } } while (0)

typedef int i32x2 __attribute__((ext_vector_type(2)));
typedef int i32x4 __attribute__((ext_vector_type(4)));
typedef unsigned u32x4 __attribute__((ext_vector_type(4)));
typedef int i32x8 __attribute__((ext_vector_type(8)));
typedef float f32x4 __attribute__((ext_vector_type(4)));
#define LAS __attribute__((address_space(3)))

constexpr int NR = 8;
constexpr int MLOC = 512, KTOT = 4096, NTOT = 16384, KSH = 512;
constexpr size_t OFF_CNT = 0;
constexpr size_t OFF_TMO = 32768;
constexpr size_t OFF_FLAG = 65536;
constexpr size_t OFF_STAGE = 1 << 20;
constexpr size_t STAGE_BYTES = 2 << 20;
constexpr unsigned long long DEADLINE = 40000000ull;

struct Params {
  const unsigned char* x; const unsigned char* w; const float* sx; const float* sw; float* out;
  unsigned char* ws[NR]; int rank; int pad;
};

__global__ __launch_bounds__(256, 1) void fused_kernel(Params p) {
  extern __shared__ __attribute__((aligned(16))) char smem[];
  const int b = blockIdx.x, tid = threadIdx.x, w = __builtin_amdgcn_readfirstlane(tid >> 6), l = tid & 63;
  unsigned char* myws = p.ws[p.rank];
  unsigned* cntp = (unsigned*)(myws + OFF_CNT + (size_t)b * 128);
  const unsigned call = __hip_atomic_load(cntp, __ATOMIC_RELAXED, __HIP_MEMORY_SCOPE_AGENT) + 1u;
  unsigned* tmo = (unsigned*)(myws + OFF_TMO);
  const unsigned par = call & 1u;
  {
    const unsigned dst = (unsigned)(p.rank + 1 + (b & 7)) & 7u;
    const int j = b >> 3, round = j >> 3, rg = j & 7, F = rg * 4 + w;
    const unsigned char* src = p.x + (size_t)(dst * 512 + F * 16 + (l & 15)) * KSH + round * 128 + (l >> 4) * 16;
    u32x4 v0 = *(const u32x4*)src, v1 = *(const u32x4*)(src + 64);
    unsigned char* d = p.ws[dst] + OFF_STAGE + par * STAGE_BYTES + (size_t)((p.rank * 4 + round) * 32 + F) * 2048 + l * 16;
    asm volatile("global_store_dwordx4 %0, %1, off sc0 sc1\n\ts_nop 1" :: "v"(d), "v"(v0) : "memory");
    asm volatile("global_store_dwordx4 %0, %1, off offset:1024 sc0 sc1\n\ts_nop 1" :: "v"(d), "v"(v1) : "memory");
    asm volatile("s_waitcnt vmcnt(0)" ::: "memory");
    unsigned* fl = (unsigned*)(p.ws[dst] + OFF_FLAG) + ((round * 8 + rg) * 8 + p.rank) * 4 + w;
    if (l == 0) __hip_atomic_store(fl, call, __ATOMIC_RELAXED, __HIP_MEMORY_SCOPE_SYSTEM);
  }
  const int xcd = b & 7, jj = b >> 3, ntile = xcd * 16 + (jj >> 1), mh = jj & 1;
  {
    const int rg = mh * 4 + w;
    const unsigned* fl = (const unsigned*)(myws + OFF_FLAG) + rg * 32 + (l & 31);
    const unsigned long long t0 = __builtin_amdgcn_s_memrealtime();
    for (;;) {
      bool ok = true;
#pragma unroll
      for (int r = 0; r < 4; ++r) ok &= __hip_atomic_load(fl + r * 256, __ATOMIC_RELAXED, __HIP_MEMORY_SCOPE_SYSTEM) >= call;
      if (__all(ok)) break;
      if (__hip_atomic_load(tmo, __ATOMIC_RELAXED, __HIP_MEMORY_SCOPE_AGENT) == call || __builtin_amdgcn_s_memrealtime() - t0 > DEADLINE) {
        __hip_atomic_store(tmo, call, __ATOMIC_RELAXED, __HIP_MEMORY_SCOPE_AGENT); break; }
      __builtin_amdgcn_s_sleep(2);
    }
    __builtin_amdgcn_fence(__ATOMIC_ACQUIRE, "");
    asm volatile("s_waitcnt vmcnt(0)" ::: "memory");
  }
  const bool dead = __hip_atomic_load(tmo, __ATOMIC_RELAXED, __HIP_MEMORY_SCOPE_AGENT) == call;
  const unsigned char* stage = myws + OFF_STAGE + par * STAGE_BYTES;
  const unsigned char* abase = stage + (size_t)(mh * 16 + w * 4) * 2048 + l * 16;
  const int kr = l >> 3, sl = l & 7, fx = ((kr >> 1) & 3) | (((w >> 1) & 1) << 2);
  const unsigned char* wsrc = p.w + (size_t)(w * 8 + kr) * NTOT + ntile * 128 + ((sl ^ fx) * 16);
  LAS char* lds = (LAS char*)smem;
  const int g = l >> 4, q = (l >> 1) & 7, pp = l & 1, xr = (q >> 1) | ((g & 1) << 2);
  const int rbase = g * 2048 + q * 128 + pp * 8;
  f32x4 acc[4][8];
#pragma unroll
  for (int m = 0; m < 4; ++m)
#pragma unroll
    for (int n = 0; n < 8; ++n) acc[m][n] = (f32x4){0, 0, 0, 0};
  auto dma = [&](int s, int buf) {
#pragma unroll
    for (int jx = 0; jx < 4; ++jx)
      __builtin_amdgcn_global_load_lds((const unsigned*)(wsrc + (size_t)(s * 128 + jx * 32) * NTOT), (LAS unsigned*)(lds + buf * 16384 + w * 1024 + jx * 4096), 16, 0, 0);
  };
  dma(0, 0);
  for (int s = 0; s < 32; ++s) {
    const int buf = s & 1;
    i32x8 a[4];
#pragma unroll
    for (int m = 0; m < 4; ++m) {
      const i32x4 lo = *(const i32x4*)(abase + (size_t)s * 65536 + m * 2048), hi = *(const i32x4*)(abase + (size_t)s * 65536 + m * 2048 + 1024);
      a[m] = (i32x8){lo[0], lo[1], lo[2], lo[3], hi[0], hi[1], hi[2], hi[3]};
    }
    asm volatile("s_waitcnt vmcnt(0)" ::: "memory");
    __syncthreads();
    if (s + 1 < 32) dma(s + 1, buf ^ 1);
#pragma unroll
    for (int n = 0; n < 8; ++n) {
      LAS char* rp = lds + buf * 16384 + rbase + ((n ^ xr) * 16);
      const i32x2 r0 = __builtin_amdgcn_ds_read_tr8_b64_v2i32((LAS i32x2*)(rp));
      const i32x2 r1 = __builtin_amdgcn_ds_read_tr8_b64_v2i32((LAS i32x2*)(rp + 1024));
      const i32x2 r2 = __builtin_amdgcn_ds_read_tr8_b64_v2i32((LAS i32x2*)(rp + 8192));
      const i32x2 r3 = __builtin_amdgcn_ds_read_tr8_b64_v2i32((LAS i32x2*)(rp + 9216));
      const i32x8 bb = {r0[0], r0[1], r1[0], r1[1], r2[0], r2[1], r3[0], r3[1]};
#pragma unroll
      for (int m = 0; m < 4; ++m) acc[m][n] = __builtin_amdgcn_mfma_scale_f32_16x16x128_f8f6f4(bb, a[m], acc[m][n], 1, 1, 0, 0, 0, 0);
    }
  }
  float sc = p.sx[0] * p.sw[0];
  if (dead) sc = __builtin_nanf("");
  float* orow = p.out + (size_t)(mh * 256 + w * 64 + (l & 15)) * NTOT + ntile * 128 + (l >> 4) * 4;
#pragma unroll
  for (int m = 0; m < 4; ++m)
#pragma unroll
    for (int n = 0; n < 8; ++n) {
      f32x4 o;
#pragma unroll
      for (int r = 0; r < 4; ++r) {
        const float y = acc[m][n][r] * sc;
        const float yc = __builtin_amdgcn_fmed3f(y, -60.f, 60.f);
        const float e = __builtin_amdgcn_exp2f(yc * -1.4426950408889634f);
        o[r] = y * __builtin_amdgcn_rcpf(1.f + e);
      }
      *(f32x4*)(orow + (size_t)m * 16 * NTOT + n * 16) = o;
    }
  if (tid == 0) __hip_atomic_store(cntp, call, __ATOMIC_RELAXED, __HIP_MEMORY_SCOPE_AGENT);
}

extern "C" void kernel_launch(int rank, int nranks, ncclComm_t comm, hipStream_t stream,
                              void* const* d_in, const int* in_sizes, int n_in,
                              void* d_out, int out_size, const int* in_dtypes, int out_dtype,
                              void* const* d_out_all, void* const* d_ws_all, size_t ws_bytes) {
  if (nranks != NR || ws_bytes < OFF_STAGE + 2 * STAGE_BYTES) { fprintf(stderr, "unexpected config\n"); return; }
  static bool init = false;
  if (!init) {
    HIP_CHECK(hipFuncSetAttribute((const void*)fused_kernel, hipFuncAttributeMaxDynamicSharedMemorySize, 160 * 1024));
    init = true;
  }
  Params p;
  p.x = (const unsigned char*)d_in[0]; p.w = (const unsigned char*)d_in[1];
  p.sx = (const float*)d_in[2]; p.sw = (const float*)d_in[3]; p.out = (float*)d_out;
  for (int i = 0; i < NR; ++i) p.ws[i] = (unsigned char*)d_ws_all[i];
  p.rank = rank; p.pad = 0;
  hipLaunchKernelGGL(fused_kernel, dim3(256), dim3(256), 160 * 1024, stream, p);
}
```

```cpp
#include <hip/hip_runtime.h>
#include <rccl/rccl.h>
#include <cstdio>

#define HIP_CHECK(expr) do { hipError_t _e = (expr); if (_e != hipSuccess) { \
    fprintf(stderr, "HIP error %s at %s:%d\n", hipGetErrorString(_e), __FILE__, __LINE__); \
    return; } } while (0)

#define CORE_ASM \
    "global_load_dword v176, v236, s[56:57] sc1\n\t" \
    "s_add_u32 m0, s45, 65536\n\t" \
    "s_add_u32 s74, s89, 0\n\t" \
    "buffer_load_dwordx4 v160, s[16:19], s74 offen lds\n\t" \
    "s_add_u32 m0, s45, 66560\n\t" \
    "s_add_u32 s74, s89, 4096\n\t" \
    "buffer_load_dwordx4 v161, s[16:19], s74 offen lds\n\t" \
    "s_mov_b64 exec, 0xffff\n\t" \
    "buffer_load_dword v179, v166, s[16:19], 0 offen\n\t" \
    "s_mov_b64 exec, -1\n\t" \
    "s_cmp_eq_u32 s47, 0\n\t" \
    "s_cselect_b32 s87, s64, s65\n\t" \
    "s_mov_b32 s85, s87\n\t" \
    "s_add_u32 m0, s44, 0\n\t" \
    "s_add_u32 s74, s85, 0\n\t" \
    "buffer_load_dwordx4 v232, s[8:11], s74 offen  lds\n\t" \
    "s_add_u32 m0, s44, 4096\n\t" \
    "s_add_u32 s74, s85, 524288\n\t" \
    "buffer_load_dwordx4 v232, s[8:11], s74 offen  lds\n\t" \
    "s_add_u32 m0, s44, 8192\n\t" \
    "s_add_u32 s74, s85, 1048576\n\t" \
    "buffer_load_dwordx4 v232, s[8:11], s74 offen  lds\n\t" \
    "s_add_u32 m0, s44, 12288\n\t" \
    "s_add_u32 s74, s85, 1572864\n\t" \
    "buffer_load_dwordx4 v232, s[8:11], s74 offen  lds\n\t" \
    "global_load_dword v253, v236, s[60:61]\n\t" \
    "global_load_dword v254, v236, s[62:63]\n\t" \
    "s_waitcnt vmcnt(7)\n\t" \
    "v_add_u32 v177, 65536, v162\n\t" \
    "v_add_u32 v178, 65536, v163\n\t" \
    "ds_read_b128 v[240:243], v177\n\t" \
    "ds_read_b128 v[244:247], v178\n\t" \
    "v_readfirstlane_b32 s75, v176\n\t" \
    "s_nop 3\n\t" \
    "s_add_u32 s75, s75, 1\n\t" \
    "s_and_b32 s74, s75, 1\n\t" \
    "s_lshl_b32 s74, s74, 21\n\t" \
    "s_add_u32 s12, s12, s74\n\t" \
    "s_addc_u32 s13, s13, 0\n\t" \
    "s_add_u32 s48, s48, s74\n\t" \
    "s_addc_u32 s49, s49, 0\n\t" \
    "v_mov_b32 v176, s75\n\t" \
    "s_waitcnt lgkmcnt(0)\n\t" \
    "s_cmp_eq_u32 s79, 0\n\t" \
    "s_cbranch_scc1 L_pay0_%=\n\t" \
    "s_mov_b64 exec, 1\n\t" \
    "global_store_dword v236, v236, s[56:57] offset:32\n\t" \
    "global_store_dword v236, v236, s[56:57] offset:32\n\t" \
    "s_mov_b64 exec, -1\n\t" \
    "s_branch L_paye0_%=\n\t" \
    "L_pay0_%=:\n\t" \
    "global_store_dwordx4 v237, v[240:243], s[48:49] sc0 sc1\n\t" \
    "global_store_dwordx4 v237, v[244:247], s[48:49] offset:1024 sc0 sc1\n\t" \
    "L_paye0_%=:\n\t" \
    "s_cmp_eq_u32 s47, 1\n\t" \
    "s_cselect_b32 s87, s64, s65\n\t" \
    "s_mov_b32 s86, s87\n\t" \
    "s_add_u32 m0, s46, 0\n\t" \
    "s_add_u32 s74, s86, 0\n\t" \
    "buffer_load_dwordx4 v232, s[8:11], s74 offen  lds\n\t" \
    "s_add_u32 m0, s46, 4096\n\t" \
    "s_add_u32 s74, s86, 524288\n\t" \
    "buffer_load_dwordx4 v232, s[8:11], s74 offen  lds\n\t" \
    "s_add_u32 m0, s46, 8192\n\t" \
    "s_add_u32 s74, s86, 1048576\n\t" \
    "buffer_load_dwordx4 v232, s[8:11], s74 offen  lds\n\t" \
    "s_add_u32 m0, s46, 12288\n\t" \
    "s_add_u32 s74, s86, 1572864\n\t" \
    "buffer_load_dwordx4 v232, s[8:11], s74 offen  lds\n\t" \
    "s_add_u32 m0, s45, 131072\n\t" \
    "s_add_u32 s74, s68, 0\n\t" \
    "buffer_load_dwordx4 v160, s[16:19], s74 offen  lds\n\t" \
    "s_add_u32 m0, s45, 132096\n\t" \
    "s_add_u32 s74, s68, 4096\n\t" \
    "buffer_load_dwordx4 v161, s[16:19], s74 offen  lds\n\t" \
    "s_cmp_eq_u32 s79, 0\n\t" \
    "s_cbranch_scc1 L_flg0_%=\n\t" \
    "s_mov_b64 exec, 1\n\t" \
    "global_store_dword v236, v236, s[56:57] offset:32\n\t" \
    "s_mov_b64 exec, -1\n\t" \
    "s_branch L_flge0_%=\n\t" \
    "L_flg0_%=:\n\t" \
    "s_waitcnt vmcnt(6)\n\t" \
    "v_mov_b32 v251, s75\n\t" \
    "s_mov_b64 exec, 1\n\t" \
    "global_store_dword v236, v251, s[52:53] sc0 sc1\n\t" \
    "s_mov_b64 exec, -1\n\t" \
    "L_flge0_%=:\n\t" \
    "s_add_u32 m0, s45, 133120\n\t" \
    "s_add_u32 s74, s68, 8192\n\t" \
    "buffer_load_dwordx4 v160, s[16:19], s74 offen  lds\n\t" \
    "s_add_u32 m0, s45, 134144\n\t" \
    "s_add_u32 s74, s68, 12288\n\t" \
    "buffer_load_dwordx4 v161, s[16:19], s74 offen  lds\n\t" \
    "s_add_u32 m0, s45, 135168\n\t" \
    "s_add_u32 s74, s68, 16384\n\t" \
    "buffer_load_dwordx4 v160, s[16:19], s74 offen  lds\n\t" \
    "s_add_u32 m0, s45, 136192\n\t" \
    "s_add_u32 s74, s68, 20480\n\t" \
    "buffer_load_dwordx4 v161, s[16:19], s74 offen  lds\n\t" \
    "s_add_u32 m0, s45, 137216\n\t" \
    "s_add_u32 s74, s68, 24576\n\t" \
    "buffer_load_dwordx4 v160, s[16:19], s74 offen  lds\n\t" \
    "s_add_u32 m0, s45, 138240\n\t" \
    "s_add_u32 s74, s68, 28672\n\t" \
    "buffer_load_dwordx4 v161, s[16:19], s74 offen  lds\n\t" \
    "s_add_u32 m0, s45, 98304\n\t" \
    "s_add_u32 s74, s69, 0\n\t" \
    "buffer_load_dwordx4 v160, s[16:19], s74 offen  lds\n\t" \
    "s_add_u32 m0, s45, 99328\n\t" \
    "s_add_u32 s74, s69, 4096\n\t" \
    "buffer_load_dwordx4 v161, s[16:19], s74 offen  lds\n\t" \
    "s_add_u32 m0, s45, 100352\n\t" \
    "s_add_u32 s74, s69, 8192\n\t" \
    "buffer_load_dwordx4 v160, s[16:19], s74 offen  lds\n\t" \
    "s_add_u32 m0, s45, 101376\n\t" \
    "s_add_u32 s74, s69, 12288\n\t" \
    "buffer_load_dwordx4 v161, s[16:19], s74 offen  lds\n\t" \
    "s_add_u32 m0, s45, 102400\n\t" \
    "s_add_u32 s74, s69, 16384\n\t" \
    "buffer_load_dwordx4 v160, s[16:19], s74 offen  lds\n\t" \
    "s_add_u32 m0, s45, 103424\n\t" \
    "s_add_u32 s74, s69, 20480\n\t" \
    "buffer_load_dwordx4 v161, s[16:19], s74 offen  lds\n\t" \
    "s_add_u32 m0, s45, 104448\n\t" \
    "s_add_u32 s74, s69, 24576\n\t" \
    "buffer_load_dwordx4 v160, s[16:19], s74 offen  lds\n\t" \
    "s_add_u32 m0, s45, 105472\n\t" \
    "s_add_u32 s74, s69, 28672\n\t" \
    "buffer_load_dwordx4 v161, s[16:19], s74 offen  lds\n\t" \
    "global_store_dword v236, v176, s[56:57] sc1\n\t" \
    "s_waitcnt vmcnt(9)\n\t" \
    "v_add_u32 v177, 131072, v162\n\t" \
    "v_add_u32 v178, 131072, v163\n\t" \
    "ds_read_b128 a[0:3], v177 offset:0\n\t" \
    "ds_read_b128 a[4:7], v178 offset:0\n\t" \
    "ds_read_b128 a[8:11], v177 offset:2048\n\t" \
    "ds_read_b128 a[12:15], v178 offset:2048\n\t" \
    "ds_read_b128 a[16:19], v177 offset:4096\n\t" \
    "ds_read_b128 a[20:23], v178 offset:4096\n\t" \
    "ds_read_b128 a[24:27], v177 offset:6144\n\t" \
    "ds_read_b128 a[28:31], v178 offset:6144\n\t" \
    "s_waitcnt lgkmcnt(0)\n\t" \
    "s_barrier\n\t" \
    "s_cmp_eq_u32 s47, 0\n\t" \
    "s_cselect_b32 s87, s66, s67\n\t" \
    "s_mov_b32 s85, s87\n\t" \
    "s_add_u32 m0, s44, 32768\n\t" \
    "s_add_u32 s74, s85, 0\n\t" \
    "buffer_load_dwordx4 v232, s[8:11], s74 offen  lds\n\t" \
    "s_add_u32 m0, s44, 36864\n\t" \
    "s_add_u32 s74, s85, 524288\n\t" \
    "buffer_load_dwordx4 v232, s[8:11], s74 offen  lds\n\t" \
    "s_add_u32 m0, s44, 40960\n\t" \
    "s_add_u32 s74, s85, 1048576\n\t" \
    "buffer_load_dwordx4 v232, s[8:11], s74 offen  lds\n\t" \
    "s_add_u32 m0, s44, 45056\n\t" \
    "s_add_u32 s74, s85, 1572864\n\t" \
    "buffer_load_dwordx4 v232, s[8:11], s74 offen  lds\n\t" \
    "s_cmp_eq_u32 s47, 0\n\t" \
    "s_cselect_b32 s87, s25, s26\n\t" \
    "s_mov_b32 s85, s87\n\t" \
    "s_add_u32 m0, s44, 65536\n\t" \
    "s_add_u32 s74, s85, 0\n\t" \
    "buffer_load_dwordx4 v232, s[8:11], s74 offen  lds\n\t" \
    "s_add_u32 m0, s44, 69632\n\t" \
    "s_add_u32 s74, s85, 524288\n\t" \
    "buffer_load_dwordx4 v232, s[8:11], s74 offen  lds\n\t" \
    "s_add_u32 m0, s44, 73728\n\t" \
    "s_add_u32 s74, s85, 1048576\n\t" \
    "buffer_load_dwordx4 v232, s[8:11], s74 offen  lds\n\t" \
    "s_add_u32 m0, s44, 77824\n\t" \
    "s_add_u32 s74, s85, 1572864\n\t" \
    "buffer_load_dwordx4 v232, s[8:11], s74 offen  lds\n\t" \
    "ds_read_b64_tr_b8 v[192:193], v224 offset:0\n\t" \
    "ds_read_b64_tr_b8 v[194:195], v224 offset:1024\n\t" \
    "ds_read_b64_tr_b8 v[196:197], v224 offset:8192\n\t" \
    "ds_read_b64_tr_b8 v[198:199], v224 offset:9216\n\t" \
    "ds_read_b64_tr_b8 v[200:201], v225 offset:0\n\t" \
    "ds_read_b64_tr_b8 v[202:203], v225 offset:1024\n\t" \
    "ds_read_b64_tr_b8 v[204:205], v225 offset:8192\n\t" \
    "ds_read_b64_tr_b8 v[206:207], v225 offset:9216\n\t" \
    "ds_read_b64_tr_b8 v[208:209], v226 offset:0\n\t" \
    "ds_read_b64_tr_b8 v[210:211], v226 offset:1024\n\t" \
    "ds_read_b64_tr_b8 v[212:213], v226 offset:8192\n\t" \
    "ds_read_b64_tr_b8 v[214:215], v226 offset:9216\n\t" \
    "s_waitcnt lgkmcnt(8)\n\t" \
    "v_mfma_f32_16x16x128_f8f6f4 v[0:3], a[0:7], v[192:199], 0 cbsz:1 blgp:1\n\t" \
    "ds_read_b64_tr_b8 v[216:217], v227 offset:0\n\t" \
    "v_mfma_f32_16x16x128_f8f6f4 v[4:7], a[8:15], v[192:199], 0 cbsz:1 blgp:1\n\t" \
    "ds_read_b64_tr_b8 v[218:219], v227 offset:1024\n\t" \
    "v_mfma_f32_16x16x128_f8f6f4 v[8:11], a[16:23], v[192:199], 0 cbsz:1 blgp:1\n\t" \
    "ds_read_b64_tr_b8 v[220:221], v227 offset:8192\n\t" \
    "s_waitcnt vmcnt(9)\n\t" \
    "v_add_u32 v177, 98304, v162\n\t" \
    "v_add_u32 v178, 98304, v163\n\t" \
    "ds_read_b128 a[32:35], v177 offset:0\n\t" \
    "ds_read_b128 a[36:39], v178 offset:0\n\t" \
    "v_mfma_f32_16x16x128_f8f6f4 v[12:15], a[24:31], v[192:199], 0 cbsz:1 blgp:1\n\t" \
    "ds_read_b64_tr_b8 v[222:223], v227 offset:9216\n\t" \
    "ds_read_b128 a[40:43], v177 offset:2048\n\t" \
    "ds_read_b128 a[44:47], v178 offset:2048\n\t" \
    "s_waitcnt lgkmcnt(12)\n\t" \
    "v_mfma_f32_16x16x128_f8f6f4 v[16:19], a[0:7], v[200:207], 0 cbsz:1 blgp:1\n\t" \
    "ds_read_b64_tr_b8 v[192:193], v228 offset:0\n\t" \
    "ds_read_b128 a[48:51], v177 offset:4096\n\t" \
    "ds_read_b128 a[52:55], v178 offset:4096\n\t" \
    "v_mfma_f32_16x16x128_f8f6f4 v[20:23], a[8:15], v[200:207], 0 cbsz:1 blgp:1\n\t" \
    "ds_read_b64_tr_b8 v[194:195], v228 offset:1024\n\t" \
    "ds_read_b128 a[56:59], v177 offset:6144\n\t" \
    "ds_read_b128 a[60:63], v178 offset:6144\n\t" \
    "v_mfma_f32_16x16x128_f8f6f4 v[24:27], a[16:23], v[200:207], 0 cbsz:1 blgp:1\n\t" \
    "ds_read_b64_tr_b8 v[196:197], v228 offset:8192\n\t" \
    "v_mfma_f32_16x16x128_f8f6f4 v[28:31], a[24:31], v[200:207], 0 cbsz:1 blgp:1\n\t" \
    "ds_read_b64_tr_b8 v[198:199], v228 offset:9216\n\t" \
    "s_waitcnt lgkmcnt(15)\n\t" \
    "v_mfma_f32_16x16x128_f8f6f4 v[32:35], a[0:7], v[208:215], 0 cbsz:1 blgp:1\n\t" \
    "ds_read_b64_tr_b8 v[200:201], v229 offset:0\n\t" \
    "v_mfma_f32_16x16x128_f8f6f4 v[36:39], a[8:15], v[208:215], 0 cbsz:1 blgp:1\n\t" \
    "ds_read_b64_tr_b8 v[202:203], v229 offset:1024\n\t" \
    "v_mfma_f32_16x16x128_f8f6f4 v[40:43], a[16:23], v[208:215], 0 cbsz:1 blgp:1\n\t" \
    "ds_read_b64_tr_b8 v[204:205], v229 offset:8192\n\t" \
    "v_mfma_f32_16x16x128_f8f6f4 v[44:47], a[24:31], v[208:215], 0 cbsz:1 blgp:1\n\t" \
    "ds_read_b64_tr_b8 v[206:207], v229 offset:9216\n\t" \
    "s_waitcnt lgkmcnt(14)\n\t" \
    "v_mfma_f32_16x16x128_f8f6f4 v[48:51], a[0:7], v[216:223], 0 cbsz:1 blgp:1\n\t" \
    "ds_read_b64_tr_b8 v[208:209], v230 offset:0\n\t" \
    "v_mfma_f32_16x16x128_f8f6f4 v[52:55], a[8:15], v[216:223], 0 cbsz:1 blgp:1\n\t" \
    "ds_read_b64_tr_b8 v[210:211], v230 offset:1024\n\t" \
    "v_mfma_f32_16x16x128_f8f6f4 v[56:59], a[16:23], v[216:223], 0 cbsz:1 blgp:1\n\t" \
    "ds_read_b64_tr_b8 v[212:213], v230 offset:8192\n\t" \
    "v_mfma_f32_16x16x128_f8f6f4 v[60:63], a[24:31], v[216:223], 0 cbsz:1 blgp:1\n\t" \
    "ds_read_b64_tr_b8 v[214:215], v230 offset:9216\n\t" \
    "s_waitcnt lgkmcnt(8)\n\t" \
    "v_mfma_f32_16x16x128_f8f6f4 v[64:67], a[0:7], v[192:199], 0 cbsz:1 blgp:1\n\t" \
    "ds_read_b64_tr_b8 v[216:217], v231 offset:0\n\t" \
    "v_mfma_f32_16x16x128_f8f6f4 v[68:71], a[8:15], v[192:199], 0 cbsz:1 blgp:1\n\t" \
    "ds_read_b64_tr_b8 v[218:219], v231 offset:1024\n\t" \
    "v_mfma_f32_16x16x128_f8f6f4 v[72:75], a[16:23], v[192:199], 0 cbsz:1 blgp:1\n\t" \
    "ds_read_b64_tr_b8 v[220:221], v231 offset:8192\n\t" \
    "v_mfma_f32_16x16x128_f8f6f4 v[76:79], a[24:31], v[192:199], 0 cbsz:1 blgp:1\n\t" \
    "ds_read_b64_tr_b8 v[222:223], v231 offset:9216\n\t" \
    "s_waitcnt lgkmcnt(8)\n\t" \
    "v_mfma_f32_16x16x128_f8f6f4 v[80:83], a[0:7], v[200:207], 0 cbsz:1 blgp:1\n\t" \
    "s_barrier\n\t" \
    "ds_read_b64_tr_b8 v[192:193], v224 offset:16384\n\t" \
    "v_mfma_f32_16x16x128_f8f6f4 v[84:87], a[8:15], v[200:207], 0 cbsz:1 blgp:1\n\t" \
    "ds_read_b64_tr_b8 v[194:195], v224 offset:17408\n\t" \
    "v_mfma_f32_16x16x128_f8f6f4 v[88:91], a[16:23], v[200:207], 0 cbsz:1 blgp:1\n\t" \
    "ds_read_b64_tr_b8 v[196:197], v224 offset:24576\n\t" \
    "v_mfma_f32_16x16x128_f8f6f4 v[92:95], a[24:31], v[200:207], 0 cbsz:1 blgp:1\n\t" \
    "ds_read_b64_tr_b8 v[198:199], v224 offset:25600\n\t" \
    "s_waitcnt lgkmcnt(8)\n\t" \
    "v_mfma_f32_16x16x128_f8f6f4 v[96:99], a[0:7], v[208:215], 0 cbsz:1 blgp:1\n\t" \
    "s_cmp_eq_u32 s47, 1\n\t" \
    "s_cselect_b32 s87, s66, s67\n\t" \
    "s_mov_b32 s86, s87\n\t" \
    "s_add_u32 m0, s46, 32768\n\t" \
    "s_add_u32 s74, s86, 0\n\t" \
    "buffer_load_dwordx4 v232, s[8:11], s74 offen  lds\n\t" \
    "ds_read_b64_tr_b8 v[200:201], v225 offset:16384\n\t" \
    "v_mfma_f32_16x16x128_f8f6f4 v[100:103], a[8:15], v[208:215], 0 cbsz:1 blgp:1\n\t" \
    "ds_read_b64_tr_b8 v[202:203], v225 offset:17408\n\t" \
    "v_mfma_f32_16x16x128_f8f6f4 v[104:107], a[16:23], v[208:215], 0 cbsz:1 blgp:1\n\t" \
    "s_add_u32 m0, s46, 36864\n\t" \
    "s_add_u32 s74, s86, 524288\n\t" \
    "buffer_load_dwordx4 v232, s[8:11], s74 offen  lds\n\t" \
    "ds_read_b64_tr_b8 v[204:205], v225 offset:24576\n\t" \
    "v_mfma_f32_16x16x128_f8f6f4 v[108:111], a[24:31], v[208:215], 0 cbsz:1 blgp:1\n\t" \
    "ds_read_b64_tr_b8 v[206:207], v225 offset:25600\n\t" \
    "s_waitcnt lgkmcnt(8)\n\t" \
    "v_mfma_f32_16x16x128_f8f6f4 v[112:115], a[0:7], v[216:223], 0 cbsz:1 blgp:1\n\t" \
    "s_add_u32 m0, s46, 40960\n\t" \
    "s_add_u32 s74, s86, 1048576\n\t" \
    "buffer_load_dwordx4 v232, s[8:11], s74 offen  lds\n\t" \
    "ds_read_b64_tr_b8 v[208:209], v226 offset:16384\n\t" \
    "v_mfma_f32_16x16x128_f8f6f4 v[116:119], a[8:15], v[216:223], 0 cbsz:1 blgp:1\n\t" \
    "ds_read_b64_tr_b8 v[210:211], v226 offset:17408\n\t" \
    "v_mfma_f32_16x16x128_f8f6f4 v[120:123], a[16:23], v[216:223], 0 cbsz:1 blgp:1\n\t" \
    "ds_read_b64_tr_b8 v[212:213], v226 offset:24576\n\t" \
    "v_mfma_f32_16x16x128_f8f6f4 v[124:127], a[24:31], v[216:223], 0 cbsz:1 blgp:1\n\t" \
    "s_add_u32 m0, s46, 45056\n\t" \
    "s_add_u32 s74, s86, 1572864\n\t" \
    "buffer_load_dwordx4 v232, s[8:11], s74 offen  lds\n\t" \
    "ds_read_b64_tr_b8 v[214:215], v226 offset:25600\n\t" \
    "s_waitcnt lgkmcnt(8)\n\t" \
    "v_mfma_f32_16x16x128_f8f6f4 v[0:3], a[32:39], v[192:199], v[0:3] cbsz:1 blgp:1\n\t" \
    "ds_read_b64_tr_b8 v[216:217], v227 offset:16384\n\t" \
    "v_mfma_f32_16x16x128_f8f6f4 v[4:7], a[40:47], v[192:199], v[4:7] cbsz:1 blgp:1\n\t" \
    "s_add_u32 m0, s45, 131072\n\t" \
    "s_add_u32 s74, s70, 0\n\t" \
    "buffer_load_dwordx4 v160, s[16:19], s74 offen  lds\n\t" \
    "ds_read_b64_tr_b8 v[218:219], v227 offset:17408\n\t" \
    "v_mfma_f32_16x16x128_f8f6f4 v[8:11], a[48:55], v[192:199], v[8:11] cbsz:1 blgp:1\n\t" \
    "ds_read_b64_tr_b8 v[220:221], v227 offset:24576\n\t" \
    "v_mfma_f32_16x16x128_f8f6f4 v[12:15], a[56:63], v[192:199], v[12:15] cbsz:1 blgp:1\n\t" \
    "s_add_u32 m0, s45, 132096\n\t" \
    "s_add_u32 s74, s70, 4096\n\t" \
    "buffer_load_dwordx4 v161, s[16:19], s74 offen  lds\n\t" \
    "ds_read_b64_tr_b8 v[222:223], v227 offset:25600\n\t" \
    "s_waitcnt lgkmcnt(8)\n\t" \
    "v_mfma_f32_16x16x128_f8f6f4 v[16:19], a[32:39], v[200:207], v[16:19] cbsz:1 blgp:1\n\t" \
    "ds_read_b64_tr_b8 v[192:193], v228 offset:16384\n\t" \
    "v_mfma_f32_16x16x128_f8f6f4 v[20:23], a[40:47], v[200:207], v[20:23] cbsz:1 blgp:1\n\t" \
    "ds_read_b64_tr_b8 v[194:195], v228 offset:17408\n\t" \
    "v_mfma_f32_16x16x128_f8f6f4 v[24:27], a[48:55], v[200:207], v[24:27] cbsz:1 blgp:1\n\t" \
    "s_add_u32 m0, s45, 133120\n\t" \
    "s_add_u32 s74, s70, 8192\n\t" \
    "buffer_load_dwordx4 v160, s[16:19], s74 offen  lds\n\t" \
    "ds_read_b64_tr_b8 v[196:197], v228 offset:24576\n\t" \
    "v_mfma_f32_16x16x128_f8f6f4 v[28:31], a[56:63], v[200:207], v[28:31] cbsz:1 blgp:1\n\t" \
    "ds_read_b64_tr_b8 v[198:199], v228 offset:25600\n\t" \
    "s_waitcnt lgkmcnt(8)\n\t" \
    "v_mfma_f32_16x16x128_f8f6f4 v[32:35], a[32:39], v[208:215], v[32:35] cbsz:1 blgp:1\n\t" \
    "s_add_u32 m0, s45, 134144\n\t" \
    "s_add_u32 s74, s70, 12288\n\t" \
    "buffer_load_dwordx4 v161, s[16:19], s74 offen  lds\n\t" \
    "ds_read_b64_tr_b8 v[200:201], v229 offset:16384\n\t" \
    "v_mfma_f32_16x16x128_f8f6f4 v[36:39], a[40:47], v[208:215], v[36:39] cbsz:1 blgp:1\n\t" \
    "ds_read_b64_tr_b8 v[202:203], v229 offset:17408\n\t" \
    "v_mfma_f32_16x16x128_f8f6f4 v[40:43], a[48:55], v[208:215], v[40:43] cbsz:1 blgp:1\n\t" \
    "s_add_u32 m0, s45, 135168\n\t" \
    "s_add_u32 s74, s70, 16384\n\t" \
    "buffer_load_dwordx4 v160, s[16:19], s74 offen  lds\n\t" \
    "ds_read_b64_tr_b8 v[204:205], v229 offset:24576\n\t" \
    "v_mfma_f32_16x16x128_f8f6f4 v[44:47], a[56:63], v[208:215], v[44:47] cbsz:1 blgp:1\n\t" \
    "ds_read_b64_tr_b8 v[206:207], v229 offset:25600\n\t" \
    "s_waitcnt lgkmcnt(8)\n\t" \
    "v_mfma_f32_16x16x128_f8f6f4 v[48:51], a[32:39], v[216:223], v[48:51] cbsz:1 blgp:1\n\t" \
    "ds_read_b64_tr_b8 v[208:209], v230 offset:16384\n\t" \
    "v_mfma_f32_16x16x128_f8f6f4 v[52:55], a[40:47], v[216:223], v[52:55] cbsz:1 blgp:1\n\t" \
    "s_add_u32 m0, s45, 136192\n\t" \
    "s_add_u32 s74, s70, 20480\n\t" \
    "buffer_load_dwordx4 v161, s[16:19], s74 offen  lds\n\t" \
    "ds_read_b64_tr_b8 v[210:211], v230 offset:17408\n\t" \
    "v_mfma_f32_16x16x128_f8f6f4 v[56:59], a[48:55], v[216:223], v[56:59] cbsz:1 blgp:1\n\t" \
    "ds_read_b64_tr_b8 v[212:213], v230 offset:24576\n\t" \
    "v_mfma_f32_16x16x128_f8f6f4 v[60:63], a[56:63], v[216:223], v[60:63] cbsz:1 blgp:1\n\t" \
    "s_add_u32 m0, s45, 137216\n\t" \
    "s_add_u32 s74, s70, 24576\n\t" \
    "buffer_load_dwordx4 v160, s[16:19], s74 offen  lds\n\t" \
    "ds_read_b64_tr_b8 v[214:215], v230 offset:25600\n\t" \
    "s_waitcnt lgkmcnt(8)\n\t" \
    "v_mfma_f32_16x16x128_f8f6f4 v[64:67], a[32:39], v[192:199], v[64:67] cbsz:1 blgp:1\n\t" \
    "ds_read_b64_tr_b8 v[216:217], v231 offset:16384\n\t" \
    "v_mfma_f32_16x16x128_f8f6f4 v[68:71], a[40:47], v[192:199], v[68:71] cbsz:1 blgp:1\n\t" \
    "s_add_u32 m0, s45, 138240\n\t" \
    "s_add_u32 s74, s70, 28672\n\t" \
    "buffer_load_dwordx4 v161, s[16:19], s74 offen  lds\n\t" \
    "ds_read_b64_tr_b8 v[218:219], v231 offset:17408\n\t" \
    "v_mfma_f32_16x16x128_f8f6f4 v[72:75], a[48:55], v[192:199], v[72:75] cbsz:1 blgp:1\n\t" \
    "ds_read_b64_tr_b8 v[220:221], v231 offset:24576\n\t" \
    "v_mfma_f32_16x16x128_f8f6f4 v[76:79], a[56:63], v[192:199], v[76:79] cbsz:1 blgp:1\n\t" \
    "ds_read_b64_tr_b8 v[222:223], v231 offset:25600\n\t" \
    "s_waitcnt lgkmcnt(8)\n\t" \
    "v_mfma_f32_16x16x128_f8f6f4 v[80:83], a[32:39], v[200:207], v[80:83] cbsz:1 blgp:1\n\t" \
    "s_waitcnt vmcnt(16)\n\t" \
    "s_waitcnt vmcnt(8)\n\t" \
    "s_barrier\n\t" \
    "s_add_u32 m0, s45, 98304\n\t" \
    "s_add_u32 s74, s71, 0\n\t" \
    "buffer_load_dwordx4 v160, s[16:19], s74 offen  lds\n\t" \
    "ds_read_b64_tr_b8 v[192:193], v224 offset:32768\n\t" \
    "v_mfma_f32_16x16x128_f8f6f4 v[84:87], a[40:47], v[200:207], v[84:87] cbsz:1 blgp:1\n\t" \
    "ds_read_b64_tr_b8 v[194:195], v224 offset:33792\n\t" \
    "v_mfma_f32_16x16x128_f8f6f4 v[88:91], a[48:55], v[200:207], v[88:91] cbsz:1 blgp:1\n\t" \
    "s_add_u32 m0, s45, 99328\n\t" \
    "s_add_u32 s74, s71, 4096\n\t" \
    "buffer_load_dwordx4 v161, s[16:19], s74 offen  lds\n\t" \
    "ds_read_b64_tr_b8 v[196:197], v224 offset:40960\n\t" \
    "v_mfma_f32_16x16x128_f8f6f4 v[92:95], a[56:63], v[200:207], v[92:95] cbsz:1 blgp:1\n\t" \
    "ds_read_b64_tr_b8 v[198:199], v224 offset:41984\n\t" \
    "s_waitcnt lgkmcnt(8)\n\t" \
    "v_mfma_f32_16x16x128_f8f6f4 v[96:99], a[32:39], v[208:215], v[96:99] cbsz:1 blgp:1\n\t" \
    "s_add_u32 m0, s45, 100352\n\t" \
    "s_add_u32 s74, s71, 8192\n\t" \
    "buffer_load_dwordx4 v160, s[16:19], s74 offen  lds\n\t" \
    "ds_read_b64_tr_b8 v[200:201], v225 offset:32768\n\t" \
    "v_mfma_f32_16x16x128_f8f6f4 v[100:103], a[40:47], v[208:215], v[100:103] cbsz:1 blgp:1\n\t" \
    "ds_read_b64_tr_b8 v[202:203], v225 offset:33792\n\t" \
    "s_waitcnt vmcnt(3)\n\t" \
    "v_add_u32 v177, 131072, v162\n\t" \
    "v_add_u32 v178, 131072, v163\n\t" \
    "ds_read_b128 a[64:67], v177 offset:0\n\t" \
    "ds_read_b128 a[68:71], v178 offset:0\n\t" \
    "v_mfma_f32_16x16x128_f8f6f4 v[104:107], a[48:55], v[208:215], v[104:107] cbsz:1 blgp:1\n\t" \
    "ds_read_b64_tr_b8 v[204:205], v225 offset:40960\n\t" \
    "ds_read_b128 a[72:75], v177 offset:2048\n\t" \
    "ds_read_b128 a[76:79], v178 offset:2048\n\t" \
    "v_mfma_f32_16x16x128_f8f6f4 v[108:111], a[56:63], v[208:215], v[108:111] cbsz:1 blgp:1\n\t" \
    "s_add_u32 m0, s45, 101376\n\t" \
    "s_add_u32 s74, s71, 12288\n\t" \
    "buffer_load_dwordx4 v161, s[16:19], s74 offen  lds\n\t" \
    "ds_read_b64_tr_b8 v[206:207], v225 offset:41984\n\t" \
    "ds_read_b128 a[80:83], v177 offset:4096\n\t" \
    "ds_read_b128 a[84:87], v178 offset:4096\n\t" \
    "s_waitcnt lgkmcnt(14)\n\t" \
    "v_mfma_f32_16x16x128_f8f6f4 v[112:115], a[32:39], v[216:223], v[112:115] cbsz:1 blgp:1\n\t" \
    "ds_read_b64_tr_b8 v[208:209], v226 offset:32768\n\t" \
    "ds_read_b128 a[88:91], v177 offset:6144\n\t" \
    "ds_read_b128 a[92:95], v178 offset:6144\n\t" \
    "v_mfma_f32_16x16x128_f8f6f4 v[116:119], a[40:47], v[216:223], v[116:119] cbsz:1 blgp:1\n\t" \
    "s_add_u32 m0, s45, 102400\n\t" \
    "s_add_u32 s74, s71, 16384\n\t" \
    "buffer_load_dwordx4 v160, s[16:19], s74 offen  lds\n\t" \
    "ds_read_b64_tr_b8 v[210:211], v226 offset:33792\n\t" \
    "v_mfma_f32_16x16x128_f8f6f4 v[120:123], a[48:55], v[216:223], v[120:123] cbsz:1 blgp:1\n\t" \
    "ds_read_b64_tr_b8 v[212:213], v226 offset:40960\n\t" \
    "v_mfma_f32_16x16x128_f8f6f4 v[124:127], a[56:63], v[216:223], v[124:127] cbsz:1 blgp:1\n\t" \
    "s_add_u32 m0, s45, 103424\n\t" \
    "s_add_u32 s74, s71, 20480\n\t" \
    "buffer_load_dwordx4 v161, s[16:19], s74 offen  lds\n\t" \
    "ds_read_b64_tr_b8 v[214:215], v226 offset:41984\n\t" \
    "s_waitcnt lgkmcnt(3)\n\t" \
    "v_mfma_f32_16x16x128_f8f6f4 v[0:3], a[64:71], v[192:199], v[0:3] cbsz:1 blgp:1\n\t" \
    "ds_read_b64_tr_b8 v[216:217], v227 offset:32768\n\t" \
    "v_mfma_f32_16x16x128_f8f6f4 v[4:7], a[72:79], v[192:199], v[4:7] cbsz:1 blgp:1\n\t" \
    "ds_read_b64_tr_b8 v[218:219], v227 offset:33792\n\t" \
    "v_mfma_f32_16x16x128_f8f6f4 v[8:11], a[80:87], v[192:199], v[8:11] cbsz:1 blgp:1\n\t" \
    "s_add_u32 m0, s45, 104448\n\t" \
    "s_add_u32 s74, s71, 24576\n\t" \
    "buffer_load_dwordx4 v160, s[16:19], s74 offen  lds\n\t" \
    "ds_read_b64_tr_b8 v[220:221], v227 offset:40960\n\t" \
    "v_mfma_f32_16x16x128_f8f6f4 v[12:15], a[88:95], v[192:199], v[12:15] cbsz:1 blgp:1\n\t" \
    "ds_read_b64_tr_b8 v[222:223], v227 offset:41984\n\t" \
    "v_mfma_f32_16x16x128_f8f6f4 v[16:19], a[64:71], v[200:207], v[16:19] cbsz:1 blgp:1\n\t" \
    "s_add_u32 m0, s45, 105472\n\t" \
    "s_add_u32 s74, s71, 28672\n\t" \
    "buffer_load_dwordx4 v161, s[16:19], s74 offen  lds\n\t" \
    "ds_read_b64_tr_b8 v[192:193], v228 offset:32768\n\t" \
    "v_mfma_f32_16x16x128_f8f6f4 v[20:23], a[72:79], v[200:207], v[20:23] cbsz:1 blgp:1\n\t" \
    "ds_read_b64_tr_b8 v[194:195], v228 offset:33792\n\t" \
    "v_mfma_f32_16x16x128_f8f6f4 v[24:27], a[80:87], v[200:207], v[24:27] cbsz:1 blgp:1\n\t" \
    "s_cmp_eq_u32 s47, 1\n\t" \
    "s_cselect_b32 s87, s25, s26\n\t" \
    "s_mov_b32 s86, s87\n\t" \
    "s_add_u32 m0, s46, 65536\n\t" \
    "s_add_u32 s74, s86, 0\n\t" \
    "buffer_load_dwordx4 v232, s[8:11], s74 offen  lds\n\t" \
    "ds_read_b64_tr_b8 v[196:197], v228 offset:40960\n\t" \
    "v_mfma_f32_16x16x128_f8f6f4 v[28:31], a[88:95], v[200:207], v[28:31] cbsz:1 blgp:1\n\t" \
    "ds_read_b64_tr_b8 v[198:199], v228 offset:41984\n\t" \
    "s_waitcnt lgkmcnt(8)\n\t" \
    "v_mfma_f32_16x16x128_f8f6f4 v[32:35], a[64:71], v[208:215], v[32:35] cbsz:1 blgp:1\n\t" \
    "ds_read_b64_tr_b8 v[200:201], v229 offset:32768\n\t" \
    "v_mfma_f32_16x16x128_f8f6f4 v[36:39], a[72:79], v[208:215], v[36:39] cbsz:1 blgp:1\n\t" \
    "s_add_u32 m0, s46, 69632\n\t" \
    "s_add_u32 s74, s86, 524288\n\t" \
    "buffer_load_dwordx4 v232, s[8:11], s74 offen  lds\n\t" \
    "ds_read_b64_tr_b8 v[202:203], v229 offset:33792\n\t" \
    "v_mfma_f32_16x16x128_f8f6f4 v[40:43], a[80:87], v[208:215], v[40:43] cbsz:1 blgp:1\n\t" \
    "ds_read_b64_tr_b8 v[204:205], v229 offset:40960\n\t" \
    "v_mfma_f32_16x16x128_f8f6f4 v[44:47], a[88:95], v[208:215], v[44:47] cbsz:1 blgp:1\n\t" \
    "s_add_u32 m0, s46, 73728\n\t" \
    "s_add_u32 s74, s86, 1048576\n\t" \
    "buffer_load_dwordx4 v232, s[8:11], s74 offen  lds\n\t" \
    "ds_read_b64_tr_b8 v[206:207], v229 offset:41984\n\t" \
    "s_waitcnt lgkmcnt(8)\n\t" \
    "v_mfma_f32_16x16x128_f8f6f4 v[48:51], a[64:71], v[216:223], v[48:51] cbsz:1 blgp:1\n\t" \
    "ds_read_b64_tr_b8 v[208:209], v230 offset:32768\n\t" \
    "v_mfma_f32_16x16x128_f8f6f4 v[52:55], a[72:79], v[216:223], v[52:55] cbsz:1 blgp:1\n\t" \
    "s_add_u32 m0, s46, 77824\n\t" \
    "s_add_u32 s74, s86, 1572864\n\t" \
    "buffer_load_dwordx4 v232, s[8:11], s74 offen  lds\n\t" \
    "ds_read_b64_tr_b8 v[210:211], v230 offset:33792\n\t" \
    "v_mfma_f32_16x16x128_f8f6f4 v[56:59], a[80:87], v[216:223], v[56:59] cbsz:1 blgp:1\n\t" \
    "ds_read_b64_tr_b8 v[212:213], v230 offset:40960\n\t" \
    "v_mfma_f32_16x16x128_f8f6f4 v[60:63], a[88:95], v[216:223], v[60:63] cbsz:1 blgp:1\n\t" \
    "ds_read_b64_tr_b8 v[214:215], v230 offset:41984\n\t" \
    "s_waitcnt lgkmcnt(8)\n\t" \
    "v_mfma_f32_16x16x128_f8f6f4 v[64:67], a[64:71], v[192:199], v[64:67] cbsz:1 blgp:1\n\t" \
    "ds_read_b64_tr_b8 v[216:217], v231 offset:32768\n\t" \
    "s_waitcnt vmcnt(4)\n\t" \
    "v_add_u32 v177, 98304, v162\n\t" \
    "v_add_u32 v178, 98304, v163\n\t" \
    "ds_read_b128 a[96:99], v177 offset:0\n\t" \
    "ds_read_b128 a[100:103], v178 offset:0\n\t" \
    "v_mfma_f32_16x16x128_f8f6f4 v[68:71], a[72:79], v[192:199], v[68:71] cbsz:1 blgp:1\n\t" \
    "ds_read_b64_tr_b8 v[218:219], v231 offset:33792\n\t" \
    "ds_read_b128 a[104:107], v177 offset:2048\n\t" \
    "ds_read_b128 a[108:111], v178 offset:2048\n\t" \
    "v_mfma_f32_16x16x128_f8f6f4 v[72:75], a[80:87], v[192:199], v[72:75] cbsz:1 blgp:1\n\t" \
    "ds_read_b64_tr_b8 v[220:221], v231 offset:40960\n\t" \
    "ds_read_b128 a[112:115], v177 offset:4096\n\t" \
    "ds_read_b128 a[116:119], v178 offset:4096\n\t" \
    "v_mfma_f32_16x16x128_f8f6f4 v[76:79], a[88:95], v[192:199], v[76:79] cbsz:1 blgp:1\n\t" \
    "ds_read_b64_tr_b8 v[222:223], v231 offset:41984\n\t" \
    "ds_read_b128 a[120:123], v177 offset:6144\n\t" \
    "ds_read_b128 a[124:127], v178 offset:6144\n\t" \
    "s_waitcnt lgkmcnt(15)\n\t" \
    "v_mfma_f32_16x16x128_f8f6f4 v[80:83], a[64:71], v[200:207], v[80:83] cbsz:1 blgp:1\n\t" \
    "s_barrier\n\t" \
    "ds_read_b64_tr_b8 v[192:193], v224 offset:49152\n\t" \
    "v_mfma_f32_16x16x128_f8f6f4 v[84:87], a[72:79], v[200:207], v[84:87] cbsz:1 blgp:1\n\t" \
    "ds_read_b64_tr_b8 v[194:195], v224 offset:50176\n\t" \
    "v_mfma_f32_16x16x128_f8f6f4 v[88:91], a[80:87], v[200:207], v[88:91] cbsz:1 blgp:1\n\t" \
    "s_waitcnt lgkmcnt(2)\n\t" \
    "s_barrier\n\t" \
    "ds_read_b64_tr_b8 v[196:197], v224 offset:57344\n\t" \
    "v_mfma_f32_16x16x128_f8f6f4 v[92:95], a[88:95], v[200:207], v[92:95] cbsz:1 blgp:1\n\t" \
    "global_load_dword v248, v235, s[54:55] offset:0 sc0 sc1\n\t" \
    "ds_read_b64_tr_b8 v[198:199], v224 offset:58368\n\t" \
    "v_mfma_f32_16x16x128_f8f6f4 v[96:99], a[64:71], v[208:215], v[96:99] cbsz:1 blgp:1\n\t" \
    "s_cmp_eq_u32 s79, 1\n\t" \
    "s_cbranch_scc1 L_pay1_%=\n\t" \
    "s_mov_b64 exec, 1\n\t" \
    "global_store_dword v236, v236, s[56:57] offset:32\n\t" \
    "global_store_dword v236, v236, s[56:57] offset:32\n\t" \
    "s_mov_b64 exec, -1\n\t" \
    "s_branch L_paye1_%=\n\t" \
    "L_pay1_%=:\n\t" \
    "global_store_dwordx4 v237, v[240:243], s[48:49] sc0 sc1\n\t" \
    "global_store_dwordx4 v237, v[244:247], s[48:49] offset:1024 sc0 sc1\n\t" \
    "L_paye1_%=:\n\t" \
    "s_cmp_eq_u32 s47, 0\n\t" \
    "s_cselect_b32 s87, s27, s28\n\t" \
    "s_mov_b32 s85, s87\n\t" \
    "s_add_u32 m0, s44, 98304\n\t" \
    "s_add_u32 s74, s85, 0\n\t" \
    "buffer_load_dwordx4 v232, s[8:11], s74 offen  lds\n\t" \
    "ds_read_b64_tr_b8 v[200:201], v225 offset:49152\n\t" \
    "v_mfma_f32_16x16x128_f8f6f4 v[100:103], a[72:79], v[208:215], v[100:103] cbsz:1 blgp:1\n\t" \
    "s_add_u32 m0, s44, 102400\n\t" \
    "s_add_u32 s74, s85, 524288\n\t" \
    "buffer_load_dwordx4 v232, s[8:11], s74 offen  lds\n\t" \
    "ds_read_b64_tr_b8 v[202:203], v225 offset:50176\n\t" \
    "v_mfma_f32_16x16x128_f8f6f4 v[104:107], a[80:87], v[208:215], v[104:107] cbsz:1 blgp:1\n\t" \
    "s_add_u32 m0, s44, 106496\n\t" \
    "s_add_u32 s74, s85, 1048576\n\t" \
    "buffer_load_dwordx4 v232, s[8:11], s74 offen  lds\n\t" \
    "ds_read_b64_tr_b8 v[204:205], v225 offset:57344\n\t" \
    "v_mfma_f32_16x16x128_f8f6f4 v[108:111], a[88:95], v[208:215], v[108:111] cbsz:1 blgp:1\n\t" \
    "s_add_u32 m0, s44, 110592\n\t" \
    "s_add_u32 s74, s85, 1572864\n\t" \
    "buffer_load_dwordx4 v232, s[8:11], s74 offen  lds\n\t" \
    "ds_read_b64_tr_b8 v[206:207], v225 offset:58368\n\t" \
    "v_mfma_f32_16x16x128_f8f6f4 v[112:115], a[64:71], v[216:223], v[112:115] cbsz:1 blgp:1\n\t" \
    "s_cmp_eq_u32 s47, 0\n\t" \
    "s_cselect_b32 s87, s29, s30\n\t" \
    "s_mov_b32 s85, s87\n\t" \
    "s_add_u32 m0, s44, 131072\n\t" \
    "s_add_u32 s74, s85, 0\n\t" \
    "buffer_load_dwordx4 v232, s[8:11], s74 offen  lds\n\t" \
    "ds_read_b64_tr_b8 v[208:209], v226 offset:49152\n\t" \
    "v_mfma_f32_16x16x128_f8f6f4 v[116:119], a[72:79], v[216:223], v[116:119] cbsz:1 blgp:1\n\t" \
    "s_add_u32 m0, s44, 135168\n\t" \
    "s_add_u32 s74, s85, 524288\n\t" \
    "buffer_load_dwordx4 v232, s[8:11], s74 offen  lds\n\t" \
    "ds_read_b64_tr_b8 v[210:211], v226 offset:50176\n\t" \
    "v_mfma_f32_16x16x128_f8f6f4 v[120:123], a[80:87], v[216:223], v[120:123] cbsz:1 blgp:1\n\t" \
    "s_add_u32 m0, s44, 139264\n\t" \
    "s_add_u32 s74, s85, 1048576\n\t" \
    "buffer_load_dwordx4 v232, s[8:11], s74 offen  lds\n\t" \
    "ds_read_b64_tr_b8 v[212:213], v226 offset:57344\n\t" \
    "v_mfma_f32_16x16x128_f8f6f4 v[124:127], a[88:95], v[216:223], v[124:127] cbsz:1 blgp:1\n\t" \
    "s_add_u32 m0, s44, 143360\n\t" \
    "s_add_u32 s74, s85, 1572864\n\t" \
    "buffer_load_dwordx4 v232, s[8:11], s74 offen  lds\n\t" \
    "ds_read_b64_tr_b8 v[214:215], v226 offset:58368\n\t" \
    "s_waitcnt lgkmcnt(8)\n\t" \
    "v_mfma_f32_16x16x128_f8f6f4 v[0:3], a[96:103], v[192:199], v[0:3] cbsz:1 blgp:1\n\t" \
    "ds_read_b64_tr_b8 v[216:217], v227 offset:49152\n\t" \
    "v_mfma_f32_16x16x128_f8f6f4 v[4:7], a[104:111], v[192:199], v[4:7] cbsz:1 blgp:1\n\t" \
    "ds_read_b64_tr_b8 v[218:219], v227 offset:50176\n\t" \
    "v_mfma_f32_16x16x128_f8f6f4 v[8:11], a[112:119], v[192:199], v[8:11] cbsz:1 blgp:1\n\t" \
    "ds_read_b64_tr_b8 v[220:221], v227 offset:57344\n\t" \
    "v_mfma_f32_16x16x128_f8f6f4 v[12:15], a[120:127], v[192:199], v[12:15] cbsz:1 blgp:1\n\t" \
    "ds_read_b64_tr_b8 v[222:223], v227 offset:58368\n\t" \
    "s_waitcnt lgkmcnt(8)\n\t" \
    "v_mfma_f32_16x16x128_f8f6f4 v[16:19], a[96:103], v[200:207], v[16:19] cbsz:1 blgp:1\n\t" \
    "ds_read_b64_tr_b8 v[192:193], v228 offset:49152\n\t" \
    "s_waitcnt vmcnt(10)\n\t" \
    "v_cmp_gt_u32 vcc, s75, v248\n\t" \
    "s_nop 0\n\t" \
    "s_cbranch_vccz L_pd0_%=\n\t" \
    "s_memrealtime s[80:81]\n\t" \
    "s_waitcnt lgkmcnt(0)\n\t" \
    "L_ps0_%=:\n\t" \
    "global_load_dword v248, v235, s[54:55] offset:0 sc0 sc1\n\t" \
    "global_load_dword v251, v236, s[58:59] sc1\n\t" \
    "s_memrealtime s[82:83]\n\t" \
    "s_waitcnt vmcnt(0) lgkmcnt(0)\n\t" \
    "v_cmp_gt_u32 vcc, s75, v248\n\t" \
    "s_nop 0\n\t" \
    "s_cbranch_vccz L_pd0_%=\n\t" \
    "v_readfirstlane_b32 s84, v251\n\t" \
    "s_sub_u32 s82, s82, s80\n\t" \
    "s_cmp_eq_u32 s84, s75\n\t" \
    "s_cbranch_scc1 L_pd0_%=\n\t" \
    "s_cmp_gt_u32 s82, 40000000\n\t" \
    "s_cbranch_scc1 L_pt0_%=\n\t" \
    "s_sleep 1\n\t" \
    "s_branch L_ps0_%=\n\t" \
    "L_pt0_%=:\n\t" \
    "v_mov_b32 v251, s75\n\t" \
    "global_store_dword v236, v251, s[58:59] sc1\n\t" \
    "s_waitcnt vmcnt(0)\n\t" \
    "L_pd0_%=:\n\t" \
    "v_mfma_f32_16x16x128_f8f6f4 v[20:23], a[104:111], v[200:207], v[20:23] cbsz:1 blgp:1\n\t" \
    "ds_read_b64_tr_b8 v[194:195], v228 offset:50176\n\t" \
    "v_mfma_f32_16x16x128_f8f6f4 v[24:27], a[112:119], v[200:207], v[24:27] cbsz:1 blgp:1\n\t" \
    "ds_read_b64_tr_b8 v[196:197], v228 offset:57344\n\t" \
    "v_mfma_f32_16x16x128_f8f6f4 v[28:31], a[120:127], v[200:207], v[28:31] cbsz:1 blgp:1\n\t" \
    "ds_read_b64_tr_b8 v[198:199], v228 offset:58368\n\t" \
    "s_waitcnt lgkmcnt(8)\n\t" \
    "v_mfma_f32_16x16x128_f8f6f4 v[32:35], a[96:103], v[208:215], v[32:35] cbsz:1 blgp:1\n\t" \
    "ds_read_b64_tr_b8 v[200:201], v229 offset:49152\n\t" \
    "v_mfma_f32_16x16x128_f8f6f4 v[36:39], a[104:111], v[208:215], v[36:39] cbsz:1 blgp:1\n\t" \
    "ds_read_b64_tr_b8 v[202:203], v229 offset:50176\n\t" \
    "v_mfma_f32_16x16x128_f8f6f4 v[40:43], a[112:119], v[208:215], v[40:43] cbsz:1 blgp:1\n\t" \
    "ds_read_b64_tr_b8 v[204:205], v229 offset:57344\n\t" \
    "v_mfma_f32_16x16x128_f8f6f4 v[44:47], a[120:127], v[208:215], v[44:47] cbsz:1 blgp:1\n\t" \
    "ds_read_b64_tr_b8 v[206:207], v229 offset:58368\n\t" \
    "s_waitcnt lgkmcnt(8)\n\t" \
    "v_mfma_f32_16x16x128_f8f6f4 v[48:51], a[96:103], v[216:223], v[48:51] cbsz:1 blgp:1\n\t" \
    "ds_read_b64_tr_b8 v[208:209], v230 offset:49152\n\t" \
    "v_mfma_f32_16x16x128_f8f6f4 v[52:55], a[104:111], v[216:223], v[52:55] cbsz:1 blgp:1\n\t" \
    "ds_read_b64_tr_b8 v[210:211], v230 offset:50176\n\t" \
    "v_mfma_f32_16x16x128_f8f6f4 v[56:59], a[112:119], v[216:223], v[56:59] cbsz:1 blgp:1\n\t" \
    "ds_read_b64_tr_b8 v[212:213], v230 offset:57344\n\t" \
    "v_mfma_f32_16x16x128_f8f6f4 v[60:63], a[120:127], v[216:223], v[60:63] cbsz:1 blgp:1\n\t" \
    "ds_read_b64_tr_b8 v[214:215], v230 offset:58368\n\t" \
    "s_waitcnt lgkmcnt(8)\n\t" \
    "v_mfma_f32_16x16x128_f8f6f4 v[64:67], a[96:103], v[192:199], v[64:67] cbsz:1 blgp:1\n\t" \
    "ds_read_b64_tr_b8 v[216:217], v231 offset:49152\n\t" \
    "v_mfma_f32_16x16x128_f8f6f4 v[68:71], a[104:111], v[192:199], v[68:71] cbsz:1 blgp:1\n\t" \
    "ds_read_b64_tr_b8 v[218:219], v231 offset:50176\n\t" \
    "v_mfma_f32_16x16x128_f8f6f4 v[72:75], a[112:119], v[192:199], v[72:75] cbsz:1 blgp:1\n\t" \
    "ds_read_b64_tr_b8 v[220:221], v231 offset:57344\n\t" \
    "v_mfma_f32_16x16x128_f8f6f4 v[76:79], a[120:127], v[192:199], v[76:79] cbsz:1 blgp:1\n\t" \
    "ds_read_b64_tr_b8 v[222:223], v231 offset:58368\n\t" \
    "s_waitcnt lgkmcnt(8)\n\t" \
    "v_mfma_f32_16x16x128_f8f6f4 v[80:83], a[96:103], v[200:207], v[80:83] cbsz:1 blgp:1\n\t" \
    "s_barrier\n\t" \
    "v_add_u32 v239, 65536, v224\n\t" \
    "ds_read_b64_tr_b8 v[192:193], v239 offset:0\n\t" \
    "v_mfma_f32_16x16x128_f8f6f4 v[84:87], a[104:111], v[200:207], v[84:87] cbsz:1 blgp:1\n\t" \
    "ds_read_b64_tr_b8 v[194:195], v239 offset:1024\n\t" \
    "v_mfma_f32_16x16x128_f8f6f4 v[88:91], a[112:119], v[200:207], v[88:91] cbsz:1 blgp:1\n\t" \
    "s_mov_b32 s72, 262144\n\t" \
    "s_mov_b32 s73, 266240\n\t" \
    "buffer_load_dwordx4 a[128:131], v233, s[12:15], s72 offen offset:0 sc0 sc1\n\t" \
    "ds_read_b64_tr_b8 v[196:197], v239 offset:8192\n\t" \
    "v_mfma_f32_16x16x128_f8f6f4 v[92:95], a[120:127], v[200:207], v[92:95] cbsz:1 blgp:1\n\t" \
    "ds_read_b64_tr_b8 v[198:199], v239 offset:9216\n\t" \
    "s_waitcnt lgkmcnt(8)\n\t" \
    "v_mfma_f32_16x16x128_f8f6f4 v[96:99], a[96:103], v[208:215], v[96:99] cbsz:1 blgp:1\n\t" \
    "buffer_load_dwordx4 a[132:135], v233, s[12:15], s72 offen offset:1024 sc0 sc1\n\t" \
    "v_add_u32 v238, 65536, v225\n\t" \
    "ds_read_b64_tr_b8 v[200:201], v238 offset:0\n\t" \
    "v_mfma_f32_16x16x128_f8f6f4 v[100:103], a[104:111], v[208:215], v[100:103] cbsz:1 blgp:1\n\t" \
    "ds_read_b64_tr_b8 v[202:203], v238 offset:1024\n\t" \
    "v_mfma_f32_16x16x128_f8f6f4 v[104:107], a[112:119], v[208:215], v[104:107] cbsz:1 blgp:1\n\t" \
    "buffer_load_dwordx4 a[136:139], v233, s[12:15], s72 offen offset:2048 sc0 sc1\n\t" \
    "ds_read_b64_tr_b8 v[204:205], v238 offset:8192\n\t" \
    "v_mfma_f32_16x16x128_f8f6f4 v[108:111], a[120:127], v[208:215], v[108:111] cbsz:1 blgp:1\n\t" \
    "buffer_load_dwordx4 a[140:143], v233, s[12:15], s72 offen offset:3072 sc0 sc1\n\t" \
    "ds_read_b64_tr_b8 v[206:207], v238 offset:9216\n\t" \
    "s_waitcnt lgkmcnt(8)\n\t" \
    "v_mfma_f32_16x16x128_f8f6f4 v[112:115], a[96:103], v[216:223], v[112:115] cbsz:1 blgp:1\n\t" \
    "buffer_load_dwordx4 a[144:147], v233, s[12:15], s73 offen offset:0 sc0 sc1\n\t" \
    "v_add_u32 v239, 65536, v226\n\t" \
    "ds_read_b64_tr_b8 v[208:209], v239 offset:0\n\t" \
    "v_mfma_f32_16x16x128_f8f6f4 v[116:119], a[104:111], v[216:223], v[116:119] cbsz:1 blgp:1\n\t" \
    "buffer_load_dwordx4 a[148:151], v233, s[12:15], s73 offen offset:1024 sc0 sc1\n\t" \
    "ds_read_b64_tr_b8 v[210:211], v239 offset:1024\n\t" \
    "v_mfma_f32_16x16x128_f8f6f4 v[120:123], a[112:119], v[216:223], v[120:123] cbsz:1 blgp:1\n\t" \
    "buffer_load_dwordx4 a[152:155], v233, s[12:15], s73 offen offset:2048 sc0 sc1\n\t" \
    "ds_read_b64_tr_b8 v[212:213], v239 offset:8192\n\t" \
    "v_mfma_f32_16x16x128_f8f6f4 v[124:127], a[120:127], v[216:223], v[124:127] cbsz:1 blgp:1\n\t" \
    "buffer_load_dwordx4 a[156:159], v233, s[12:15], s73 offen offset:3072 sc0 sc1\n\t" \
    "ds_read_b64_tr_b8 v[214:215], v239 offset:9216\n\t" \
    "s_waitcnt vmcnt(0)\n\t" \
    "s_waitcnt lgkmcnt(8)\n\t" \
    "v_mfma_f32_16x16x128_f8f6f4 v[0:3], a[128:135], v[192:199], v[0:3] cbsz:1 blgp:1\n\t" \
    "s_cmp_eq_u32 s47, 1\n\t" \
    "s_cselect_b32 s87, s27, s28\n\t" \
    "s_mov_b32 s86, s87\n\t" \
    "s_add_u32 m0, s46, 98304\n\t" \
    "s_add_u32 s74, s86, 0\n\t" \
    "buffer_load_dwordx4 v232, s[8:11], s74 offen  lds\n\t" \
    "v_add_u32 v238, 65536, v227\n\t" \
    "ds_read_b64_tr_b8 v[216:217], v238 offset:0\n\t" \
    "v_mfma_f32_16x16x128_f8f6f4 v[4:7], a[136:143], v[192:199], v[4:7] cbsz:1 blgp:1\n\t" \
    "ds_read_b64_tr_b8 v[218:219], v238 offset:1024\n\t" \
    "v_mfma_f32_16x16x128_f8f6f4 v[8:11], a[144:151], v[192:199], v[8:11] cbsz:1 blgp:1\n\t" \
    "ds_read_b64_tr_b8 v[220:221], v238 offset:8192\n\t" \
    "v_mfma_f32_16x16x128_f8f6f4 v[12:15], a[152:159], v[192:199], v[12:15] cbsz:1 blgp:1\n\t" \
    "s_add_u32 m0, s46, 102400\n\t" \
    "s_add_u32 s74, s86, 524288\n\t" \
    "buffer_load_dwordx4 v232, s[8:11], s74 offen  lds\n\t" \
    "ds_read_b64_tr_b8 v[222:223], v238 offset:9216\n\t" \
    "s_waitcnt lgkmcnt(8)\n\t" \
    "v_mfma_f32_16x16x128_f8f6f4 v[16:19], a[128:135], v[200:207], v[16:19] cbsz:1 blgp:1\n\t" \
    "v_add_u32 v239, 65536, v228\n\t" \
    "ds_read_b64_tr_b8 v[192:193], v239 offset:0\n\t" \
    "v_mfma_f32_16x16x128_f8f6f4 v[20:23], a[136:143], v[200:207], v[20:23] cbsz:1 blgp:1\n\t" \
    "s_add_u32 m0, s46, 106496\n\t" \
    "s_add_u32 s74, s86, 1048576\n\t" \
    "buffer_load_dwordx4 v232, s[8:11], s74 offen  lds\n\t" \
    "ds_read_b64_tr_b8 v[194:195], v239 offset:1024\n\t" \
    "v_mfma_f32_16x16x128_f8f6f4 v[24:27], a[144:151], v[200:207], v[24:27] cbsz:1 blgp:1\n\t" \
    "ds_read_b64_tr_b8 v[196:197], v239 offset:8192\n\t" \
    "v_mfma_f32_16x16x128_f8f6f4 v[28:31], a[152:159], v[200:207], v[28:31] cbsz:1 blgp:1\n\t" \
    "s_add_u32 m0, s46, 110592\n\t" \
    "s_add_u32 s74, s86, 1572864\n\t" \
    "buffer_load_dwordx4 v232, s[8:11], s74 offen  lds\n\t" \
    "ds_read_b64_tr_b8 v[198:199], v239 offset:9216\n\t" \
    "s_waitcnt lgkmcnt(8)\n\t" \
    "v_mfma_f32_16x16x128_f8f6f4 v[32:35], a[128:135], v[208:215], v[32:35] cbsz:1 blgp:1\n\t" \
    "v_add_u32 v238, 65536, v229\n\t" \
    "ds_read_b64_tr_b8 v[200:201], v238 offset:0\n\t" \
    "v_mfma_f32_16x16x128_f8f6f4 v[36:39], a[136:143], v[208:215], v[36:39] cbsz:1 blgp:1\n\t" \
    "ds_read_b64_tr_b8 v[202:203], v238 offset:1024\n\t" \
    "v_mfma_f32_16x16x128_f8f6f4 v[40:43], a[144:151], v[208:215], v[40:43] cbsz:1 blgp:1\n\t" \
    "s_mov_b32 s72, 327680\n\t" \
    "s_mov_b32 s73, 331776\n\t" \
    "buffer_load_dwordx4 a[160:163], v233, s[12:15], s72 offen offset:0 sc0 sc1\n\t" \
    "ds_read_b64_tr_b8 v[204:205], v238 offset:8192\n\t" \
    "v_mfma_f32_16x16x128_f8f6f4 v[44:47], a[152:159], v[208:215], v[44:47] cbsz:1 blgp:1\n\t" \
    "ds_read_b64_tr_b8 v[206:207], v238 offset:9216\n\t" \
    "s_waitcnt lgkmcnt(8)\n\t" \
    "v_mfma_f32_16x16x128_f8f6f4 v[48:51], a[128:135], v[216:223], v[48:51] cbsz:1 blgp:1\n\t" \
    "buffer_load_dwordx4 a[164:167], v233, s[12:15], s72 offen offset:1024 sc0 sc1\n\t" \
    "v_add_u32 v239, 65536, v230\n\t" \
    "ds_read_b64_tr_b8 v[208:209], v239 offset:0\n\t" \
    "v_mfma_f32_16x16x128_f8f6f4 v[52:55], a[136:143], v[216:223], v[52:55] cbsz:1 blgp:1\n\t" \
    "ds_read_b64_tr_b8 v[210:211], v239 offset:1024\n\t" \
    "v_mfma_f32_16x16x128_f8f6f4 v[56:59], a[144:151], v[216:223], v[56:59] cbsz:1 blgp:1\n\t" \
    "buffer_load_dwordx4 a[168:171], v233, s[12:15], s72 offen offset:2048 sc0 sc1\n\t" \
    "ds_read_b64_tr_b8 v[212:213], v239 offset:8192\n\t" \
    "v_mfma_f32_16x16x128_f8f6f4 v[60:63], a[152:159], v[216:223], v[60:63] cbsz:1 blgp:1\n\t" \
    "ds_read_b64_tr_b8 v[214:215], v239 offset:9216\n\t" \
    "s_waitcnt lgkmcnt(8)\n\t" \
    "v_mfma_f32_16x16x128_f8f6f4 v[64:67], a[128:135], v[192:199], v[64:67] cbsz:1 blgp:1\n\t" \
    "v_add_u32 v238, 65536, v231\n\t" \
    "ds_read_b64_tr_b8 v[216:217], v238 offset:0\n\t" \
    "v_mfma_f32_16x16x128_f8f6f4 v[68:71], a[136:143], v[192:199], v[68:71] cbsz:1 blgp:1\n\t" \
    "buffer_load_dwordx4 a[172:175], v233, s[12:15], s72 offen offset:3072 sc0 sc1\n\t" \
    "ds_read_b64_tr_b8 v[218:219], v238 offset:1024\n\t" \
    "v_mfma_f32_16x16x128_f8f6f4 v[72:75], a[144:151], v[192:199], v[72:75] cbsz:1 blgp:1\n\t" \
    "ds_read_b64_tr_b8 v[220:221], v238 offset:8192\n\t" \
    "v_mfma_f32_16x16x128_f8f6f4 v[76:79], a[152:159], v[192:199], v[76:79] cbsz:1 blgp:1\n\t" \
    "buffer_load_dwordx4 a[176:179], v233, s[12:15], s73 offen offset:0 sc0 sc1\n\t" \
    "ds_read_b64_tr_b8 v[222:223], v238 offset:9216\n\t" \
    "s_waitcnt lgkmcnt(8)\n\t" \
    "v_mfma_f32_16x16x128_f8f6f4 v[80:83], a[128:135], v[200:207], v[80:83] cbsz:1 blgp:1\n\t" \
    "s_barrier\n\t" \
    "v_add_u32 v239, 65536, v224\n\t" \
    "ds_read_b64_tr_b8 v[192:193], v239 offset:16384\n\t" \
    "v_mfma_f32_16x16x128_f8f6f4 v[84:87], a[136:143], v[200:207], v[84:87] cbsz:1 blgp:1\n\t" \
    "buffer_load_dwordx4 a[180:183], v233, s[12:15], s73 offen offset:1024 sc0 sc1\n\t" \
    "ds_read_b64_tr_b8 v[194:195], v239 offset:17408\n\t" \
    "v_mfma_f32_16x16x128_f8f6f4 v[88:91], a[144:151], v[200:207], v[88:91] cbsz:1 blgp:1\n\t" \
    "ds_read_b64_tr_b8 v[196:197], v239 offset:24576\n\t" \
    "v_mfma_f32_16x16x128_f8f6f4 v[92:95], a[152:159], v[200:207], v[92:95] cbsz:1 blgp:1\n\t" \
    "ds_read_b64_tr_b8 v[198:199], v239 offset:25600\n\t" \
    "s_waitcnt lgkmcnt(8)\n\t" \
    "v_mfma_f32_16x16x128_f8f6f4 v[96:99], a[128:135], v[208:215], v[96:99] cbsz:1 blgp:1\n\t" \
    "buffer_load_dwordx4 a[184:187], v233, s[12:15], s73 offen offset:2048 sc0 sc1\n\t" \
    "v_add_u32 v238, 65536, v225\n\t" \
    "ds_read_b64_tr_b8 v[200:201], v238 offset:16384\n\t" \
    "v_mfma_f32_16x16x128_f8f6f4 v[100:103], a[136:143], v[208:215], v[100:103] cbsz:1 blgp:1\n\t" \
    "ds_read_b64_tr_b8 v[202:203], v238 offset:17408\n\t" \
    "v_mfma_f32_16x16x128_f8f6f4 v[104:107], a[144:151], v[208:215], v[104:107] cbsz:1 blgp:1\n\t" \
    "buffer_load_dwordx4 a[188:191], v233, s[12:15], s73 offen offset:3072 sc0 sc1\n\t" \
    "ds_read_b64_tr_b8 v[204:205], v238 offset:24576\n\t" \
    "v_mfma_f32_16x16x128_f8f6f4 v[108:111], a[152:159], v[208:215], v[108:111] cbsz:1 blgp:1\n\t" \
    "ds_read_b64_tr_b8 v[206:207], v238 offset:25600\n\t" \
    "s_waitcnt lgkmcnt(8)\n\t" \
    "v_mfma_f32_16x16x128_f8f6f4 v[112:115], a[128:135], v[216:223], v[112:115] cbsz:1 blgp:1\n\t" \
    "s_cmp_eq_u32 s47, 1\n\t" \
    "s_cselect_b32 s87, s29, s30\n\t" \
    "s_mov_b32 s86, s87\n\t" \
    "s_add_u32 m0, s46, 131072\n\t" \
    "s_add_u32 s74, s86, 0\n\t" \
    "buffer_load_dwordx4 v232, s[8:11], s74 offen  lds\n\t" \
    "v_add_u32 v239, 65536, v226\n\t" \
    "ds_read_b64_tr_b8 v[208:209], v239 offset:16384\n\t" \
    "v_mfma_f32_16x16x128_f8f6f4 v[116:119], a[136:143], v[216:223], v[116:119] cbsz:1 blgp:1\n\t" \
    "ds_read_b64_tr_b8 v[210:211], v239 offset:17408\n\t" \
    "v_mfma_f32_16x16x128_f8f6f4 v[120:123], a[144:151], v[216:223], v[120:123] cbsz:1 blgp:1\n\t" \
    "ds_read_b64_tr_b8 v[212:213], v239 offset:24576\n\t" \
    "v_mfma_f32_16x16x128_f8f6f4 v[124:127], a[152:159], v[216:223], v[124:127] cbsz:1 blgp:1\n\t" \
    "s_add_u32 m0, s46, 135168\n\t" \
    "s_add_u32 s74, s86, 524288\n\t" \
    "buffer_load_dwordx4 v232, s[8:11], s74 offen  lds\n\t" \
    "ds_read_b64_tr_b8 v[214:215], v239 offset:25600\n\t" \
    "s_waitcnt vmcnt(2)\n\t" \
    "s_waitcnt lgkmcnt(8)\n\t" \
    "v_mfma_f32_16x16x128_f8f6f4 v[0:3], a[160:167], v[192:199], v[0:3] cbsz:1 blgp:1\n\t" \
    "v_add_u32 v238, 65536, v227\n\t" \
    "ds_read_b64_tr_b8 v[216:217], v238 offset:16384\n\t" \
    "v_mfma_f32_16x16x128_f8f6f4 v[4:7], a[168:175], v[192:199], v[4:7] cbsz:1 blgp:1\n\t" \
    "s_add_u32 m0, s46, 139264\n\t" \
    "s_add_u32 s74, s86, 1048576\n\t" \
    "buffer_load_dwordx4 v232, s[8:11], s74 offen  lds\n\t" \
    "ds_read_b64_tr_b8 v[218:219], v238 offset:17408\n\t" \
    "v_mfma_f32_16x16x128_f8f6f4 v[8:11], a[176:183], v[192:199], v[8:11] cbsz:1 blgp:1\n\t" \
    "ds_read_b64_tr_b8 v[220:221], v238 offset:24576\n\t" \
    "v_mfma_f32_16x16x128_f8f6f4 v[12:15], a[184:191], v[192:199], v[12:15] cbsz:1 blgp:1\n\t" \
    "s_add_u32 m0, s46, 143360\n\t" \
    "s_add_u32 s74, s86, 1572864\n\t" \
    "buffer_load_dwordx4 v232, s[8:11], s74 offen  lds\n\t" \
    "ds_read_b64_tr_b8 v[222:223], v238 offset:25600\n\t" \
    "s_waitcnt lgkmcnt(8)\n\t" \
    "v_mfma_f32_16x16x128_f8f6f4 v[16:19], a[160:167], v[200:207], v[16:19] cbsz:1 blgp:1\n\t" \
    "v_add_u32 v239, 65536, v228\n\t" \
    "ds_read_b64_tr_b8 v[192:193], v239 offset:16384\n\t" \
    "v_mfma_f32_16x16x128_f8f6f4 v[20:23], a[168:175], v[200:207], v[20:23] cbsz:1 blgp:1\n\t" \
    "ds_read_b64_tr_b8 v[194:195], v239 offset:17408\n\t" \
    "v_mfma_f32_16x16x128_f8f6f4 v[24:27], a[176:183], v[200:207], v[24:27] cbsz:1 blgp:1\n\t" \
    "s_mov_b32 s72, 393216\n\t" \
    "s_mov_b32 s73, 397312\n\t" \
    "buffer_load_dwordx4 a[192:195], v233, s[12:15], s72 offen offset:0 sc0 sc1\n\t" \
    "ds_read_b64_tr_b8 v[196:197], v239 offset:24576\n\t" \
    "v_mfma_f32_16x16x128_f8f6f4 v[28:31], a[184:191], v[200:207], v[28:31] cbsz:1 blgp:1\n\t" \
    "ds_read_b64_tr_b8 v[198:199], v239 offset:25600\n\t" \
    "s_waitcnt lgkmcnt(8)\n\t" \
    "v_mfma_f32_16x16x128_f8f6f4 v[32:35], a[160:167], v[208:215], v[32:35] cbsz:1 blgp:1\n\t" \
    "buffer_load_dwordx4 a[196:199], v233, s[12:15], s72 offen offset:1024 sc0 sc1\n\t" \
    "v_add_u32 v238, 65536, v229\n\t" \
    "ds_read_b64_tr_b8 v[200:201], v238 offset:16384\n\t" \
    "v_mfma_f32_16x16x128_f8f6f4 v[36:39], a[168:175], v[208:215], v[36:39] cbsz:1 blgp:1\n\t" \
    "ds_read_b64_tr_b8 v[202:203], v238 offset:17408\n\t" \
    "v_mfma_f32_16x16x128_f8f6f4 v[40:43], a[176:183], v[208:215], v[40:43] cbsz:1 blgp:1\n\t" \
    "buffer_load_dwordx4 a[200:203], v233, s[12:15], s72 offen offset:2048 sc0 sc1\n\t" \
    "ds_read_b64_tr_b8 v[204:205], v238 offset:24576\n\t" \
    "v_mfma_f32_16x16x128_f8f6f4 v[44:47], a[184:191], v[208:215], v[44:47] cbsz:1 blgp:1\n\t" \
    "ds_read_b64_tr_b8 v[206:207], v238 offset:25600\n\t" \
    "s_waitcnt lgkmcnt(8)\n\t" \
    "v_mfma_f32_16x16x128_f8f6f4 v[48:51], a[160:167], v[216:223], v[48:51] cbsz:1 blgp:1\n\t" \
    "v_add_u32 v239, 65536, v230\n\t" \
    "ds_read_b64_tr_b8 v[208:209], v239 offset:16384\n\t" \
    "v_mfma_f32_16x16x128_f8f6f4 v[52:55], a[168:175], v[216:223], v[52:55] cbsz:1 blgp:1\n\t" \
    "buffer_load_dwordx4 a[204:207], v233, s[12:15], s72 offen offset:3072 sc0 sc1\n\t" \
    "ds_read_b64_tr_b8 v[210:211], v239 offset:17408\n\t" \
    "v_mfma_f32_16x16x128_f8f6f4 v[56:59], a[176:183], v[216:223], v[56:59] cbsz:1 blgp:1\n\t" \
    "ds_read_b64_tr_b8 v[212:213], v239 offset:24576\n\t" \
    "v_mfma_f32_16x16x128_f8f6f4 v[60:63], a[184:191], v[216:223], v[60:63] cbsz:1 blgp:1\n\t" \
    "buffer_load_dwordx4 a[208:211], v233, s[12:15], s73 offen offset:0 sc0 sc1\n\t" \
    "ds_read_b64_tr_b8 v[214:215], v239 offset:25600\n\t" \
    "s_waitcnt lgkmcnt(8)\n\t" \
    "v_mfma_f32_16x16x128_f8f6f4 v[64:67], a[160:167], v[192:199], v[64:67] cbsz:1 blgp:1\n\t" \
    "v_add_u32 v238, 65536, v231\n\t" \
    "ds_read_b64_tr_b8 v[216:217], v238 offset:16384\n\t" \
    "v_mfma_f32_16x16x128_f8f6f4 v[68:71], a[168:175], v[192:199], v[68:71] cbsz:1 blgp:1\n\t" \
    "buffer_load_dwordx4 a[212:215], v233, s[12:15], s73 offen offset:1024 sc0 sc1\n\t" \
    "ds_read_b64_tr_b8 v[218:219], v238 offset:17408\n\t" \
    "v_mfma_f32_16x16x128_f8f6f4 v[72:75], a[176:183], v[192:199], v[72:75] cbsz:1 blgp:1\n\t" \
    "ds_read_b64_tr_b8 v[220:221], v238 offset:24576\n\t" \
    "v_mfma_f32_16x16x128_f8f6f4 v[76:79], a[184:191], v[192:199], v[76:79] cbsz:1 blgp:1\n\t" \
    "ds_read_b64_tr_b8 v[222:223], v238 offset:25600\n\t" \
    "s_waitcnt lgkmcnt(8)\n\t" \
    "v_mfma_f32_16x16x128_f8f6f4 v[80:83], a[160:167], v[200:207], v[80:83] cbsz:1 blgp:1\n\t" \
    "s_barrier\n\t" \
    "buffer_load_dwordx4 a[216:219], v233, s[12:15], s73 offen offset:2048 sc0 sc1\n\t" \
    "v_add_u32 v239, 65536, v224\n\t" \
    "ds_read_b64_tr_b8 v[192:193], v239 offset:32768\n\t" \
    "v_mfma_f32_16x16x128_f8f6f4 v[84:87], a[168:175], v[200:207], v[84:87] cbsz:1 blgp:1\n\t" \
    "ds_read_b64_tr_b8 v[194:195], v239 offset:33792\n\t" \
    "v_mfma_f32_16x16x128_f8f6f4 v[88:91], a[176:183], v[200:207], v[88:91] cbsz:1 blgp:1\n\t" \
    "buffer_load_dwordx4 a[220:223], v233, s[12:15], s73 offen offset:3072 sc0 sc1\n\t" \
    "ds_read_b64_tr_b8 v[196:197], v239 offset:40960\n\t" \
    "v_mfma_f32_16x16x128_f8f6f4 v[92:95], a[184:191], v[200:207], v[92:95] cbsz:1 blgp:1\n\t" \
    "ds_read_b64_tr_b8 v[198:199], v239 offset:41984\n\t" \
    "s_waitcnt lgkmcnt(8)\n\t" \
    "v_mfma_f32_16x16x128_f8f6f4 v[96:99], a[160:167], v[208:215], v[96:99] cbsz:1 blgp:1\n\t" \
    "s_mov_b32 s72, 458752\n\t" \
    "s_mov_b32 s73, 462848\n\t" \
    "buffer_load_dwordx4 a[224:227], v233, s[12:15], s72 offen offset:0 sc0 sc1\n\t" \
    "v_add_u32 v238, 65536, v225\n\t" \
    "ds_read_b64_tr_b8 v[200:201], v238 offset:32768\n\t" \
    "v_mfma_f32_16x16x128_f8f6f4 v[100:103], a[168:175], v[208:215], v[100:103] cbsz:1 blgp:1\n\t" \
    "ds_read_b64_tr_b8 v[202:203], v238 offset:33792\n\t" \
    "v_mfma_f32_16x16x128_f8f6f4 v[104:107], a[176:183], v[208:215], v[104:107] cbsz:1 blgp:1\n\t" \
    "ds_read_b64_tr_b8 v[204:205], v238 offset:40960\n\t" \
    "v_mfma_f32_16x16x128_f8f6f4 v[108:111], a[184:191], v[208:215], v[108:111] cbsz:1 blgp:1\n\t" \
    "buffer_load_dwordx4 a[228:231], v233, s[12:15], s72 offen offset:1024 sc0 sc1\n\t" \
    "ds_read_b64_tr_b8 v[206:207], v238 offset:41984\n\t" \
    "s_waitcnt lgkmcnt(8)\n\t" \
    "v_mfma_f32_16x16x128_f8f6f4 v[112:115], a[160:167], v[216:223], v[112:115] cbsz:1 blgp:1\n\t" \
    "v_add_u32 v239, 65536, v226\n\t" \
    "ds_read_b64_tr_b8 v[208:209], v239 offset:32768\n\t" \
    "v_mfma_f32_16x16x128_f8f6f4 v[116:119], a[168:175], v[216:223], v[116:119] cbsz:1 blgp:1\n\t" \
    "buffer_load_dwordx4 a[232:235], v233, s[12:15], s72 offen offset:2048 sc0 sc1\n\t" \
    "ds_read_b64_tr_b8 v[210:211], v239 offset:33792\n\t" \
    "v_mfma_f32_16x16x128_f8f6f4 v[120:123], a[176:183], v[216:223], v[120:123] cbsz:1 blgp:1\n\t" \
    "ds_read_b64_tr_b8 v[212:213], v239 offset:40960\n\t" \
    "v_mfma_f32_16x16x128_f8f6f4 v[124:127], a[184:191], v[216:223], v[124:127] cbsz:1 blgp:1\n\t" \
    "buffer_load_dwordx4 a[236:239], v233, s[12:15], s72 offen offset:3072 sc0 sc1\n\t" \
    "ds_read_b64_tr_b8 v[214:215], v239 offset:41984\n\t" \
    "s_waitcnt vmcnt(4)\n\t" \
    "s_waitcnt lgkmcnt(8)\n\t" \
    "v_mfma_f32_16x16x128_f8f6f4 v[0:3], a[192:199], v[192:199], v[0:3] cbsz:1 blgp:1\n\t" \
    "v_add_u32 v238, 65536, v227\n\t" \
    "ds_read_b64_tr_b8 v[216:217], v238 offset:32768\n\t" \
    "v_mfma_f32_16x16x128_f8f6f4 v[4:7], a[200:207], v[192:199], v[4:7] cbsz:1 blgp:1\n\t" \
    "ds_read_b64_tr_b8 v[218:219], v238 offset:33792\n\t" \
    "v_mfma_f32_16x16x128_f8f6f4 v[8:11], a[208:215], v[192:199], v[8:11] cbsz:1 blgp:1\n\t" \
    "buffer_load_dwordx4 a[240:243], v233, s[12:15], s73 offen offset:0 sc0 sc1\n\t" \
    "ds_read_b64_tr_b8 v[220:221], v238 offset:40960\n\t" \
    "v_mfma_f32_16x16x128_f8f6f4 v[12:15], a[216:223], v[192:199], v[12:15] cbsz:1 blgp:1\n\t" \
    "ds_read_b64_tr_b8 v[222:223], v238 offset:41984\n\t" \
    "s_waitcnt lgkmcnt(8)\n\t" \
    "v_mfma_f32_16x16x128_f8f6f4 v[16:19], a[192:199], v[200:207], v[16:19] cbsz:1 blgp:1\n\t" \
    "buffer_load_dwordx4 a[244:247], v233, s[12:15], s73 offen offset:1024 sc0 sc1\n\t" \
    "v_add_u32 v239, 65536, v228\n\t" \
    "ds_read_b64_tr_b8 v[192:193], v239 offset:32768\n\t" \
    "v_mfma_f32_16x16x128_f8f6f4 v[20:23], a[200:207], v[200:207], v[20:23] cbsz:1 blgp:1\n\t" \
    "ds_read_b64_tr_b8 v[194:195], v239 offset:33792\n\t" \
    "v_mfma_f32_16x16x128_f8f6f4 v[24:27], a[208:215], v[200:207], v[24:27] cbsz:1 blgp:1\n\t" \
    "buffer_load_dwordx4 a[248:251], v233, s[12:15], s73 offen offset:2048 sc0 sc1\n\t" \
    "ds_read_b64_tr_b8 v[196:197], v239 offset:40960\n\t" \
    "v_mfma_f32_16x16x128_f8f6f4 v[28:31], a[216:223], v[200:207], v[28:31] cbsz:1 blgp:1\n\t" \
    "ds_read_b64_tr_b8 v[198:199], v239 offset:41984\n\t" \
    "s_waitcnt lgkmcnt(8)\n\t" \
    "v_mfma_f32_16x16x128_f8f6f4 v[32:35], a[192:199], v[208:215], v[32:35] cbsz:1 blgp:1\n\t" \
    "v_add_u32 v238, 65536, v229\n\t" \
    "ds_read_b64_tr_b8 v[200:201], v238 offset:32768\n\t" \
    "v_mfma_f32_16x16x128_f8f6f4 v[36:39], a[200:207], v[208:215], v[36:39] cbsz:1 blgp:1\n\t" \
    "buffer_load_dwordx4 a[252:255], v233, s[12:15], s73 offen offset:3072 sc0 sc1\n\t" \
    "ds_read_b64_tr_b8 v[202:203], v238 offset:33792\n\t" \
    "v_mfma_f32_16x16x128_f8f6f4 v[40:43], a[208:215], v[208:215], v[40:43] cbsz:1 blgp:1\n\t" \
    "ds_read_b64_tr_b8 v[204:205], v238 offset:40960\n\t" \
    "v_mfma_f32_16x16x128_f8f6f4 v[44:47], a[216:223], v[208:215], v[44:47] cbsz:1 blgp:1\n\t" \
    "s_mov_b32 s72, 524288\n\t" \
    "s_mov_b32 s73, 528384\n\t" \
    "buffer_load_dwordx4 v[128:131], v233, s[12:15], s72 offen offset:0 sc0 sc1\n\t" \
    "ds_read_b64_tr_b8 v[206:207], v238 offset:41984\n\t" \
    "s_waitcnt lgkmcnt(8)\n\t" \
    "v_mfma_f32_16x16x128_f8f6f4 v[48:51], a[192:199], v[216:223], v[48:51] cbsz:1 blgp:1\n\t" \
    "v_add_u32 v239, 65536, v230\n\t" \
    "ds_read_b64_tr_b8 v[208:209], v239 offset:32768\n\t" \
    "v_mfma_f32_16x16x128_f8f6f4 v[52:55], a[200:207], v[216:223], v[52:55] cbsz:1 blgp:1\n\t" \
    "s_mov_b32 s72, 589824\n\t" \
    "s_mov_b32 s73, 593920\n\t" \
    "buffer_load_dwordx4 v[160:163], v233, s[12:15], s72 offen offset:0 sc0 sc1\n\t" \
    "ds_read_b64_tr_b8 v[210:211], v239 offset:33792\n\t" \
    "v_mfma_f32_16x16x128_f8f6f4 v[56:59], a[208:215], v[216:223], v[56:59] cbsz:1 blgp:1\n\t" \
    "s_mov_b32 s72, 524288\n\t" \
    "s_mov_b32 s73, 528384\n\t" \
    "buffer_load_dwordx4 v[132:135], v233, s[12:15], s72 offen offset:1024 sc0 sc1\n\t" \
    "s_mov_b32 s72, 589824\n\t" \
    "s_mov_b32 s73, 593920\n\t" \
    "buffer_load_dwordx4 v[164:167], v233, s[12:15], s72 offen offset:1024 sc0 sc1\n\t" \
    "ds_read_b64_tr_b8 v[212:213], v239 offset:40960\n\t" \
    "v_mfma_f32_16x16x128_f8f6f4 v[60:63], a[216:223], v[216:223], v[60:63] cbsz:1 blgp:1\n\t" \
    "s_mov_b32 s72, 524288\n\t" \
    "s_mov_b32 s73, 528384\n\t" \
    "buffer_load_dwordx4 v[136:139], v233, s[12:15], s72 offen offset:2048 sc0 sc1\n\t" \
    "s_mov_b32 s72, 589824\n\t" \
    "s_mov_b32 s73, 593920\n\t" \
    "buffer_load_dwordx4 v[168:171], v233, s[12:15], s72 offen offset:2048 sc0 sc1\n\t" \
    "ds_read_b64_tr_b8 v[214:215], v239 offset:41984\n\t" \
    "s_waitcnt lgkmcnt(8)\n\t" \
    "v_mfma_f32_16x16x128_f8f6f4 v[64:67], a[192:199], v[192:199], v[64:67] cbsz:1 blgp:1\n\t" \
    "s_mov_b32 s72, 524288\n\t" \
    "s_mov_b32 s73, 528384\n\t" \
    "buffer_load_dwordx4 v[140:143], v233, s[12:15], s72 offen offset:3072 sc0 sc1\n\t" \
    "s_mov_b32 s72, 589824\n\t" \
    "s_mov_b32 s73, 593920\n\t" \
    "buffer_load_dwordx4 v[172:175], v233, s[12:15], s72 offen offset:3072 sc0 sc1\n\t" \
    "v_add_u32 v238, 65536, v231\n\t" \
    "ds_read_b64_tr_b8 v[216:217], v238 offset:32768\n\t" \
    "v_mfma_f32_16x16x128_f8f6f4 v[68:71], a[200:207], v[192:199], v[68:71] cbsz:1 blgp:1\n\t" \
    "s_mov_b32 s72, 524288\n\t" \
    "s_mov_b32 s73, 528384\n\t" \
    "buffer_load_dwordx4 v[144:147], v233, s[12:15], s73 offen offset:0 sc0 sc1\n\t" \
    "s_mov_b32 s72, 589824\n\t" \
    "s_mov_b32 s73, 593920\n\t" \
    "buffer_load_dwordx4 v[176:179], v233, s[12:15], s73 offen offset:0 sc0 sc1\n\t" \
    "ds_read_b64_tr_b8 v[218:219], v238 offset:33792\n\t" \
    "v_mfma_f32_16x16x128_f8f6f4 v[72:75], a[208:215], v[192:199], v[72:75] cbsz:1 blgp:1\n\t" \
    "s_mov_b32 s72, 524288\n\t" \
    "s_mov_b32 s73, 528384\n\t" \
    "buffer_load_dwordx4 v[148:151], v233, s[12:15], s73 offen offset:1024 sc0 sc1\n\t" \
    "s_mov_b32 s72, 589824\n\t" \
    "s_mov_b32 s73, 593920\n\t" \
    "buffer_load_dwordx4 v[180:183], v233, s[12:15], s73 offen offset:1024 sc0 sc1\n\t" \
    "ds_read_b64_tr_b8 v[220:221], v238 offset:40960\n\t" \
    "v_mfma_f32_16x16x128_f8f6f4 v[76:79], a[216:223], v[192:199], v[76:79] cbsz:1 blgp:1\n\t" \
    "s_mov_b32 s72, 524288\n\t" \
    "s_mov_b32 s73, 528384\n\t" \
    "buffer_load_dwordx4 v[152:155], v233, s[12:15], s73 offen offset:2048 sc0 sc1\n\t" \
    "s_mov_b32 s72, 589824\n\t" \
    "s_mov_b32 s73, 593920\n\t" \
    "buffer_load_dwordx4 v[184:187], v233, s[12:15], s73 offen offset:2048 sc0 sc1\n\t" \
    "ds_read_b64_tr_b8 v[222:223], v238 offset:41984\n\t" \
    "s_waitcnt lgkmcnt(8)\n\t" \
    "v_mfma_f32_16x16x128_f8f6f4 v[80:83], a[192:199], v[200:207], v[80:83] cbsz:1 blgp:1\n\t" \
    "s_barrier\n\t" \
    "s_mov_b32 s72, 524288\n\t" \
    "s_mov_b32 s73, 528384\n\t" \
    "buffer_load_dwordx4 v[156:159], v233, s[12:15], s73 offen offset:3072 sc0 sc1\n\t" \
    "s_mov_b32 s72, 589824\n\t" \
    "s_mov_b32 s73, 593920\n\t" \
    "buffer_load_dwordx4 v[188:191], v233, s[12:15], s73 offen offset:3072 sc0 sc1\n\t" \
    "v_add_u32 v239, 65536, v224\n\t" \
    "ds_read_b64_tr_b8 v[192:193], v239 offset:49152\n\t" \
    "v_mfma_f32_16x16x128_f8f6f4 v[84:87], a[200:207], v[200:207], v[84:87] cbsz:1 blgp:1\n\t" \
    "s_cmp_eq_u32 s79, 1\n\t" \
    "s_cbranch_scc1 L_flg1_%=\n\t" \
    "s_mov_b64 exec, 1\n\t" \
    "global_store_dword v236, v236, s[56:57] offset:32\n\t" \
    "s_mov_b64 exec, -1\n\t" \
    "s_branch L_flge1_%=\n\t" \
    "L_flg1_%=:\n\t" \
    "s_waitcnt vmcnt(63)\n\t" \
    "v_mov_b32 v251, s75\n\t" \
    "s_mov_b64 exec, 1\n\t" \
    "global_store_dword v236, v251, s[52:53] sc0 sc1\n\t" \
    "s_mov_b64 exec, -1\n\t" \
    "L_flge1_%=:\n\t" \
    "s_cmp_eq_u32 s79, 2\n\t" \
    "s_cbranch_scc1 L_pay2_%=\n\t" \
    "s_mov_b64 exec, 1\n\t" \
    "global_store_dword v236, v236, s[56:57] offset:32\n\t" \
    "global_store_dword v236, v236, s[56:57] offset:32\n\t" \
    "s_mov_b64 exec, -1\n\t" \
    "s_branch L_paye2_%=\n\t" \
    "L_pay2_%=:\n\t" \
    "global_store_dwordx4 v237, v[240:243], s[48:49] sc0 sc1\n\t" \
    "global_store_dwordx4 v237, v[244:247], s[48:49] offset:1024 sc0 sc1\n\t" \
    "L_paye2_%=:\n\t" \
    "s_cmp_eq_u32 s47, 0\n\t" \
    "s_cselect_b32 s87, s31, s25\n\t" \
    "s_mov_b32 s88, 0\n\t" \
    "s_cselect_b32 s88, s88, 2097152\n\t" \
    "s_add_u32 s85, s87, s88\n\t" \
    "s_add_u32 m0, s44, 0\n\t" \
    "s_add_u32 s74, s85, 0\n\t" \
    "buffer_load_dwordx4 v232, s[8:11], s74 offen  lds\n\t" \
    "ds_read_b64_tr_b8 v[194:195], v239 offset:50176\n\t" \
    "v_mfma_f32_16x16x128_f8f6f4 v[88:91], a[208:215], v[200:207], v[88:91] cbsz:1 blgp:1\n\t" \
    "s_add_u32 m0, s44, 4096\n\t" \
    "s_add_u32 s74, s85, 524288\n\t" \
    "buffer_load_dwordx4 v232, s[8:11], s74 offen  lds\n\t" \
    "ds_read_b64_tr_b8 v[196:197], v239 offset:57344\n\t" \
    "v_mfma_f32_16x16x128_f8f6f4 v[92:95], a[216:223], v[200:207], v[92:95] cbsz:1 blgp:1\n\t" \
    "s_add_u32 m0, s44, 8192\n\t" \
    "s_add_u32 s74, s85, 1048576\n\t" \
    "buffer_load_dwordx4 v232, s[8:11], s74 offen  lds\n\t" \
    "ds_read_b64_tr_b8 v[198:199], v239 offset:58368\n\t" \
    "s_waitcnt lgkmcnt(8)\n\t" \
    "v_mfma_f32_16x16x128_f8f6f4 v[96:99], a[192:199], v[208:215], v[96:99] cbsz:1 blgp:1\n\t" \
    "s_add_u32 m0, s44, 12288\n\t" \
    "s_add_u32 s74, s85, 1572864\n\t" \
    "buffer_load_dwordx4 v232, s[8:11], s74 offen  lds\n\t" \
    "v_add_u32 v238, 65536, v225\n\t" \
    "ds_read_b64_tr_b8 v[200:201], v238 offset:49152\n\t" \
    "v_mfma_f32_16x16x128_f8f6f4 v[100:103], a[200:207], v[208:215], v[100:103] cbsz:1 blgp:1\n\t" \
    "s_cmp_eq_u32 s47, 0\n\t" \
    "s_cselect_b32 s87, s26, s27\n\t" \
    "s_add_u32 s85, s87, 2097152\n\t" \
    "s_add_u32 m0, s44, 32768\n\t" \
    "s_add_u32 s74, s85, 0\n\t" \
    "buffer_load_dwordx4 v232, s[8:11], s74 offen  lds\n\t" \
    "ds_read_b64_tr_b8 v[202:203], v238 offset:50176\n\t" \
    "v_mfma_f32_16x16x128_f8f6f4 v[104:107], a[208:215], v[208:215], v[104:107] cbsz:1 blgp:1\n\t" \
    "s_add_u32 m0, s44, 36864\n\t" \
    "s_add_u32 s74, s85, 524288\n\t" \
    "buffer_load_dwordx4 v232, s[8:11], s74 offen  lds\n\t" \
    "ds_read_b64_tr_b8 v[204:205], v238 offset:57344\n\t" \
    "v_mfma_f32_16x16x128_f8f6f4 v[108:111], a[216:223], v[208:215], v[108:111] cbsz:1 blgp:1\n\t" \
    "s_add_u32 m0, s44, 40960\n\t" \
    "s_add_u32 s74, s85, 1048576\n\t" \
    "buffer_load_dwordx4 v232, s[8:11], s74 offen  lds\n\t" \
    "ds_read_b64_tr_b8 v[206:207], v238 offset:58368\n\t" \
    "s_waitcnt lgkmcnt(8)\n\t" \
    "v_mfma_f32_16x16x128_f8f6f4 v[112:115], a[192:199], v[216:223], v[112:115] cbsz:1 blgp:1\n\t" \
    "s_add_u32 m0, s44, 45056\n\t" \
    "s_add_u32 s74, s85, 1572864\n\t" \
    "buffer_load_dwordx4 v232, s[8:11], s74 offen  lds\n\t" \
    "v_add_u32 v239, 65536, v226\n\t" \
    "ds_read_b64_tr_b8 v[208:209], v239 offset:49152\n\t" \
    "v_mfma_f32_16x16x128_f8f6f4 v[116:119], a[200:207], v[216:223], v[116:119] cbsz:1 blgp:1\n\t" \
    "ds_read_b64_tr_b8 v[210:211], v239 offset:50176\n\t" \
    "v_mfma_f32_16x16x128_f8f6f4 v[120:123], a[208:215], v[216:223], v[120:123] cbsz:1 blgp:1\n\t" \
    "ds_read_b64_tr_b8 v[212:213], v239 offset:57344\n\t" \
    "v_mfma_f32_16x16x128_f8f6f4 v[124:127], a[216:223], v[216:223], v[124:127] cbsz:1 blgp:1\n\t" \
    "ds_read_b64_tr_b8 v[214:215], v239 offset:58368\n\t" \
    "s_waitcnt vmcnt(27)\n\t" \
    "s_waitcnt lgkmcnt(8)\n\t" \
    "v_mfma_f32_16x16x128_f8f6f4 v[0:3], a[224:231], v[192:199], v[0:3] cbsz:1 blgp:1\n\t" \
    "v_add_u32 v238, 65536, v227\n\t" \
    "ds_read_b64_tr_b8 v[216:217], v238 offset:49152\n\t" \
    "v_mfma_f32_16x16x128_f8f6f4 v[4:7], a[232:239], v[192:199], v[4:7] cbsz:1 blgp:1\n\t" \
    "ds_read_b64_tr_b8 v[218:219], v238 offset:50176\n\t" \
    "v_mfma_f32_16x16x128_f8f6f4 v[8:11], a[240:247], v[192:199], v[8:11] cbsz:1 blgp:1\n\t" \
    "ds_read_b64_tr_b8 v[220:221], v238 offset:57344\n\t" \
    "v_mfma_f32_16x16x128_f8f6f4 v[12:15], a[248:255], v[192:199], v[12:15] cbsz:1 blgp:1\n\t" \
    "ds_read_b64_tr_b8 v[222:223], v238 offset:58368\n\t" \
    "s_waitcnt lgkmcnt(8)\n\t" \
    "v_mfma_f32_16x16x128_f8f6f4 v[16:19], a[224:231], v[200:207], v[16:19] cbsz:1 blgp:1\n\t" \
    "v_add_u32 v239, 65536, v228\n\t" \
    "ds_read_b64_tr_b8 v[192:193], v239 offset:49152\n\t" \
    "v_mfma_f32_16x16x128_f8f6f4 v[20:23], a[232:239], v[200:207], v[20:23] cbsz:1 blgp:1\n\t" \
    "ds_read_b64_tr_b8 v[194:195], v239 offset:50176\n\t" \
    "v_mfma_f32_16x16x128_f8f6f4 v[24:27], a[240:247], v[200:207], v[24:27] cbsz:1 blgp:1\n\t" \
    "ds_read_b64_tr_b8 v[196:197], v239 offset:57344\n\t" \
    "v_mfma_f32_16x16x128_f8f6f4 v[28:31], a[248:255], v[200:207], v[28:31] cbsz:1 blgp:1\n\t" \
    "ds_read_b64_tr_b8 v[198:199], v239 offset:58368\n\t" \
    "s_waitcnt lgkmcnt(8)\n\t" \
    "v_mfma_f32_16x16x128_f8f6f4 v[32:35], a[224:231], v[208:215], v[32:35] cbsz:1 blgp:1\n\t" \
    "v_add_u32 v238, 65536, v229\n\t" \
    "ds_read_b64_tr_b8 v[200:201], v238 offset:49152\n\t" \
    "v_mfma_f32_16x16x128_f8f6f4 v[36:39], a[232:239], v[208:215], v[36:39] cbsz:1 blgp:1\n\t" \
    "ds_read_b64_tr_b8 v[202:203], v238 offset:50176\n\t" \
    "v_mfma_f32_16x16x128_f8f6f4 v[40:43], a[240:247], v[208:215], v[40:43] cbsz:1 blgp:1\n\t" \
    "ds_read_b64_tr_b8 v[204:205], v238 offset:57344\n\t" \
    "v_mfma_f32_16x16x128_f8f6f4 v[44:47], a[248:255], v[208:215], v[44:47] cbsz:1 blgp:1\n\t" \
    "ds_read_b64_tr_b8 v[206:207], v238 offset:58368\n\t" \
    "s_waitcnt lgkmcnt(8)\n\t" \
    "v_mfma_f32_16x16x128_f8f6f4 v[48:51], a[224:231], v[216:223], v[48:51] cbsz:1 blgp:1\n\t" \
    "v_add_u32 v239, 65536, v230\n\t" \
    "ds_read_b64_tr_b8 v[208:209], v239 offset:49152\n\t" \
    "v_mfma_f32_16x16x128_f8f6f4 v[52:55], a[232:239], v[216:223], v[52:55] cbsz:1 blgp:1\n\t" \
    "ds_read_b64_tr_b8 v[210:211], v239 offset:50176\n\t" \
    "v_mfma_f32_16x16x128_f8f6f4 v[56:59], a[240:247], v[216:223], v[56:59] cbsz:1 blgp:1\n\t" \
    "ds_read_b64_tr_b8 v[212:213], v239 offset:57344\n\t" \
    "v_mfma_f32_16x16x128_f8f6f4 v[60:63], a[248:255], v[216:223], v[60:63] cbsz:1 blgp:1\n\t" \
    "ds_read_b64_tr_b8 v[214:215], v239 offset:58368\n\t" \
    "s_waitcnt lgkmcnt(8)\n\t" \
    "v_mfma_f32_16x16x128_f8f6f4 v[64:67], a[224:231], v[192:199], v[64:67] cbsz:1 blgp:1\n\t" \
    "v_add_u32 v238, 65536, v231\n\t" \
    "ds_read_b64_tr_b8 v[216:217], v238 offset:49152\n\t" \
    "v_mfma_f32_16x16x128_f8f6f4 v[68:71], a[232:239], v[192:199], v[68:71] cbsz:1 blgp:1\n\t" \
    "ds_read_b64_tr_b8 v[218:219], v238 offset:50176\n\t" \
    "v_mfma_f32_16x16x128_f8f6f4 v[72:75], a[240:247], v[192:199], v[72:75] cbsz:1 blgp:1\n\t" \
    "ds_read_b64_tr_b8 v[220:221], v238 offset:57344\n\t" \
    "v_mfma_f32_16x16x128_f8f6f4 v[76:79], a[248:255], v[192:199], v[76:79] cbsz:1 blgp:1\n\t" \
    "s_cmp_eq_u32 s47, 1\n\t" \
    "s_cselect_b32 s87, s31, s25\n\t" \
    "s_mov_b32 s88, 0\n\t" \
    "s_cselect_b32 s88, s88, 2097152\n\t" \
    "s_add_u32 s86, s87, s88\n\t" \
    "s_add_u32 m0, s46, 0\n\t" \
    "s_add_u32 s74, s86, 0\n\t" \
    "buffer_load_dwordx4 v232, s[8:11], s74 offen  lds\n\t" \
    "ds_read_b64_tr_b8 v[222:223], v238 offset:58368\n\t" \
    "s_waitcnt lgkmcnt(8)\n\t" \
    "v_mfma_f32_16x16x128_f8f6f4 v[80:83], a[224:231], v[200:207], v[80:83] cbsz:1 blgp:1\n\t" \
    "s_barrier\n\t" \
    "v_add_u32 v239, 131072, v224\n\t" \
    "ds_read_b64_tr_b8 v[192:193], v239 offset:0\n\t" \
    "v_mfma_f32_16x16x128_f8f6f4 v[84:87], a[232:239], v[200:207], v[84:87] cbsz:1 blgp:1\n\t" \
    "ds_read_b64_tr_b8 v[194:195], v239 offset:1024\n\t" \
    "v_mfma_f32_16x16x128_f8f6f4 v[88:91], a[240:247], v[200:207], v[88:91] cbsz:1 blgp:1\n\t" \
    "s_add_u32 m0, s46, 4096\n\t" \
    "s_add_u32 s74, s86, 524288\n\t" \
    "buffer_load_dwordx4 v232, s[8:11], s74 offen  lds\n\t" \
    "ds_read_b64_tr_b8 v[196:197], v239 offset:8192\n\t" \
    "v_mfma_f32_16x16x128_f8f6f4 v[92:95], a[248:255], v[200:207], v[92:95] cbsz:1 blgp:1\n\t" \
    "ds_read_b64_tr_b8 v[198:199], v239 offset:9216\n\t" \
    "s_waitcnt lgkmcnt(8)\n\t" \
    "v_mfma_f32_16x16x128_f8f6f4 v[96:99], a[224:231], v[208:215], v[96:99] cbsz:1 blgp:1\n\t" \
    "s_add_u32 m0, s46, 8192\n\t" \
    "s_add_u32 s74, s86, 1048576\n\t" \
    "buffer_load_dwordx4 v232, s[8:11], s74 offen  lds\n\t" \
    "v_add_u32 v238, 131072, v225\n\t" \
    "ds_read_b64_tr_b8 v[200:201], v238 offset:0\n\t" \
    "v_mfma_f32_16x16x128_f8f6f4 v[100:103], a[232:239], v[208:215], v[100:103] cbsz:1 blgp:1\n\t" \
    "ds_read_b64_tr_b8 v[202:203], v238 offset:1024\n\t" \
    "v_mfma_f32_16x16x128_f8f6f4 v[104:107], a[240:247], v[208:215], v[104:107] cbsz:1 blgp:1\n\t" \
    "s_add_u32 m0, s46, 12288\n\t" \
    "s_add_u32 s74, s86, 1572864\n\t" \
    "buffer_load_dwordx4 v232, s[8:11], s74 offen  lds\n\t" \
    "ds_read_b64_tr_b8 v[204:205], v238 offset:8192\n\t" \
    "v_mfma_f32_16x16x128_f8f6f4 v[108:111], a[248:255], v[208:215], v[108:111] cbsz:1 blgp:1\n\t" \
    "ds_read_b64_tr_b8 v[206:207], v238 offset:9216\n\t" \
    "s_waitcnt lgkmcnt(8)\n\t" \
    "v_mfma_f32_16x16x128_f8f6f4 v[112:115], a[224:231], v[216:223], v[112:115] cbsz:1 blgp:1\n\t" \
    "v_add_u32 v239, 131072, v226\n\t" \
    "ds_read_b64_tr_b8 v[208:209], v239 offset:0\n\t" \
    "v_mfma_f32_16x16x128_f8f6f4 v[116:119], a[232:239], v[216:223], v[116:119] cbsz:1 blgp:1\n\t" \
    "s_cmp_eq_u32 s47, 1\n\t" \
    "s_cselect_b32 s87, s26, s27\n\t" \
    "s_add_u32 s86, s87, 2097152\n\t" \
    "s_add_u32 m0, s46, 32768\n\t" \
    "s_add_u32 s74, s86, 0\n\t" \
    "buffer_load_dwordx4 v232, s[8:11], s74 offen  lds\n\t" \
    "ds_read_b64_tr_b8 v[210:211], v239 offset:1024\n\t" \
    "v_mfma_f32_16x16x128_f8f6f4 v[120:123], a[240:247], v[216:223], v[120:123] cbsz:1 blgp:1\n\t" \
    "ds_read_b64_tr_b8 v[212:213], v239 offset:8192\n\t" \
    "v_mfma_f32_16x16x128_f8f6f4 v[124:127], a[248:255], v[216:223], v[124:127] cbsz:1 blgp:1\n\t" \
    "s_add_u32 m0, s46, 36864\n\t" \
    "s_add_u32 s74, s86, 524288\n\t" \
    "buffer_load_dwordx4 v232, s[8:11], s74 offen  lds\n\t" \
    "ds_read_b64_tr_b8 v[214:215], v239 offset:9216\n\t" \
    "s_waitcnt vmcnt(18)\n\t" \
    "s_waitcnt lgkmcnt(8)\n\t" \
    "v_mfma_f32_16x16x128_f8f6f4 v[0:3], v[128:135], v[192:199], v[0:3] cbsz:1 blgp:1\n\t" \
    "v_add_u32 v238, 131072, v227\n\t" \
    "ds_read_b64_tr_b8 v[216:217], v238 offset:0\n\t" \
    "v_mfma_f32_16x16x128_f8f6f4 v[4:7], v[136:143], v[192:199], v[4:7] cbsz:1 blgp:1\n\t" \
    "s_add_u32 m0, s46, 40960\n\t" \
    "s_add_u32 s74, s86, 1048576\n\t" \
    "buffer_load_dwordx4 v232, s[8:11], s74 offen  lds\n\t" \
    "ds_read_b64_tr_b8 v[218:219], v238 offset:1024\n\t" \
    "v_mfma_f32_16x16x128_f8f6f4 v[8:11], v[144:151], v[192:199], v[8:11] cbsz:1 blgp:1\n\t" \
    "ds_read_b64_tr_b8 v[220:221], v238 offset:8192\n\t" \
    "v_mfma_f32_16x16x128_f8f6f4 v[12:15], v[152:159], v[192:199], v[12:15] cbsz:1 blgp:1\n\t" \
    "ds_read_b64_tr_b8 v[222:223], v238 offset:9216\n\t" \
    "s_waitcnt lgkmcnt(8)\n\t" \
    "v_mfma_f32_16x16x128_f8f6f4 v[16:19], v[128:135], v[200:207], v[16:19] cbsz:1 blgp:1\n\t" \
    "s_add_u32 m0, s46, 45056\n\t" \
    "s_add_u32 s74, s86, 1572864\n\t" \
    "buffer_load_dwordx4 v232, s[8:11], s74 offen  lds\n\t" \
    "v_add_u32 v239, 131072, v228\n\t" \
    "ds_read_b64_tr_b8 v[192:193], v239 offset:0\n\t" \
    "v_mfma_f32_16x16x128_f8f6f4 v[20:23], v[136:143], v[200:207], v[20:23] cbsz:1 blgp:1\n\t" \
    "ds_read_b64_tr_b8 v[194:195], v239 offset:1024\n\t" \
    "v_mfma_f32_16x16x128_f8f6f4 v[24:27], v[144:151], v[200:207], v[24:27] cbsz:1 blgp:1\n\t" \
    "s_mov_b32 s72, 655360\n\t" \
    "s_mov_b32 s73, 659456\n\t" \
    "buffer_load_dwordx4 a[0:3], v233, s[12:15], s72 offen offset:0 sc0 sc1\n\t" \
    "ds_read_b64_tr_b8 v[196:197], v239 offset:8192\n\t" \
    "v_mfma_f32_16x16x128_f8f6f4 v[28:31], v[152:159], v[200:207], v[28:31] cbsz:1 blgp:1\n\t" \
    "ds_read_b64_tr_b8 v[198:199], v239 offset:9216\n\t" \
    "s_waitcnt lgkmcnt(8)\n\t" \
    "v_mfma_f32_16x16x128_f8f6f4 v[32:35], v[128:135], v[208:215], v[32:35] cbsz:1 blgp:1\n\t" \
    "buffer_load_dwordx4 a[4:7], v233, s[12:15], s72 offen offset:1024 sc0 sc1\n\t" \
    "v_add_u32 v238, 131072, v229\n\t" \
    "ds_read_b64_tr_b8 v[200:201], v238 offset:0\n\t" \
    "v_mfma_f32_16x16x128_f8f6f4 v[36:39], v[136:143], v[208:215], v[36:39] cbsz:1 blgp:1\n\t" \
    "ds_read_b64_tr_b8 v[202:203], v238 offset:1024\n\t" \
    "v_mfma_f32_16x16x128_f8f6f4 v[40:43], v[144:151], v[208:215], v[40:43] cbsz:1 blgp:1\n\t" \
    "ds_read_b64_tr_b8 v[204:205], v238 offset:8192\n\t" \
    "v_mfma_f32_16x16x128_f8f6f4 v[44:47], v[152:159], v[208:215], v[44:47] cbsz:1 blgp:1\n\t" \
    "buffer_load_dwordx4 a[8:11], v233, s[12:15], s72 offen offset:2048 sc0 sc1\n\t" \
    "ds_read_b64_tr_b8 v[206:207], v238 offset:9216\n\t" \
    "s_waitcnt lgkmcnt(8)\n\t" \
    "v_mfma_f32_16x16x128_f8f6f4 v[48:51], v[128:135], v[216:223], v[48:51] cbsz:1 blgp:1\n\t" \
    "v_add_u32 v239, 131072, v230\n\t" \
    "ds_read_b64_tr_b8 v[208:209], v239 offset:0\n\t" \
    "v_mfma_f32_16x16x128_f8f6f4 v[52:55], v[136:143], v[216:223], v[52:55] cbsz:1 blgp:1\n\t" \
    "buffer_load_dwordx4 a[12:15], v233, s[12:15], s72 offen offset:3072 sc0 sc1\n\t" \
    "ds_read_b64_tr_b8 v[210:211], v239 offset:1024\n\t" \
    "v_mfma_f32_16x16x128_f8f6f4 v[56:59], v[144:151], v[216:223], v[56:59] cbsz:1 blgp:1\n\t" \
    "global_load_dword v248, v235, s[54:55] offset:1024 sc0 sc1\n\t" \
    "ds_read_b64_tr_b8 v[212:213], v239 offset:8192\n\t" \
    "v_mfma_f32_16x16x128_f8f6f4 v[60:63], v[152:159], v[216:223], v[60:63] cbsz:1 blgp:1\n\t" \
    "buffer_load_dwordx4 a[16:19], v233, s[12:15], s73 offen offset:0 sc0 sc1\n\t" \
    "ds_read_b64_tr_b8 v[214:215], v239 offset:9216\n\t" \
    "s_waitcnt lgkmcnt(8)\n\t" \
    "v_mfma_f32_16x16x128_f8f6f4 v[64:67], v[128:135], v[192:199], v[64:67] cbsz:1 blgp:1\n\t" \
    "v_add_u32 v238, 131072, v231\n\t" \
    "ds_read_b64_tr_b8 v[216:217], v238 offset:0\n\t" \
    "v_mfma_f32_16x16x128_f8f6f4 v[68:71], v[136:143], v[192:199], v[68:71] cbsz:1 blgp:1\n\t" \
    "ds_read_b64_tr_b8 v[218:219], v238 offset:1024\n\t" \
    "v_mfma_f32_16x16x128_f8f6f4 v[72:75], v[144:151], v[192:199], v[72:75] cbsz:1 blgp:1\n\t" \
    "buffer_load_dwordx4 a[20:23], v233, s[12:15], s73 offen offset:1024 sc0 sc1\n\t" \
    "ds_read_b64_tr_b8 v[220:221], v238 offset:8192\n\t" \
    "v_mfma_f32_16x16x128_f8f6f4 v[76:79], v[152:159], v[192:199], v[76:79] cbsz:1 blgp:1\n\t" \
    "ds_read_b64_tr_b8 v[222:223], v238 offset:9216\n\t" \
    "s_waitcnt lgkmcnt(8)\n\t" \
    "v_mfma_f32_16x16x128_f8f6f4 v[80:83], v[128:135], v[200:207], v[80:83] cbsz:1 blgp:1\n\t" \
    "s_barrier\n\t" \
    "buffer_load_dwordx4 a[24:27], v233, s[12:15], s73 offen offset:2048 sc0 sc1\n\t" \
    "v_add_u32 v239, 131072, v224\n\t" \
    "ds_read_b64_tr_b8 v[192:193], v239 offset:16384\n\t" \
    "v_mfma_f32_16x16x128_f8f6f4 v[84:87], v[136:143], v[200:207], v[84:87] cbsz:1 blgp:1\n\t" \
    "ds_read_b64_tr_b8 v[194:195], v239 offset:17408\n\t" \
    "v_mfma_f32_16x16x128_f8f6f4 v[88:91], v[144:151], v[200:207], v[88:91] cbsz:1 blgp:1\n\t" \
    "buffer_load_dwordx4 a[28:31], v233, s[12:15], s73 offen offset:3072 sc0 sc1\n\t" \
    "ds_read_b64_tr_b8 v[196:197], v239 offset:24576\n\t" \
    "v_mfma_f32_16x16x128_f8f6f4 v[92:95], v[152:159], v[200:207], v[92:95] cbsz:1 blgp:1\n\t" \
    "ds_read_b64_tr_b8 v[198:199], v239 offset:25600\n\t" \
    "s_waitcnt lgkmcnt(8)\n\t" \
    "v_mfma_f32_16x16x128_f8f6f4 v[96:99], v[128:135], v[208:215], v[96:99] cbsz:1 blgp:1\n\t" \
    "v_add_u32 v238, 131072, v225\n\t" \
    "ds_read_b64_tr_b8 v[200:201], v238 offset:16384\n\t" \
    "v_mfma_f32_16x16x128_f8f6f4 v[100:103], v[136:143], v[208:215], v[100:103] cbsz:1 blgp:1\n\t" \
    "ds_read_b64_tr_b8 v[202:203], v238 offset:17408\n\t" \
    "v_mfma_f32_16x16x128_f8f6f4 v[104:107], v[144:151], v[208:215], v[104:107] cbsz:1 blgp:1\n\t" \
    "ds_read_b64_tr_b8 v[204:205], v238 offset:24576\n\t" \
    "v_mfma_f32_16x16x128_f8f6f4 v[108:111], v[152:159], v[208:215], v[108:111] cbsz:1 blgp:1\n\t" \
    "ds_read_b64_tr_b8 v[206:207], v238 offset:25600\n\t" \
    "s_waitcnt lgkmcnt(8)\n\t" \
    "v_mfma_f32_16x16x128_f8f6f4 v[112:115], v[128:135], v[216:223], v[112:115] cbsz:1 blgp:1\n\t" \
    "v_add_u32 v239, 131072, v226\n\t" \
    "ds_read_b64_tr_b8 v[208:209], v239 offset:16384\n\t" \
    "v_mfma_f32_16x16x128_f8f6f4 v[116:119], v[136:143], v[216:223], v[116:119] cbsz:1 blgp:1\n\t" \
    "ds_read_b64_tr_b8 v[210:211], v239 offset:17408\n\t" \
    "v_mfma_f32_16x16x128_f8f6f4 v[120:123], v[144:151], v[216:223], v[120:123] cbsz:1 blgp:1\n\t" \
    "ds_read_b64_tr_b8 v[212:213], v239 offset:24576\n\t" \
    "v_mfma_f32_16x16x128_f8f6f4 v[124:127], v[152:159], v[216:223], v[124:127] cbsz:1 blgp:1\n\t" \
    "ds_read_b64_tr_b8 v[214:215], v239 offset:25600\n\t" \
    "s_waitcnt vmcnt(28)\n\t" \
    "s_waitcnt lgkmcnt(8)\n\t" \
    "v_mfma_f32_16x16x128_f8f6f4 v[0:3], v[160:167], v[192:199], v[0:3] cbsz:1 blgp:1\n\t" \
    "v_add_u32 v238, 131072, v227\n\t" \
    "ds_read_b64_tr_b8 v[216:217], v238 offset:16384\n\t" \
    "v_mfma_f32_16x16x128_f8f6f4 v[4:7], v[168:175], v[192:199], v[4:7] cbsz:1 blgp:1\n\t" \
    "ds_read_b64_tr_b8 v[218:219], v238 offset:17408\n\t" \
    "v_mfma_f32_16x16x128_f8f6f4 v[8:11], v[176:183], v[192:199], v[8:11] cbsz:1 blgp:1\n\t" \
    "ds_read_b64_tr_b8 v[220:221], v238 offset:24576\n\t" \
    "v_mfma_f32_16x16x128_f8f6f4 v[12:15], v[184:191], v[192:199], v[12:15] cbsz:1 blgp:1\n\t" \
    "ds_read_b64_tr_b8 v[222:223], v238 offset:25600\n\t" \
    "s_waitcnt lgkmcnt(8)\n\t" \
    "v_mfma_f32_16x16x128_f8f6f4 v[16:19], v[160:167], v[200:207], v[16:19] cbsz:1 blgp:1\n\t" \
    "v_add_u32 v239, 131072, v228\n\t" \
    "ds_read_b64_tr_b8 v[192:193], v239 offset:16384\n\t" \
    "v_mfma_f32_16x16x128_f8f6f4 v[20:23], v[168:175], v[200:207], v[20:23] cbsz:1 blgp:1\n\t" \
    "ds_read_b64_tr_b8 v[194:195], v239 offset:17408\n\t" \
    "v_mfma_f32_16x16x128_f8f6f4 v[24:27], v[176:183], v[200:207], v[24:27] cbsz:1 blgp:1\n\t" \
    "ds_read_b64_tr_b8 v[196:197], v239 offset:24576\n\t" \
    "v_mfma_f32_16x16x128_f8f6f4 v[28:31], v[184:191], v[200:207], v[28:31] cbsz:1 blgp:1\n\t" \
    "ds_read_b64_tr_b8 v[198:199], v239 offset:25600\n\t" \
    "s_waitcnt lgkmcnt(8)\n\t" \
    "v_mfma_f32_16x16x128_f8f6f4 v[32:35], v[160:167], v[208:215], v[32:35] cbsz:1 blgp:1\n\t" \
    "v_add_u32 v238, 131072, v229\n\t" \
    "ds_read_b64_tr_b8 v[200:201], v238 offset:16384\n\t" \
    "v_mfma_f32_16x16x128_f8f6f4 v[36:39], v[168:175], v[208:215], v[36:39] cbsz:1 blgp:1\n\t" \
    "ds_read_b64_tr_b8 v[202:203], v238 offset:17408\n\t" \
    "v_mfma_f32_16x16x128_f8f6f4 v[40:43], v[176:183], v[208:215], v[40:43] cbsz:1 blgp:1\n\t" \
    "ds_read_b64_tr_b8 v[204:205], v238 offset:24576\n\t" \
    "v_mfma_f32_16x16x128_f8f6f4 v[44:47], v[184:191], v[208:215], v[44:47] cbsz:1 blgp:1\n\t" \
    "ds_read_b64_tr_b8 v[206:207], v238 offset:25600\n\t" \
    "s_waitcnt lgkmcnt(8)\n\t" \
    "v_mfma_f32_16x16x128_f8f6f4 v[48:51], v[160:167], v[216:223], v[48:51] cbsz:1 blgp:1\n\t" \
    "v_add_u32 v239, 131072, v230\n\t" \
    "ds_read_b64_tr_b8 v[208:209], v239 offset:16384\n\t" \
    "s_waitcnt vmcnt(4)\n\t" \
    "v_cmp_gt_u32 vcc, s75, v248\n\t" \
    "s_nop 0\n\t" \
    "s_cbranch_vccz L_pd1_%=\n\t" \
    "s_memrealtime s[80:81]\n\t" \
    "s_waitcnt lgkmcnt(0)\n\t" \
    "L_ps1_%=:\n\t" \
    "global_load_dword v248, v235, s[54:55] offset:1024 sc0 sc1\n\t" \
    "global_load_dword v251, v236, s[58:59] sc1\n\t" \
    "s_memrealtime s[82:83]\n\t" \
    "s_waitcnt vmcnt(0) lgkmcnt(0)\n\t" \
    "v_cmp_gt_u32 vcc, s75, v248\n\t" \
    "s_nop 0\n\t" \
    "s_cbranch_vccz L_pd1_%=\n\t" \
    "v_readfirstlane_b32 s84, v251\n\t" \
    "s_sub_u32 s82, s82, s80\n\t" \
    "s_cmp_eq_u32 s84, s75\n\t" \
    "s_cbranch_scc1 L_pd1_%=\n\t" \
    "s_cmp_gt_u32 s82, 40000000\n\t" \
    "s_cbranch_scc1 L_pt1_%=\n\t" \
    "s_sleep 1\n\t" \
    "s_branch L_ps1_%=\n\t" \
    "L_pt1_%=:\n\t" \
    "v_mov_b32 v251, s75\n\t" \
    "global_store_dword v236, v251, s[58:59] sc1\n\t" \
    "s_waitcnt vmcnt(0)\n\t" \
    "L_pd1_%=:\n\t" \
    "v_mfma_f32_16x16x128_f8f6f4 v[52:55], v[168:175], v[216:223], v[52:55] cbsz:1 blgp:1\n\t" \
    "s_mov_b32 s72, 720896\n\t" \
    "s_mov_b32 s73, 724992\n\t" \
    "buffer_load_dwordx4 a[32:35], v233, s[12:15], s72 offen offset:0 sc0 sc1\n\t" \
    "ds_read_b64_tr_b8 v[210:211], v239 offset:17408\n\t" \
    "v_mfma_f32_16x16x128_f8f6f4 v[56:59], v[176:183], v[216:223], v[56:59] cbsz:1 blgp:1\n\t" \
    "ds_read_b64_tr_b8 v[212:213], v239 offset:24576\n\t" \
    "v_mfma_f32_16x16x128_f8f6f4 v[60:63], v[184:191], v[216:223], v[60:63] cbsz:1 blgp:1\n\t" \
    "buffer_load_dwordx4 a[36:39], v233, s[12:15], s72 offen offset:1024 sc0 sc1\n\t" \
    "ds_read_b64_tr_b8 v[214:215], v239 offset:25600\n\t" \
    "s_waitcnt lgkmcnt(8)\n\t" \
    "v_mfma_f32_16x16x128_f8f6f4 v[64:67], v[160:167], v[192:199], v[64:67] cbsz:1 blgp:1\n\t" \
    "v_add_u32 v238, 131072, v231\n\t" \
    "ds_read_b64_tr_b8 v[216:217], v238 offset:16384\n\t" \
    "v_mfma_f32_16x16x128_f8f6f4 v[68:71], v[168:175], v[192:199], v[68:71] cbsz:1 blgp:1\n\t" \
    "buffer_load_dwordx4 a[40:43], v233, s[12:15], s72 offen offset:2048 sc0 sc1\n\t" \
    "ds_read_b64_tr_b8 v[218:219], v238 offset:17408\n\t" \
    "v_mfma_f32_16x16x128_f8f6f4 v[72:75], v[176:183], v[192:199], v[72:75] cbsz:1 blgp:1\n\t" \
    "ds_read_b64_tr_b8 v[220:221], v238 offset:24576\n\t" \
    "v_mfma_f32_16x16x128_f8f6f4 v[76:79], v[184:191], v[192:199], v[76:79] cbsz:1 blgp:1\n\t" \
    "ds_read_b64_tr_b8 v[222:223], v238 offset:25600\n\t" \
    "s_waitcnt lgkmcnt(8)\n\t" \
    "v_mfma_f32_16x16x128_f8f6f4 v[80:83], v[160:167], v[200:207], v[80:83] cbsz:1 blgp:1\n\t" \
    "s_barrier\n\t" \
    "buffer_load_dwordx4 a[44:47], v233, s[12:15], s72 offen offset:3072 sc0 sc1\n\t" \
    "ds_read_b64_tr_b8 v[192:193], v224 offset:0\n\t" \
    "v_mfma_f32_16x16x128_f8f6f4 v[84:87], v[168:175], v[200:207], v[84:87] cbsz:1 blgp:1\n\t" \
    "ds_read_b64_tr_b8 v[194:195], v224 offset:1024\n\t" \
    "v_mfma_f32_16x16x128_f8f6f4 v[88:91], v[176:183], v[200:207], v[88:91] cbsz:1 blgp:1\n\t" \
    "buffer_load_dwordx4 a[48:51], v233, s[12:15], s73 offen offset:0 sc0 sc1\n\t" \
    "ds_read_b64_tr_b8 v[196:197], v224 offset:8192\n\t" \
    "v_mfma_f32_16x16x128_f8f6f4 v[92:95], v[184:191], v[200:207], v[92:95] cbsz:1 blgp:1\n\t" \
    "ds_read_b64_tr_b8 v[198:199], v224 offset:9216\n\t" \
    "s_waitcnt lgkmcnt(8)\n\t" \
    "v_mfma_f32_16x16x128_f8f6f4 v[96:99], v[160:167], v[208:215], v[96:99] cbsz:1 blgp:1\n\t" \
    "buffer_load_dwordx4 a[52:55], v233, s[12:15], s73 offen offset:1024 sc0 sc1\n\t" \
    "ds_read_b64_tr_b8 v[200:201], v225 offset:0\n\t" \
    "v_mfma_f32_16x16x128_f8f6f4 v[100:103], v[168:175], v[208:215], v[100:103] cbsz:1 blgp:1\n\t" \
    "ds_read_b64_tr_b8 v[202:203], v225 offset:1024\n\t" \
    "v_mfma_f32_16x16x128_f8f6f4 v[104:107], v[176:183], v[208:215], v[104:107] cbsz:1 blgp:1\n\t" \
    "ds_read_b64_tr_b8 v[204:205], v225 offset:8192\n\t" \
    "v_mfma_f32_16x16x128_f8f6f4 v[108:111], v[184:191], v[208:215], v[108:111] cbsz:1 blgp:1\n\t" \
    "buffer_load_dwordx4 a[56:59], v233, s[12:15], s73 offen offset:2048 sc0 sc1\n\t" \
    "ds_read_b64_tr_b8 v[206:207], v225 offset:9216\n\t" \
    "s_waitcnt lgkmcnt(8)\n\t" \
    "v_mfma_f32_16x16x128_f8f6f4 v[112:115], v[160:167], v[216:223], v[112:115] cbsz:1 blgp:1\n\t" \
    "ds_read_b64_tr_b8 v[208:209], v226 offset:0\n\t" \
    "v_mfma_f32_16x16x128_f8f6f4 v[116:119], v[168:175], v[216:223], v[116:119] cbsz:1 blgp:1\n\t" \
    "buffer_load_dwordx4 a[60:63], v233, s[12:15], s73 offen offset:3072 sc0 sc1\n\t" \
    "ds_read_b64_tr_b8 v[210:211], v226 offset:1024\n\t" \
    "v_mfma_f32_16x16x128_f8f6f4 v[120:123], v[176:183], v[216:223], v[120:123] cbsz:1 blgp:1\n\t" \
    "ds_read_b64_tr_b8 v[212:213], v226 offset:8192\n\t" \
    "v_mfma_f32_16x16x128_f8f6f4 v[124:127], v[184:191], v[216:223], v[124:127] cbsz:1 blgp:1\n\t" \
    "s_mov_b32 s72, 786432\n\t" \
    "s_mov_b32 s73, 790528\n\t" \
    "buffer_load_dwordx4 a[64:67], v233, s[12:15], s72 offen offset:0 sc0 sc1\n\t" \
    "ds_read_b64_tr_b8 v[214:215], v226 offset:9216\n\t" \
    "s_waitcnt vmcnt(9)\n\t" \
    "s_waitcnt lgkmcnt(8)\n\t" \
    "v_mfma_f32_16x16x128_f8f6f4 v[0:3], a[0:7], v[192:199], v[0:3] cbsz:1 blgp:1\n\t" \
    "ds_read_b64_tr_b8 v[216:217], v227 offset:0\n\t" \
    "v_mfma_f32_16x16x128_f8f6f4 v[4:7], a[8:15], v[192:199], v[4:7] cbsz:1 blgp:1\n\t" \
    "ds_read_b64_tr_b8 v[218:219], v227 offset:1024\n\t" \
    "v_mfma_f32_16x16x128_f8f6f4 v[8:11], a[16:23], v[192:199], v[8:11] cbsz:1 blgp:1\n\t" \
    "buffer_load_dwordx4 a[68:71], v233, s[12:15], s72 offen offset:1024 sc0 sc1\n\t" \
    "ds_read_b64_tr_b8 v[220:221], v227 offset:8192\n\t" \
    "v_mul_f32 v253, v253, v254\n\t" \
    "v_mul_f32 v254, 0xbfb8aa3b, v253\n\t" \
    "v_rcp_f32 v251, v253\n\t" \
    "v_mul_f32 v252, 0x3bbd4680, v254\n\t" \
    "s_nop 1\n\t" \
    "v_readfirstlane_b32 s76, v253\n\t" \
    "v_readfirstlane_b32 s77, v254\n\t" \
    "v_readfirstlane_b32 s78, v251\n\t" \
    "v_readfirstlane_b32 s71, v252\n\t" \
    "s_nop 3\n\t" \
    "v_mfma_f32_16x16x128_f8f6f4 v[12:15], a[24:31], v[192:199], v[12:15] cbsz:1 blgp:1\n\t" \
    "ds_read_b64_tr_b8 v[222:223], v227 offset:9216\n\t" \
    "s_waitcnt lgkmcnt(8)\n\t" \
    "v_mfma_f32_16x16x128_f8f6f4 v[16:19], a[0:7], v[200:207], v[16:19] cbsz:1 blgp:1\n\t" \
    "buffer_load_dwordx4 a[72:75], v233, s[12:15], s72 offen offset:2048 sc0 sc1\n\t" \
    "ds_read_b64_tr_b8 v[192:193], v228 offset:0\n\t" \
    "v_mfma_f32_16x16x128_f8f6f4 v[20:23], a[8:15], v[200:207], v[20:23] cbsz:1 blgp:1\n\t" \
    "ds_read_b64_tr_b8 v[194:195], v228 offset:1024\n\t" \
    "v_mfma_f32_16x16x128_f8f6f4 v[24:27], a[16:23], v[200:207], v[24:27] cbsz:1 blgp:1\n\t" \
    "buffer_load_dwordx4 a[76:79], v233, s[12:15], s72 offen offset:3072 sc0 sc1\n\t" \
    "ds_read_b64_tr_b8 v[196:197], v228 offset:8192\n\t" \
    "v_mfma_f32_16x16x128_f8f6f4 v[28:31], a[24:31], v[200:207], v[28:31] cbsz:1 blgp:1\n\t" \
    "ds_read_b64_tr_b8 v[198:199], v228 offset:9216\n\t" \
    "s_waitcnt lgkmcnt(8)\n\t" \
    "v_mfma_f32_16x16x128_f8f6f4 v[32:35], a[0:7], v[208:215], v[32:35] cbsz:1 blgp:1\n\t" \
    "ds_read_b64_tr_b8 v[200:201], v229 offset:0\n\t" \
    "v_mfma_f32_16x16x128_f8f6f4 v[36:39], a[8:15], v[208:215], v[36:39] cbsz:1 blgp:1\n\t" \
    "buffer_load_dwordx4 a[80:83], v233, s[12:15], s73 offen offset:0 sc0 sc1\n\t" \
    "ds_read_b64_tr_b8 v[202:203], v229 offset:1024\n\t" \
    "v_mfma_f32_16x16x128_f8f6f4 v[40:43], a[16:23], v[208:215], v[40:43] cbsz:1 blgp:1\n\t" \
    "ds_read_b64_tr_b8 v[204:205], v229 offset:8192\n\t" \
    "v_mfma_f32_16x16x128_f8f6f4 v[44:47], a[24:31], v[208:215], v[44:47] cbsz:1 blgp:1\n\t" \
    "buffer_load_dwordx4 a[84:87], v233, s[12:15], s73 offen offset:1024 sc0 sc1\n\t" \
    "ds_read_b64_tr_b8 v[206:207], v229 offset:9216\n\t" \
    "s_waitcnt lgkmcnt(8)\n\t" \
    "v_mfma_f32_16x16x128_f8f6f4 v[48:51], a[0:7], v[216:223], v[48:51] cbsz:1 blgp:1\n\t" \
    "ds_read_b64_tr_b8 v[208:209], v230 offset:0\n\t" \
    "v_mfma_f32_16x16x128_f8f6f4 v[52:55], a[8:15], v[216:223], v[52:55] cbsz:1 blgp:1\n\t" \
    "s_mov_b32 s72, 851968\n\t" \
    "s_mov_b32 s73, 856064\n\t" \
    "buffer_load_dwordx4 a[96:99], v233, s[12:15], s72 offen offset:0 sc0 sc1\n\t" \
    "ds_read_b64_tr_b8 v[210:211], v230 offset:1024\n\t" \
    "v_mfma_f32_16x16x128_f8f6f4 v[56:59], a[16:23], v[216:223], v[56:59] cbsz:1 blgp:1\n\t" \
    "buffer_load_dwordx4 a[100:103], v233, s[12:15], s72 offen offset:1024 sc0 sc1\n\t" \
    "ds_read_b64_tr_b8 v[212:213], v230 offset:8192\n\t" \
    "v_mfma_f32_16x16x128_f8f6f4 v[60:63], a[24:31], v[216:223], v[60:63] cbsz:1 blgp:1\n\t" \
    "buffer_load_dwordx4 a[104:107], v233, s[12:15], s72 offen offset:2048 sc0 sc1\n\t" \
    "ds_read_b64_tr_b8 v[214:215], v230 offset:9216\n\t" \
    "s_waitcnt lgkmcnt(8)\n\t" \
    "v_mfma_f32_16x16x128_f8f6f4 v[64:67], a[0:7], v[192:199], v[64:67] cbsz:1 blgp:1\n\t" \
    "buffer_load_dwordx4 a[108:111], v233, s[12:15], s72 offen offset:3072 sc0 sc1\n\t" \
    "ds_read_b64_tr_b8 v[216:217], v231 offset:0\n\t" \
    "v_mfma_f32_16x16x128_f8f6f4 v[68:71], a[8:15], v[192:199], v[68:71] cbsz:1 blgp:1\n\t" \
    "buffer_load_dwordx4 a[112:115], v233, s[12:15], s73 offen offset:0 sc0 sc1\n\t" \
    "ds_read_b64_tr_b8 v[218:219], v231 offset:1024\n\t" \
    "v_mfma_f32_16x16x128_f8f6f4 v[72:75], a[16:23], v[192:199], v[72:75] cbsz:1 blgp:1\n\t" \
    "buffer_load_dwordx4 a[116:119], v233, s[12:15], s73 offen offset:1024 sc0 sc1\n\t" \
    "ds_read_b64_tr_b8 v[220:221], v231 offset:8192\n\t" \
    "v_mfma_f32_16x16x128_f8f6f4 v[76:79], a[24:31], v[192:199], v[76:79] cbsz:1 blgp:1\n\t" \
    "s_mov_b32 s72, 786432\n\t" \
    "s_mov_b32 s73, 790528\n\t" \
    "buffer_load_dwordx4 a[88:91], v233, s[12:15], s73 offen offset:2048 sc0 sc1\n\t" \
    "s_mov_b32 s72, 851968\n\t" \
    "s_mov_b32 s73, 856064\n\t" \
    "buffer_load_dwordx4 a[120:123], v233, s[12:15], s73 offen offset:2048 sc0 sc1\n\t" \
    "ds_read_b64_tr_b8 v[222:223], v231 offset:9216\n\t" \
    "s_waitcnt lgkmcnt(8)\n\t" \
    "v_mfma_f32_16x16x128_f8f6f4 v[80:83], a[0:7], v[200:207], v[80:83] cbsz:1 blgp:1\n\t" \
    "s_barrier\n\t" \
    "s_mov_b32 s72, 786432\n\t" \
    "s_mov_b32 s73, 790528\n\t" \
    "buffer_load_dwordx4 a[92:95], v233, s[12:15], s73 offen offset:3072 sc0 sc1\n\t" \
    "s_mov_b32 s72, 851968\n\t" \
    "s_mov_b32 s73, 856064\n\t" \
    "buffer_load_dwordx4 a[124:127], v233, s[12:15], s73 offen offset:3072 sc0 sc1\n\t" \
    "ds_read_b64_tr_b8 v[192:193], v224 offset:16384\n\t" \
    "v_mfma_f32_16x16x128_f8f6f4 v[84:87], a[8:15], v[200:207], v[84:87] cbsz:1 blgp:1\n\t" \
    "s_cmp_eq_u32 s79, 2\n\t" \
    "s_cbranch_scc1 L_flg2_%=\n\t" \
    "s_mov_b64 exec, 1\n\t" \
    "global_store_dword v236, v236, s[56:57] offset:32\n\t" \
    "s_mov_b64 exec, -1\n\t" \
    "s_branch L_flge2_%=\n\t" \
    "L_flg2_%=:\n\t" \
    "s_waitcnt vmcnt(49)\n\t" \
    "v_mov_b32 v251, s75\n\t" \
    "s_mov_b64 exec, 1\n\t" \
    "global_store_dword v236, v251, s[52:53] sc0 sc1\n\t" \
    "s_mov_b64 exec, -1\n\t" \
    "L_flge2_%=:\n\t" \
    "s_cmp_eq_u32 s79, 3\n\t" \
    "s_cbranch_scc1 L_pay3_%=\n\t" \
    "s_mov_b64 exec, 1\n\t" \
    "global_store_dword v236, v236, s[56:57] offset:32\n\t" \
    "global_store_dword v236, v236, s[56:57] offset:32\n\t" \
    "s_mov_b64 exec, -1\n\t" \
    "s_branch L_paye3_%=\n\t" \
    "L_pay3_%=:\n\t" \
    "global_store_dwordx4 v237, v[240:243], s[48:49] sc0 sc1\n\t" \
    "global_store_dwordx4 v237, v[244:247], s[48:49] offset:1024 sc0 sc1\n\t" \
    "L_paye3_%=:\n\t" \
    "s_cmp_eq_u32 s47, 0\n\t" \
    "s_cselect_b32 s87, s28, s29\n\t" \
    "s_add_u32 s85, s87, 2097152\n\t" \
    "s_add_u32 m0, s44, 65536\n\t" \
    "s_add_u32 s74, s85, 0\n\t" \
    "buffer_load_dwordx4 v232, s[8:11], s74 offen  lds\n\t" \
    "ds_read_b64_tr_b8 v[194:195], v224 offset:17408\n\t" \
    "v_mfma_f32_16x16x128_f8f6f4 v[88:91], a[16:23], v[200:207], v[88:91] cbsz:1 blgp:1\n\t" \
    "s_add_u32 m0, s44, 69632\n\t" \
    "s_add_u32 s74, s85, 524288\n\t" \
    "buffer_load_dwordx4 v232, s[8:11], s74 offen  lds\n\t" \
    "ds_read_b64_tr_b8 v[196:197], v224 offset:24576\n\t" \
    "v_mfma_f32_16x16x128_f8f6f4 v[92:95], a[24:31], v[200:207], v[92:95] cbsz:1 blgp:1\n\t" \
    "s_add_u32 m0, s44, 73728\n\t" \
    "s_add_u32 s74, s85, 1048576\n\t" \
    "buffer_load_dwordx4 v232, s[8:11], s74 offen  lds\n\t" \
    "ds_read_b64_tr_b8 v[198:199], v224 offset:25600\n\t" \
    "s_waitcnt lgkmcnt(8)\n\t" \
    "v_mfma_f32_16x16x128_f8f6f4 v[96:99], a[0:7], v[208:215], v[96:99] cbsz:1 blgp:1\n\t" \
    "s_add_u32 m0, s44, 77824\n\t" \
    "s_add_u32 s74, s85, 1572864\n\t" \
    "buffer_load_dwordx4 v232, s[8:11], s74 offen  lds\n\t" \
    "ds_read_b64_tr_b8 v[200:201], v225 offset:16384\n\t" \
    "v_mfma_f32_16x16x128_f8f6f4 v[100:103], a[8:15], v[208:215], v[100:103] cbsz:1 blgp:1\n\t" \
    "s_cmp_eq_u32 s47, 0\n\t" \
    "s_cselect_b32 s87, s30, s31\n\t" \
    "s_add_u32 s85, s87, 2097152\n\t" \
    "s_add_u32 m0, s44, 98304\n\t" \
    "s_add_u32 s74, s85, 0\n\t" \
    "buffer_load_dwordx4 v232, s[8:11], s74 offen  lds\n\t" \
    "ds_read_b64_tr_b8 v[202:203], v225 offset:17408\n\t" \
    "v_mfma_f32_16x16x128_f8f6f4 v[104:107], a[16:23], v[208:215], v[104:107] cbsz:1 blgp:1\n\t" \
    "s_add_u32 m0, s44, 102400\n\t" \
    "s_add_u32 s74, s85, 524288\n\t" \
    "buffer_load_dwordx4 v232, s[8:11], s74 offen  lds\n\t" \
    "ds_read_b64_tr_b8 v[204:205], v225 offset:24576\n\t" \
    "v_mfma_f32_16x16x128_f8f6f4 v[108:111], a[24:31], v[208:215], v[108:111] cbsz:1 blgp:1\n\t" \
    "s_add_u32 m0, s44, 106496\n\t" \
    "s_add_u32 s74, s85, 1048576\n\t" \
    "buffer_load_dwordx4 v232, s[8:11], s74 offen  lds\n\t" \
    "ds_read_b64_tr_b8 v[206:207], v225 offset:25600\n\t" \
    "s_waitcnt lgkmcnt(8)\n\t" \
    "v_mfma_f32_16x16x128_f8f6f4 v[112:115], a[0:7], v[216:223], v[112:115] cbsz:1 blgp:1\n\t" \
    "s_add_u32 m0, s44, 110592\n\t" \
    "s_add_u32 s74, s85, 1572864\n\t" \
    "buffer_load_dwordx4 v232, s[8:11], s74 offen  lds\n\t" \
    "ds_read_b64_tr_b8 v[208:209], v226 offset:16384\n\t" \
    "v_mfma_f32_16x16x128_f8f6f4 v[116:119], a[8:15], v[216:223], v[116:119] cbsz:1 blgp:1\n\t" \
    "ds_read_b64_tr_b8 v[210:211], v226 offset:17408\n\t" \
    "v_mfma_f32_16x16x128_f8f6f4 v[120:123], a[16:23], v[216:223], v[120:123] cbsz:1 blgp:1\n\t" \
    "ds_read_b64_tr_b8 v[212:213], v226 offset:24576\n\t" \
    "v_mfma_f32_16x16x128_f8f6f4 v[124:127], a[24:31], v[216:223], v[124:127] cbsz:1 blgp:1\n\t" \
    "ds_read_b64_tr_b8 v[214:215], v226 offset:25600\n\t" \
    "s_waitcnt vmcnt(27)\n\t" \
    "s_waitcnt lgkmcnt(8)\n\t" \
    "v_mfma_f32_16x16x128_f8f6f4 v[0:3], a[32:39], v[192:199], v[0:3] cbsz:1 blgp:1\n\t" \
    "ds_read_b64_tr_b8 v[216:217], v227 offset:16384\n\t" \
    "v_mfma_f32_16x16x128_f8f6f4 v[4:7], a[40:47], v[192:199], v[4:7] cbsz:1 blgp:1\n\t" \
    "ds_read_b64_tr_b8 v[218:219], v227 offset:17408\n\t" \
    "v_mfma_f32_16x16x128_f8f6f4 v[8:11], a[48:55], v[192:199], v[8:11] cbsz:1 blgp:1\n\t" \
    "ds_read_b64_tr_b8 v[220:221], v227 offset:24576\n\t" \
    "v_mfma_f32_16x16x128_f8f6f4 v[12:15], a[56:63], v[192:199], v[12:15] cbsz:1 blgp:1\n\t" \
    "ds_read_b64_tr_b8 v[222:223], v227 offset:25600\n\t" \
    "s_waitcnt lgkmcnt(8)\n\t" \
    "v_mfma_f32_16x16x128_f8f6f4 v[16:19], a[32:39], v[200:207], v[16:19] cbsz:1 blgp:1\n\t" \
    "ds_read_b64_tr_b8 v[192:193], v228 offset:16384\n\t" \
    "v_mfma_f32_16x16x128_f8f6f4 v[20:23], a[40:47], v[200:207], v[20:23] cbsz:1 blgp:1\n\t" \
    "ds_read_b64_tr_b8 v[194:195], v228 offset:17408\n\t" \
    "v_mfma_f32_16x16x128_f8f6f4 v[24:27], a[48:55], v[200:207], v[24:27] cbsz:1 blgp:1\n\t" \
    "ds_read_b64_tr_b8 v[196:197], v228 offset:24576\n\t" \
    "v_mfma_f32_16x16x128_f8f6f4 v[28:31], a[56:63], v[200:207], v[28:31] cbsz:1 blgp:1\n\t" \
    "ds_read_b64_tr_b8 v[198:199], v228 offset:25600\n\t" \
    "s_waitcnt lgkmcnt(8)\n\t" \
    "v_mfma_f32_16x16x128_f8f6f4 v[32:35], a[32:39], v[208:215], v[32:35] cbsz:1 blgp:1\n\t" \
    "ds_read_b64_tr_b8 v[200:201], v229 offset:16384\n\t" \
    "v_mfma_f32_16x16x128_f8f6f4 v[36:39], a[40:47], v[208:215], v[36:39] cbsz:1 blgp:1\n\t" \
    "ds_read_b64_tr_b8 v[202:203], v229 offset:17408\n\t" \
    "v_mfma_f32_16x16x128_f8f6f4 v[40:43], a[48:55], v[208:215], v[40:43] cbsz:1 blgp:1\n\t" \
    "ds_read_b64_tr_b8 v[204:205], v229 offset:24576\n\t" \
    "v_mfma_f32_16x16x128_f8f6f4 v[44:47], a[56:63], v[208:215], v[44:47] cbsz:1 blgp:1\n\t" \
    "ds_read_b64_tr_b8 v[206:207], v229 offset:25600\n\t" \
    "s_waitcnt lgkmcnt(8)\n\t" \
    "v_mfma_f32_16x16x128_f8f6f4 v[48:51], a[32:39], v[216:223], v[48:51] cbsz:1 blgp:1\n\t" \
    "ds_read_b64_tr_b8 v[208:209], v230 offset:16384\n\t" \
    "v_mfma_f32_16x16x128_f8f6f4 v[52:55], a[40:47], v[216:223], v[52:55] cbsz:1 blgp:1\n\t" \
    "ds_read_b64_tr_b8 v[210:211], v230 offset:17408\n\t" \
    "v_mfma_f32_16x16x128_f8f6f4 v[56:59], a[48:55], v[216:223], v[56:59] cbsz:1 blgp:1\n\t" \
    "ds_read_b64_tr_b8 v[212:213], v230 offset:24576\n\t" \
    "v_mfma_f32_16x16x128_f8f6f4 v[60:63], a[56:63], v[216:223], v[60:63] cbsz:1 blgp:1\n\t" \
    "ds_read_b64_tr_b8 v[214:215], v230 offset:25600\n\t" \
    "s_waitcnt lgkmcnt(8)\n\t" \
    "v_mfma_f32_16x16x128_f8f6f4 v[64:67], a[32:39], v[192:199], v[64:67] cbsz:1 blgp:1\n\t" \
    "ds_read_b64_tr_b8 v[216:217], v231 offset:16384\n\t" \
    "v_mfma_f32_16x16x128_f8f6f4 v[68:71], a[40:47], v[192:199], v[68:71] cbsz:1 blgp:1\n\t" \
    "ds_read_b64_tr_b8 v[218:219], v231 offset:17408\n\t" \
    "v_mfma_f32_16x16x128_f8f6f4 v[72:75], a[48:55], v[192:199], v[72:75] cbsz:1 blgp:1\n\t" \
    "ds_read_b64_tr_b8 v[220:221], v231 offset:24576\n\t" \
    "v_mfma_f32_16x16x128_f8f6f4 v[76:79], a[56:63], v[192:199], v[76:79] cbsz:1 blgp:1\n\t" \
    "s_cmp_eq_u32 s47, 1\n\t" \
    "s_cselect_b32 s87, s28, s29\n\t" \
    "s_add_u32 s86, s87, 2097152\n\t" \
    "s_add_u32 m0, s46, 65536\n\t" \
    "s_add_u32 s74, s86, 0\n\t" \
    "buffer_load_dwordx4 v232, s[8:11], s74 offen  lds\n\t" \
    "ds_read_b64_tr_b8 v[222:223], v231 offset:25600\n\t" \
    "s_waitcnt lgkmcnt(8)\n\t" \
    "v_mfma_f32_16x16x128_f8f6f4 v[80:83], a[32:39], v[200:207], v[80:83] cbsz:1 blgp:1\n\t" \
    "s_barrier\n\t" \
    "ds_read_b64_tr_b8 v[192:193], v224 offset:32768\n\t" \
    "v_mfma_f32_16x16x128_f8f6f4 v[84:87], a[40:47], v[200:207], v[84:87] cbsz:1 blgp:1\n\t" \
    "ds_read_b64_tr_b8 v[194:195], v224 offset:33792\n\t" \
    "v_mfma_f32_16x16x128_f8f6f4 v[88:91], a[48:55], v[200:207], v[88:91] cbsz:1 blgp:1\n\t" \
    "s_add_u32 m0, s46, 69632\n\t" \
    "s_add_u32 s74, s86, 524288\n\t" \
    "buffer_load_dwordx4 v232, s[8:11], s74 offen  lds\n\t" \
    "ds_read_b64_tr_b8 v[196:197], v224 offset:40960\n\t" \
    "v_mfma_f32_16x16x128_f8f6f4 v[92:95], a[56:63], v[200:207], v[92:95] cbsz:1 blgp:1\n\t" \
    "ds_read_b64_tr_b8 v[198:199], v224 offset:41984\n\t" \
    "s_waitcnt lgkmcnt(8)\n\t" \
    "v_mfma_f32_16x16x128_f8f6f4 v[96:99], a[32:39], v[208:215], v[96:99] cbsz:1 blgp:1\n\t" \
    "s_add_u32 m0, s46, 73728\n\t" \
    "s_add_u32 s74, s86, 1048576\n\t" \
    "buffer_load_dwordx4 v232, s[8:11], s74 offen  lds\n\t" \
    "ds_read_b64_tr_b8 v[200:201], v225 offset:32768\n\t" \
    "v_mfma_f32_16x16x128_f8f6f4 v[100:103], a[40:47], v[208:215], v[100:103] cbsz:1 blgp:1\n\t" \
    "ds_read_b64_tr_b8 v[202:203], v225 offset:33792\n\t" \
    "v_mfma_f32_16x16x128_f8f6f4 v[104:107], a[48:55], v[208:215], v[104:107] cbsz:1 blgp:1\n\t" \
    "s_add_u32 m0, s46, 77824\n\t" \
    "s_add_u32 s74, s86, 1572864\n\t" \
    "buffer_load_dwordx4 v232, s[8:11], s74 offen  lds\n\t" \
    "ds_read_b64_tr_b8 v[204:205], v225 offset:40960\n\t" \
    "v_mfma_f32_16x16x128_f8f6f4 v[108:111], a[56:63], v[208:215], v[108:111] cbsz:1 blgp:1\n\t" \
    "ds_read_b64_tr_b8 v[206:207], v225 offset:41984\n\t" \
    "s_waitcnt lgkmcnt(8)\n\t" \
    "v_mfma_f32_16x16x128_f8f6f4 v[112:115], a[32:39], v[216:223], v[112:115] cbsz:1 blgp:1\n\t" \
    "ds_read_b64_tr_b8 v[208:209], v226 offset:32768\n\t" \
    "v_mfma_f32_16x16x128_f8f6f4 v[116:119], a[40:47], v[216:223], v[116:119] cbsz:1 blgp:1\n\t" \
    "s_cmp_eq_u32 s47, 1\n\t" \
    "s_cselect_b32 s87, s30, s31\n\t" \
    "s_add_u32 s86, s87, 2097152\n\t" \
    "s_add_u32 m0, s46, 98304\n\t" \
    "s_add_u32 s74, s86, 0\n\t" \
    "buffer_load_dwordx4 v232, s[8:11], s74 offen  lds\n\t" \
    "ds_read_b64_tr_b8 v[210:211], v226 offset:33792\n\t" \
    "v_mfma_f32_16x16x128_f8f6f4 v[120:123], a[48:55], v[216:223], v[120:123] cbsz:1 blgp:1\n\t" \
    "ds_read_b64_tr_b8 v[212:213], v226 offset:40960\n\t" \
    "v_mfma_f32_16x16x128_f8f6f4 v[124:127], a[56:63], v[216:223], v[124:127] cbsz:1 blgp:1\n\t" \
    "s_add_u32 m0, s46, 102400\n\t" \
    "s_add_u32 s74, s86, 524288\n\t" \
    "buffer_load_dwordx4 v232, s[8:11], s74 offen  lds\n\t" \
    "ds_read_b64_tr_b8 v[214:215], v226 offset:41984\n\t" \
    "s_waitcnt vmcnt(18)\n\t" \
    "s_waitcnt lgkmcnt(8)\n\t" \
    "v_mfma_f32_16x16x128_f8f6f4 v[0:3], a[64:71], v[192:199], v[0:3] cbsz:1 blgp:1\n\t" \
    "ds_read_b64_tr_b8 v[216:217], v227 offset:32768\n\t" \
    "v_mfma_f32_16x16x128_f8f6f4 v[4:7], a[72:79], v[192:199], v[4:7] cbsz:1 blgp:1\n\t" \
    "s_add_u32 m0, s46, 106496\n\t" \
    "s_add_u32 s74, s86, 1048576\n\t" \
    "buffer_load_dwordx4 v232, s[8:11], s74 offen  lds\n\t" \
    "ds_read_b64_tr_b8 v[218:219], v227 offset:33792\n\t" \
    "v_mfma_f32_16x16x128_f8f6f4 v[8:11], a[80:87], v[192:199], v[8:11] cbsz:1 blgp:1\n\t" \
    "ds_read_b64_tr_b8 v[220:221], v227 offset:40960\n\t" \
    "v_mfma_f32_16x16x128_f8f6f4 v[12:15], a[88:95], v[192:199], v[12:15] cbsz:1 blgp:1\n\t" \
    "ds_read_b64_tr_b8 v[222:223], v227 offset:41984\n\t" \
    "s_waitcnt lgkmcnt(8)\n\t" \
    "v_mfma_f32_16x16x128_f8f6f4 v[16:19], a[64:71], v[200:207], v[16:19] cbsz:1 blgp:1\n\t" \
    "s_add_u32 m0, s46, 110592\n\t" \
    "s_add_u32 s74, s86, 1572864\n\t" \
    "buffer_load_dwordx4 v232, s[8:11], s74 offen  lds\n\t" \
    "ds_read_b64_tr_b8 v[192:193], v228 offset:32768\n\t" \
    "v_mfma_f32_16x16x128_f8f6f4 v[20:23], a[72:79], v[200:207], v[20:23] cbsz:1 blgp:1\n\t" \
    "ds_read_b64_tr_b8 v[194:195], v228 offset:33792\n\t" \
    "v_mfma_f32_16x16x128_f8f6f4 v[24:27], a[80:87], v[200:207], v[24:27] cbsz:1 blgp:1\n\t" \
    "s_mov_b32 s72, 917504\n\t" \
    "s_mov_b32 s73, 921600\n\t" \
    "buffer_load_dwordx4 a[128:131], v233, s[12:15], s72 offen offset:0 sc0 sc1\n\t" \
    "ds_read_b64_tr_b8 v[196:197], v228 offset:40960\n\t" \
    "v_mfma_f32_16x16x128_f8f6f4 v[28:31], a[88:95], v[200:207], v[28:31] cbsz:1 blgp:1\n\t" \
    "ds_read_b64_tr_b8 v[198:199], v228 offset:41984\n\t" \
    "s_waitcnt lgkmcnt(8)\n\t" \
    "v_mfma_f32_16x16x128_f8f6f4 v[32:35], a[64:71], v[208:215], v[32:35] cbsz:1 blgp:1\n\t" \
    "buffer_load_dwordx4 a[132:135], v233, s[12:15], s72 offen offset:1024 sc0 sc1\n\t" \
    "ds_read_b64_tr_b8 v[200:201], v229 offset:32768\n\t" \
    "v_mfma_f32_16x16x128_f8f6f4 v[36:39], a[72:79], v[208:215], v[36:39] cbsz:1 blgp:1\n\t" \
    "ds_read_b64_tr_b8 v[202:203], v229 offset:33792\n\t" \
    "v_mfma_f32_16x16x128_f8f6f4 v[40:43], a[80:87], v[208:215], v[40:43] cbsz:1 blgp:1\n\t" \
    "ds_read_b64_tr_b8 v[204:205], v229 offset:40960\n\t" \
    "v_mfma_f32_16x16x128_f8f6f4 v[44:47], a[88:95], v[208:215], v[44:47] cbsz:1 blgp:1\n\t" \
    "buffer_load_dwordx4 a[136:139], v233, s[12:15], s72 offen offset:2048 sc0 sc1\n\t" \
    "ds_read_b64_tr_b8 v[206:207], v229 offset:41984\n\t" \
    "s_waitcnt lgkmcnt(8)\n\t" \
    "v_mfma_f32_16x16x128_f8f6f4 v[48:51], a[64:71], v[216:223], v[48:51] cbsz:1 blgp:1\n\t" \
    "ds_read_b64_tr_b8 v[208:209], v230 offset:32768\n\t" \
    "v_mfma_f32_16x16x128_f8f6f4 v[52:55], a[72:79], v[216:223], v[52:55] cbsz:1 blgp:1\n\t" \
    "buffer_load_dwordx4 a[140:143], v233, s[12:15], s72 offen offset:3072 sc0 sc1\n\t" \
    "ds_read_b64_tr_b8 v[210:211], v230 offset:33792\n\t" \
    "v_mfma_f32_16x16x128_f8f6f4 v[56:59], a[80:87], v[216:223], v[56:59] cbsz:1 blgp:1\n\t" \
    "ds_read_b64_tr_b8 v[212:213], v230 offset:40960\n\t" \
    "v_mfma_f32_16x16x128_f8f6f4 v[60:63], a[88:95], v[216:223], v[60:63] cbsz:1 blgp:1\n\t" \
    "buffer_load_dwordx4 a[144:147], v233, s[12:15], s73 offen offset:0 sc0 sc1\n\t" \
    "ds_read_b64_tr_b8 v[214:215], v230 offset:41984\n\t" \
    "s_waitcnt lgkmcnt(8)\n\t" \
    "v_mfma_f32_16x16x128_f8f6f4 v[64:67], a[64:71], v[192:199], v[64:67] cbsz:1 blgp:1\n\t" \
    "ds_read_b64_tr_b8 v[216:217], v231 offset:32768\n\t" \
    "v_mfma_f32_16x16x128_f8f6f4 v[68:71], a[72:79], v[192:199], v[68:71] cbsz:1 blgp:1\n\t" \
    "ds_read_b64_tr_b8 v[218:219], v231 offset:33792\n\t" \
    "v_mfma_f32_16x16x128_f8f6f4 v[72:75], a[80:87], v[192:199], v[72:75] cbsz:1 blgp:1\n\t" \
    "buffer_load_dwordx4 a[148:151], v233, s[12:15], s73 offen offset:1024 sc0 sc1\n\t" \
    "ds_read_b64_tr_b8 v[220:221], v231 offset:40960\n\t" \
    "v_mfma_f32_16x16x128_f8f6f4 v[76:79], a[88:95], v[192:199], v[76:79] cbsz:1 blgp:1\n\t" \
    "ds_read_b64_tr_b8 v[222:223], v231 offset:41984\n\t" \
    "s_waitcnt lgkmcnt(8)\n\t" \
    "v_mfma_f32_16x16x128_f8f6f4 v[80:83], a[64:71], v[200:207], v[80:83] cbsz:1 blgp:1\n\t" \
    "s_barrier\n\t" \
    "buffer_load_dwordx4 a[152:155], v233, s[12:15], s73 offen offset:2048 sc0 sc1\n\t" \
    "ds_read_b64_tr_b8 v[192:193], v224 offset:49152\n\t" \
    "v_mfma_f32_16x16x128_f8f6f4 v[84:87], a[72:79], v[200:207], v[84:87] cbsz:1 blgp:1\n\t" \
    "ds_read_b64_tr_b8 v[194:195], v224 offset:50176\n\t" \
    "v_mfma_f32_16x16x128_f8f6f4 v[88:91], a[80:87], v[200:207], v[88:91] cbsz:1 blgp:1\n\t" \
    "buffer_load_dwordx4 a[156:159], v233, s[12:15], s73 offen offset:3072 sc0 sc1\n\t" \
    "ds_read_b64_tr_b8 v[196:197], v224 offset:57344\n\t" \
    "v_mfma_f32_16x16x128_f8f6f4 v[92:95], a[88:95], v[200:207], v[92:95] cbsz:1 blgp:1\n\t" \
    "ds_read_b64_tr_b8 v[198:199], v224 offset:58368\n\t" \
    "s_waitcnt lgkmcnt(8)\n\t" \
    "v_mfma_f32_16x16x128_f8f6f4 v[96:99], a[64:71], v[208:215], v[96:99] cbsz:1 blgp:1\n\t" \
    "ds_read_b64_tr_b8 v[200:201], v225 offset:49152\n\t" \
    "v_mfma_f32_16x16x128_f8f6f4 v[100:103], a[72:79], v[208:215], v[100:103] cbsz:1 blgp:1\n\t" \
    "s_mov_b32 s72, 983040\n\t" \
    "s_mov_b32 s73, 987136\n\t" \
    "buffer_load_dwordx4 a[160:163], v233, s[12:15], s72 offen offset:0 sc0 sc1\n\t" \
    "ds_read_b64_tr_b8 v[202:203], v225 offset:50176\n\t" \
    "v_mfma_f32_16x16x128_f8f6f4 v[104:107], a[80:87], v[208:215], v[104:107] cbsz:1 blgp:1\n\t" \
    "ds_read_b64_tr_b8 v[204:205], v225 offset:57344\n\t" \
    "v_mfma_f32_16x16x128_f8f6f4 v[108:111], a[88:95], v[208:215], v[108:111] cbsz:1 blgp:1\n\t" \
    "buffer_load_dwordx4 a[164:167], v233, s[12:15], s72 offen offset:1024 sc0 sc1\n\t" \
    "ds_read_b64_tr_b8 v[206:207], v225 offset:58368\n\t" \
    "s_waitcnt lgkmcnt(8)\n\t" \
    "v_mfma_f32_16x16x128_f8f6f4 v[112:115], a[64:71], v[216:223], v[112:115] cbsz:1 blgp:1\n\t" \
    "ds_read_b64_tr_b8 v[208:209], v226 offset:49152\n\t" \
    "v_mfma_f32_16x16x128_f8f6f4 v[116:119], a[72:79], v[216:223], v[116:119] cbsz:1 blgp:1\n\t" \
    "buffer_load_dwordx4 a[168:171], v233, s[12:15], s72 offen offset:2048 sc0 sc1\n\t" \
    "ds_read_b64_tr_b8 v[210:211], v226 offset:50176\n\t" \
    "v_mfma_f32_16x16x128_f8f6f4 v[120:123], a[80:87], v[216:223], v[120:123] cbsz:1 blgp:1\n\t" \
    "ds_read_b64_tr_b8 v[212:213], v226 offset:57344\n\t" \
    "v_mfma_f32_16x16x128_f8f6f4 v[124:127], a[88:95], v[216:223], v[124:127] cbsz:1 blgp:1\n\t" \
    "ds_read_b64_tr_b8 v[214:215], v226 offset:58368\n\t" \
    "s_waitcnt vmcnt(30)\n\t" \
    "s_waitcnt lgkmcnt(8)\n\t" \
    "v_mfma_f32_16x16x128_f8f6f4 v[0:3], a[96:103], v[192:199], v[0:3] cbsz:1 blgp:1\n\t" \
    "buffer_load_dwordx4 a[172:175], v233, s[12:15], s72 offen offset:3072 sc0 sc1\n\t" \
    "ds_read_b64_tr_b8 v[216:217], v227 offset:49152\n\t" \
    "v_mfma_f32_16x16x128_f8f6f4 v[4:7], a[104:111], v[192:199], v[4:7] cbsz:1 blgp:1\n\t" \
    "ds_read_b64_tr_b8 v[218:219], v227 offset:50176\n\t" \
    "v_mfma_f32_16x16x128_f8f6f4 v[8:11], a[112:119], v[192:199], v[8:11] cbsz:1 blgp:1\n\t" \
    "buffer_load_dwordx4 a[176:179], v233, s[12:15], s73 offen offset:0 sc0 sc1\n\t" \
    "ds_read_b64_tr_b8 v[220:221], v227 offset:57344\n\t" \
    "v_mfma_f32_16x16x128_f8f6f4 v[12:15], a[120:127], v[192:199], v[12:15] cbsz:1 blgp:1\n\t" \
    "ds_read_b64_tr_b8 v[222:223], v227 offset:58368\n\t" \
    "s_waitcnt lgkmcnt(8)\n\t" \
    "v_mfma_f32_16x16x128_f8f6f4 v[16:19], a[96:103], v[200:207], v[16:19] cbsz:1 blgp:1\n\t" \
    "global_load_dword v248, v235, s[54:55] offset:2048 sc0 sc1\n\t" \
    "buffer_load_dwordx4 a[180:183], v233, s[12:15], s73 offen offset:1024 sc0 sc1\n\t" \
    "ds_read_b64_tr_b8 v[192:193], v228 offset:49152\n\t" \
    "v_mfma_f32_16x16x128_f8f6f4 v[20:23], a[104:111], v[200:207], v[20:23] cbsz:1 blgp:1\n\t" \
    "ds_read_b64_tr_b8 v[194:195], v228 offset:50176\n\t" \
    "v_mfma_f32_16x16x128_f8f6f4 v[24:27], a[112:119], v[200:207], v[24:27] cbsz:1 blgp:1\n\t" \
    "ds_read_b64_tr_b8 v[196:197], v228 offset:57344\n\t" \
    "v_mfma_f32_16x16x128_f8f6f4 v[28:31], a[120:127], v[200:207], v[28:31] cbsz:1 blgp:1\n\t" \
    "buffer_load_dwordx4 a[184:187], v233, s[12:15], s73 offen offset:2048 sc0 sc1\n\t" \
    "ds_read_b64_tr_b8 v[198:199], v228 offset:58368\n\t" \
    "s_waitcnt lgkmcnt(8)\n\t" \
    "v_mfma_f32_16x16x128_f8f6f4 v[32:35], a[96:103], v[208:215], v[32:35] cbsz:1 blgp:1\n\t" \
    "ds_read_b64_tr_b8 v[200:201], v229 offset:49152\n\t" \
    "v_mfma_f32_16x16x128_f8f6f4 v[36:39], a[104:111], v[208:215], v[36:39] cbsz:1 blgp:1\n\t" \
    "buffer_load_dwordx4 a[188:191], v233, s[12:15], s73 offen offset:3072 sc0 sc1\n\t" \
    "ds_read_b64_tr_b8 v[202:203], v229 offset:50176\n\t" \
    "v_mfma_f32_16x16x128_f8f6f4 v[40:43], a[112:119], v[208:215], v[40:43] cbsz:1 blgp:1\n\t" \
    "ds_read_b64_tr_b8 v[204:205], v229 offset:57344\n\t" \
    "v_mfma_f32_16x16x128_f8f6f4 v[44:47], a[120:127], v[208:215], v[44:47] cbsz:1 blgp:1\n\t" \
    "s_mov_b32 s72, 1048576\n\t" \
    "s_mov_b32 s73, 1052672\n\t" \
    "buffer_load_dwordx4 a[192:195], v233, s[12:15], s72 offen offset:0 sc0 sc1\n\t" \
    "ds_read_b64_tr_b8 v[206:207], v229 offset:58368\n\t" \
    "s_waitcnt lgkmcnt(8)\n\t" \
    "v_mfma_f32_16x16x128_f8f6f4 v[48:51], a[96:103], v[216:223], v[48:51] cbsz:1 blgp:1\n\t" \
    "ds_read_b64_tr_b8 v[208:209], v230 offset:49152\n\t" \
    "v_mfma_f32_16x16x128_f8f6f4 v[52:55], a[104:111], v[216:223], v[52:55] cbsz:1 blgp:1\n\t" \
    "ds_read_b64_tr_b8 v[210:211], v230 offset:50176\n\t" \
    "v_mfma_f32_16x16x128_f8f6f4 v[56:59], a[112:119], v[216:223], v[56:59] cbsz:1 blgp:1\n\t" \
    "buffer_load_dwordx4 a[196:199], v233, s[12:15], s72 offen offset:1024 sc0 sc1\n\t" \
    "ds_read_b64_tr_b8 v[212:213], v230 offset:57344\n\t" \
    "v_mfma_f32_16x16x128_f8f6f4 v[60:63], a[120:127], v[216:223], v[60:63] cbsz:1 blgp:1\n\t" \
    "ds_read_b64_tr_b8 v[214:215], v230 offset:58368\n\t" \
    "s_waitcnt lgkmcnt(8)\n\t" \
    "v_mfma_f32_16x16x128_f8f6f4 v[64:67], a[96:103], v[192:199], v[64:67] cbsz:1 blgp:1\n\t" \
    "buffer_load_dwordx4 a[200:203], v233, s[12:15], s72 offen offset:2048 sc0 sc1\n\t" \
    "ds_read_b64_tr_b8 v[216:217], v231 offset:49152\n\t" \
    "v_mfma_f32_16x16x128_f8f6f4 v[68:71], a[104:111], v[192:199], v[68:71] cbsz:1 blgp:1\n\t" \
    "ds_read_b64_tr_b8 v[218:219], v231 offset:50176\n\t" \
    "v_mfma_f32_16x16x128_f8f6f4 v[72:75], a[112:119], v[192:199], v[72:75] cbsz:1 blgp:1\n\t" \
    "buffer_load_dwordx4 a[204:207], v233, s[12:15], s72 offen offset:3072 sc0 sc1\n\t" \
    "ds_read_b64_tr_b8 v[220:221], v231 offset:57344\n\t" \
    "v_mfma_f32_16x16x128_f8f6f4 v[76:79], a[120:127], v[192:199], v[76:79] cbsz:1 blgp:1\n\t" \
    "ds_read_b64_tr_b8 v[222:223], v231 offset:58368\n\t" \
    "s_waitcnt lgkmcnt(8)\n\t" \
    "v_mfma_f32_16x16x128_f8f6f4 v[80:83], a[96:103], v[200:207], v[80:83] cbsz:1 blgp:1\n\t" \
    "s_waitcnt vmcnt(33)\n\t" \
    "s_waitcnt vmcnt(25)\n\t" \
    "s_barrier\n\t" \
    "v_add_u32 v239, 65536, v224\n\t" \
    "ds_read_b64_tr_b8 v[192:193], v239 offset:0\n\t" \
    "v_mfma_f32_16x16x128_f8f6f4 v[84:87], a[104:111], v[200:207], v[84:87] cbsz:1 blgp:1\n\t" \
    "buffer_load_dwordx4 a[208:211], v233, s[12:15], s73 offen offset:0 sc0 sc1\n\t" \
    "ds_read_b64_tr_b8 v[194:195], v239 offset:1024\n\t" \
    "v_mfma_f32_16x16x128_f8f6f4 v[88:91], a[112:119], v[200:207], v[88:91] cbsz:1 blgp:1\n\t" \
    "ds_read_b64_tr_b8 v[196:197], v239 offset:8192\n\t" \
    "v_mfma_f32_16x16x128_f8f6f4 v[92:95], a[120:127], v[200:207], v[92:95] cbsz:1 blgp:1\n\t" \
    "buffer_load_dwordx4 a[212:215], v233, s[12:15], s73 offen offset:1024 sc0 sc1\n\t" \
    "ds_read_b64_tr_b8 v[198:199], v239 offset:9216\n\t" \
    "s_waitcnt lgkmcnt(8)\n\t" \
    "v_mfma_f32_16x16x128_f8f6f4 v[96:99], a[96:103], v[208:215], v[96:99] cbsz:1 blgp:1\n\t" \
    "v_add_u32 v238, 65536, v225\n\t" \
    "ds_read_b64_tr_b8 v[200:201], v238 offset:0\n\t" \
    "v_mfma_f32_16x16x128_f8f6f4 v[100:103], a[104:111], v[208:215], v[100:103] cbsz:1 blgp:1\n\t" \
    "buffer_load_dwordx4 a[216:219], v233, s[12:15], s73 offen offset:2048 sc0 sc1\n\t" \
    "ds_read_b64_tr_b8 v[202:203], v238 offset:1024\n\t" \
    "v_mfma_f32_16x16x128_f8f6f4 v[104:107], a[112:119], v[208:215], v[104:107] cbsz:1 blgp:1\n\t" \
    "ds_read_b64_tr_b8 v[204:205], v238 offset:8192\n\t" \
    "v_mfma_f32_16x16x128_f8f6f4 v[108:111], a[120:127], v[208:215], v[108:111] cbsz:1 blgp:1\n\t" \
    "ds_read_b64_tr_b8 v[206:207], v238 offset:9216\n\t" \
    "s_waitcnt lgkmcnt(8)\n\t" \
    "v_mfma_f32_16x16x128_f8f6f4 v[112:115], a[96:103], v[216:223], v[112:115] cbsz:1 blgp:1\n\t" \
    "buffer_load_dwordx4 a[220:223], v233, s[12:15], s73 offen offset:3072 sc0 sc1\n\t" \
    "v_add_u32 v239, 65536, v226\n\t" \
    "ds_read_b64_tr_b8 v[208:209], v239 offset:0\n\t" \
    "v_mfma_f32_16x16x128_f8f6f4 v[116:119], a[104:111], v[216:223], v[116:119] cbsz:1 blgp:1\n\t" \
    "ds_read_b64_tr_b8 v[210:211], v239 offset:1024\n\t" \
    "v_mfma_f32_16x16x128_f8f6f4 v[120:123], a[112:119], v[216:223], v[120:123] cbsz:1 blgp:1\n\t" \
    "s_mov_b32 s72, 1114112\n\t" \
    "s_mov_b32 s73, 1118208\n\t" \
    "buffer_load_dwordx4 a[224:227], v233, s[12:15], s72 offen offset:0 sc0 sc1\n\t" \
    "ds_read_b64_tr_b8 v[212:213], v239 offset:8192\n\t" \
    "v_mfma_f32_16x16x128_f8f6f4 v[124:127], a[120:127], v[216:223], v[124:127] cbsz:1 blgp:1\n\t" \
    "ds_read_b64_tr_b8 v[214:215], v239 offset:9216\n\t" \
    "s_waitcnt vmcnt(18)\n\t" \
    "s_waitcnt lgkmcnt(8)\n\t" \
    "v_mfma_f32_16x16x128_f8f6f4 v[0:3], a[128:135], v[192:199], v[0:3] cbsz:1 blgp:1\n\t" \
    "buffer_load_dwordx4 a[228:231], v233, s[12:15], s72 offen offset:1024 sc0 sc1\n\t" \
    "v_add_u32 v238, 65536, v227\n\t" \
    "ds_read_b64_tr_b8 v[216:217], v238 offset:0\n\t" \
    "v_mfma_f32_16x16x128_f8f6f4 v[4:7], a[136:143], v[192:199], v[4:7] cbsz:1 blgp:1\n\t" \
    "ds_read_b64_tr_b8 v[218:219], v238 offset:1024\n\t" \
    "v_mfma_f32_16x16x128_f8f6f4 v[8:11], a[144:151], v[192:199], v[8:11] cbsz:1 blgp:1\n\t" \
    "ds_read_b64_tr_b8 v[220:221], v238 offset:8192\n\t" \
    "v_mfma_f32_16x16x128_f8f6f4 v[12:15], a[152:159], v[192:199], v[12:15] cbsz:1 blgp:1\n\t" \
    "buffer_load_dwordx4 a[232:235], v233, s[12:15], s72 offen offset:2048 sc0 sc1\n\t" \
    "ds_read_b64_tr_b8 v[222:223], v238 offset:9216\n\t" \
    "s_waitcnt lgkmcnt(8)\n\t" \
    "v_mfma_f32_16x16x128_f8f6f4 v[16:19], a[128:135], v[200:207], v[16:19] cbsz:1 blgp:1\n\t" \
    "v_add_u32 v239, 65536, v228\n\t" \
    "ds_read_b64_tr_b8 v[192:193], v239 offset:0\n\t" \
    "v_mfma_f32_16x16x128_f8f6f4 v[20:23], a[136:143], v[200:207], v[20:23] cbsz:1 blgp:1\n\t" \
    "buffer_load_dwordx4 a[236:239], v233, s[12:15], s72 offen offset:3072 sc0 sc1\n\t" \
    "ds_read_b64_tr_b8 v[194:195], v239 offset:1024\n\t" \
    "v_mfma_f32_16x16x128_f8f6f4 v[24:27], a[144:151], v[200:207], v[24:27] cbsz:1 blgp:1\n\t" \
    "ds_read_b64_tr_b8 v[196:197], v239 offset:8192\n\t" \
    "v_mfma_f32_16x16x128_f8f6f4 v[28:31], a[152:159], v[200:207], v[28:31] cbsz:1 blgp:1\n\t" \
    "buffer_load_dwordx4 a[240:243], v233, s[12:15], s73 offen offset:0 sc0 sc1\n\t" \
    "ds_read_b64_tr_b8 v[198:199], v239 offset:9216\n\t" \
    "s_waitcnt lgkmcnt(8)\n\t" \
    "v_mfma_f32_16x16x128_f8f6f4 v[32:35], a[128:135], v[208:215], v[32:35] cbsz:1 blgp:1\n\t" \
    "v_add_u32 v238, 65536, v229\n\t" \
    "ds_read_b64_tr_b8 v[200:201], v238 offset:0\n\t" \
    "v_mfma_f32_16x16x128_f8f6f4 v[36:39], a[136:143], v[208:215], v[36:39] cbsz:1 blgp:1\n\t" \
    "ds_read_b64_tr_b8 v[202:203], v238 offset:1024\n\t" \
    "v_mfma_f32_16x16x128_f8f6f4 v[40:43], a[144:151], v[208:215], v[40:43] cbsz:1 blgp:1\n\t" \
    "buffer_load_dwordx4 a[244:247], v233, s[12:15], s73 offen offset:1024 sc0 sc1\n\t" \
    "ds_read_b64_tr_b8 v[204:205], v238 offset:8192\n\t" \
    "v_mfma_f32_16x16x128_f8f6f4 v[44:47], a[152:159], v[208:215], v[44:47] cbsz:1 blgp:1\n\t" \
    "ds_read_b64_tr_b8 v[206:207], v238 offset:9216\n\t" \
    "s_waitcnt lgkmcnt(8)\n\t" \
    "v_mfma_f32_16x16x128_f8f6f4 v[48:51], a[128:135], v[216:223], v[48:51] cbsz:1 blgp:1\n\t" \
    "buffer_load_dwordx4 a[248:251], v233, s[12:15], s73 offen offset:2048 sc0 sc1\n\t" \
    "v_add_u32 v239, 65536, v230\n\t" \
    "ds_read_b64_tr_b8 v[208:209], v239 offset:0\n\t" \
    "s_waitcnt vmcnt(18)\n\t" \
    "v_cmp_gt_u32 vcc, s75, v248\n\t" \
    "s_nop 0\n\t" \
    "s_cbranch_vccz L_pd2_%=\n\t" \
    "s_memrealtime s[80:81]\n\t" \
    "s_waitcnt lgkmcnt(0)\n\t" \
    "L_ps2_%=:\n\t" \
    "global_load_dword v248, v235, s[54:55] offset:2048 sc0 sc1\n\t" \
    "global_load_dword v251, v236, s[58:59] sc1\n\t" \
    "s_memrealtime s[82:83]\n\t" \
    "s_waitcnt vmcnt(0) lgkmcnt(0)\n\t" \
    "v_cmp_gt_u32 vcc, s75, v248\n\t" \
    "s_nop 0\n\t" \
    "s_cbranch_vccz L_pd2_%=\n\t" \
    "v_readfirstlane_b32 s84, v251\n\t" \
    "s_sub_u32 s82, s82, s80\n\t" \
    "s_cmp_eq_u32 s84, s75\n\t" \
    "s_cbranch_scc1 L_pd2_%=\n\t" \
    "s_cmp_gt_u32 s82, 40000000\n\t" \
    "s_cbranch_scc1 L_pt2_%=\n\t" \
    "s_sleep 1\n\t" \
    "s_branch L_ps2_%=\n\t" \
    "L_pt2_%=:\n\t" \
    "v_mov_b32 v251, s75\n\t" \
    "global_store_dword v236, v251, s[58:59] sc1\n\t" \
    "s_waitcnt vmcnt(0)\n\t" \
    "L_pd2_%=:\n\t" \
    "v_mfma_f32_16x16x128_f8f6f4 v[52:55], a[136:143], v[216:223], v[52:55] cbsz:1 blgp:1\n\t" \
    "ds_read_b64_tr_b8 v[210:211], v239 offset:1024\n\t" \
    "v_mfma_f32_16x16x128_f8f6f4 v[56:59], a[144:151], v[216:223], v[56:59] cbsz:1 blgp:1\n\t" \
    "buffer_load_dwordx4 a[252:255], v233, s[12:15], s73 offen offset:3072 sc0 sc1\n\t" \
    "ds_read_b64_tr_b8 v[212:213], v239 offset:8192\n\t" \
    "v_mfma_f32_16x16x128_f8f6f4 v[60:63], a[152:159], v[216:223], v[60:63] cbsz:1 blgp:1\n\t" \
    "ds_read_b64_tr_b8 v[214:215], v239 offset:9216\n\t" \
    "s_waitcnt lgkmcnt(8)\n\t" \
    "v_mfma_f32_16x16x128_f8f6f4 v[64:67], a[128:135], v[192:199], v[64:67] cbsz:1 blgp:1\n\t" \
    "v_add_u32 v238, 65536, v231\n\t" \
    "ds_read_b64_tr_b8 v[216:217], v238 offset:0\n\t" \
    "v_mfma_f32_16x16x128_f8f6f4 v[68:71], a[136:143], v[192:199], v[68:71] cbsz:1 blgp:1\n\t" \
    "s_mov_b32 s72, 1179648\n\t" \
    "s_mov_b32 s73, 1183744\n\t" \
    "buffer_load_dwordx4 v[128:131], v233, s[12:15], s72 offen offset:0 sc0 sc1\n\t" \
    "ds_read_b64_tr_b8 v[218:219], v238 offset:1024\n\t" \
    "v_mfma_f32_16x16x128_f8f6f4 v[72:75], a[144:151], v[192:199], v[72:75] cbsz:1 blgp:1\n\t" \
    "ds_read_b64_tr_b8 v[220:221], v238 offset:8192\n\t" \
    "v_mfma_f32_16x16x128_f8f6f4 v[76:79], a[152:159], v[192:199], v[76:79] cbsz:1 blgp:1\n\t" \
    "buffer_load_dwordx4 v[132:135], v233, s[12:15], s72 offen offset:1024 sc0 sc1\n\t" \
    "ds_read_b64_tr_b8 v[222:223], v238 offset:9216\n\t" \
    "s_waitcnt lgkmcnt(8)\n\t" \
    "v_mfma_f32_16x16x128_f8f6f4 v[80:83], a[128:135], v[200:207], v[80:83] cbsz:1 blgp:1\n\t" \
    "s_barrier\n\t" \
    "v_add_u32 v239, 65536, v224\n\t" \
    "ds_read_b64_tr_b8 v[192:193], v239 offset:16384\n\t" \
    "v_mfma_f32_16x16x128_f8f6f4 v[84:87], a[136:143], v[200:207], v[84:87] cbsz:1 blgp:1\n\t" \
    "s_cmp_eq_u32 s79, 3\n\t" \
    "s_cbranch_scc1 L_flg3_%=\n\t" \
    "s_mov_b64 exec, 1\n\t" \
    "global_store_dword v236, v236, s[56:57] offset:32\n\t" \
    "s_mov_b64 exec, -1\n\t" \
    "s_branch L_flge3_%=\n\t" \
    "L_flg3_%=:\n\t" \
    "s_waitcnt vmcnt(51)\n\t" \
    "v_mov_b32 v251, s75\n\t" \
    "s_mov_b64 exec, 1\n\t" \
    "global_store_dword v236, v251, s[52:53] sc0 sc1\n\t" \
    "s_mov_b64 exec, -1\n\t" \
    "L_flge3_%=:\n\t" \
    "s_cmp_eq_u32 s47, 0\n\t" \
    "s_cselect_b32 s87, s25, s26\n\t" \
    "s_add_u32 s85, s87, 4194304\n\t" \
    "s_add_u32 m0, s44, 131072\n\t" \
    "s_add_u32 s74, s85, 0\n\t" \
    "buffer_load_dwordx4 v232, s[8:11], s74 offen  lds\n\t" \
    "ds_read_b64_tr_b8 v[194:195], v239 offset:17408\n\t" \
    "v_mfma_f32_16x16x128_f8f6f4 v[88:91], a[144:151], v[200:207], v[88:91] cbsz:1 blgp:1\n\t" \
    "s_add_u32 m0, s44, 135168\n\t" \
    "s_add_u32 s74, s85, 524288\n\t" \
    "buffer_load_dwordx4 v232, s[8:11], s74 offen  lds\n\t" \
    "ds_read_b64_tr_b8 v[196:197], v239 offset:24576\n\t" \
    "v_mfma_f32_16x16x128_f8f6f4 v[92:95], a[152:159], v[200:207], v[92:95] cbsz:1 blgp:1\n\t" \
    "s_add_u32 m0, s44, 139264\n\t" \
    "s_add_u32 s74, s85, 1048576\n\t" \
    "buffer_load_dwordx4 v232, s[8:11], s74 offen  lds\n\t" \
    "ds_read_b64_tr_b8 v[198:199], v239 offset:25600\n\t" \
    "s_waitcnt lgkmcnt(8)\n\t" \
    "v_mfma_f32_16x16x128_f8f6f4 v[96:99], a[128:135], v[208:215], v[96:99] cbsz:1 blgp:1\n\t" \
    "s_add_u32 m0, s44, 143360\n\t" \
    "s_add_u32 s74, s85, 1572864\n\t" \
    "buffer_load_dwordx4 v232, s[8:11], s74 offen  lds\n\t" \
    "v_add_u32 v238, 65536, v225\n\t" \
    "ds_read_b64_tr_b8 v[200:201], v238 offset:16384\n\t" \
    "v_mfma_f32_16x16x128_f8f6f4 v[100:103], a[136:143], v[208:215], v[100:103] cbsz:1 blgp:1\n\t" \
    "s_cmp_eq_u32 s47, 0\n\t" \
    "s_cselect_b32 s87, s27, s28\n\t" \
    "s_add_u32 s85, s87, 4194304\n\t" \
    "s_add_u32 m0, s44, 0\n\t" \
    "s_add_u32 s74, s85, 0\n\t" \
    "buffer_load_dwordx4 v232, s[8:11], s74 offen  lds\n\t" \
    "ds_read_b64_tr_b8 v[202:203], v238 offset:17408\n\t" \
    "v_mfma_f32_16x16x128_f8f6f4 v[104:107], a[144:151], v[208:215], v[104:107] cbsz:1 blgp:1\n\t" \
    "s_add_u32 m0, s44, 4096\n\t" \
    "s_add_u32 s74, s85, 524288\n\t" \
    "buffer_load_dwordx4 v232, s[8:11], s74 offen  lds\n\t" \
    "ds_read_b64_tr_b8 v[204:205], v238 offset:24576\n\t" \
    "v_mfma_f32_16x16x128_f8f6f4 v[108:111], a[152:159], v[208:215], v[108:111] cbsz:1 blgp:1\n\t" \
    "s_add_u32 m0, s44, 8192\n\t" \
    "s_add_u32 s74, s85, 1048576\n\t" \
    "buffer_load_dwordx4 v232, s[8:11], s74 offen  lds\n\t" \
    "ds_read_b64_tr_b8 v[206:207], v238 offset:25600\n\t" \
    "s_waitcnt lgkmcnt(8)\n\t" \
    "v_mfma_f32_16x16x128_f8f6f4 v[112:115], a[128:135], v[216:223], v[112:115] cbsz:1 blgp:1\n\t" \
    "s_add_u32 m0, s44, 12288\n\t" \
    "s_add_u32 s74, s85, 1572864\n\t" \
    "buffer_load_dwordx4 v232, s[8:11], s74 offen  lds\n\t" \
    "v_add_u32 v239, 65536, v226\n\t" \
    "ds_read_b64_tr_b8 v[208:209], v239 offset:16384\n\t" \
    "v_mfma_f32_16x16x128_f8f6f4 v[116:119], a[136:143], v[216:223], v[116:119] cbsz:1 blgp:1\n\t" \
    "ds_read_b64_tr_b8 v[210:211], v239 offset:17408\n\t" \
    "v_mfma_f32_16x16x128_f8f6f4 v[120:123], a[144:151], v[216:223], v[120:123] cbsz:1 blgp:1\n\t" \
    "ds_read_b64_tr_b8 v[212:213], v239 offset:24576\n\t" \
    "v_mfma_f32_16x16x128_f8f6f4 v[124:127], a[152:159], v[216:223], v[124:127] cbsz:1 blgp:1\n\t" \
    "ds_read_b64_tr_b8 v[214:215], v239 offset:25600\n\t" \
    "s_waitcnt vmcnt(27)\n\t" \
    "s_waitcnt lgkmcnt(8)\n\t" \
    "v_mfma_f32_16x16x128_f8f6f4 v[0:3], a[160:167], v[192:199], v[0:3] cbsz:1 blgp:1\n\t" \
    "v_add_u32 v238, 65536, v227\n\t" \
    "ds_read_b64_tr_b8 v[216:217], v238 offset:16384\n\t" \
    "v_mfma_f32_16x16x128_f8f6f4 v[4:7], a[168:175], v[192:199], v[4:7] cbsz:1 blgp:1\n\t" \
    "ds_read_b64_tr_b8 v[218:219], v238 offset:17408\n\t" \
    "v_mfma_f32_16x16x128_f8f6f4 v[8:11], a[176:183], v[192:199], v[8:11] cbsz:1 blgp:1\n\t" \
    "ds_read_b64_tr_b8 v[220:221], v238 offset:24576\n\t" \
    "v_mfma_f32_16x16x128_f8f6f4 v[12:15], a[184:191], v[192:199], v[12:15] cbsz:1 blgp:1\n\t" \
    "ds_read_b64_tr_b8 v[222:223], v238 offset:25600\n\t" \
    "s_waitcnt lgkmcnt(8)\n\t" \
    "v_mfma_f32_16x16x128_f8f6f4 v[16:19], a[160:167], v[200:207], v[16:19] cbsz:1 blgp:1\n\t" \
    "v_add_u32 v239, 65536, v228\n\t" \
    "ds_read_b64_tr_b8 v[192:193], v239 offset:16384\n\t" \
    "v_mfma_f32_16x16x128_f8f6f4 v[20:23], a[168:175], v[200:207], v[20:23] cbsz:1 blgp:1\n\t" \
    "ds_read_b64_tr_b8 v[194:195], v239 offset:17408\n\t" \
    "v_mfma_f32_16x16x128_f8f6f4 v[24:27], a[176:183], v[200:207], v[24:27] cbsz:1 blgp:1\n\t" \
    "ds_read_b64_tr_b8 v[196:197], v239 offset:24576\n\t" \
    "v_mfma_f32_16x16x128_f8f6f4 v[28:31], a[184:191], v[200:207], v[28:31] cbsz:1 blgp:1\n\t" \
    "ds_read_b64_tr_b8 v[198:199], v239 offset:25600\n\t" \
    "s_waitcnt lgkmcnt(8)\n\t" \
    "v_mfma_f32_16x16x128_f8f6f4 v[32:35], a[160:167], v[208:215], v[32:35] cbsz:1 blgp:1\n\t" \
    "v_add_u32 v238, 65536, v229\n\t" \
    "ds_read_b64_tr_b8 v[200:201], v238 offset:16384\n\t" \
    "v_mfma_f32_16x16x128_f8f6f4 v[36:39], a[168:175], v[208:215], v[36:39] cbsz:1 blgp:1\n\t" \
    "ds_read_b64_tr_b8 v[202:203], v238 offset:17408\n\t" \
    "v_mfma_f32_16x16x128_f8f6f4 v[40:43], a[176:183], v[208:215], v[40:43] cbsz:1 blgp:1\n\t" \
    "ds_read_b64_tr_b8 v[204:205], v238 offset:24576\n\t" \
    "v_mfma_f32_16x16x128_f8f6f4 v[44:47], a[184:191], v[208:215], v[44:47] cbsz:1 blgp:1\n\t" \
    "ds_read_b64_tr_b8 v[206:207], v238 offset:25600\n\t" \
    "s_waitcnt lgkmcnt(8)\n\t" \
    "v_mfma_f32_16x16x128_f8f6f4 v[48:51], a[160:167], v[216:223], v[48:51] cbsz:1 blgp:1\n\t" \
    "v_add_u32 v239, 65536, v230\n\t" \
    "ds_read_b64_tr_b8 v[208:209], v239 offset:16384\n\t" \
    "v_mfma_f32_16x16x128_f8f6f4 v[52:55], a[168:175], v[216:223], v[52:55] cbsz:1 blgp:1\n\t" \
    "ds_read_b64_tr_b8 v[210:211], v239 offset:17408\n\t" \
    "v_mfma_f32_16x16x128_f8f6f4 v[56:59], a[176:183], v[216:223], v[56:59] cbsz:1 blgp:1\n\t" \
    "ds_read_b64_tr_b8 v[212:213], v239 offset:24576\n\t" \
    "v_mfma_f32_16x16x128_f8f6f4 v[60:63], a[184:191], v[216:223], v[60:63] cbsz:1 blgp:1\n\t" \
    "ds_read_b64_tr_b8 v[214:215], v239 offset:25600\n\t" \
    "s_waitcnt lgkmcnt(8)\n\t" \
    "v_mfma_f32_16x16x128_f8f6f4 v[64:67], a[160:167], v[192:199], v[64:67] cbsz:1 blgp:1\n\t" \
    "v_add_u32 v238, 65536, v231\n\t" \
    "ds_read_b64_tr_b8 v[216:217], v238 offset:16384\n\t" \
    "v_mfma_f32_16x16x128_f8f6f4 v[68:71], a[168:175], v[192:199], v[68:71] cbsz:1 blgp:1\n\t" \
    "ds_read_b64_tr_b8 v[218:219], v238 offset:17408\n\t" \
    "v_mfma_f32_16x16x128_f8f6f4 v[72:75], a[176:183], v[192:199], v[72:75] cbsz:1 blgp:1\n\t" \
    "ds_read_b64_tr_b8 v[220:221], v238 offset:24576\n\t" \
    "v_mfma_f32_16x16x128_f8f6f4 v[76:79], a[184:191], v[192:199], v[76:79] cbsz:1 blgp:1\n\t" \
    "s_cmp_eq_u32 s47, 1\n\t" \
    "s_cselect_b32 s87, s25, s26\n\t" \
    "s_add_u32 s86, s87, 4194304\n\t" \
    "s_add_u32 m0, s46, 131072\n\t" \
    "s_add_u32 s74, s86, 0\n\t" \
    "buffer_load_dwordx4 v232, s[8:11], s74 offen  lds\n\t" \
    "ds_read_b64_tr_b8 v[222:223], v238 offset:25600\n\t" \
    "s_waitcnt lgkmcnt(8)\n\t" \
    "v_mfma_f32_16x16x128_f8f6f4 v[80:83], a[160:167], v[200:207], v[80:83] cbsz:1 blgp:1\n\t" \
    "s_barrier\n\t" \
    "v_add_u32 v239, 65536, v224\n\t" \
    "ds_read_b64_tr_b8 v[192:193], v239 offset:32768\n\t" \
    "v_mfma_f32_16x16x128_f8f6f4 v[84:87], a[168:175], v[200:207], v[84:87] cbsz:1 blgp:1\n\t" \
    "ds_read_b64_tr_b8 v[194:195], v239 offset:33792\n\t" \
    "v_mfma_f32_16x16x128_f8f6f4 v[88:91], a[176:183], v[200:207], v[88:91] cbsz:1 blgp:1\n\t" \
    "s_add_u32 m0, s46, 135168\n\t" \
    "s_add_u32 s74, s86, 524288\n\t" \
    "buffer_load_dwordx4 v232, s[8:11], s74 offen  lds\n\t" \
    "ds_read_b64_tr_b8 v[196:197], v239 offset:40960\n\t" \
    "v_mfma_f32_16x16x128_f8f6f4 v[92:95], a[184:191], v[200:207], v[92:95] cbsz:1 blgp:1\n\t" \
    "ds_read_b64_tr_b8 v[198:199], v239 offset:41984\n\t" \
    "s_waitcnt lgkmcnt(8)\n\t" \
    "v_mfma_f32_16x16x128_f8f6f4 v[96:99], a[160:167], v[208:215], v[96:99] cbsz:1 blgp:1\n\t" \
    "s_add_u32 m0, s46, 139264\n\t" \
    "s_add_u32 s74, s86, 1048576\n\t" \
    "buffer_load_dwordx4 v232, s[8:11], s74 offen  lds\n\t" \
    "v_add_u32 v238, 65536, v225\n\t" \
    "ds_read_b64_tr_b8 v[200:201], v238 offset:32768\n\t" \
    "v_mfma_f32_16x16x128_f8f6f4 v[100:103], a[168:175], v[208:215], v[100:103] cbsz:1 blgp:1\n\t" \
    "ds_read_b64_tr_b8 v[202:203], v238 offset:33792\n\t" \
    "v_mfma_f32_16x16x128_f8f6f4 v[104:107], a[176:183], v[208:215], v[104:107] cbsz:1 blgp:1\n\t" \
    "s_add_u32 m0, s46, 143360\n\t" \
    "s_add_u32 s74, s86, 1572864\n\t" \
    "buffer_load_dwordx4 v232, s[8:11], s74 offen  lds\n\t" \
    "ds_read_b64_tr_b8 v[204:205], v238 offset:40960\n\t" \
    "v_mfma_f32_16x16x128_f8f6f4 v[108:111], a[184:191], v[208:215], v[108:111] cbsz:1 blgp:1\n\t" \
    "ds_read_b64_tr_b8 v[206:207], v238 offset:41984\n\t" \
    "s_waitcnt lgkmcnt(8)\n\t" \
    "v_mfma_f32_16x16x128_f8f6f4 v[112:115], a[160:167], v[216:223], v[112:115] cbsz:1 blgp:1\n\t" \
    "v_add_u32 v239, 65536, v226\n\t" \
    "ds_read_b64_tr_b8 v[208:209], v239 offset:32768\n\t" \
    "v_mfma_f32_16x16x128_f8f6f4 v[116:119], a[168:175], v[216:223], v[116:119] cbsz:1 blgp:1\n\t" \
    "s_cmp_eq_u32 s47, 1\n\t" \
    "s_cselect_b32 s87, s27, s28\n\t" \
    "s_add_u32 s86, s87, 4194304\n\t" \
    "s_add_u32 m0, s46, 0\n\t" \
    "s_add_u32 s74, s86, 0\n\t" \
    "buffer_load_dwordx4 v232, s[8:11], s74 offen  lds\n\t" \
    "ds_read_b64_tr_b8 v[210:211], v239 offset:33792\n\t" \
    "v_mfma_f32_16x16x128_f8f6f4 v[120:123], a[176:183], v[216:223], v[120:123] cbsz:1 blgp:1\n\t" \
    "ds_read_b64_tr_b8 v[212:213], v239 offset:40960\n\t" \
    "v_mfma_f32_16x16x128_f8f6f4 v[124:127], a[184:191], v[216:223], v[124:127] cbsz:1 blgp:1\n\t" \
    "s_add_u32 m0, s46, 4096\n\t" \
    "s_add_u32 s74, s86, 524288\n\t" \
    "buffer_load_dwordx4 v232, s[8:11], s74 offen  lds\n\t" \
    "ds_read_b64_tr_b8 v[214:215], v239 offset:41984\n\t" \
    "s_waitcnt vmcnt(25)\n\t" \
    "s_waitcnt lgkmcnt(8)\n\t" \
    "v_mfma_f32_16x16x128_f8f6f4 v[0:3], a[192:199], v[192:199], v[0:3] cbsz:1 blgp:1\n\t" \
    "v_add_u32 v238, 65536, v227\n\t" \
    "ds_read_b64_tr_b8 v[216:217], v238 offset:32768\n\t" \
    "v_mfma_f32_16x16x128_f8f6f4 v[4:7], a[200:207], v[192:199], v[4:7] cbsz:1 blgp:1\n\t" \
    "s_add_u32 m0, s46, 8192\n\t" \
    "s_add_u32 s74, s86, 1048576\n\t" \
    "buffer_load_dwordx4 v232, s[8:11], s74 offen  lds\n\t" \
    "ds_read_b64_tr_b8 v[218:219], v238 offset:33792\n\t" \
    "v_mfma_f32_16x16x128_f8f6f4 v[8:11], a[208:215], v[192:199], v[8:11] cbsz:1 blgp:1\n\t" \
    "ds_read_b64_tr_b8 v[220:221], v238 offset:40960\n\t" \
    "v_mfma_f32_16x16x128_f8f6f4 v[12:15], a[216:223], v[192:199], v[12:15] cbsz:1 blgp:1\n\t" \
    "ds_read_b64_tr_b8 v[222:223], v238 offset:41984\n\t" \
    "s_waitcnt lgkmcnt(8)\n\t" \
    "v_mfma_f32_16x16x128_f8f6f4 v[16:19], a[192:199], v[200:207], v[16:19] cbsz:1 blgp:1\n\t" \
    "s_add_u32 m0, s46, 12288\n\t" \
    "s_add_u32 s74, s86, 1572864\n\t" \
    "buffer_load_dwordx4 v232, s[8:11], s74 offen  lds\n\t" \
    "v_add_u32 v239, 65536, v228\n\t" \
    "ds_read_b64_tr_b8 v[192:193], v239 offset:32768\n\t" \
    "v_mfma_f32_16x16x128_f8f6f4 v[20:23], a[200:207], v[200:207], v[20:23] cbsz:1 blgp:1\n\t" \
    "ds_read_b64_tr_b8 v[194:195], v239 offset:33792\n\t" \
    "v_mfma_f32_16x16x128_f8f6f4 v[24:27], a[208:215], v[200:207], v[24:27] cbsz:1 blgp:1\n\t" \
    "buffer_load_dwordx4 v[136:139], v233, s[12:15], s72 offen offset:2048 sc0 sc1\n\t" \
    "ds_read_b64_tr_b8 v[196:197], v239 offset:40960\n\t" \
    "v_mfma_f32_16x16x128_f8f6f4 v[28:31], a[216:223], v[200:207], v[28:31] cbsz:1 blgp:1\n\t" \
    "ds_read_b64_tr_b8 v[198:199], v239 offset:41984\n\t" \
    "s_waitcnt lgkmcnt(8)\n\t" \
    "v_mfma_f32_16x16x128_f8f6f4 v[32:35], a[192:199], v[208:215], v[32:35] cbsz:1 blgp:1\n\t" \
    "buffer_load_dwordx4 v[140:143], v233, s[12:15], s72 offen offset:3072 sc0 sc1\n\t" \
    "v_add_u32 v238, 65536, v229\n\t" \
    "ds_read_b64_tr_b8 v[200:201], v238 offset:32768\n\t" \
    "v_mfma_f32_16x16x128_f8f6f4 v[36:39], a[200:207], v[208:215], v[36:39] cbsz:1 blgp:1\n\t" \
    "ds_read_b64_tr_b8 v[202:203], v238 offset:33792\n\t" \
    "v_mfma_f32_16x16x128_f8f6f4 v[40:43], a[208:215], v[208:215], v[40:43] cbsz:1 blgp:1\n\t" \
    "ds_read_b64_tr_b8 v[204:205], v238 offset:40960\n\t" \
    "v_mfma_f32_16x16x128_f8f6f4 v[44:47], a[216:223], v[208:215], v[44:47] cbsz:1 blgp:1\n\t" \
    "buffer_load_dwordx4 v[144:147], v233, s[12:15], s73 offen offset:0 sc0 sc1\n\t" \
    "ds_read_b64_tr_b8 v[206:207], v238 offset:41984\n\t" \
    "s_waitcnt lgkmcnt(8)\n\t" \
    "v_mfma_f32_16x16x128_f8f6f4 v[48:51], a[192:199], v[216:223], v[48:51] cbsz:1 blgp:1\n\t" \
    "v_add_u32 v239, 65536, v230\n\t" \
    "ds_read_b64_tr_b8 v[208:209], v239 offset:32768\n\t" \
    "v_mfma_f32_16x16x128_f8f6f4 v[52:55], a[200:207], v[216:223], v[52:55] cbsz:1 blgp:1\n\t" \
    "buffer_load_dwordx4 v[148:151], v233, s[12:15], s73 offen offset:1024 sc0 sc1\n\t" \
    "ds_read_b64_tr_b8 v[210:211], v239 offset:33792\n\t" \
    "v_mfma_f32_16x16x128_f8f6f4 v[56:59], a[208:215], v[216:223], v[56:59] cbsz:1 blgp:1\n\t" \
    "ds_read_b64_tr_b8 v[212:213], v239 offset:40960\n\t" \
    "v_mfma_f32_16x16x128_f8f6f4 v[60:63], a[216:223], v[216:223], v[60:63] cbsz:1 blgp:1\n\t" \
    "buffer_load_dwordx4 v[152:155], v233, s[12:15], s73 offen offset:2048 sc0 sc1\n\t" \
    "ds_read_b64_tr_b8 v[214:215], v239 offset:41984\n\t" \
    "s_waitcnt lgkmcnt(8)\n\t" \
    "v_mfma_f32_16x16x128_f8f6f4 v[64:67], a[192:199], v[192:199], v[64:67] cbsz:1 blgp:1\n\t" \
    "v_add_u32 v238, 65536, v231\n\t" \
    "ds_read_b64_tr_b8 v[216:217], v238 offset:32768\n\t" \
    "v_mfma_f32_16x16x128_f8f6f4 v[68:71], a[200:207], v[192:199], v[68:71] cbsz:1 blgp:1\n\t" \
    "ds_read_b64_tr_b8 v[218:219], v238 offset:33792\n\t" \
    "v_mfma_f32_16x16x128_f8f6f4 v[72:75], a[208:215], v[192:199], v[72:75] cbsz:1 blgp:1\n\t" \
    "buffer_load_dwordx4 v[156:159], v233, s[12:15], s73 offen offset:3072 sc0 sc1\n\t" \
    "ds_read_b64_tr_b8 v[220:221], v238 offset:40960\n\t" \
    "v_mfma_f32_16x16x128_f8f6f4 v[76:79], a[216:223], v[192:199], v[76:79] cbsz:1 blgp:1\n\t" \
    "ds_read_b64_tr_b8 v[222:223], v238 offset:41984\n\t" \
    "s_waitcnt lgkmcnt(8)\n\t" \
    "v_mfma_f32_16x16x128_f8f6f4 v[80:83], a[192:199], v[200:207], v[80:83] cbsz:1 blgp:1\n\t" \
    "s_barrier\n\t" \
    "s_mov_b32 s72, 1245184\n\t" \
    "s_mov_b32 s73, 1249280\n\t" \
    "buffer_load_dwordx4 v[160:163], v233, s[12:15], s72 offen offset:0 sc0 sc1\n\t" \
    "v_add_u32 v239, 65536, v224\n\t" \
    "ds_read_b64_tr_b8 v[192:193], v239 offset:49152\n\t" \
    "v_mfma_f32_16x16x128_f8f6f4 v[84:87], a[200:207], v[200:207], v[84:87] cbsz:1 blgp:1\n\t" \
    "ds_read_b64_tr_b8 v[194:195], v239 offset:50176\n\t" \
    "v_mfma_f32_16x16x128_f8f6f4 v[88:91], a[208:215], v[200:207], v[88:91] cbsz:1 blgp:1\n\t" \
    "buffer_load_dwordx4 v[164:167], v233, s[12:15], s72 offen offset:1024 sc0 sc1\n\t" \
    "ds_read_b64_tr_b8 v[196:197], v239 offset:57344\n\t" \
    "v_mfma_f32_16x16x128_f8f6f4 v[92:95], a[216:223], v[200:207], v[92:95] cbsz:1 blgp:1\n\t" \
    "ds_read_b64_tr_b8 v[198:199], v239 offset:58368\n\t" \
    "s_waitcnt lgkmcnt(8)\n\t" \
    "v_mfma_f32_16x16x128_f8f6f4 v[96:99], a[192:199], v[208:215], v[96:99] cbsz:1 blgp:1\n\t" \
    "v_add_u32 v238, 65536, v225\n\t" \
    "ds_read_b64_tr_b8 v[200:201], v238 offset:49152\n\t" \
    "v_mfma_f32_16x16x128_f8f6f4 v[100:103], a[200:207], v[208:215], v[100:103] cbsz:1 blgp:1\n\t" \
    "buffer_load_dwordx4 v[168:171], v233, s[12:15], s72 offen offset:2048 sc0 sc1\n\t" \
    "ds_read_b64_tr_b8 v[202:203], v238 offset:50176\n\t" \
    "v_mfma_f32_16x16x128_f8f6f4 v[104:107], a[208:215], v[208:215], v[104:107] cbsz:1 blgp:1\n\t" \
    "ds_read_b64_tr_b8 v[204:205], v238 offset:57344\n\t" \
    "v_mfma_f32_16x16x128_f8f6f4 v[108:111], a[216:223], v[208:215], v[108:111] cbsz:1 blgp:1\n\t" \
    "buffer_load_dwordx4 v[172:175], v233, s[12:15], s72 offen offset:3072 sc0 sc1\n\t" \
    "ds_read_b64_tr_b8 v[206:207], v238 offset:58368\n\t" \
    "s_waitcnt lgkmcnt(8)\n\t" \
    "v_mfma_f32_16x16x128_f8f6f4 v[112:115], a[192:199], v[216:223], v[112:115] cbsz:1 blgp:1\n\t" \
    "v_add_u32 v239, 65536, v226\n\t" \
    "ds_read_b64_tr_b8 v[208:209], v239 offset:49152\n\t" \
    "v_mfma_f32_16x16x128_f8f6f4 v[116:119], a[200:207], v[216:223], v[116:119] cbsz:1 blgp:1\n\t" \
    "buffer_load_dwordx4 v[176:179], v233, s[12:15], s73 offen offset:0 sc0 sc1\n\t" \
    "ds_read_b64_tr_b8 v[210:211], v239 offset:50176\n\t" \
    "v_mfma_f32_16x16x128_f8f6f4 v[120:123], a[208:215], v[216:223], v[120:123] cbsz:1 blgp:1\n\t" \
    "ds_read_b64_tr_b8 v[212:213], v239 offset:57344\n\t" \
    "v_mfma_f32_16x16x128_f8f6f4 v[124:127], a[216:223], v[216:223], v[124:127] cbsz:1 blgp:1\n\t" \
    "ds_read_b64_tr_b8 v[214:215], v239 offset:58368\n\t" \
    "s_waitcnt vmcnt(30)\n\t" \
    "s_waitcnt lgkmcnt(8)\n\t" \
    "v_mfma_f32_16x16x128_f8f6f4 v[0:3], a[224:231], v[192:199], v[0:3] cbsz:1 blgp:1\n\t" \
    "buffer_load_dwordx4 v[180:183], v233, s[12:15], s73 offen offset:1024 sc0 sc1\n\t" \
    "v_add_u32 v238, 65536, v227\n\t" \
    "ds_read_b64_tr_b8 v[216:217], v238 offset:49152\n\t" \
    "v_mfma_f32_16x16x128_f8f6f4 v[4:7], a[232:239], v[192:199], v[4:7] cbsz:1 blgp:1\n\t" \
    "ds_read_b64_tr_b8 v[218:219], v238 offset:50176\n\t" \
    "v_mfma_f32_16x16x128_f8f6f4 v[8:11], a[240:247], v[192:199], v[8:11] cbsz:1 blgp:1\n\t" \
    "buffer_load_dwordx4 v[184:187], v233, s[12:15], s73 offen offset:2048 sc0 sc1\n\t" \
    "ds_read_b64_tr_b8 v[220:221], v238 offset:57344\n\t" \
    "v_mfma_f32_16x16x128_f8f6f4 v[12:15], a[248:255], v[192:199], v[12:15] cbsz:1 blgp:1\n\t" \
    "ds_read_b64_tr_b8 v[222:223], v238 offset:58368\n\t" \
    "s_waitcnt lgkmcnt(8)\n\t" \
    "v_mfma_f32_16x16x128_f8f6f4 v[16:19], a[224:231], v[200:207], v[16:19] cbsz:1 blgp:1\n\t" \
    "buffer_load_dwordx4 v[188:191], v233, s[12:15], s73 offen offset:3072 sc0 sc1\n\t" \
    "v_add_u32 v239, 65536, v228\n\t" \
    "ds_read_b64_tr_b8 v[192:193], v239 offset:49152\n\t" \
    "v_mfma_f32_16x16x128_f8f6f4 v[20:23], a[232:239], v[200:207], v[20:23] cbsz:1 blgp:1\n\t" \
    "ds_read_b64_tr_b8 v[194:195], v239 offset:50176\n\t" \
    "v_mfma_f32_16x16x128_f8f6f4 v[24:27], a[240:247], v[200:207], v[24:27] cbsz:1 blgp:1\n\t" \
    "ds_read_b64_tr_b8 v[196:197], v239 offset:57344\n\t" \
    "v_mfma_f32_16x16x128_f8f6f4 v[28:31], a[248:255], v[200:207], v[28:31] cbsz:1 blgp:1\n\t" \
    "s_mov_b32 s72, 1310720\n\t" \
    "s_mov_b32 s73, 1314816\n\t" \
    "buffer_load_dwordx4 a[0:3], v233, s[12:15], s72 offen offset:0 sc0 sc1\n\t" \
    "ds_read_b64_tr_b8 v[198:199], v239 offset:58368\n\t" \
    "s_waitcnt lgkmcnt(8)\n\t" \
    "v_mfma_f32_16x16x128_f8f6f4 v[32:35], a[224:231], v[208:215], v[32:35] cbsz:1 blgp:1\n\t" \
    "v_add_u32 v238, 65536, v229\n\t" \
    "ds_read_b64_tr_b8 v[200:201], v238 offset:49152\n\t" \
    "v_mfma_f32_16x16x128_f8f6f4 v[36:39], a[232:239], v[208:215], v[36:39] cbsz:1 blgp:1\n\t" \
    "buffer_load_dwordx4 a[4:7], v233, s[12:15], s72 offen offset:1024 sc0 sc1\n\t" \
    "ds_read_b64_tr_b8 v[202:203], v238 offset:50176\n\t" \
    "v_mfma_f32_16x16x128_f8f6f4 v[40:43], a[240:247], v[208:215], v[40:43] cbsz:1 blgp:1\n\t" \
    "ds_read_b64_tr_b8 v[204:205], v238 offset:57344\n\t" \
    "v_mfma_f32_16x16x128_f8f6f4 v[44:47], a[248:255], v[208:215], v[44:47] cbsz:1 blgp:1\n\t" \
    "buffer_load_dwordx4 a[8:11], v233, s[12:15], s72 offen offset:2048 sc0 sc1\n\t" \
    "ds_read_b64_tr_b8 v[206:207], v238 offset:58368\n\t" \
    "s_waitcnt lgkmcnt(8)\n\t" \
    "v_mfma_f32_16x16x128_f8f6f4 v[48:51], a[224:231], v[216:223], v[48:51] cbsz:1 blgp:1\n\t" \
    "v_add_u32 v239, 65536, v230\n\t" \
    "ds_read_b64_tr_b8 v[208:209], v239 offset:49152\n\t" \
    "v_mfma_f32_16x16x128_f8f6f4 v[52:55], a[232:239], v[216:223], v[52:55] cbsz:1 blgp:1\n\t" \
    "ds_read_b64_tr_b8 v[210:211], v239 offset:50176\n\t" \
    "v_mfma_f32_16x16x128_f8f6f4 v[56:59], a[240:247], v[216:223], v[56:59] cbsz:1 blgp:1\n\t" \
    "buffer_load_dwordx4 a[12:15], v233, s[12:15], s72 offen offset:3072 sc0 sc1\n\t" \
    "ds_read_b64_tr_b8 v[212:213], v239 offset:57344\n\t" \
    "v_mfma_f32_16x16x128_f8f6f4 v[60:63], a[248:255], v[216:223], v[60:63] cbsz:1 blgp:1\n\t" \
    "ds_read_b64_tr_b8 v[214:215], v239 offset:58368\n\t" \
    "s_waitcnt lgkmcnt(8)\n\t" \
    "v_mfma_f32_16x16x128_f8f6f4 v[64:67], a[224:231], v[192:199], v[64:67] cbsz:1 blgp:1\n\t" \
    "global_load_dword v248, v235, s[54:55] offset:3072 sc0 sc1\n\t" \
    "buffer_load_dwordx4 a[16:19], v233, s[12:15], s73 offen offset:0 sc0 sc1\n\t" \
    "v_add_u32 v238, 65536, v231\n\t" \
    "ds_read_b64_tr_b8 v[216:217], v238 offset:49152\n\t" \
    "v_mfma_f32_16x16x128_f8f6f4 v[68:71], a[232:239], v[192:199], v[68:71] cbsz:1 blgp:1\n\t" \
    "ds_read_b64_tr_b8 v[218:219], v238 offset:50176\n\t" \
    "v_mfma_f32_16x16x128_f8f6f4 v[72:75], a[240:247], v[192:199], v[72:75] cbsz:1 blgp:1\n\t" \
    "buffer_load_dwordx4 a[20:23], v233, s[12:15], s73 offen offset:1024 sc0 sc1\n\t" \
    "ds_read_b64_tr_b8 v[220:221], v238 offset:57344\n\t" \
    "v_mfma_f32_16x16x128_f8f6f4 v[76:79], a[248:255], v[192:199], v[76:79] cbsz:1 blgp:1\n\t" \
    "ds_read_b64_tr_b8 v[222:223], v238 offset:58368\n\t" \
    "s_waitcnt lgkmcnt(8)\n\t" \
    "v_mfma_f32_16x16x128_f8f6f4 v[80:83], a[224:231], v[200:207], v[80:83] cbsz:1 blgp:1\n\t" \
    "s_waitcnt vmcnt(33)\n\t" \
    "s_waitcnt vmcnt(25)\n\t" \
    "s_barrier\n\t" \
    "v_add_u32 v239, 131072, v224\n\t" \
    "ds_read_b64_tr_b8 v[192:193], v239 offset:0\n\t" \
    "v_mfma_f32_16x16x128_f8f6f4 v[84:87], a[232:239], v[200:207], v[84:87] cbsz:1 blgp:1\n\t" \
    "buffer_load_dwordx4 a[24:27], v233, s[12:15], s73 offen offset:2048 sc0 sc1\n\t" \
    "ds_read_b64_tr_b8 v[194:195], v239 offset:1024\n\t" \
    "v_mfma_f32_16x16x128_f8f6f4 v[88:91], a[240:247], v[200:207], v[88:91] cbsz:1 blgp:1\n\t" \
    "ds_read_b64_tr_b8 v[196:197], v239 offset:8192\n\t" \
    "v_mfma_f32_16x16x128_f8f6f4 v[92:95], a[248:255], v[200:207], v[92:95] cbsz:1 blgp:1\n\t" \
    "buffer_load_dwordx4 a[28:31], v233, s[12:15], s73 offen offset:3072 sc0 sc1\n\t" \
    "ds_read_b64_tr_b8 v[198:199], v239 offset:9216\n\t" \
    "s_waitcnt lgkmcnt(8)\n\t" \
    "v_mfma_f32_16x16x128_f8f6f4 v[96:99], a[224:231], v[208:215], v[96:99] cbsz:1 blgp:1\n\t" \
    "v_add_u32 v238, 131072, v225\n\t" \
    "ds_read_b64_tr_b8 v[200:201], v238 offset:0\n\t" \
    "v_mfma_f32_16x16x128_f8f6f4 v[100:103], a[232:239], v[208:215], v[100:103] cbsz:1 blgp:1\n\t" \
    "s_mov_b32 s72, 1376256\n\t" \
    "s_mov_b32 s73, 1380352\n\t" \
    "buffer_load_dwordx4 a[32:35], v233, s[12:15], s72 offen offset:0 sc0 sc1\n\t" \
    "ds_read_b64_tr_b8 v[202:203], v238 offset:1024\n\t" \
    "v_mfma_f32_16x16x128_f8f6f4 v[104:107], a[240:247], v[208:215], v[104:107] cbsz:1 blgp:1\n\t" \
    "ds_read_b64_tr_b8 v[204:205], v238 offset:8192\n\t" \
    "v_mfma_f32_16x16x128_f8f6f4 v[108:111], a[248:255], v[208:215], v[108:111] cbsz:1 blgp:1\n\t" \
    "ds_read_b64_tr_b8 v[206:207], v238 offset:9216\n\t" \
    "s_waitcnt lgkmcnt(8)\n\t" \
    "v_mfma_f32_16x16x128_f8f6f4 v[112:115], a[224:231], v[216:223], v[112:115] cbsz:1 blgp:1\n\t" \
    "buffer_load_dwordx4 a[36:39], v233, s[12:15], s72 offen offset:1024 sc0 sc1\n\t" \
    "v_add_u32 v239, 131072, v226\n\t" \
    "ds_read_b64_tr_b8 v[208:209], v239 offset:0\n\t" \
    "v_mfma_f32_16x16x128_f8f6f4 v[116:119], a[232:239], v[216:223], v[116:119] cbsz:1 blgp:1\n\t" \
    "ds_read_b64_tr_b8 v[210:211], v239 offset:1024\n\t" \
    "v_mfma_f32_16x16x128_f8f6f4 v[120:123], a[240:247], v[216:223], v[120:123] cbsz:1 blgp:1\n\t" \
    "buffer_load_dwordx4 a[40:43], v233, s[12:15], s72 offen offset:2048 sc0 sc1\n\t" \
    "ds_read_b64_tr_b8 v[212:213], v239 offset:8192\n\t" \
    "v_mfma_f32_16x16x128_f8f6f4 v[124:127], a[248:255], v[216:223], v[124:127] cbsz:1 blgp:1\n\t" \
    "ds_read_b64_tr_b8 v[214:215], v239 offset:9216\n\t" \
    "s_waitcnt vmcnt(20)\n\t" \
    "s_waitcnt lgkmcnt(8)\n\t" \
    "v_mfma_f32_16x16x128_f8f6f4 v[0:3], v[128:135], v[192:199], v[0:3] cbsz:1 blgp:1\n\t" \
    "buffer_load_dwordx4 a[44:47], v233, s[12:15], s72 offen offset:3072 sc0 sc1\n\t" \
    "v_add_u32 v238, 131072, v227\n\t" \
    "ds_read_b64_tr_b8 v[216:217], v238 offset:0\n\t" \
    "v_mfma_f32_16x16x128_f8f6f4 v[4:7], v[136:143], v[192:199], v[4:7] cbsz:1 blgp:1\n\t" \
    "ds_read_b64_tr_b8 v[218:219], v238 offset:1024\n\t" \
    "v_mfma_f32_16x16x128_f8f6f4 v[8:11], v[144:151], v[192:199], v[8:11] cbsz:1 blgp:1\n\t" \
    "ds_read_b64_tr_b8 v[220:221], v238 offset:8192\n\t" \
    "v_mfma_f32_16x16x128_f8f6f4 v[12:15], v[152:159], v[192:199], v[12:15] cbsz:1 blgp:1\n\t" \
    "buffer_load_dwordx4 a[48:51], v233, s[12:15], s73 offen offset:0 sc0 sc1\n\t" \
    "ds_read_b64_tr_b8 v[222:223], v238 offset:9216\n\t" \
    "s_waitcnt lgkmcnt(8)\n\t" \
    "v_mfma_f32_16x16x128_f8f6f4 v[16:19], v[128:135], v[200:207], v[16:19] cbsz:1 blgp:1\n\t" \
    "v_add_u32 v239, 131072, v228\n\t" \
    "ds_read_b64_tr_b8 v[192:193], v239 offset:0\n\t" \
    "v_mfma_f32_16x16x128_f8f6f4 v[20:23], v[136:143], v[200:207], v[20:23] cbsz:1 blgp:1\n\t" \
    "buffer_load_dwordx4 a[52:55], v233, s[12:15], s73 offen offset:1024 sc0 sc1\n\t" \
    "ds_read_b64_tr_b8 v[194:195], v239 offset:1024\n\t" \
    "v_mfma_f32_16x16x128_f8f6f4 v[24:27], v[144:151], v[200:207], v[24:27] cbsz:1 blgp:1\n\t" \
    "ds_read_b64_tr_b8 v[196:197], v239 offset:8192\n\t" \
    "v_mfma_f32_16x16x128_f8f6f4 v[28:31], v[152:159], v[200:207], v[28:31] cbsz:1 blgp:1\n\t" \
    "buffer_load_dwordx4 a[56:59], v233, s[12:15], s73 offen offset:2048 sc0 sc1\n\t" \
    "ds_read_b64_tr_b8 v[198:199], v239 offset:9216\n\t" \
    "s_waitcnt lgkmcnt(8)\n\t" \
    "v_mfma_f32_16x16x128_f8f6f4 v[32:35], v[128:135], v[208:215], v[32:35] cbsz:1 blgp:1\n\t" \
    "v_add_u32 v238, 131072, v229\n\t" \
    "ds_read_b64_tr_b8 v[200:201], v238 offset:0\n\t" \
    "v_mfma_f32_16x16x128_f8f6f4 v[36:39], v[136:143], v[208:215], v[36:39] cbsz:1 blgp:1\n\t" \
    "ds_read_b64_tr_b8 v[202:203], v238 offset:1024\n\t" \
    "v_mfma_f32_16x16x128_f8f6f4 v[40:43], v[144:151], v[208:215], v[40:43] cbsz:1 blgp:1\n\t" \
    "buffer_load_dwordx4 a[60:63], v233, s[12:15], s73 offen offset:3072 sc0 sc1\n\t" \
    "ds_read_b64_tr_b8 v[204:205], v238 offset:8192\n\t" \
    "v_mfma_f32_16x16x128_f8f6f4 v[44:47], v[152:159], v[208:215], v[44:47] cbsz:1 blgp:1\n\t" \
    "ds_read_b64_tr_b8 v[206:207], v238 offset:9216\n\t" \
    "s_waitcnt lgkmcnt(8)\n\t" \
    "v_mfma_f32_16x16x128_f8f6f4 v[48:51], v[128:135], v[216:223], v[48:51] cbsz:1 blgp:1\n\t" \
    "s_mov_b32 s72, 1441792\n\t" \
    "s_mov_b32 s73, 1445888\n\t" \
    "buffer_load_dwordx4 a[64:67], v233, s[12:15], s72 offen offset:0 sc0 sc1\n\t" \
    "v_add_u32 v239, 131072, v230\n\t" \
    "ds_read_b64_tr_b8 v[208:209], v239 offset:0\n\t" \
    "v_mfma_f32_16x16x128_f8f6f4 v[52:55], v[136:143], v[216:223], v[52:55] cbsz:1 blgp:1\n\t" \
    "ds_read_b64_tr_b8 v[210:211], v239 offset:1024\n\t" \
    "v_mfma_f32_16x16x128_f8f6f4 v[56:59], v[144:151], v[216:223], v[56:59] cbsz:1 blgp:1\n\t" \
    "buffer_load_dwordx4 a[68:71], v233, s[12:15], s72 offen offset:1024 sc0 sc1\n\t" \
    "ds_read_b64_tr_b8 v[212:213], v239 offset:8192\n\t" \
    "v_mfma_f32_16x16x128_f8f6f4 v[60:63], v[152:159], v[216:223], v[60:63] cbsz:1 blgp:1\n\t" \
    "ds_read_b64_tr_b8 v[214:215], v239 offset:9216\n\t" \
    "s_waitcnt lgkmcnt(8)\n\t" \
    "v_mfma_f32_16x16x128_f8f6f4 v[64:67], v[128:135], v[192:199], v[64:67] cbsz:1 blgp:1\n\t" \
    "v_add_u32 v238, 131072, v231\n\t" \
    "ds_read_b64_tr_b8 v[216:217], v238 offset:0\n\t" \
    "v_mfma_f32_16x16x128_f8f6f4 v[68:71], v[136:143], v[192:199], v[68:71] cbsz:1 blgp:1\n\t" \
    "buffer_load_dwordx4 a[72:75], v233, s[12:15], s72 offen offset:2048 sc0 sc1\n\t" \
    "ds_read_b64_tr_b8 v[218:219], v238 offset:1024\n\t" \
    "v_mfma_f32_16x16x128_f8f6f4 v[72:75], v[144:151], v[192:199], v[72:75] cbsz:1 blgp:1\n\t" \
    "ds_read_b64_tr_b8 v[220:221], v238 offset:8192\n\t" \
    "v_mfma_f32_16x16x128_f8f6f4 v[76:79], v[152:159], v[192:199], v[76:79] cbsz:1 blgp:1\n\t" \
    "buffer_load_dwordx4 a[76:79], v233, s[12:15], s72 offen offset:3072 sc0 sc1\n\t" \
    "ds_read_b64_tr_b8 v[222:223], v238 offset:9216\n\t" \
    "s_waitcnt lgkmcnt(8)\n\t" \
    "v_mfma_f32_16x16x128_f8f6f4 v[80:83], v[128:135], v[200:207], v[80:83] cbsz:1 blgp:1\n\t" \
    "s_barrier\n\t" \
    "v_add_u32 v239, 131072, v224\n\t" \
    "ds_read_b64_tr_b8 v[192:193], v239 offset:16384\n\t" \
    "v_mfma_f32_16x16x128_f8f6f4 v[84:87], v[136:143], v[200:207], v[84:87] cbsz:1 blgp:1\n\t" \
    "s_cmp_eq_u32 s47, 0\n\t" \
    "s_cselect_b32 s87, s29, s30\n\t" \
    "s_add_u32 s85, s87, 4194304\n\t" \
    "s_add_u32 m0, s44, 32768\n\t" \
    "s_add_u32 s74, s85, 0\n\t" \
    "buffer_load_dwordx4 v232, s[8:11], s74 offen  lds\n\t" \
    "ds_read_b64_tr_b8 v[194:195], v239 offset:17408\n\t" \
    "v_mfma_f32_16x16x128_f8f6f4 v[88:91], v[144:151], v[200:207], v[88:91] cbsz:1 blgp:1\n\t" \
    "s_add_u32 m0, s44, 36864\n\t" \
    "s_add_u32 s74, s85, 524288\n\t" \
    "buffer_load_dwordx4 v232, s[8:11], s74 offen  lds\n\t" \
    "ds_read_b64_tr_b8 v[196:197], v239 offset:24576\n\t" \
    "v_mfma_f32_16x16x128_f8f6f4 v[92:95], v[152:159], v[200:207], v[92:95] cbsz:1 blgp:1\n\t" \
    "s_add_u32 m0, s44, 40960\n\t" \
    "s_add_u32 s74, s85, 1048576\n\t" \
    "buffer_load_dwordx4 v232, s[8:11], s74 offen  lds\n\t" \
    "ds_read_b64_tr_b8 v[198:199], v239 offset:25600\n\t" \
    "s_waitcnt lgkmcnt(8)\n\t" \
    "v_mfma_f32_16x16x128_f8f6f4 v[96:99], v[128:135], v[208:215], v[96:99] cbsz:1 blgp:1\n\t" \
    "s_add_u32 m0, s44, 45056\n\t" \
    "s_add_u32 s74, s85, 1572864\n\t" \
    "buffer_load_dwordx4 v232, s[8:11], s74 offen  lds\n\t" \
    "v_add_u32 v238, 131072, v225\n\t" \
    "ds_read_b64_tr_b8 v[200:201], v238 offset:16384\n\t" \
    "s_waitcnt vmcnt(20)\n\t" \
    "v_cmp_gt_u32 vcc, s75, v248\n\t" \
    "s_nop 0\n\t" \
    "s_cbranch_vccz L_pd3_%=\n\t" \
    "s_memrealtime s[80:81]\n\t" \
    "s_waitcnt lgkmcnt(0)\n\t" \
    "L_ps3_%=:\n\t" \
    "global_load_dword v248, v235, s[54:55] offset:3072 sc0 sc1\n\t" \
    "global_load_dword v251, v236, s[58:59] sc1\n\t" \
    "s_memrealtime s[82:83]\n\t" \
    "s_waitcnt vmcnt(0) lgkmcnt(0)\n\t" \
    "v_cmp_gt_u32 vcc, s75, v248\n\t" \
    "s_nop 0\n\t" \
    "s_cbranch_vccz L_pd3_%=\n\t" \
    "v_readfirstlane_b32 s84, v251\n\t" \
    "s_sub_u32 s82, s82, s80\n\t" \
    "s_cmp_eq_u32 s84, s75\n\t" \
    "s_cbranch_scc1 L_pd3_%=\n\t" \
    "s_cmp_gt_u32 s82, 40000000\n\t" \
    "s_cbranch_scc1 L_pt3_%=\n\t" \
    "s_sleep 1\n\t" \
    "s_branch L_ps3_%=\n\t" \
    "L_pt3_%=:\n\t" \
    "v_mov_b32 v251, s75\n\t" \
    "global_store_dword v236, v251, s[58:59] sc1\n\t" \
    "s_waitcnt vmcnt(0)\n\t" \
    "L_pd3_%=:\n\t" \
    "global_load_dword v250, v236, s[58:59] sc1\n\t" \
    "v_mfma_f32_16x16x128_f8f6f4 v[100:103], v[136:143], v[208:215], v[100:103] cbsz:1 blgp:1\n\t" \
    "s_cmp_eq_u32 s47, 0\n\t" \
    "s_cselect_b32 s87, s31, s25\n\t" \
    "s_mov_b32 s88, 4194304\n\t" \
    "s_cselect_b32 s88, s88, 6291456\n\t" \
    "s_add_u32 s85, s87, s88\n\t" \
    "s_add_u32 m0, s44, 65536\n\t" \
    "s_add_u32 s74, s85, 0\n\t" \
    "buffer_load_dwordx4 v232, s[8:11], s74 offen  lds\n\t" \
    "ds_read_b64_tr_b8 v[202:203], v238 offset:17408\n\t" \
    "v_mfma_f32_16x16x128_f8f6f4 v[104:107], v[144:151], v[208:215], v[104:107] cbsz:1 blgp:1\n\t" \
    "s_add_u32 m0, s44, 69632\n\t" \
    "s_add_u32 s74, s85, 524288\n\t" \
    "buffer_load_dwordx4 v232, s[8:11], s74 offen  lds\n\t" \
    "ds_read_b64_tr_b8 v[204:205], v238 offset:24576\n\t" \
    "v_mfma_f32_16x16x128_f8f6f4 v[108:111], v[152:159], v[208:215], v[108:111] cbsz:1 blgp:1\n\t" \
    "s_add_u32 m0, s44, 73728\n\t" \
    "s_add_u32 s74, s85, 1048576\n\t" \
    "buffer_load_dwordx4 v232, s[8:11], s74 offen  lds\n\t" \
    "ds_read_b64_tr_b8 v[206:207], v238 offset:25600\n\t" \
    "s_waitcnt lgkmcnt(8)\n\t" \
    "v_mfma_f32_16x16x128_f8f6f4 v[112:115], v[128:135], v[216:223], v[112:115] cbsz:1 blgp:1\n\t" \
    "s_add_u32 m0, s44, 77824\n\t" \
    "s_add_u32 s74, s85, 1572864\n\t" \
    "buffer_load_dwordx4 v232, s[8:11], s74 offen  lds\n\t" \
    "v_add_u32 v239, 131072, v226\n\t" \
    "ds_read_b64_tr_b8 v[208:209], v239 offset:16384\n\t" \
    "v_mfma_f32_16x16x128_f8f6f4 v[116:119], v[136:143], v[216:223], v[116:119] cbsz:1 blgp:1\n\t" \
    "ds_read_b64_tr_b8 v[210:211], v239 offset:17408\n\t" \
    "v_mfma_f32_16x16x128_f8f6f4 v[120:123], v[144:151], v[216:223], v[120:123] cbsz:1 blgp:1\n\t" \
    "ds_read_b64_tr_b8 v[212:213], v239 offset:24576\n\t" \
    "v_mfma_f32_16x16x128_f8f6f4 v[124:127], v[152:159], v[216:223], v[124:127] cbsz:1 blgp:1\n\t" \
    "ds_read_b64_tr_b8 v[214:215], v239 offset:25600\n\t" \
    "s_waitcnt lgkmcnt(8)\n\t" \
    "v_mfma_f32_16x16x128_f8f6f4 v[0:3], v[160:167], v[192:199], v[0:3] cbsz:1 blgp:1\n\t" \
    "v_add_u32 v238, 131072, v227\n\t" \
    "ds_read_b64_tr_b8 v[216:217], v238 offset:16384\n\t" \
    "v_mfma_f32_16x16x128_f8f6f4 v[4:7], v[168:175], v[192:199], v[4:7] cbsz:1 blgp:1\n\t" \
    "ds_read_b64_tr_b8 v[218:219], v238 offset:17408\n\t" \
    "v_mfma_f32_16x16x128_f8f6f4 v[8:11], v[176:183], v[192:199], v[8:11] cbsz:1 blgp:1\n\t" \
    "ds_read_b64_tr_b8 v[220:221], v238 offset:24576\n\t" \
    "v_mfma_f32_16x16x128_f8f6f4 v[12:15], v[184:191], v[192:199], v[12:15] cbsz:1 blgp:1\n\t" \
    "ds_read_b64_tr_b8 v[222:223], v238 offset:25600\n\t" \
    "s_waitcnt lgkmcnt(8)\n\t" \
    "v_mfma_f32_16x16x128_f8f6f4 v[16:19], v[160:167], v[200:207], v[16:19] cbsz:1 blgp:1\n\t" \
    "v_add_u32 v239, 131072, v228\n\t" \
    "ds_read_b64_tr_b8 v[192:193], v239 offset:16384\n\t" \
    "v_mfma_f32_16x16x128_f8f6f4 v[20:23], v[168:175], v[200:207], v[20:23] cbsz:1 blgp:1\n\t" \
    "ds_read_b64_tr_b8 v[194:195], v239 offset:17408\n\t" \
    "v_mfma_f32_16x16x128_f8f6f4 v[24:27], v[176:183], v[200:207], v[24:27] cbsz:1 blgp:1\n\t" \
    "ds_read_b64_tr_b8 v[196:197], v239 offset:24576\n\t" \
    "v_mfma_f32_16x16x128_f8f6f4 v[28:31], v[184:191], v[200:207], v[28:31] cbsz:1 blgp:1\n\t" \
    "ds_read_b64_tr_b8 v[198:199], v239 offset:25600\n\t" \
    "s_waitcnt lgkmcnt(8)\n\t" \
    "v_mfma_f32_16x16x128_f8f6f4 v[32:35], v[160:167], v[208:215], v[32:35] cbsz:1 blgp:1\n\t" \
    "v_add_u32 v238, 131072, v229\n\t" \
    "ds_read_b64_tr_b8 v[200:201], v238 offset:16384\n\t" \
    "v_mfma_f32_16x16x128_f8f6f4 v[36:39], v[168:175], v[208:215], v[36:39] cbsz:1 blgp:1\n\t" \
    "ds_read_b64_tr_b8 v[202:203], v238 offset:17408\n\t" \
    "v_mfma_f32_16x16x128_f8f6f4 v[40:43], v[176:183], v[208:215], v[40:43] cbsz:1 blgp:1\n\t" \
    "ds_read_b64_tr_b8 v[204:205], v238 offset:24576\n\t" \
    "v_mfma_f32_16x16x128_f8f6f4 v[44:47], v[184:191], v[208:215], v[44:47] cbsz:1 blgp:1\n\t" \
    "ds_read_b64_tr_b8 v[206:207], v238 offset:25600\n\t" \
    "s_waitcnt lgkmcnt(8)\n\t" \
    "v_mfma_f32_16x16x128_f8f6f4 v[48:51], v[160:167], v[216:223], v[48:51] cbsz:1 blgp:1\n\t" \
    "v_add_u32 v239, 131072, v230\n\t" \
    "ds_read_b64_tr_b8 v[208:209], v239 offset:16384\n\t" \
    "v_mfma_f32_16x16x128_f8f6f4 v[52:55], v[168:175], v[216:223], v[52:55] cbsz:1 blgp:1\n\t" \
    "ds_read_b64_tr_b8 v[210:211], v239 offset:17408\n\t" \
    "v_mfma_f32_16x16x128_f8f6f4 v[56:59], v[176:183], v[216:223], v[56:59] cbsz:1 blgp:1\n\t" \
    "ds_read_b64_tr_b8 v[212:213], v239 offset:24576\n\t" \
    "v_mfma_f32_16x16x128_f8f6f4 v[60:63], v[184:191], v[216:223], v[60:63] cbsz:1 blgp:1\n\t" \
    "ds_read_b64_tr_b8 v[214:215], v239 offset:25600\n\t" \
    "s_waitcnt lgkmcnt(8)\n\t" \
    "v_mfma_f32_16x16x128_f8f6f4 v[64:67], v[160:167], v[192:199], v[64:67] cbsz:1 blgp:1\n\t" \
    "v_add_u32 v238, 131072, v231\n\t" \
    "ds_read_b64_tr_b8 v[216:217], v238 offset:16384\n\t" \
    "v_mfma_f32_16x16x128_f8f6f4 v[68:71], v[168:175], v[192:199], v[68:71] cbsz:1 blgp:1\n\t" \
    "ds_read_b64_tr_b8 v[218:219], v238 offset:17408\n\t" \
    "v_mfma_f32_16x16x128_f8f6f4 v[72:75], v[176:183], v[192:199], v[72:75] cbsz:1 blgp:1\n\t" \
    "ds_read_b64_tr_b8 v[220:221], v238 offset:24576\n\t" \
    "v_mfma_f32_16x16x128_f8f6f4 v[76:79], v[184:191], v[192:199], v[76:79] cbsz:1 blgp:1\n\t" \
    "s_cmp_eq_u32 s47, 1\n\t" \
    "s_cselect_b32 s87, s29, s30\n\t" \
    "s_add_u32 s86, s87, 4194304\n\t" \
    "s_add_u32 m0, s46, 32768\n\t" \
    "s_add_u32 s74, s86, 0\n\t" \
    "buffer_load_dwordx4 v232, s[8:11], s74 offen  lds\n\t" \
    "ds_read_b64_tr_b8 v[222:223], v238 offset:25600\n\t" \
    "s_waitcnt lgkmcnt(8)\n\t" \
    "v_mfma_f32_16x16x128_f8f6f4 v[80:83], v[160:167], v[200:207], v[80:83] cbsz:1 blgp:1\n\t" \
    "s_barrier\n\t" \
    "ds_read_b64_tr_b8 v[192:193], v224 offset:0\n\t" \
    "v_mfma_f32_16x16x128_f8f6f4 v[84:87], v[168:175], v[200:207], v[84:87] cbsz:1 blgp:1\n\t" \
    "ds_read_b64_tr_b8 v[194:195], v224 offset:1024\n\t" \
    "v_mfma_f32_16x16x128_f8f6f4 v[88:91], v[176:183], v[200:207], v[88:91] cbsz:1 blgp:1\n\t" \
    "s_add_u32 m0, s46, 36864\n\t" \
    "s_add_u32 s74, s86, 524288\n\t" \
    "buffer_load_dwordx4 v232, s[8:11], s74 offen  lds\n\t" \
    "ds_read_b64_tr_b8 v[196:197], v224 offset:8192\n\t" \
    "v_mfma_f32_16x16x128_f8f6f4 v[92:95], v[184:191], v[200:207], v[92:95] cbsz:1 blgp:1\n\t" \
    "ds_read_b64_tr_b8 v[198:199], v224 offset:9216\n\t" \
    "s_waitcnt lgkmcnt(8)\n\t" \
    "v_mfma_f32_16x16x128_f8f6f4 v[96:99], v[160:167], v[208:215], v[96:99] cbsz:1 blgp:1\n\t" \
    "s_add_u32 m0, s46, 40960\n\t" \
    "s_add_u32 s74, s86, 1048576\n\t" \
    "buffer_load_dwordx4 v232, s[8:11], s74 offen  lds\n\t" \
    "ds_read_b64_tr_b8 v[200:201], v225 offset:0\n\t" \
    "v_mfma_f32_16x16x128_f8f6f4 v[100:103], v[168:175], v[208:215], v[100:103] cbsz:1 blgp:1\n\t" \
    "ds_read_b64_tr_b8 v[202:203], v225 offset:1024\n\t" \
    "v_mfma_f32_16x16x128_f8f6f4 v[104:107], v[176:183], v[208:215], v[104:107] cbsz:1 blgp:1\n\t" \
    "s_add_u32 m0, s46, 45056\n\t" \
    "s_add_u32 s74, s86, 1572864\n\t" \
    "buffer_load_dwordx4 v232, s[8:11], s74 offen  lds\n\t" \
    "ds_read_b64_tr_b8 v[204:205], v225 offset:8192\n\t" \
    "v_mfma_f32_16x16x128_f8f6f4 v[108:111], v[184:191], v[208:215], v[108:111] cbsz:1 blgp:1\n\t" \
    "ds_read_b64_tr_b8 v[206:207], v225 offset:9216\n\t" \
    "s_waitcnt lgkmcnt(8)\n\t" \
    "v_mfma_f32_16x16x128_f8f6f4 v[112:115], v[160:167], v[216:223], v[112:115] cbsz:1 blgp:1\n\t" \
    "ds_read_b64_tr_b8 v[208:209], v226 offset:0\n\t" \
    "v_mfma_f32_16x16x128_f8f6f4 v[116:119], v[168:175], v[216:223], v[116:119] cbsz:1 blgp:1\n\t" \
    "s_cmp_eq_u32 s47, 1\n\t" \
    "s_cselect_b32 s87, s31, s25\n\t" \
    "s_mov_b32 s88, 4194304\n\t" \
    "s_cselect_b32 s88, s88, 6291456\n\t" \
    "s_add_u32 s86, s87, s88\n\t" \
    "s_add_u32 m0, s46, 65536\n\t" \
    "s_add_u32 s74, s86, 0\n\t" \
    "buffer_load_dwordx4 v232, s[8:11], s74 offen  lds\n\t" \
    "ds_read_b64_tr_b8 v[210:211], v226 offset:1024\n\t" \
    "v_mfma_f32_16x16x128_f8f6f4 v[120:123], v[176:183], v[216:223], v[120:123] cbsz:1 blgp:1\n\t" \
    "ds_read_b64_tr_b8 v[212:213], v226 offset:8192\n\t" \
    "v_mfma_f32_16x16x128_f8f6f4 v[124:127], v[184:191], v[216:223], v[124:127] cbsz:1 blgp:1\n\t" \
    "s_add_u32 m0, s46, 69632\n\t" \
    "s_add_u32 s74, s86, 524288\n\t" \
    "buffer_load_dwordx4 v232, s[8:11], s74 offen  lds\n\t" \
    "ds_read_b64_tr_b8 v[214:215], v226 offset:9216\n\t" \
    "s_waitcnt vmcnt(27)\n\t" \
    "s_waitcnt lgkmcnt(8)\n\t" \
    "v_mfma_f32_16x16x128_f8f6f4 v[0:3], a[0:7], v[192:199], v[0:3] cbsz:1 blgp:1\n\t" \
    "ds_read_b64_tr_b8 v[216:217], v227 offset:0\n\t" \
    "v_mfma_f32_16x16x128_f8f6f4 v[4:7], a[8:15], v[192:199], v[4:7] cbsz:1 blgp:1\n\t" \
    "s_add_u32 m0, s46, 73728\n\t" \
    "s_add_u32 s74, s86, 1048576\n\t" \
    "buffer_load_dwordx4 v232, s[8:11], s74 offen  lds\n\t" \
    "ds_read_b64_tr_b8 v[218:219], v227 offset:1024\n\t" \
    "v_mfma_f32_16x16x128_f8f6f4 v[8:11], a[16:23], v[192:199], v[8:11] cbsz:1 blgp:1\n\t" \
    "ds_read_b64_tr_b8 v[220:221], v227 offset:8192\n\t" \
    "v_mfma_f32_16x16x128_f8f6f4 v[12:15], a[24:31], v[192:199], v[12:15] cbsz:1 blgp:1\n\t" \
    "ds_read_b64_tr_b8 v[222:223], v227 offset:9216\n\t" \
    "s_waitcnt lgkmcnt(8)\n\t" \
    "v_mfma_f32_16x16x128_f8f6f4 v[16:19], a[0:7], v[200:207], v[16:19] cbsz:1 blgp:1\n\t" \
    "s_add_u32 m0, s46, 77824\n\t" \
    "s_add_u32 s74, s86, 1572864\n\t" \
    "buffer_load_dwordx4 v232, s[8:11], s74 offen  lds\n\t" \
    "ds_read_b64_tr_b8 v[192:193], v228 offset:0\n\t" \
    "v_mfma_f32_16x16x128_f8f6f4 v[20:23], a[8:15], v[200:207], v[20:23] cbsz:1 blgp:1\n\t" \
    "ds_read_b64_tr_b8 v[194:195], v228 offset:1024\n\t" \
    "v_mfma_f32_16x16x128_f8f6f4 v[24:27], a[16:23], v[200:207], v[24:27] cbsz:1 blgp:1\n\t" \
    "buffer_load_dwordx4 a[80:83], v233, s[12:15], s73 offen offset:0 sc0 sc1\n\t" \
    "ds_read_b64_tr_b8 v[196:197], v228 offset:8192\n\t" \
    "v_mfma_f32_16x16x128_f8f6f4 v[28:31], a[24:31], v[200:207], v[28:31] cbsz:1 blgp:1\n\t" \
    "ds_read_b64_tr_b8 v[198:199], v228 offset:9216\n\t" \
    "s_waitcnt lgkmcnt(8)\n\t" \
    "v_mfma_f32_16x16x128_f8f6f4 v[32:35], a[0:7], v[208:215], v[32:35] cbsz:1 blgp:1\n\t" \
    "buffer_load_dwordx4 a[84:87], v233, s[12:15], s73 offen offset:1024 sc0 sc1\n\t" \
    "ds_read_b64_tr_b8 v[200:201], v229 offset:0\n\t" \
    "v_mfma_f32_16x16x128_f8f6f4 v[36:39], a[8:15], v[208:215], v[36:39] cbsz:1 blgp:1\n\t" \
    "ds_read_b64_tr_b8 v[202:203], v229 offset:1024\n\t" \
    "v_mfma_f32_16x16x128_f8f6f4 v[40:43], a[16:23], v[208:215], v[40:43] cbsz:1 blgp:1\n\t" \
    "ds_read_b64_tr_b8 v[204:205], v229 offset:8192\n\t" \
    "v_mfma_f32_16x16x128_f8f6f4 v[44:47], a[24:31], v[208:215], v[44:47] cbsz:1 blgp:1\n\t" \
    "buffer_load_dwordx4 a[88:91], v233, s[12:15], s73 offen offset:2048 sc0 sc1\n\t" \
    "ds_read_b64_tr_b8 v[206:207], v229 offset:9216\n\t" \
    "s_waitcnt lgkmcnt(8)\n\t" \
    "v_mfma_f32_16x16x128_f8f6f4 v[48:51], a[0:7], v[216:223], v[48:51] cbsz:1 blgp:1\n\t" \
    "ds_read_b64_tr_b8 v[208:209], v230 offset:0\n\t" \
    "v_mfma_f32_16x16x128_f8f6f4 v[52:55], a[8:15], v[216:223], v[52:55] cbsz:1 blgp:1\n\t" \
    "buffer_load_dwordx4 a[92:95], v233, s[12:15], s73 offen offset:3072 sc0 sc1\n\t" \
    "ds_read_b64_tr_b8 v[210:211], v230 offset:1024\n\t" \
    "v_mfma_f32_16x16x128_f8f6f4 v[56:59], a[16:23], v[216:223], v[56:59] cbsz:1 blgp:1\n\t" \
    "ds_read_b64_tr_b8 v[212:213], v230 offset:8192\n\t" \
    "v_mfma_f32_16x16x128_f8f6f4 v[60:63], a[24:31], v[216:223], v[60:63] cbsz:1 blgp:1\n\t" \
    "s_mov_b32 s72, 1507328\n\t" \
    "s_mov_b32 s73, 1511424\n\t" \
    "buffer_load_dwordx4 a[96:99], v233, s[12:15], s72 offen offset:0 sc0 sc1\n\t" \
    "ds_read_b64_tr_b8 v[214:215], v230 offset:9216\n\t" \
    "s_waitcnt lgkmcnt(8)\n\t" \
    "v_mfma_f32_16x16x128_f8f6f4 v[64:67], a[0:7], v[192:199], v[64:67] cbsz:1 blgp:1\n\t" \
    "ds_read_b64_tr_b8 v[216:217], v231 offset:0\n\t" \
    "v_mfma_f32_16x16x128_f8f6f4 v[68:71], a[8:15], v[192:199], v[68:71] cbsz:1 blgp:1\n\t" \
    "ds_read_b64_tr_b8 v[218:219], v231 offset:1024\n\t" \
    "v_mfma_f32_16x16x128_f8f6f4 v[72:75], a[16:23], v[192:199], v[72:75] cbsz:1 blgp:1\n\t" \
    "buffer_load_dwordx4 a[100:103], v233, s[12:15], s72 offen offset:1024 sc0 sc1\n\t" \
    "ds_read_b64_tr_b8 v[220:221], v231 offset:8192\n\t" \
    "v_mfma_f32_16x16x128_f8f6f4 v[76:79], a[24:31], v[192:199], v[76:79] cbsz:1 blgp:1\n\t" \
    "ds_read_b64_tr_b8 v[222:223], v231 offset:9216\n\t" \
    "s_waitcnt lgkmcnt(8)\n\t" \
    "v_mfma_f32_16x16x128_f8f6f4 v[80:83], a[0:7], v[200:207], v[80:83] cbsz:1 blgp:1\n\t" \
    "s_barrier\n\t" \
    "buffer_load_dwordx4 a[104:107], v233, s[12:15], s72 offen offset:2048 sc0 sc1\n\t" \
    "ds_read_b64_tr_b8 v[192:193], v224 offset:16384\n\t" \
    "v_mfma_f32_16x16x128_f8f6f4 v[84:87], a[8:15], v[200:207], v[84:87] cbsz:1 blgp:1\n\t" \
    "ds_read_b64_tr_b8 v[194:195], v224 offset:17408\n\t" \
    "v_mfma_f32_16x16x128_f8f6f4 v[88:91], a[16:23], v[200:207], v[88:91] cbsz:1 blgp:1\n\t" \
    "buffer_load_dwordx4 a[108:111], v233, s[12:15], s72 offen offset:3072 sc0 sc1\n\t" \
    "ds_read_b64_tr_b8 v[196:197], v224 offset:24576\n\t" \
    "v_mfma_f32_16x16x128_f8f6f4 v[92:95], a[24:31], v[200:207], v[92:95] cbsz:1 blgp:1\n\t" \
    "ds_read_b64_tr_b8 v[198:199], v224 offset:25600\n\t" \
    "s_waitcnt lgkmcnt(8)\n\t" \
    "v_mfma_f32_16x16x128_f8f6f4 v[96:99], a[0:7], v[208:215], v[96:99] cbsz:1 blgp:1\n\t" \
    "ds_read_b64_tr_b8 v[200:201], v225 offset:16384\n\t" \
    "v_mfma_f32_16x16x128_f8f6f4 v[100:103], a[8:15], v[208:215], v[100:103] cbsz:1 blgp:1\n\t" \
    "buffer_load_dwordx4 a[112:115], v233, s[12:15], s73 offen offset:0 sc0 sc1\n\t" \
    "ds_read_b64_tr_b8 v[202:203], v225 offset:17408\n\t" \
    "v_mfma_f32_16x16x128_f8f6f4 v[104:107], a[16:23], v[208:215], v[104:107] cbsz:1 blgp:1\n\t" \
    "ds_read_b64_tr_b8 v[204:205], v225 offset:24576\n\t" \
    "v_mfma_f32_16x16x128_f8f6f4 v[108:111], a[24:31], v[208:215], v[108:111] cbsz:1 blgp:1\n\t" \
    "buffer_load_dwordx4 a[116:119], v233, s[12:15], s73 offen offset:1024 sc0 sc1\n\t" \
    "ds_read_b64_tr_b8 v[206:207], v225 offset:25600\n\t" \
    "s_waitcnt lgkmcnt(8)\n\t" \
    "v_mfma_f32_16x16x128_f8f6f4 v[112:115], a[0:7], v[216:223], v[112:115] cbsz:1 blgp:1\n\t" \
    "ds_read_b64_tr_b8 v[208:209], v226 offset:16384\n\t" \
    "v_mfma_f32_16x16x128_f8f6f4 v[116:119], a[8:15], v[216:223], v[116:119] cbsz:1 blgp:1\n\t" \
    "buffer_load_dwordx4 a[120:123], v233, s[12:15], s73 offen offset:2048 sc0 sc1\n\t" \
    "ds_read_b64_tr_b8 v[210:211], v226 offset:17408\n\t" \
    "v_mfma_f32_16x16x128_f8f6f4 v[120:123], a[16:23], v[216:223], v[120:123] cbsz:1 blgp:1\n\t" \
    "ds_read_b64_tr_b8 v[212:213], v226 offset:24576\n\t" \
    "v_mfma_f32_16x16x128_f8f6f4 v[124:127], a[24:31], v[216:223], v[124:127] cbsz:1 blgp:1\n\t" \
    "ds_read_b64_tr_b8 v[214:215], v226 offset:25600\n\t" \
    "s_waitcnt vmcnt(32)\n\t" \
    "s_waitcnt lgkmcnt(8)\n\t" \
    "v_mfma_f32_16x16x128_f8f6f4 v[0:3], a[32:39], v[192:199], v[0:3] cbsz:1 blgp:1\n\t" \
    "buffer_load_dwordx4 a[124:127], v233, s[12:15], s73 offen offset:3072 sc0 sc1\n\t" \
    "ds_read_b64_tr_b8 v[216:217], v227 offset:16384\n\t" \
    "v_mfma_f32_16x16x128_f8f6f4 v[4:7], a[40:47], v[192:199], v[4:7] cbsz:1 blgp:1\n\t" \
    "ds_read_b64_tr_b8 v[218:219], v227 offset:17408\n\t" \
    "v_mfma_f32_16x16x128_f8f6f4 v[8:11], a[48:55], v[192:199], v[8:11] cbsz:1 blgp:1\n\t" \
    "s_mov_b32 s72, 1572864\n\t" \
    "s_mov_b32 s73, 1576960\n\t" \
    "buffer_load_dwordx4 a[128:131], v233, s[12:15], s72 offen offset:0 sc0 sc1\n\t" \
    "ds_read_b64_tr_b8 v[220:221], v227 offset:24576\n\t" \
    "v_mfma_f32_16x16x128_f8f6f4 v[12:15], a[56:63], v[192:199], v[12:15] cbsz:1 blgp:1\n\t" \
    "ds_read_b64_tr_b8 v[222:223], v227 offset:25600\n\t" \
    "s_waitcnt lgkmcnt(8)\n\t" \
    "v_mfma_f32_16x16x128_f8f6f4 v[16:19], a[32:39], v[200:207], v[16:19] cbsz:1 blgp:1\n\t" \
    "buffer_load_dwordx4 a[132:135], v233, s[12:15], s72 offen offset:1024 sc0 sc1\n\t" \
    "ds_read_b64_tr_b8 v[192:193], v228 offset:16384\n\t" \
    "v_mfma_f32_16x16x128_f8f6f4 v[20:23], a[40:47], v[200:207], v[20:23] cbsz:1 blgp:1\n\t" \
    "ds_read_b64_tr_b8 v[194:195], v228 offset:17408\n\t" \
    "v_mfma_f32_16x16x128_f8f6f4 v[24:27], a[48:55], v[200:207], v[24:27] cbsz:1 blgp:1\n\t" \
    "ds_read_b64_tr_b8 v[196:197], v228 offset:24576\n\t" \
    "v_mfma_f32_16x16x128_f8f6f4 v[28:31], a[56:63], v[200:207], v[28:31] cbsz:1 blgp:1\n\t" \
    "buffer_load_dwordx4 a[136:139], v233, s[12:15], s72 offen offset:2048 sc0 sc1\n\t" \
    "ds_read_b64_tr_b8 v[198:199], v228 offset:25600\n\t" \
    "s_waitcnt lgkmcnt(8)\n\t" \
    "v_mfma_f32_16x16x128_f8f6f4 v[32:35], a[32:39], v[208:215], v[32:35] cbsz:1 blgp:1\n\t" \
    "ds_read_b64_tr_b8 v[200:201], v229 offset:16384\n\t" \
    "v_mfma_f32_16x16x128_f8f6f4 v[36:39], a[40:47], v[208:215], v[36:39] cbsz:1 blgp:1\n\t" \
    "buffer_load_dwordx4 a[140:143], v233, s[12:15], s72 offen offset:3072 sc0 sc1\n\t" \
    "ds_read_b64_tr_b8 v[202:203], v229 offset:17408\n\t" \
    "v_mfma_f32_16x16x128_f8f6f4 v[40:43], a[48:55], v[208:215], v[40:43] cbsz:1 blgp:1\n\t" \
    "ds_read_b64_tr_b8 v[204:205], v229 offset:24576\n\t" \
    "v_mfma_f32_16x16x128_f8f6f4 v[44:47], a[56:63], v[208:215], v[44:47] cbsz:1 blgp:1\n\t" \
    "buffer_load_dwordx4 a[144:147], v233, s[12:15], s73 offen offset:0 sc0 sc1\n\t" \
    "ds_read_b64_tr_b8 v[206:207], v229 offset:25600\n\t" \
    "s_waitcnt lgkmcnt(8)\n\t" \
    "v_mfma_f32_16x16x128_f8f6f4 v[48:51], a[32:39], v[216:223], v[48:51] cbsz:1 blgp:1\n\t" \
    "ds_read_b64_tr_b8 v[208:209], v230 offset:16384\n\t" \
    "v_mfma_f32_16x16x128_f8f6f4 v[52:55], a[40:47], v[216:223], v[52:55] cbsz:1 blgp:1\n\t" \
    "ds_read_b64_tr_b8 v[210:211], v230 offset:17408\n\t" \
    "v_mfma_f32_16x16x128_f8f6f4 v[56:59], a[48:55], v[216:223], v[56:59] cbsz:1 blgp:1\n\t" \
    "buffer_load_dwordx4 a[148:151], v233, s[12:15], s73 offen offset:1024 sc0 sc1\n\t" \
    "ds_read_b64_tr_b8 v[212:213], v230 offset:24576\n\t" \
    "v_mfma_f32_16x16x128_f8f6f4 v[60:63], a[56:63], v[216:223], v[60:63] cbsz:1 blgp:1\n\t" \
    "ds_read_b64_tr_b8 v[214:215], v230 offset:25600\n\t" \
    "s_waitcnt lgkmcnt(8)\n\t" \
    "v_mfma_f32_16x16x128_f8f6f4 v[64:67], a[32:39], v[192:199], v[64:67] cbsz:1 blgp:1\n\t" \
    "buffer_load_dwordx4 a[152:155], v233, s[12:15], s73 offen offset:2048 sc0 sc1\n\t" \
    "ds_read_b64_tr_b8 v[216:217], v231 offset:16384\n\t" \
    "v_mfma_f32_16x16x128_f8f6f4 v[68:71], a[40:47], v[192:199], v[68:71] cbsz:1 blgp:1\n\t" \
    "ds_read_b64_tr_b8 v[218:219], v231 offset:17408\n\t" \
    "v_mfma_f32_16x16x128_f8f6f4 v[72:75], a[48:55], v[192:199], v[72:75] cbsz:1 blgp:1\n\t" \
    "buffer_load_dwordx4 a[156:159], v233, s[12:15], s73 offen offset:3072 sc0 sc1\n\t" \
    "ds_read_b64_tr_b8 v[220:221], v231 offset:24576\n\t" \
    "v_mfma_f32_16x16x128_f8f6f4 v[76:79], a[56:63], v[192:199], v[76:79] cbsz:1 blgp:1\n\t" \
    "ds_read_b64_tr_b8 v[222:223], v231 offset:25600\n\t" \
    "s_waitcnt lgkmcnt(8)\n\t" \
    "v_mfma_f32_16x16x128_f8f6f4 v[80:83], a[32:39], v[200:207], v[80:83] cbsz:1 blgp:1\n\t" \
    "s_waitcnt vmcnt(33)\n\t" \
    "s_waitcnt vmcnt(24)\n\t" \
    "s_barrier\n\t" \
    "ds_read_b64_tr_b8 v[192:193], v224 offset:32768\n\t" \
    "v_mfma_f32_16x16x128_f8f6f4 v[84:87], a[40:47], v[200:207], v[84:87] cbsz:1 blgp:1\n\t" \
    "s_mov_b32 s72, 1638400\n\t" \
    "s_mov_b32 s73, 1642496\n\t" \
    "buffer_load_dwordx4 a[160:163], v233, s[12:15], s72 offen offset:0 sc0 sc1\n\t" \
    "ds_read_b64_tr_b8 v[194:195], v224 offset:33792\n\t" \
    "v_mfma_f32_16x16x128_f8f6f4 v[88:91], a[48:55], v[200:207], v[88:91] cbsz:1 blgp:1\n\t" \
    "ds_read_b64_tr_b8 v[196:197], v224 offset:40960\n\t" \
    "v_mfma_f32_16x16x128_f8f6f4 v[92:95], a[56:63], v[200:207], v[92:95] cbsz:1 blgp:1\n\t" \
    "buffer_load_dwordx4 a[164:167], v233, s[12:15], s72 offen offset:1024 sc0 sc1\n\t" \
    "ds_read_b64_tr_b8 v[198:199], v224 offset:41984\n\t" \
    "s_waitcnt lgkmcnt(8)\n\t" \
    "v_mfma_f32_16x16x128_f8f6f4 v[96:99], a[32:39], v[208:215], v[96:99] cbsz:1 blgp:1\n\t" \
    "ds_read_b64_tr_b8 v[200:201], v225 offset:32768\n\t" \
    "v_mfma_f32_16x16x128_f8f6f4 v[100:103], a[40:47], v[208:215], v[100:103] cbsz:1 blgp:1\n\t" \
    "buffer_load_dwordx4 a[168:171], v233, s[12:15], s72 offen offset:2048 sc0 sc1\n\t" \
    "ds_read_b64_tr_b8 v[202:203], v225 offset:33792\n\t" \
    "v_mfma_f32_16x16x128_f8f6f4 v[104:107], a[48:55], v[208:215], v[104:107] cbsz:1 blgp:1\n\t" \
    "ds_read_b64_tr_b8 v[204:205], v225 offset:40960\n\t" \
    "v_mfma_f32_16x16x128_f8f6f4 v[108:111], a[56:63], v[208:215], v[108:111] cbsz:1 blgp:1\n\t" \
    "ds_read_b64_tr_b8 v[206:207], v225 offset:41984\n\t" \
    "s_waitcnt lgkmcnt(8)\n\t" \
    "v_mfma_f32_16x16x128_f8f6f4 v[112:115], a[32:39], v[216:223], v[112:115] cbsz:1 blgp:1\n\t" \
    "buffer_load_dwordx4 a[172:175], v233, s[12:15], s72 offen offset:3072 sc0 sc1\n\t" \
    "ds_read_b64_tr_b8 v[208:209], v226 offset:32768\n\t" \
    "v_mfma_f32_16x16x128_f8f6f4 v[116:119], a[40:47], v[216:223], v[116:119] cbsz:1 blgp:1\n\t" \
    "ds_read_b64_tr_b8 v[210:211], v226 offset:33792\n\t" \
    "v_mfma_f32_16x16x128_f8f6f4 v[120:123], a[48:55], v[216:223], v[120:123] cbsz:1 blgp:1\n\t" \
    "buffer_load_dwordx4 a[176:179], v233, s[12:15], s73 offen offset:0 sc0 sc1\n\t" \
    "ds_read_b64_tr_b8 v[212:213], v226 offset:40960\n\t" \
    "v_mfma_f32_16x16x128_f8f6f4 v[124:127], a[56:63], v[216:223], v[124:127] cbsz:1 blgp:1\n\t" \
    "ds_read_b64_tr_b8 v[214:215], v226 offset:41984\n\t" \
    "s_waitcnt vmcnt(21)\n\t" \
    "s_waitcnt lgkmcnt(8)\n\t" \
    "v_mfma_f32_16x16x128_f8f6f4 v[0:3], a[64:71], v[192:199], v[0:3] cbsz:1 blgp:1\n\t" \
    "buffer_load_dwordx4 a[180:183], v233, s[12:15], s73 offen offset:1024 sc0 sc1\n\t" \
    "ds_read_b64_tr_b8 v[216:217], v227 offset:32768\n\t" \
    "v_mfma_f32_16x16x128_f8f6f4 v[4:7], a[72:79], v[192:199], v[4:7] cbsz:1 blgp:1\n\t" \
    "ds_read_b64_tr_b8 v[218:219], v227 offset:33792\n\t" \
    "v_mfma_f32_16x16x128_f8f6f4 v[8:11], a[80:87], v[192:199], v[8:11] cbsz:1 blgp:1\n\t" \
    "ds_read_b64_tr_b8 v[220:221], v227 offset:40960\n\t" \
    "v_mfma_f32_16x16x128_f8f6f4 v[12:15], a[88:95], v[192:199], v[12:15] cbsz:1 blgp:1\n\t" \
    "buffer_load_dwordx4 a[184:187], v233, s[12:15], s73 offen offset:2048 sc0 sc1\n\t" \
    "ds_read_b64_tr_b8 v[222:223], v227 offset:41984\n\t" \
    "s_waitcnt lgkmcnt(8)\n\t" \
    "v_mfma_f32_16x16x128_f8f6f4 v[16:19], a[64:71], v[200:207], v[16:19] cbsz:1 blgp:1\n\t" \
    "ds_read_b64_tr_b8 v[192:193], v228 offset:32768\n\t" \
    "v_mfma_f32_16x16x128_f8f6f4 v[20:23], a[72:79], v[200:207], v[20:23] cbsz:1 blgp:1\n\t" \
    "buffer_load_dwordx4 a[188:191], v233, s[12:15], s73 offen offset:3072 sc0 sc1\n\t" \
    "ds_read_b64_tr_b8 v[194:195], v228 offset:33792\n\t" \
    "v_mfma_f32_16x16x128_f8f6f4 v[24:27], a[80:87], v[200:207], v[24:27] cbsz:1 blgp:1\n\t" \
    "ds_read_b64_tr_b8 v[196:197], v228 offset:40960\n\t" \
    "v_mfma_f32_16x16x128_f8f6f4 v[28:31], a[88:95], v[200:207], v[28:31] cbsz:1 blgp:1\n\t" \
    "s_mov_b32 s72, 1703936\n\t" \
    "s_mov_b32 s73, 1708032\n\t" \
    "buffer_load_dwordx4 a[192:195], v233, s[12:15], s72 offen offset:0 sc0 sc1\n\t" \
    "ds_read_b64_tr_b8 v[198:199], v228 offset:41984\n\t" \
    "s_waitcnt lgkmcnt(8)\n\t" \
    "v_mfma_f32_16x16x128_f8f6f4 v[32:35], a[64:71], v[208:215], v[32:35] cbsz:1 blgp:1\n\t" \
    "ds_read_b64_tr_b8 v[200:201], v229 offset:32768\n\t" \
    "v_mfma_f32_16x16x128_f8f6f4 v[36:39], a[72:79], v[208:215], v[36:39] cbsz:1 blgp:1\n\t" \
    "ds_read_b64_tr_b8 v[202:203], v229 offset:33792\n\t" \
    "v_mfma_f32_16x16x128_f8f6f4 v[40:43], a[80:87], v[208:215], v[40:43] cbsz:1 blgp:1\n\t" \
    "buffer_load_dwordx4 a[196:199], v233, s[12:15], s72 offen offset:1024 sc0 sc1\n\t" \
    "ds_read_b64_tr_b8 v[204:205], v229 offset:40960\n\t" \
    "v_mfma_f32_16x16x128_f8f6f4 v[44:47], a[88:95], v[208:215], v[44:47] cbsz:1 blgp:1\n\t" \
    "ds_read_b64_tr_b8 v[206:207], v229 offset:41984\n\t" \
    "s_waitcnt lgkmcnt(8)\n\t" \
    "v_mfma_f32_16x16x128_f8f6f4 v[48:51], a[64:71], v[216:223], v[48:51] cbsz:1 blgp:1\n\t" \
    "buffer_load_dwordx4 a[200:203], v233, s[12:15], s72 offen offset:2048 sc0 sc1\n\t" \
    "ds_read_b64_tr_b8 v[208:209], v230 offset:32768\n\t" \
    "v_mfma_f32_16x16x128_f8f6f4 v[52:55], a[72:79], v[216:223], v[52:55] cbsz:1 blgp:1\n\t" \
    "ds_read_b64_tr_b8 v[210:211], v230 offset:33792\n\t" \
    "v_mfma_f32_16x16x128_f8f6f4 v[56:59], a[80:87], v[216:223], v[56:59] cbsz:1 blgp:1\n\t" \
    "buffer_load_dwordx4 a[204:207], v233, s[12:15], s72 offen offset:3072 sc0 sc1\n\t" \
    "ds_read_b64_tr_b8 v[212:213], v230 offset:40960\n\t" \
    "v_mfma_f32_16x16x128_f8f6f4 v[60:63], a[88:95], v[216:223], v[60:63] cbsz:1 blgp:1\n\t" \
    "ds_read_b64_tr_b8 v[214:215], v230 offset:41984\n\t" \
    "s_waitcnt lgkmcnt(8)\n\t" \
    "v_mfma_f32_16x16x128_f8f6f4 v[64:67], a[64:71], v[192:199], v[64:67] cbsz:1 blgp:1\n\t" \
    "ds_read_b64_tr_b8 v[216:217], v231 offset:32768\n\t" \
    "v_mfma_f32_16x16x128_f8f6f4 v[68:71], a[72:79], v[192:199], v[68:71] cbsz:1 blgp:1\n\t" \
    "buffer_load_dwordx4 a[208:211], v233, s[12:15], s73 offen offset:0 sc0 sc1\n\t" \
    "ds_read_b64_tr_b8 v[218:219], v231 offset:33792\n\t" \
    "v_mfma_f32_16x16x128_f8f6f4 v[72:75], a[80:87], v[192:199], v[72:75] cbsz:1 blgp:1\n\t" \
    "ds_read_b64_tr_b8 v[220:221], v231 offset:40960\n\t" \
    "v_mfma_f32_16x16x128_f8f6f4 v[76:79], a[88:95], v[192:199], v[76:79] cbsz:1 blgp:1\n\t" \
    "buffer_load_dwordx4 a[212:215], v233, s[12:15], s73 offen offset:1024 sc0 sc1\n\t" \
    "ds_read_b64_tr_b8 v[222:223], v231 offset:41984\n\t" \
    "s_waitcnt lgkmcnt(8)\n\t" \
    "v_mfma_f32_16x16x128_f8f6f4 v[80:83], a[64:71], v[200:207], v[80:83] cbsz:1 blgp:1\n\t" \
    "s_barrier\n\t" \
    "ds_read_b64_tr_b8 v[192:193], v224 offset:49152\n\t" \
    "v_mfma_f32_16x16x128_f8f6f4 v[84:87], a[72:79], v[200:207], v[84:87] cbsz:1 blgp:1\n\t" \
    "s_cmp_eq_u32 s47, 0\n\t" \
    "s_cselect_b32 s87, s26, s27\n\t" \
    "s_add_u32 s85, s87, 6291456\n\t" \
    "s_add_u32 m0, s44, 98304\n\t" \
    "s_add_u32 s74, s85, 0\n\t" \
    "buffer_load_dwordx4 v232, s[8:11], s74 offen  lds\n\t" \
    "ds_read_b64_tr_b8 v[194:195], v224 offset:50176\n\t" \
    "v_mfma_f32_16x16x128_f8f6f4 v[88:91], a[80:87], v[200:207], v[88:91] cbsz:1 blgp:1\n\t" \
    "s_add_u32 m0, s44, 102400\n\t" \
    "s_add_u32 s74, s85, 524288\n\t" \
    "buffer_load_dwordx4 v232, s[8:11], s74 offen  lds\n\t" \
    "ds_read_b64_tr_b8 v[196:197], v224 offset:57344\n\t" \
    "v_mfma_f32_16x16x128_f8f6f4 v[92:95], a[88:95], v[200:207], v[92:95] cbsz:1 blgp:1\n\t" \
    "s_add_u32 m0, s44, 106496\n\t" \
    "s_add_u32 s74, s85, 1048576\n\t" \
    "buffer_load_dwordx4 v232, s[8:11], s74 offen  lds\n\t" \
    "ds_read_b64_tr_b8 v[198:199], v224 offset:58368\n\t" \
    "s_waitcnt lgkmcnt(8)\n\t" \
    "v_mfma_f32_16x16x128_f8f6f4 v[96:99], a[64:71], v[208:215], v[96:99] cbsz:1 blgp:1\n\t" \
    "s_add_u32 m0, s44, 110592\n\t" \
    "s_add_u32 s74, s85, 1572864\n\t" \
    "buffer_load_dwordx4 v232, s[8:11], s74 offen  lds\n\t" \
    "ds_read_b64_tr_b8 v[200:201], v225 offset:49152\n\t" \
    "v_mfma_f32_16x16x128_f8f6f4 v[100:103], a[72:79], v[208:215], v[100:103] cbsz:1 blgp:1\n\t" \
    "s_cmp_eq_u32 s47, 0\n\t" \
    "s_cselect_b32 s87, s28, s29\n\t" \
    "s_add_u32 s85, s87, 6291456\n\t" \
    "s_add_u32 m0, s44, 131072\n\t" \
    "s_add_u32 s74, s85, 0\n\t" \
    "buffer_load_dwordx4 v232, s[8:11], s74 offen  lds\n\t" \
    "ds_read_b64_tr_b8 v[202:203], v225 offset:50176\n\t" \
    "v_mfma_f32_16x16x128_f8f6f4 v[104:107], a[80:87], v[208:215], v[104:107] cbsz:1 blgp:1\n\t" \
    "s_add_u32 m0, s44, 135168\n\t" \
    "s_add_u32 s74, s85, 524288\n\t" \
    "buffer_load_dwordx4 v232, s[8:11], s74 offen  lds\n\t" \
    "ds_read_b64_tr_b8 v[204:205], v225 offset:57344\n\t" \
    "v_mfma_f32_16x16x128_f8f6f4 v[108:111], a[88:95], v[208:215], v[108:111] cbsz:1 blgp:1\n\t" \
    "s_add_u32 m0, s44, 139264\n\t" \
    "s_add_u32 s74, s85, 1048576\n\t" \
    "buffer_load_dwordx4 v232, s[8:11], s74 offen  lds\n\t" \
    "ds_read_b64_tr_b8 v[206:207], v225 offset:58368\n\t" \
    "s_waitcnt lgkmcnt(8)\n\t" \
    "v_mfma_f32_16x16x128_f8f6f4 v[112:115], a[64:71], v[216:223], v[112:115] cbsz:1 blgp:1\n\t" \
    "s_add_u32 m0, s44, 143360\n\t" \
    "s_add_u32 s74, s85, 1572864\n\t" \
    "buffer_load_dwordx4 v232, s[8:11], s74 offen  lds\n\t" \
    "ds_read_b64_tr_b8 v[208:209], v226 offset:49152\n\t" \
    "v_mfma_f32_16x16x128_f8f6f4 v[116:119], a[72:79], v[216:223], v[116:119] cbsz:1 blgp:1\n\t" \
    "ds_read_b64_tr_b8 v[210:211], v226 offset:50176\n\t" \
    "v_mfma_f32_16x16x128_f8f6f4 v[120:123], a[80:87], v[216:223], v[120:123] cbsz:1 blgp:1\n\t" \
    "ds_read_b64_tr_b8 v[212:213], v226 offset:57344\n\t" \
    "v_mfma_f32_16x16x128_f8f6f4 v[124:127], a[88:95], v[216:223], v[124:127] cbsz:1 blgp:1\n\t" \
    "ds_read_b64_tr_b8 v[214:215], v226 offset:58368\n\t" \
    "s_waitcnt vmcnt(30)\n\t" \
    "s_waitcnt lgkmcnt(8)\n\t" \
    "v_mfma_f32_16x16x128_f8f6f4 v[0:3], a[96:103], v[192:199], v[0:3] cbsz:1 blgp:1\n\t" \
    "ds_read_b64_tr_b8 v[216:217], v227 offset:49152\n\t" \
    "v_mfma_f32_16x16x128_f8f6f4 v[4:7], a[104:111], v[192:199], v[4:7] cbsz:1 blgp:1\n\t" \
    "ds_read_b64_tr_b8 v[218:219], v227 offset:50176\n\t" \
    "v_mfma_f32_16x16x128_f8f6f4 v[8:11], a[112:119], v[192:199], v[8:11] cbsz:1 blgp:1\n\t" \
    "ds_read_b64_tr_b8 v[220:221], v227 offset:57344\n\t" \
    "v_mfma_f32_16x16x128_f8f6f4 v[12:15], a[120:127], v[192:199], v[12:15] cbsz:1 blgp:1\n\t" \
    "ds_read_b64_tr_b8 v[222:223], v227 offset:58368\n\t" \
    "s_waitcnt lgkmcnt(8)\n\t" \
    "v_mfma_f32_16x16x128_f8f6f4 v[16:19], a[96:103], v[200:207], v[16:19] cbsz:1 blgp:1\n\t" \
    "ds_read_b64_tr_b8 v[192:193], v228 offset:49152\n\t" \
    "v_mfma_f32_16x16x128_f8f6f4 v[20:23], a[104:111], v[200:207], v[20:23] cbsz:1 blgp:1\n\t" \
    "ds_read_b64_tr_b8 v[194:195], v228 offset:50176\n\t" \
    "v_mfma_f32_16x16x128_f8f6f4 v[24:27], a[112:119], v[200:207], v[24:27] cbsz:1 blgp:1\n\t" \
    "ds_read_b64_tr_b8 v[196:197], v228 offset:57344\n\t" \
    "v_mfma_f32_16x16x128_f8f6f4 v[28:31], a[120:127], v[200:207], v[28:31] cbsz:1 blgp:1\n\t" \
    "ds_read_b64_tr_b8 v[198:199], v228 offset:58368\n\t" \
    "s_waitcnt lgkmcnt(8)\n\t" \
    "v_mfma_f32_16x16x128_f8f6f4 v[32:35], a[96:103], v[208:215], v[32:35] cbsz:1 blgp:1\n\t" \
    "ds_read_b64_tr_b8 v[200:201], v229 offset:49152\n\t" \
    "v_mfma_f32_16x16x128_f8f6f4 v[36:39], a[104:111], v[208:215], v[36:39] cbsz:1 blgp:1\n\t" \
    "ds_read_b64_tr_b8 v[202:203], v229 offset:50176\n\t" \
    "v_mfma_f32_16x16x128_f8f6f4 v[40:43], a[112:119], v[208:215], v[40:43] cbsz:1 blgp:1\n\t" \
    "ds_read_b64_tr_b8 v[204:205], v229 offset:57344\n\t" \
    "v_mfma_f32_16x16x128_f8f6f4 v[44:47], a[120:127], v[208:215], v[44:47] cbsz:1 blgp:1\n\t" \
    "ds_read_b64_tr_b8 v[206:207], v229 offset:58368\n\t" \
    "s_waitcnt lgkmcnt(8)\n\t" \
    "v_mfma_f32_16x16x128_f8f6f4 v[48:51], a[96:103], v[216:223], v[48:51] cbsz:1 blgp:1\n\t" \
    "ds_read_b64_tr_b8 v[208:209], v230 offset:49152\n\t" \
    "v_mfma_f32_16x16x128_f8f6f4 v[52:55], a[104:111], v[216:223], v[52:55] cbsz:1 blgp:1\n\t" \
    "ds_read_b64_tr_b8 v[210:211], v230 offset:50176\n\t" \
    "v_mfma_f32_16x16x128_f8f6f4 v[56:59], a[112:119], v[216:223], v[56:59] cbsz:1 blgp:1\n\t" \
    "ds_read_b64_tr_b8 v[212:213], v230 offset:57344\n\t" \
    "v_mfma_f32_16x16x128_f8f6f4 v[60:63], a[120:127], v[216:223], v[60:63] cbsz:1 blgp:1\n\t" \
    "ds_read_b64_tr_b8 v[214:215], v230 offset:58368\n\t" \
    "s_waitcnt lgkmcnt(8)\n\t" \
    "v_mfma_f32_16x16x128_f8f6f4 v[64:67], a[96:103], v[192:199], v[64:67] cbsz:1 blgp:1\n\t" \
    "ds_read_b64_tr_b8 v[216:217], v231 offset:49152\n\t" \
    "v_mfma_f32_16x16x128_f8f6f4 v[68:71], a[104:111], v[192:199], v[68:71] cbsz:1 blgp:1\n\t" \
    "ds_read_b64_tr_b8 v[218:219], v231 offset:50176\n\t" \
    "v_mfma_f32_16x16x128_f8f6f4 v[72:75], a[112:119], v[192:199], v[72:75] cbsz:1 blgp:1\n\t" \
    "ds_read_b64_tr_b8 v[220:221], v231 offset:57344\n\t" \
    "v_mfma_f32_16x16x128_f8f6f4 v[76:79], a[120:127], v[192:199], v[76:79] cbsz:1 blgp:1\n\t" \
    "s_cmp_eq_u32 s47, 1\n\t" \
    "s_cselect_b32 s87, s26, s27\n\t" \
    "s_add_u32 s86, s87, 6291456\n\t" \
    "s_add_u32 m0, s46, 98304\n\t" \
    "s_add_u32 s74, s86, 0\n\t" \
    "buffer_load_dwordx4 v232, s[8:11], s74 offen  lds\n\t" \
    "ds_read_b64_tr_b8 v[222:223], v231 offset:58368\n\t" \
    "s_waitcnt lgkmcnt(8)\n\t" \
    "v_mfma_f32_16x16x128_f8f6f4 v[80:83], a[96:103], v[200:207], v[80:83] cbsz:1 blgp:1\n\t" \
    "s_barrier\n\t" \
    "v_add_u32 v239, 65536, v224\n\t" \
    "ds_read_b64_tr_b8 v[192:193], v239 offset:0\n\t" \
    "v_mfma_f32_16x16x128_f8f6f4 v[84:87], a[104:111], v[200:207], v[84:87] cbsz:1 blgp:1\n\t" \
    "ds_read_b64_tr_b8 v[194:195], v239 offset:1024\n\t" \
    "v_mfma_f32_16x16x128_f8f6f4 v[88:91], a[112:119], v[200:207], v[88:91] cbsz:1 blgp:1\n\t" \
    "s_add_u32 m0, s46, 102400\n\t" \
    "s_add_u32 s74, s86, 524288\n\t" \
    "buffer_load_dwordx4 v232, s[8:11], s74 offen  lds\n\t" \
    "ds_read_b64_tr_b8 v[196:197], v239 offset:8192\n\t" \
    "v_mfma_f32_16x16x128_f8f6f4 v[92:95], a[120:127], v[200:207], v[92:95] cbsz:1 blgp:1\n\t" \
    "ds_read_b64_tr_b8 v[198:199], v239 offset:9216\n\t" \
    "s_waitcnt lgkmcnt(8)\n\t" \
    "v_mfma_f32_16x16x128_f8f6f4 v[96:99], a[96:103], v[208:215], v[96:99] cbsz:1 blgp:1\n\t" \
    "s_add_u32 m0, s46, 106496\n\t" \
    "s_add_u32 s74, s86, 1048576\n\t" \
    "buffer_load_dwordx4 v232, s[8:11], s74 offen  lds\n\t" \
    "v_add_u32 v238, 65536, v225\n\t" \
    "ds_read_b64_tr_b8 v[200:201], v238 offset:0\n\t" \
    "v_mfma_f32_16x16x128_f8f6f4 v[100:103], a[104:111], v[208:215], v[100:103] cbsz:1 blgp:1\n\t" \
    "ds_read_b64_tr_b8 v[202:203], v238 offset:1024\n\t" \
    "v_mfma_f32_16x16x128_f8f6f4 v[104:107], a[112:119], v[208:215], v[104:107] cbsz:1 blgp:1\n\t" \
    "s_add_u32 m0, s46, 110592\n\t" \
    "s_add_u32 s74, s86, 1572864\n\t" \
    "buffer_load_dwordx4 v232, s[8:11], s74 offen  lds\n\t" \
    "ds_read_b64_tr_b8 v[204:205], v238 offset:8192\n\t" \
    "v_mfma_f32_16x16x128_f8f6f4 v[108:111], a[120:127], v[208:215], v[108:111] cbsz:1 blgp:1\n\t" \
    "ds_read_b64_tr_b8 v[206:207], v238 offset:9216\n\t" \
    "s_waitcnt lgkmcnt(8)\n\t" \
    "v_mfma_f32_16x16x128_f8f6f4 v[112:115], a[96:103], v[216:223], v[112:115] cbsz:1 blgp:1\n\t" \
    "v_add_u32 v239, 65536, v226\n\t" \
    "ds_read_b64_tr_b8 v[208:209], v239 offset:0\n\t" \
    "v_mfma_f32_16x16x128_f8f6f4 v[116:119], a[104:111], v[216:223], v[116:119] cbsz:1 blgp:1\n\t" \
    "s_cmp_eq_u32 s47, 1\n\t" \
    "s_cselect_b32 s87, s28, s29\n\t" \
    "s_add_u32 s86, s87, 6291456\n\t" \
    "s_add_u32 m0, s46, 131072\n\t" \
    "s_add_u32 s74, s86, 0\n\t" \
    "buffer_load_dwordx4 v232, s[8:11], s74 offen  lds\n\t" \
    "ds_read_b64_tr_b8 v[210:211], v239 offset:1024\n\t" \
    "v_mfma_f32_16x16x128_f8f6f4 v[120:123], a[112:119], v[216:223], v[120:123] cbsz:1 blgp:1\n\t" \
    "ds_read_b64_tr_b8 v[212:213], v239 offset:8192\n\t" \
    "v_mfma_f32_16x16x128_f8f6f4 v[124:127], a[120:127], v[216:223], v[124:127] cbsz:1 blgp:1\n\t" \
    "s_add_u32 m0, s46, 135168\n\t" \
    "s_add_u32 s74, s86, 524288\n\t" \
    "buffer_load_dwordx4 v232, s[8:11], s74 offen  lds\n\t" \
    "ds_read_b64_tr_b8 v[214:215], v239 offset:9216\n\t" \
    "s_waitcnt vmcnt(28)\n\t" \
    "s_waitcnt lgkmcnt(8)\n\t" \
    "v_mfma_f32_16x16x128_f8f6f4 v[0:3], a[128:135], v[192:199], v[0:3] cbsz:1 blgp:1\n\t" \
    "v_add_u32 v238, 65536, v227\n\t" \
    "ds_read_b64_tr_b8 v[216:217], v238 offset:0\n\t" \
    "v_mfma_f32_16x16x128_f8f6f4 v[4:7], a[136:143], v[192:199], v[4:7] cbsz:1 blgp:1\n\t" \
    "s_add_u32 m0, s46, 139264\n\t" \
    "s_add_u32 s74, s86, 1048576\n\t" \
    "buffer_load_dwordx4 v232, s[8:11], s74 offen  lds\n\t" \
    "ds_read_b64_tr_b8 v[218:219], v238 offset:1024\n\t" \
    "v_mfma_f32_16x16x128_f8f6f4 v[8:11], a[144:151], v[192:199], v[8:11] cbsz:1 blgp:1\n\t" \
    "ds_read_b64_tr_b8 v[220:221], v238 offset:8192\n\t" \
    "v_mfma_f32_16x16x128_f8f6f4 v[12:15], a[152:159], v[192:199], v[12:15] cbsz:1 blgp:1\n\t" \
    "ds_read_b64_tr_b8 v[222:223], v238 offset:9216\n\t" \
    "s_waitcnt lgkmcnt(8)\n\t" \
    "v_mfma_f32_16x16x128_f8f6f4 v[16:19], a[128:135], v[200:207], v[16:19] cbsz:1 blgp:1\n\t" \
    "s_add_u32 m0, s46, 143360\n\t" \
    "s_add_u32 s74, s86, 1572864\n\t" \
    "buffer_load_dwordx4 v232, s[8:11], s74 offen  lds\n\t" \
    "v_add_u32 v239, 65536, v228\n\t" \
    "ds_read_b64_tr_b8 v[192:193], v239 offset:0\n\t" \
    "v_mfma_f32_16x16x128_f8f6f4 v[20:23], a[136:143], v[200:207], v[20:23] cbsz:1 blgp:1\n\t" \
    "ds_read_b64_tr_b8 v[194:195], v239 offset:1024\n\t" \
    "v_mfma_f32_16x16x128_f8f6f4 v[24:27], a[144:151], v[200:207], v[24:27] cbsz:1 blgp:1\n\t" \
    "buffer_load_dwordx4 a[216:219], v233, s[12:15], s73 offen offset:2048 sc0 sc1\n\t" \
    "ds_read_b64_tr_b8 v[196:197], v239 offset:8192\n\t" \
    "v_mfma_f32_16x16x128_f8f6f4 v[28:31], a[152:159], v[200:207], v[28:31] cbsz:1 blgp:1\n\t" \
    "ds_read_b64_tr_b8 v[198:199], v239 offset:9216\n\t" \
    "s_waitcnt lgkmcnt(8)\n\t" \
    "v_mfma_f32_16x16x128_f8f6f4 v[32:35], a[128:135], v[208:215], v[32:35] cbsz:1 blgp:1\n\t" \
    "buffer_load_dwordx4 a[220:223], v233, s[12:15], s73 offen offset:3072 sc0 sc1\n\t" \
    "v_add_u32 v238, 65536, v229\n\t" \
    "ds_read_b64_tr_b8 v[200:201], v238 offset:0\n\t" \
    "v_mfma_f32_16x16x128_f8f6f4 v[36:39], a[136:143], v[208:215], v[36:39] cbsz:1 blgp:1\n\t" \
    "ds_read_b64_tr_b8 v[202:203], v238 offset:1024\n\t" \
    "v_mfma_f32_16x16x128_f8f6f4 v[40:43], a[144:151], v[208:215], v[40:43] cbsz:1 blgp:1\n\t" \
    "ds_read_b64_tr_b8 v[204:205], v238 offset:8192\n\t" \
    "v_mfma_f32_16x16x128_f8f6f4 v[44:47], a[152:159], v[208:215], v[44:47] cbsz:1 blgp:1\n\t" \
    "s_mov_b32 s72, 1769472\n\t" \
    "s_mov_b32 s73, 1773568\n\t" \
    "buffer_load_dwordx4 a[224:227], v233, s[12:15], s72 offen offset:0 sc0 sc1\n\t" \
    "ds_read_b64_tr_b8 v[206:207], v238 offset:9216\n\t" \
    "s_waitcnt lgkmcnt(8)\n\t" \
    "v_mfma_f32_16x16x128_f8f6f4 v[48:51], a[128:135], v[216:223], v[48:51] cbsz:1 blgp:1\n\t" \
    "v_add_u32 v239, 65536, v230\n\t" \
    "ds_read_b64_tr_b8 v[208:209], v239 offset:0\n\t" \
    "v_mfma_f32_16x16x128_f8f6f4 v[52:55], a[136:143], v[216:223], v[52:55] cbsz:1 blgp:1\n\t" \
    "buffer_load_dwordx4 a[228:231], v233, s[12:15], s72 offen offset:1024 sc0 sc1\n\t" \
    "ds_read_b64_tr_b8 v[210:211], v239 offset:1024\n\t" \
    "v_mfma_f32_16x16x128_f8f6f4 v[56:59], a[144:151], v[216:223], v[56:59] cbsz:1 blgp:1\n\t" \
    "ds_read_b64_tr_b8 v[212:213], v239 offset:8192\n\t" \
    "v_mfma_f32_16x16x128_f8f6f4 v[60:63], a[152:159], v[216:223], v[60:63] cbsz:1 blgp:1\n\t" \
    "buffer_load_dwordx4 a[232:235], v233, s[12:15], s72 offen offset:2048 sc0 sc1\n\t" \
    "ds_read_b64_tr_b8 v[214:215], v239 offset:9216\n\t" \
    "s_waitcnt lgkmcnt(8)\n\t" \
    "v_mfma_f32_16x16x128_f8f6f4 v[64:67], a[128:135], v[192:199], v[64:67] cbsz:1 blgp:1\n\t" \
    "v_add_u32 v238, 65536, v231\n\t" \
    "ds_read_b64_tr_b8 v[216:217], v238 offset:0\n\t" \
    "v_mfma_f32_16x16x128_f8f6f4 v[68:71], a[136:143], v[192:199], v[68:71] cbsz:1 blgp:1\n\t" \
    "ds_read_b64_tr_b8 v[218:219], v238 offset:1024\n\t" \
    "v_mfma_f32_16x16x128_f8f6f4 v[72:75], a[144:151], v[192:199], v[72:75] cbsz:1 blgp:1\n\t" \
    "buffer_load_dwordx4 a[236:239], v233, s[12:15], s72 offen offset:3072 sc0 sc1\n\t" \
    "ds_read_b64_tr_b8 v[220:221], v238 offset:8192\n\t" \
    "v_mfma_f32_16x16x128_f8f6f4 v[76:79], a[152:159], v[192:199], v[76:79] cbsz:1 blgp:1\n\t" \
    "ds_read_b64_tr_b8 v[222:223], v238 offset:9216\n\t" \
    "s_waitcnt lgkmcnt(8)\n\t" \
    "v_mfma_f32_16x16x128_f8f6f4 v[80:83], a[128:135], v[200:207], v[80:83] cbsz:1 blgp:1\n\t" \
    "s_barrier\n\t" \
    "buffer_load_dwordx4 a[240:243], v233, s[12:15], s73 offen offset:0 sc0 sc1\n\t" \
    "v_add_u32 v239, 65536, v224\n\t" \
    "ds_read_b64_tr_b8 v[192:193], v239 offset:16384\n\t" \
    "v_mfma_f32_16x16x128_f8f6f4 v[84:87], a[136:143], v[200:207], v[84:87] cbsz:1 blgp:1\n\t" \
    "ds_read_b64_tr_b8 v[194:195], v239 offset:17408\n\t" \
    "v_mfma_f32_16x16x128_f8f6f4 v[88:91], a[144:151], v[200:207], v[88:91] cbsz:1 blgp:1\n\t" \
    "buffer_load_dwordx4 a[244:247], v233, s[12:15], s73 offen offset:1024 sc0 sc1\n\t" \
    "ds_read_b64_tr_b8 v[196:197], v239 offset:24576\n\t" \
    "v_mfma_f32_16x16x128_f8f6f4 v[92:95], a[152:159], v[200:207], v[92:95] cbsz:1 blgp:1\n\t" \
    "ds_read_b64_tr_b8 v[198:199], v239 offset:25600\n\t" \
    "s_waitcnt lgkmcnt(8)\n\t" \
    "v_mfma_f32_16x16x128_f8f6f4 v[96:99], a[128:135], v[208:215], v[96:99] cbsz:1 blgp:1\n\t" \
    "v_add_u32 v238, 65536, v225\n\t" \
    "ds_read_b64_tr_b8 v[200:201], v238 offset:16384\n\t" \
    "v_mfma_f32_16x16x128_f8f6f4 v[100:103], a[136:143], v[208:215], v[100:103] cbsz:1 blgp:1\n\t" \
    "buffer_load_dwordx4 a[248:251], v233, s[12:15], s73 offen offset:2048 sc0 sc1\n\t" \
    "ds_read_b64_tr_b8 v[202:203], v238 offset:17408\n\t" \
    "v_mfma_f32_16x16x128_f8f6f4 v[104:107], a[144:151], v[208:215], v[104:107] cbsz:1 blgp:1\n\t" \
    "ds_read_b64_tr_b8 v[204:205], v238 offset:24576\n\t" \
    "v_mfma_f32_16x16x128_f8f6f4 v[108:111], a[152:159], v[208:215], v[108:111] cbsz:1 blgp:1\n\t" \
    "buffer_load_dwordx4 a[252:255], v233, s[12:15], s73 offen offset:3072 sc0 sc1\n\t" \
    "ds_read_b64_tr_b8 v[206:207], v238 offset:25600\n\t" \
    "s_waitcnt lgkmcnt(8)\n\t" \
    "v_mfma_f32_16x16x128_f8f6f4 v[112:115], a[128:135], v[216:223], v[112:115] cbsz:1 blgp:1\n\t" \
    "v_add_u32 v239, 65536, v226\n\t" \
    "ds_read_b64_tr_b8 v[208:209], v239 offset:16384\n\t" \
    "v_mfma_f32_16x16x128_f8f6f4 v[116:119], a[136:143], v[216:223], v[116:119] cbsz:1 blgp:1\n\t" \
    "s_mov_b32 s72, 1835008\n\t" \
    "s_mov_b32 s73, 1839104\n\t" \
    "buffer_load_dwordx4 v[128:131], v233, s[12:15], s72 offen offset:0 sc0 sc1\n\t" \
    "ds_read_b64_tr_b8 v[210:211], v239 offset:17408\n\t" \
    "v_mfma_f32_16x16x128_f8f6f4 v[120:123], a[144:151], v[216:223], v[120:123] cbsz:1 blgp:1\n\t" \
    "ds_read_b64_tr_b8 v[212:213], v239 offset:24576\n\t" \
    "v_mfma_f32_16x16x128_f8f6f4 v[124:127], a[152:159], v[216:223], v[124:127] cbsz:1 blgp:1\n\t" \
    "ds_read_b64_tr_b8 v[214:215], v239 offset:25600\n\t" \
    "s_waitcnt vmcnt(33)\n\t" \
    "s_waitcnt lgkmcnt(8)\n\t" \
    "v_mfma_f32_16x16x128_f8f6f4 v[0:3], a[160:167], v[192:199], v[0:3] cbsz:1 blgp:1\n\t" \
    "buffer_load_dwordx4 v[132:135], v233, s[12:15], s72 offen offset:1024 sc0 sc1\n\t" \
    "v_add_u32 v238, 65536, v227\n\t" \
    "ds_read_b64_tr_b8 v[216:217], v238 offset:16384\n\t" \
    "v_mfma_f32_16x16x128_f8f6f4 v[4:7], a[168:175], v[192:199], v[4:7] cbsz:1 blgp:1\n\t" \
    "ds_read_b64_tr_b8 v[218:219], v238 offset:17408\n\t" \
    "v_mfma_f32_16x16x128_f8f6f4 v[8:11], a[176:183], v[192:199], v[8:11] cbsz:1 blgp:1\n\t" \
    "buffer_load_dwordx4 v[136:139], v233, s[12:15], s72 offen offset:2048 sc0 sc1\n\t" \
    "ds_read_b64_tr_b8 v[220:221], v238 offset:24576\n\t" \
    "v_mfma_f32_16x16x128_f8f6f4 v[12:15], a[184:191], v[192:199], v[12:15] cbsz:1 blgp:1\n\t" \
    "ds_read_b64_tr_b8 v[222:223], v238 offset:25600\n\t" \
    "s_waitcnt lgkmcnt(8)\n\t" \
    "v_mfma_f32_16x16x128_f8f6f4 v[16:19], a[160:167], v[200:207], v[16:19] cbsz:1 blgp:1\n\t" \
    "buffer_load_dwordx4 v[140:143], v233, s[12:15], s72 offen offset:3072 sc0 sc1\n\t" \
    "v_add_u32 v239, 65536, v228\n\t" \
    "ds_read_b64_tr_b8 v[192:193], v239 offset:16384\n\t" \
    "v_mfma_f32_16x16x128_f8f6f4 v[20:23], a[168:175], v[200:207], v[20:23] cbsz:1 blgp:1\n\t" \
    "ds_read_b64_tr_b8 v[194:195], v239 offset:17408\n\t" \
    "v_mfma_f32_16x16x128_f8f6f4 v[24:27], a[176:183], v[200:207], v[24:27] cbsz:1 blgp:1\n\t" \
    "ds_read_b64_tr_b8 v[196:197], v239 offset:24576\n\t" \
    "v_mfma_f32_16x16x128_f8f6f4 v[28:31], a[184:191], v[200:207], v[28:31] cbsz:1 blgp:1\n\t" \
    "buffer_load_dwordx4 v[144:147], v233, s[12:15], s73 offen offset:0 sc0 sc1\n\t" \
    "ds_read_b64_tr_b8 v[198:199], v239 offset:25600\n\t" \
    "s_waitcnt lgkmcnt(8)\n\t" \
    "v_mfma_f32_16x16x128_f8f6f4 v[32:35], a[160:167], v[208:215], v[32:35] cbsz:1 blgp:1\n\t" \
    "v_add_u32 v238, 65536, v229\n\t" \
    "ds_read_b64_tr_b8 v[200:201], v238 offset:16384\n\t" \
    "v_mfma_f32_16x16x128_f8f6f4 v[36:39], a[168:175], v[208:215], v[36:39] cbsz:1 blgp:1\n\t" \
    "buffer_load_dwordx4 v[148:151], v233, s[12:15], s73 offen offset:1024 sc0 sc1\n\t" \
    "ds_read_b64_tr_b8 v[202:203], v238 offset:17408\n\t" \
    "v_mfma_f32_16x16x128_f8f6f4 v[40:43], a[176:183], v[208:215], v[40:43] cbsz:1 blgp:1\n\t" \
    "ds_read_b64_tr_b8 v[204:205], v238 offset:24576\n\t" \
    "v_mfma_f32_16x16x128_f8f6f4 v[44:47], a[184:191], v[208:215], v[44:47] cbsz:1 blgp:1\n\t" \
    "buffer_load_dwordx4 v[152:155], v233, s[12:15], s73 offen offset:2048 sc0 sc1\n\t" \
    "ds_read_b64_tr_b8 v[206:207], v238 offset:25600\n\t" \
    "s_waitcnt lgkmcnt(8)\n\t" \
    "v_mfma_f32_16x16x128_f8f6f4 v[48:51], a[160:167], v[216:223], v[48:51] cbsz:1 blgp:1\n\t" \
    "v_add_u32 v239, 65536, v230\n\t" \
    "ds_read_b64_tr_b8 v[208:209], v239 offset:16384\n\t" \
    "v_mfma_f32_16x16x128_f8f6f4 v[52:55], a[168:175], v[216:223], v[52:55] cbsz:1 blgp:1\n\t" \
    "ds_read_b64_tr_b8 v[210:211], v239 offset:17408\n\t" \
    "v_mfma_f32_16x16x128_f8f6f4 v[56:59], a[176:183], v[216:223], v[56:59] cbsz:1 blgp:1\n\t" \
    "buffer_load_dwordx4 v[156:159], v233, s[12:15], s73 offen offset:3072 sc0 sc1\n\t" \
    "ds_read_b64_tr_b8 v[212:213], v239 offset:24576\n\t" \
    "v_mfma_f32_16x16x128_f8f6f4 v[60:63], a[184:191], v[216:223], v[60:63] cbsz:1 blgp:1\n\t" \
    "ds_read_b64_tr_b8 v[214:215], v239 offset:25600\n\t" \
    "s_waitcnt lgkmcnt(8)\n\t" \
    "v_mfma_f32_16x16x128_f8f6f4 v[64:67], a[160:167], v[192:199], v[64:67] cbsz:1 blgp:1\n\t" \
    "s_mov_b32 s72, 1900544\n\t" \
    "s_mov_b32 s73, 1904640\n\t" \
    "buffer_load_dwordx4 v[160:163], v233, s[12:15], s72 offen offset:0 sc0 sc1\n\t" \
    "v_add_u32 v238, 65536, v231\n\t" \
    "ds_read_b64_tr_b8 v[216:217], v238 offset:16384\n\t" \
    "v_mfma_f32_16x16x128_f8f6f4 v[68:71], a[168:175], v[192:199], v[68:71] cbsz:1 blgp:1\n\t" \
    "ds_read_b64_tr_b8 v[218:219], v238 offset:17408\n\t" \
    "v_mfma_f32_16x16x128_f8f6f4 v[72:75], a[176:183], v[192:199], v[72:75] cbsz:1 blgp:1\n\t" \
    "buffer_load_dwordx4 v[164:167], v233, s[12:15], s72 offen offset:1024 sc0 sc1\n\t" \
    "ds_read_b64_tr_b8 v[220:221], v238 offset:24576\n\t" \
    "v_mfma_f32_16x16x128_f8f6f4 v[76:79], a[184:191], v[192:199], v[76:79] cbsz:1 blgp:1\n\t" \
    "ds_read_b64_tr_b8 v[222:223], v238 offset:25600\n\t" \
    "s_waitcnt lgkmcnt(8)\n\t" \
    "v_mfma_f32_16x16x128_f8f6f4 v[80:83], a[160:167], v[200:207], v[80:83] cbsz:1 blgp:1\n\t" \
    "s_waitcnt vmcnt(32)\n\t" \
    "s_waitcnt vmcnt(24)\n\t" \
    "s_barrier\n\t" \
    "v_add_u32 v239, 65536, v224\n\t" \
    "ds_read_b64_tr_b8 v[192:193], v239 offset:32768\n\t" \
    "v_mfma_f32_16x16x128_f8f6f4 v[84:87], a[168:175], v[200:207], v[84:87] cbsz:1 blgp:1\n\t" \
    "buffer_load_dwordx4 v[168:171], v233, s[12:15], s72 offen offset:2048 sc0 sc1\n\t" \
    "ds_read_b64_tr_b8 v[194:195], v239 offset:33792\n\t" \
    "v_mfma_f32_16x16x128_f8f6f4 v[88:91], a[176:183], v[200:207], v[88:91] cbsz:1 blgp:1\n\t" \
    "ds_read_b64_tr_b8 v[196:197], v239 offset:40960\n\t" \
    "v_mfma_f32_16x16x128_f8f6f4 v[92:95], a[184:191], v[200:207], v[92:95] cbsz:1 blgp:1\n\t" \
    "buffer_load_dwordx4 v[172:175], v233, s[12:15], s72 offen offset:3072 sc0 sc1\n\t" \
    "ds_read_b64_tr_b8 v[198:199], v239 offset:41984\n\t" \
    "s_waitcnt lgkmcnt(8)\n\t" \
    "v_mfma_f32_16x16x128_f8f6f4 v[96:99], a[160:167], v[208:215], v[96:99] cbsz:1 blgp:1\n\t" \
    "v_add_u32 v238, 65536, v225\n\t" \
    "ds_read_b64_tr_b8 v[200:201], v238 offset:32768\n\t" \
    "v_mfma_f32_16x16x128_f8f6f4 v[100:103], a[168:175], v[208:215], v[100:103] cbsz:1 blgp:1\n\t" \
    "buffer_load_dwordx4 v[176:179], v233, s[12:15], s73 offen offset:0 sc0 sc1\n\t" \
    "ds_read_b64_tr_b8 v[202:203], v238 offset:33792\n\t" \
    "v_mfma_f32_16x16x128_f8f6f4 v[104:107], a[176:183], v[208:215], v[104:107] cbsz:1 blgp:1\n\t" \
    "ds_read_b64_tr_b8 v[204:205], v238 offset:40960\n\t" \
    "v_mfma_f32_16x16x128_f8f6f4 v[108:111], a[184:191], v[208:215], v[108:111] cbsz:1 blgp:1\n\t" \
    "ds_read_b64_tr_b8 v[206:207], v238 offset:41984\n\t" \
    "s_waitcnt lgkmcnt(8)\n\t" \
    "v_mfma_f32_16x16x128_f8f6f4 v[112:115], a[160:167], v[216:223], v[112:115] cbsz:1 blgp:1\n\t" \
    "buffer_load_dwordx4 v[180:183], v233, s[12:15], s73 offen offset:1024 sc0 sc1\n\t" \
    "v_add_u32 v239, 65536, v226\n\t" \
    "ds_read_b64_tr_b8 v[208:209], v239 offset:32768\n\t" \
    "v_mfma_f32_16x16x128_f8f6f4 v[116:119], a[168:175], v[216:223], v[116:119] cbsz:1 blgp:1\n\t" \
    "ds_read_b64_tr_b8 v[210:211], v239 offset:33792\n\t" \
    "v_mfma_f32_16x16x128_f8f6f4 v[120:123], a[176:183], v[216:223], v[120:123] cbsz:1 blgp:1\n\t" \
    "buffer_load_dwordx4 v[184:187], v233, s[12:15], s73 offen offset:2048 sc0 sc1\n\t" \
    "ds_read_b64_tr_b8 v[212:213], v239 offset:40960\n\t" \
    "v_mfma_f32_16x16x128_f8f6f4 v[124:127], a[184:191], v[216:223], v[124:127] cbsz:1 blgp:1\n\t" \
    "ds_read_b64_tr_b8 v[214:215], v239 offset:41984\n\t" \
    "s_waitcnt vmcnt(23)\n\t" \
    "s_waitcnt lgkmcnt(8)\n\t" \
    "v_mfma_f32_16x16x128_f8f6f4 v[0:3], a[192:199], v[192:199], v[0:3] cbsz:1 blgp:1\n\t" \
    "buffer_load_dwordx4 v[188:191], v233, s[12:15], s73 offen offset:3072 sc0 sc1\n\t" \
    "v_add_u32 v238, 65536, v227\n\t" \
    "ds_read_b64_tr_b8 v[216:217], v238 offset:32768\n\t" \
    "v_mfma_f32_16x16x128_f8f6f4 v[4:7], a[200:207], v[192:199], v[4:7] cbsz:1 blgp:1\n\t" \
    "ds_read_b64_tr_b8 v[218:219], v238 offset:33792\n\t" \
    "v_mfma_f32_16x16x128_f8f6f4 v[8:11], a[208:215], v[192:199], v[8:11] cbsz:1 blgp:1\n\t" \
    "ds_read_b64_tr_b8 v[220:221], v238 offset:40960\n\t" \
    "v_mfma_f32_16x16x128_f8f6f4 v[12:15], a[216:223], v[192:199], v[12:15] cbsz:1 blgp:1\n\t" \
    "s_mov_b32 s72, 1966080\n\t" \
    "s_mov_b32 s73, 1970176\n\t" \
    "buffer_load_dwordx4 a[0:3], v233, s[12:15], s72 offen offset:0 sc0 sc1\n\t" \
    "ds_read_b64_tr_b8 v[222:223], v238 offset:41984\n\t" \
    "s_waitcnt lgkmcnt(8)\n\t" \
    "v_mfma_f32_16x16x128_f8f6f4 v[16:19], a[192:199], v[200:207], v[16:19] cbsz:1 blgp:1\n\t" \
    "v_add_u32 v239, 65536, v228\n\t" \
    "ds_read_b64_tr_b8 v[192:193], v239 offset:32768\n\t" \
    "v_mfma_f32_16x16x128_f8f6f4 v[20:23], a[200:207], v[200:207], v[20:23] cbsz:1 blgp:1\n\t" \
    "buffer_load_dwordx4 a[4:7], v233, s[12:15], s72 offen offset:1024 sc0 sc1\n\t" \
    "ds_read_b64_tr_b8 v[194:195], v239 offset:33792\n\t" \
    "v_mfma_f32_16x16x128_f8f6f4 v[24:27], a[208:215], v[200:207], v[24:27] cbsz:1 blgp:1\n\t" \
    "ds_read_b64_tr_b8 v[196:197], v239 offset:40960\n\t" \
    "v_mfma_f32_16x16x128_f8f6f4 v[28:31], a[216:223], v[200:207], v[28:31] cbsz:1 blgp:1\n\t" \
    "buffer_load_dwordx4 a[8:11], v233, s[12:15], s72 offen offset:2048 sc0 sc1\n\t" \
    "ds_read_b64_tr_b8 v[198:199], v239 offset:41984\n\t" \
    "s_waitcnt lgkmcnt(8)\n\t" \
    "v_mfma_f32_16x16x128_f8f6f4 v[32:35], a[192:199], v[208:215], v[32:35] cbsz:1 blgp:1\n\t" \
    "v_add_u32 v238, 65536, v229\n\t" \
    "ds_read_b64_tr_b8 v[200:201], v238 offset:32768\n\t" \
    "v_mfma_f32_16x16x128_f8f6f4 v[36:39], a[200:207], v[208:215], v[36:39] cbsz:1 blgp:1\n\t" \
    "ds_read_b64_tr_b8 v[202:203], v238 offset:33792\n\t" \
    "v_mfma_f32_16x16x128_f8f6f4 v[40:43], a[208:215], v[208:215], v[40:43] cbsz:1 blgp:1\n\t" \
    "buffer_load_dwordx4 a[12:15], v233, s[12:15], s72 offen offset:3072 sc0 sc1\n\t" \
    "ds_read_b64_tr_b8 v[204:205], v238 offset:40960\n\t" \
    "v_mfma_f32_16x16x128_f8f6f4 v[44:47], a[216:223], v[208:215], v[44:47] cbsz:1 blgp:1\n\t" \
    "ds_read_b64_tr_b8 v[206:207], v238 offset:41984\n\t" \
    "s_waitcnt lgkmcnt(8)\n\t" \
    "v_mfma_f32_16x16x128_f8f6f4 v[48:51], a[192:199], v[216:223], v[48:51] cbsz:1 blgp:1\n\t" \
    "buffer_load_dwordx4 a[16:19], v233, s[12:15], s73 offen offset:0 sc0 sc1\n\t" \
    "v_add_u32 v239, 65536, v230\n\t" \
    "ds_read_b64_tr_b8 v[208:209], v239 offset:32768\n\t" \
    "v_mfma_f32_16x16x128_f8f6f4 v[52:55], a[200:207], v[216:223], v[52:55] cbsz:1 blgp:1\n\t" \
    "ds_read_b64_tr_b8 v[210:211], v239 offset:33792\n\t" \
    "v_mfma_f32_16x16x128_f8f6f4 v[56:59], a[208:215], v[216:223], v[56:59] cbsz:1 blgp:1\n\t" \
    "buffer_load_dwordx4 a[20:23], v233, s[12:15], s73 offen offset:1024 sc0 sc1\n\t" \
    "ds_read_b64_tr_b8 v[212:213], v239 offset:40960\n\t" \
    "v_mfma_f32_16x16x128_f8f6f4 v[60:63], a[216:223], v[216:223], v[60:63] cbsz:1 blgp:1\n\t" \
    "ds_read_b64_tr_b8 v[214:215], v239 offset:41984\n\t" \
    "s_waitcnt lgkmcnt(8)\n\t" \
    "v_mfma_f32_16x16x128_f8f6f4 v[64:67], a[192:199], v[192:199], v[64:67] cbsz:1 blgp:1\n\t" \
    "v_add_u32 v238, 65536, v231\n\t" \
    "ds_read_b64_tr_b8 v[216:217], v238 offset:32768\n\t" \
    "v_mfma_f32_16x16x128_f8f6f4 v[68:71], a[200:207], v[192:199], v[68:71] cbsz:1 blgp:1\n\t" \
    "buffer_load_dwordx4 a[24:27], v233, s[12:15], s73 offen offset:2048 sc0 sc1\n\t" \
    "ds_read_b64_tr_b8 v[218:219], v238 offset:33792\n\t" \
    "v_mfma_f32_16x16x128_f8f6f4 v[72:75], a[208:215], v[192:199], v[72:75] cbsz:1 blgp:1\n\t" \
    "ds_read_b64_tr_b8 v[220:221], v238 offset:40960\n\t" \
    "v_mfma_f32_16x16x128_f8f6f4 v[76:79], a[216:223], v[192:199], v[76:79] cbsz:1 blgp:1\n\t" \
    "buffer_load_dwordx4 a[28:31], v233, s[12:15], s73 offen offset:3072 sc0 sc1\n\t" \
    "ds_read_b64_tr_b8 v[222:223], v238 offset:41984\n\t" \
    "s_waitcnt lgkmcnt(8)\n\t" \
    "v_mfma_f32_16x16x128_f8f6f4 v[80:83], a[192:199], v[200:207], v[80:83] cbsz:1 blgp:1\n\t" \
    "s_barrier\n\t" \
    "v_add_u32 v239, 65536, v224\n\t" \
    "ds_read_b64_tr_b8 v[192:193], v239 offset:49152\n\t" \
    "v_mfma_f32_16x16x128_f8f6f4 v[84:87], a[200:207], v[200:207], v[84:87] cbsz:1 blgp:1\n\t" \
    "s_cmp_eq_u32 s47, 0\n\t" \
    "s_cselect_b32 s87, s30, s31\n\t" \
    "s_add_u32 s85, s87, 6291456\n\t" \
    "s_add_u32 m0, s44, 0\n\t" \
    "s_add_u32 s74, s85, 0\n\t" \
    "buffer_load_dwordx4 v232, s[8:11], s74 offen  lds\n\t" \
    "ds_read_b64_tr_b8 v[194:195], v239 offset:50176\n\t" \
    "v_mfma_f32_16x16x128_f8f6f4 v[88:91], a[208:215], v[200:207], v[88:91] cbsz:1 blgp:1\n\t" \
    "s_add_u32 m0, s44, 4096\n\t" \
    "s_add_u32 s74, s85, 524288\n\t" \
    "buffer_load_dwordx4 v232, s[8:11], s74 offen  lds\n\t" \
    "ds_read_b64_tr_b8 v[196:197], v239 offset:57344\n\t" \
    "v_mfma_f32_16x16x128_f8f6f4 v[92:95], a[216:223], v[200:207], v[92:95] cbsz:1 blgp:1\n\t" \
    "s_add_u32 m0, s44, 8192\n\t" \
    "s_add_u32 s74, s85, 1048576\n\t" \
    "buffer_load_dwordx4 v232, s[8:11], s74 offen  lds\n\t" \
    "ds_read_b64_tr_b8 v[198:199], v239 offset:58368\n\t" \
    "s_waitcnt lgkmcnt(8)\n\t" \
    "v_mfma_f32_16x16x128_f8f6f4 v[96:99], a[192:199], v[208:215], v[96:99] cbsz:1 blgp:1\n\t" \
    "s_add_u32 m0, s44, 12288\n\t" \
    "s_add_u32 s74, s85, 1572864\n\t" \
    "buffer_load_dwordx4 v232, s[8:11], s74 offen  lds\n\t" \
    "v_add_u32 v238, 65536, v225\n\t" \
    "ds_read_b64_tr_b8 v[200:201], v238 offset:49152\n\t" \
    "v_mfma_f32_16x16x128_f8f6f4 v[100:103], a[200:207], v[208:215], v[100:103] cbsz:1 blgp:1\n\t" \
    "ds_read_b64_tr_b8 v[202:203], v238 offset:50176\n\t" \
    "v_mfma_f32_16x16x128_f8f6f4 v[104:107], a[208:215], v[208:215], v[104:107] cbsz:1 blgp:1\n\t" \
    "ds_read_b64_tr_b8 v[204:205], v238 offset:57344\n\t" \
    "v_mfma_f32_16x16x128_f8f6f4 v[108:111], a[216:223], v[208:215], v[108:111] cbsz:1 blgp:1\n\t" \
    "ds_read_b64_tr_b8 v[206:207], v238 offset:58368\n\t" \
    "s_waitcnt lgkmcnt(8)\n\t" \
    "v_mfma_f32_16x16x128_f8f6f4 v[112:115], a[192:199], v[216:223], v[112:115] cbsz:1 blgp:1\n\t" \
    "v_add_u32 v239, 65536, v226\n\t" \
    "ds_read_b64_tr_b8 v[208:209], v239 offset:49152\n\t" \
    "v_mfma_f32_16x16x128_f8f6f4 v[116:119], a[200:207], v[216:223], v[116:119] cbsz:1 blgp:1\n\t" \
    "ds_read_b64_tr_b8 v[210:211], v239 offset:50176\n\t" \
    "v_mfma_f32_16x16x128_f8f6f4 v[120:123], a[208:215], v[216:223], v[120:123] cbsz:1 blgp:1\n\t" \
    "ds_read_b64_tr_b8 v[212:213], v239 offset:57344\n\t" \
    "v_mfma_f32_16x16x128_f8f6f4 v[124:127], a[216:223], v[216:223], v[124:127] cbsz:1 blgp:1\n\t" \
    "ds_read_b64_tr_b8 v[214:215], v239 offset:58368\n\t" \
    "s_waitcnt vmcnt(28)\n\t" \
    "s_waitcnt lgkmcnt(8)\n\t" \
    "v_mfma_f32_16x16x128_f8f6f4 v[0:3], a[224:231], v[192:199], v[0:3] cbsz:1 blgp:1\n\t" \
    "v_add_u32 v238, 65536, v227\n\t" \
    "ds_read_b64_tr_b8 v[216:217], v238 offset:49152\n\t" \
    "v_mfma_f32_16x16x128_f8f6f4 v[4:7], a[232:239], v[192:199], v[4:7] cbsz:1 blgp:1\n\t" \
    "ds_read_b64_tr_b8 v[218:219], v238 offset:50176\n\t" \
    "v_mfma_f32_16x16x128_f8f6f4 v[8:11], a[240:247], v[192:199], v[8:11] cbsz:1 blgp:1\n\t" \
    "ds_read_b64_tr_b8 v[220:221], v238 offset:57344\n\t" \
    "v_mfma_f32_16x16x128_f8f6f4 v[12:15], a[248:255], v[192:199], v[12:15] cbsz:1 blgp:1\n\t" \
    "ds_read_b64_tr_b8 v[222:223], v238 offset:58368\n\t" \
    "s_waitcnt lgkmcnt(8)\n\t" \
    "v_mfma_f32_16x16x128_f8f6f4 v[16:19], a[224:231], v[200:207], v[16:19] cbsz:1 blgp:1\n\t" \
    "v_add_u32 v239, 65536, v228\n\t" \
    "ds_read_b64_tr_b8 v[192:193], v239 offset:49152\n\t" \
    "v_mfma_f32_16x16x128_f8f6f4 v[20:23], a[232:239], v[200:207], v[20:23] cbsz:1 blgp:1\n\t" \
    "ds_read_b64_tr_b8 v[194:195], v239 offset:50176\n\t" \
    "v_mfma_f32_16x16x128_f8f6f4 v[24:27], a[240:247], v[200:207], v[24:27] cbsz:1 blgp:1\n\t" \
    "ds_read_b64_tr_b8 v[196:197], v239 offset:57344\n\t" \
    "v_mfma_f32_16x16x128_f8f6f4 v[28:31], a[248:255], v[200:207], v[28:31] cbsz:1 blgp:1\n\t" \
    "ds_read_b64_tr_b8 v[198:199], v239 offset:58368\n\t" \
    "s_waitcnt lgkmcnt(8)\n\t" \
    "v_mfma_f32_16x16x128_f8f6f4 v[32:35], a[224:231], v[208:215], v[32:35] cbsz:1 blgp:1\n\t" \
    "v_add_u32 v238, 65536, v229\n\t" \
    "ds_read_b64_tr_b8 v[200:201], v238 offset:49152\n\t" \
    "v_mfma_f32_16x16x128_f8f6f4 v[36:39], a[232:239], v[208:215], v[36:39] cbsz:1 blgp:1\n\t" \
    "ds_read_b64_tr_b8 v[202:203], v238 offset:50176\n\t" \
    "v_mfma_f32_16x16x128_f8f6f4 v[40:43], a[240:247], v[208:215], v[40:43] cbsz:1 blgp:1\n\t" \
    "ds_read_b64_tr_b8 v[204:205], v238 offset:57344\n\t" \
    "v_mfma_f32_16x16x128_f8f6f4 v[44:47], a[248:255], v[208:215], v[44:47] cbsz:1 blgp:1\n\t" \
    "ds_read_b64_tr_b8 v[206:207], v238 offset:58368\n\t" \
    "s_waitcnt lgkmcnt(8)\n\t" \
    "v_mfma_f32_16x16x128_f8f6f4 v[48:51], a[224:231], v[216:223], v[48:51] cbsz:1 blgp:1\n\t" \
    "v_add_u32 v239, 65536, v230\n\t" \
    "ds_read_b64_tr_b8 v[208:209], v239 offset:49152\n\t" \
    "v_mfma_f32_16x16x128_f8f6f4 v[52:55], a[232:239], v[216:223], v[52:55] cbsz:1 blgp:1\n\t" \
    "ds_read_b64_tr_b8 v[210:211], v239 offset:50176\n\t" \
    "v_mfma_f32_16x16x128_f8f6f4 v[56:59], a[240:247], v[216:223], v[56:59] cbsz:1 blgp:1\n\t" \
    "ds_read_b64_tr_b8 v[212:213], v239 offset:57344\n\t" \
    "v_mfma_f32_16x16x128_f8f6f4 v[60:63], a[248:255], v[216:223], v[60:63] cbsz:1 blgp:1\n\t" \
    "ds_read_b64_tr_b8 v[214:215], v239 offset:58368\n\t" \
    "s_waitcnt lgkmcnt(8)\n\t" \
    "v_mfma_f32_16x16x128_f8f6f4 v[64:67], a[224:231], v[192:199], v[64:67] cbsz:1 blgp:1\n\t" \
    "v_add_u32 v238, 65536, v231\n\t" \
    "ds_read_b64_tr_b8 v[216:217], v238 offset:49152\n\t" \
    "v_mfma_f32_16x16x128_f8f6f4 v[68:71], a[232:239], v[192:199], v[68:71] cbsz:1 blgp:1\n\t" \
    "ds_read_b64_tr_b8 v[218:219], v238 offset:50176\n\t" \
    "v_mfma_f32_16x16x128_f8f6f4 v[72:75], a[240:247], v[192:199], v[72:75] cbsz:1 blgp:1\n\t" \
    "ds_read_b64_tr_b8 v[220:221], v238 offset:57344\n\t" \
    "v_mfma_f32_16x16x128_f8f6f4 v[76:79], a[248:255], v[192:199], v[76:79] cbsz:1 blgp:1\n\t" \
    "s_cmp_eq_u32 s47, 1\n\t" \
    "s_cselect_b32 s87, s30, s31\n\t" \
    "s_add_u32 s86, s87, 6291456\n\t" \
    "s_add_u32 m0, s46, 0\n\t" \
    "s_add_u32 s74, s86, 0\n\t" \
    "buffer_load_dwordx4 v232, s[8:11], s74 offen  lds\n\t" \
    "ds_read_b64_tr_b8 v[222:223], v238 offset:58368\n\t" \
    "s_waitcnt lgkmcnt(8)\n\t" \
    "v_mfma_f32_16x16x128_f8f6f4 v[80:83], a[224:231], v[200:207], v[80:83] cbsz:1 blgp:1\n\t" \
    "s_barrier\n\t" \
    "v_add_u32 v239, 131072, v224\n\t" \
    "ds_read_b64_tr_b8 v[192:193], v239 offset:0\n\t" \
    "v_mfma_f32_16x16x128_f8f6f4 v[84:87], a[232:239], v[200:207], v[84:87] cbsz:1 blgp:1\n\t" \
    "ds_read_b64_tr_b8 v[194:195], v239 offset:1024\n\t" \
    "v_mfma_f32_16x16x128_f8f6f4 v[88:91], a[240:247], v[200:207], v[88:91] cbsz:1 blgp:1\n\t" \
    "s_add_u32 m0, s46, 4096\n\t" \
    "s_add_u32 s74, s86, 524288\n\t" \
    "buffer_load_dwordx4 v232, s[8:11], s74 offen  lds\n\t" \
    "ds_read_b64_tr_b8 v[196:197], v239 offset:8192\n\t" \
    "v_mfma_f32_16x16x128_f8f6f4 v[92:95], a[248:255], v[200:207], v[92:95] cbsz:1 blgp:1\n\t" \
    "ds_read_b64_tr_b8 v[198:199], v239 offset:9216\n\t" \
    "s_waitcnt lgkmcnt(8)\n\t" \
    "v_mfma_f32_16x16x128_f8f6f4 v[96:99], a[224:231], v[208:215], v[96:99] cbsz:1 blgp:1\n\t" \
    "s_add_u32 m0, s46, 8192\n\t" \
    "s_add_u32 s74, s86, 1048576\n\t" \
    "buffer_load_dwordx4 v232, s[8:11], s74 offen  lds\n\t" \
    "v_add_u32 v238, 131072, v225\n\t" \
    "ds_read_b64_tr_b8 v[200:201], v238 offset:0\n\t" \
    "v_mfma_f32_16x16x128_f8f6f4 v[100:103], a[232:239], v[208:215], v[100:103] cbsz:1 blgp:1\n\t" \
    "ds_read_b64_tr_b8 v[202:203], v238 offset:1024\n\t" \
    "v_mfma_f32_16x16x128_f8f6f4 v[104:107], a[240:247], v[208:215], v[104:107] cbsz:1 blgp:1\n\t" \
    "s_add_u32 m0, s46, 12288\n\t" \
    "s_add_u32 s74, s86, 1572864\n\t" \
    "buffer_load_dwordx4 v232, s[8:11], s74 offen  lds\n\t" \
    "ds_read_b64_tr_b8 v[204:205], v238 offset:8192\n\t" \
    "v_mfma_f32_16x16x128_f8f6f4 v[108:111], a[248:255], v[208:215], v[108:111] cbsz:1 blgp:1\n\t" \
    "ds_read_b64_tr_b8 v[206:207], v238 offset:9216\n\t" \
    "s_waitcnt lgkmcnt(8)\n\t" \
    "v_mfma_f32_16x16x128_f8f6f4 v[112:115], a[224:231], v[216:223], v[112:115] cbsz:1 blgp:1\n\t" \
    "v_add_u32 v239, 131072, v226\n\t" \
    "ds_read_b64_tr_b8 v[208:209], v239 offset:0\n\t" \
    "v_mfma_f32_16x16x128_f8f6f4 v[116:119], a[232:239], v[216:223], v[116:119] cbsz:1 blgp:1\n\t" \
    "s_mov_b32 s72, 2031616\n\t" \
    "s_mov_b32 s73, 2035712\n\t" \
    "buffer_load_dwordx4 a[32:35], v233, s[12:15], s72 offen offset:0 sc0 sc1\n\t" \
    "ds_read_b64_tr_b8 v[210:211], v239 offset:1024\n\t" \
    "v_mfma_f32_16x16x128_f8f6f4 v[120:123], a[240:247], v[216:223], v[120:123] cbsz:1 blgp:1\n\t" \
    "ds_read_b64_tr_b8 v[212:213], v239 offset:8192\n\t" \
    "v_mfma_f32_16x16x128_f8f6f4 v[124:127], a[248:255], v[216:223], v[124:127] cbsz:1 blgp:1\n\t" \
    "buffer_load_dwordx4 a[36:39], v233, s[12:15], s72 offen offset:1024 sc0 sc1\n\t" \
    "ds_read_b64_tr_b8 v[214:215], v239 offset:9216\n\t" \
    "s_waitcnt vmcnt(26)\n\t" \
    "s_waitcnt lgkmcnt(8)\n\t" \
    "v_mfma_f32_16x16x128_f8f6f4 v[0:3], v[128:135], v[192:199], v[0:3] cbsz:1 blgp:1\n\t" \
    "v_add_u32 v238, 131072, v227\n\t" \
    "ds_read_b64_tr_b8 v[216:217], v238 offset:0\n\t" \
    "v_mfma_f32_16x16x128_f8f6f4 v[4:7], v[136:143], v[192:199], v[4:7] cbsz:1 blgp:1\n\t" \
    "buffer_load_dwordx4 a[40:43], v233, s[12:15], s72 offen offset:2048 sc0 sc1\n\t" \
    "ds_read_b64_tr_b8 v[218:219], v238 offset:1024\n\t" \
    "v_mfma_f32_16x16x128_f8f6f4 v[8:11], v[144:151], v[192:199], v[8:11] cbsz:1 blgp:1\n\t" \
    "ds_read_b64_tr_b8 v[220:221], v238 offset:8192\n\t" \
    "v_mfma_f32_16x16x128_f8f6f4 v[12:15], v[152:159], v[192:199], v[12:15] cbsz:1 blgp:1\n\t" \
    "ds_read_b64_tr_b8 v[222:223], v238 offset:9216\n\t" \
    "s_waitcnt lgkmcnt(8)\n\t" \
    "v_mfma_f32_16x16x128_f8f6f4 v[16:19], v[128:135], v[200:207], v[16:19] cbsz:1 blgp:1\n\t" \
    "buffer_load_dwordx4 a[44:47], v233, s[12:15], s72 offen offset:3072 sc0 sc1\n\t" \
    "v_add_u32 v239, 131072, v228\n\t" \
    "ds_read_b64_tr_b8 v[192:193], v239 offset:0\n\t" \
    "v_mfma_f32_16x16x128_f8f6f4 v[20:23], v[136:143], v[200:207], v[20:23] cbsz:1 blgp:1\n\t" \
    "ds_read_b64_tr_b8 v[194:195], v239 offset:1024\n\t" \
    "v_mfma_f32_16x16x128_f8f6f4 v[24:27], v[144:151], v[200:207], v[24:27] cbsz:1 blgp:1\n\t" \
    "buffer_load_dwordx4 a[48:51], v233, s[12:15], s73 offen offset:0 sc0 sc1\n\t" \
    "ds_read_b64_tr_b8 v[196:197], v239 offset:8192\n\t" \
    "v_mfma_f32_16x16x128_f8f6f4 v[28:31], v[152:159], v[200:207], v[28:31] cbsz:1 blgp:1\n\t" \
    "ds_read_b64_tr_b8 v[198:199], v239 offset:9216\n\t" \
    "s_waitcnt lgkmcnt(8)\n\t" \
    "v_mfma_f32_16x16x128_f8f6f4 v[32:35], v[128:135], v[208:215], v[32:35] cbsz:1 blgp:1\n\t" \
    "buffer_load_dwordx4 a[52:55], v233, s[12:15], s73 offen offset:1024 sc0 sc1\n\t" \
    "v_add_u32 v238, 131072, v229\n\t" \
    "ds_read_b64_tr_b8 v[200:201], v238 offset:0\n\t" \
    "v_mfma_f32_16x16x128_f8f6f4 v[36:39], v[136:143], v[208:215], v[36:39] cbsz:1 blgp:1\n\t" \
    "ds_read_b64_tr_b8 v[202:203], v238 offset:1024\n\t" \
    "v_mfma_f32_16x16x128_f8f6f4 v[40:43], v[144:151], v[208:215], v[40:43] cbsz:1 blgp:1\n\t" \
    "ds_read_b64_tr_b8 v[204:205], v238 offset:8192\n\t" \
    "v_mfma_f32_16x16x128_f8f6f4 v[44:47], v[152:159], v[208:215], v[44:47] cbsz:1 blgp:1\n\t" \
    "buffer_load_dwordx4 a[56:59], v233, s[12:15], s73 offen offset:2048 sc0 sc1\n\t" \
    "ds_read_b64_tr_b8 v[206:207], v238 offset:9216\n\t" \
    "s_waitcnt lgkmcnt(8)\n\t" \
    "v_mfma_f32_16x16x128_f8f6f4 v[48:51], v[128:135], v[216:223], v[48:51] cbsz:1 blgp:1\n\t" \
    "v_add_u32 v239, 131072, v230\n\t" \
    "ds_read_b64_tr_b8 v[208:209], v239 offset:0\n\t" \
    "v_mfma_f32_16x16x128_f8f6f4 v[52:55], v[136:143], v[216:223], v[52:55] cbsz:1 blgp:1\n\t" \
    "buffer_load_dwordx4 a[60:63], v233, s[12:15], s73 offen offset:3072 sc0 sc1\n\t" \
    "ds_read_b64_tr_b8 v[210:211], v239 offset:1024\n\t" \
    "v_mfma_f32_16x16x128_f8f6f4 v[56:59], v[144:151], v[216:223], v[56:59] cbsz:1 blgp:1\n\t" \
    "ds_read_b64_tr_b8 v[212:213], v239 offset:8192\n\t" \
    "v_mfma_f32_16x16x128_f8f6f4 v[60:63], v[152:159], v[216:223], v[60:63] cbsz:1 blgp:1\n\t" \
    "ds_read_b64_tr_b8 v[214:215], v239 offset:9216\n\t" \
    "s_waitcnt lgkmcnt(8)\n\t" \
    "v_mfma_f32_16x16x128_f8f6f4 v[64:67], v[128:135], v[192:199], v[64:67] cbsz:1 blgp:1\n\t" \
    "v_add_u32 v238, 131072, v231\n\t" \
    "ds_read_b64_tr_b8 v[216:217], v238 offset:0\n\t" \
    "v_mfma_f32_16x16x128_f8f6f4 v[68:71], v[136:143], v[192:199], v[68:71] cbsz:1 blgp:1\n\t" \
    "ds_read_b64_tr_b8 v[218:219], v238 offset:1024\n\t" \
    "v_mfma_f32_16x16x128_f8f6f4 v[72:75], v[144:151], v[192:199], v[72:75] cbsz:1 blgp:1\n\t" \
    "ds_read_b64_tr_b8 v[220:221], v238 offset:8192\n\t" \
    "v_mfma_f32_16x16x128_f8f6f4 v[76:79], v[152:159], v[192:199], v[76:79] cbsz:1 blgp:1\n\t" \
    "ds_read_b64_tr_b8 v[222:223], v238 offset:9216\n\t" \
    "s_waitcnt lgkmcnt(8)\n\t" \
    "v_mfma_f32_16x16x128_f8f6f4 v[80:83], v[128:135], v[200:207], v[80:83] cbsz:1 blgp:1\n\t" \
    "s_barrier\n\t" \
    "v_add_u32 v239, 131072, v224\n\t" \
    "ds_read_b64_tr_b8 v[192:193], v239 offset:16384\n\t" \
    "v_mfma_f32_16x16x128_f8f6f4 v[84:87], v[136:143], v[200:207], v[84:87] cbsz:1 blgp:1\n\t" \
    "ds_read_b64_tr_b8 v[194:195], v239 offset:17408\n\t" \
    "v_mfma_f32_16x16x128_f8f6f4 v[88:91], v[144:151], v[200:207], v[88:91] cbsz:1 blgp:1\n\t" \
    "ds_read_b64_tr_b8 v[196:197], v239 offset:24576\n\t" \
    "v_mfma_f32_16x16x128_f8f6f4 v[92:95], v[152:159], v[200:207], v[92:95] cbsz:1 blgp:1\n\t" \
    "ds_read_b64_tr_b8 v[198:199], v239 offset:25600\n\t" \
    "s_waitcnt lgkmcnt(8)\n\t" \
    "v_mfma_f32_16x16x128_f8f6f4 v[96:99], v[128:135], v[208:215], v[96:99] cbsz:1 blgp:1\n\t" \
    "v_add_u32 v238, 131072, v225\n\t" \
    "ds_read_b64_tr_b8 v[200:201], v238 offset:16384\n\t" \
    "v_mfma_f32_16x16x128_f8f6f4 v[100:103], v[136:143], v[208:215], v[100:103] cbsz:1 blgp:1\n\t" \
    "ds_read_b64_tr_b8 v[202:203], v238 offset:17408\n\t" \
    "v_mfma_f32_16x16x128_f8f6f4 v[104:107], v[144:151], v[208:215], v[104:107] cbsz:1 blgp:1\n\t" \
    "ds_read_b64_tr_b8 v[204:205], v238 offset:24576\n\t" \
    "v_mfma_f32_16x16x128_f8f6f4 v[108:111], v[152:159], v[208:215], v[108:111] cbsz:1 blgp:1\n\t" \
    "ds_read_b64_tr_b8 v[206:207], v238 offset:25600\n\t" \
    "s_waitcnt lgkmcnt(8)\n\t" \
    "v_mfma_f32_16x16x128_f8f6f4 v[112:115], v[128:135], v[216:223], v[112:115] cbsz:1 blgp:1\n\t" \
    "v_add_u32 v239, 131072, v226\n\t" \
    "ds_read_b64_tr_b8 v[208:209], v239 offset:16384\n\t" \
    "v_mfma_f32_16x16x128_f8f6f4 v[116:119], v[136:143], v[216:223], v[116:119] cbsz:1 blgp:1\n\t" \
    "ds_read_b64_tr_b8 v[210:211], v239 offset:17408\n\t" \
    "v_mfma_f32_16x16x128_f8f6f4 v[120:123], v[144:151], v[216:223], v[120:123] cbsz:1 blgp:1\n\t" \
    "ds_read_b64_tr_b8 v[212:213], v239 offset:24576\n\t" \
    "v_mfma_f32_16x16x128_f8f6f4 v[124:127], v[152:159], v[216:223], v[124:127] cbsz:1 blgp:1\n\t" \
    "ds_read_b64_tr_b8 v[214:215], v239 offset:25600\n\t" \
    "s_waitcnt vmcnt(24)\n\t" \
    "s_waitcnt lgkmcnt(8)\n\t" \
    "v_mfma_f32_16x16x128_f8f6f4 v[0:3], v[160:167], v[192:199], v[0:3] cbsz:1 blgp:1\n\t" \
    "v_add_u32 v238, 131072, v227\n\t" \
    "ds_read_b64_tr_b8 v[216:217], v238 offset:16384\n\t" \
    "v_mfma_f32_16x16x128_f8f6f4 v[4:7], v[168:175], v[192:199], v[4:7] cbsz:1 blgp:1\n\t" \
    "ds_read_b64_tr_b8 v[218:219], v238 offset:17408\n\t" \
    "v_mfma_f32_16x16x128_f8f6f4 v[8:11], v[176:183], v[192:199], v[8:11] cbsz:1 blgp:1\n\t" \
    "ds_read_b64_tr_b8 v[220:221], v238 offset:24576\n\t" \
    "v_mfma_f32_16x16x128_f8f6f4 v[12:15], v[184:191], v[192:199], v[12:15] cbsz:1 blgp:1\n\t" \
    "ds_read_b64_tr_b8 v[222:223], v238 offset:25600\n\t" \
    "s_waitcnt lgkmcnt(8)\n\t" \
    "v_mfma_f32_16x16x128_f8f6f4 v[16:19], v[160:167], v[200:207], v[16:19] cbsz:1 blgp:1\n\t" \
    "v_add_u32 v239, 131072, v228\n\t" \
    "ds_read_b64_tr_b8 v[192:193], v239 offset:16384\n\t" \
    "v_mfma_f32_16x16x128_f8f6f4 v[20:23], v[168:175], v[200:207], v[20:23] cbsz:1 blgp:1\n\t" \
    "ds_read_b64_tr_b8 v[194:195], v239 offset:17408\n\t" \
    "v_mfma_f32_16x16x128_f8f6f4 v[24:27], v[176:183], v[200:207], v[24:27] cbsz:1 blgp:1\n\t" \
    "ds_read_b64_tr_b8 v[196:197], v239 offset:24576\n\t" \
    "v_mfma_f32_16x16x128_f8f6f4 v[28:31], v[184:191], v[200:207], v[28:31] cbsz:1 blgp:1\n\t" \
    "ds_read_b64_tr_b8 v[198:199], v239 offset:25600\n\t" \
    "s_waitcnt lgkmcnt(8)\n\t" \
    "v_mfma_f32_16x16x128_f8f6f4 v[32:35], v[160:167], v[208:215], v[32:35] cbsz:1 blgp:1\n\t" \
    "v_add_u32 v238, 131072, v229\n\t" \
    "ds_read_b64_tr_b8 v[200:201], v238 offset:16384\n\t" \
    "v_mfma_f32_16x16x128_f8f6f4 v[36:39], v[168:175], v[208:215], v[36:39] cbsz:1 blgp:1\n\t" \
    "ds_read_b64_tr_b8 v[202:203], v238 offset:17408\n\t" \
    "v_mfma_f32_16x16x128_f8f6f4 v[40:43], v[176:183], v[208:215], v[40:43] cbsz:1 blgp:1\n\t" \
    "ds_read_b64_tr_b8 v[204:205], v238 offset:24576\n\t" \
    "v_mfma_f32_16x16x128_f8f6f4 v[44:47], v[184:191], v[208:215], v[44:47] cbsz:1 blgp:1\n\t" \
    "ds_read_b64_tr_b8 v[206:207], v238 offset:25600\n\t" \
    "s_waitcnt lgkmcnt(8)\n\t" \
    "v_mfma_f32_16x16x128_f8f6f4 v[48:51], v[160:167], v[216:223], v[48:51] cbsz:1 blgp:1\n\t" \
    "v_add_u32 v239, 131072, v230\n\t" \
    "ds_read_b64_tr_b8 v[208:209], v239 offset:16384\n\t" \
    "v_mfma_f32_16x16x128_f8f6f4 v[52:55], v[168:175], v[216:223], v[52:55] cbsz:1 blgp:1\n\t" \
    "ds_read_b64_tr_b8 v[210:211], v239 offset:17408\n\t" \
    "v_mfma_f32_16x16x128_f8f6f4 v[56:59], v[176:183], v[216:223], v[56:59] cbsz:1 blgp:1\n\t" \
    "ds_read_b64_tr_b8 v[212:213], v239 offset:24576\n\t" \
    "v_mfma_f32_16x16x128_f8f6f4 v[60:63], v[184:191], v[216:223], v[60:63] cbsz:1 blgp:1\n\t" \
    "ds_read_b64_tr_b8 v[214:215], v239 offset:25600\n\t" \
    "s_waitcnt lgkmcnt(8)\n\t" \
    "v_mfma_f32_16x16x128_f8f6f4 v[64:67], v[160:167], v[192:199], v[64:67] cbsz:1 blgp:1\n\t" \
    "v_add_u32 v238, 131072, v231\n\t" \
    "ds_read_b64_tr_b8 v[216:217], v238 offset:16384\n\t" \
    "v_mfma_f32_16x16x128_f8f6f4 v[68:71], v[168:175], v[192:199], v[68:71] cbsz:1 blgp:1\n\t" \
    "ds_read_b64_tr_b8 v[218:219], v238 offset:17408\n\t" \
    "v_mfma_f32_16x16x128_f8f6f4 v[72:75], v[176:183], v[192:199], v[72:75] cbsz:1 blgp:1\n\t" \
    "ds_read_b64_tr_b8 v[220:221], v238 offset:24576\n\t" \
    "v_mfma_f32_16x16x128_f8f6f4 v[76:79], v[184:191], v[192:199], v[76:79] cbsz:1 blgp:1\n\t" \
    "ds_read_b64_tr_b8 v[222:223], v238 offset:25600\n\t" \
    "s_waitcnt lgkmcnt(8)\n\t" \
    "v_mfma_f32_16x16x128_f8f6f4 v[80:83], v[160:167], v[200:207], v[80:83] cbsz:1 blgp:1\n\t" \
    "s_waitcnt vmcnt(12)\n\t" \
    "s_waitcnt vmcnt(8)\n\t" \
    "s_barrier\n\t" \
    "ds_read_b64_tr_b8 v[192:193], v224 offset:0\n\t" \
    "v_mfma_f32_16x16x128_f8f6f4 v[84:87], v[168:175], v[200:207], v[84:87] cbsz:1 blgp:1\n\t" \
    "ds_read_b64_tr_b8 v[194:195], v224 offset:1024\n\t" \
    "v_mfma_f32_16x16x128_f8f6f4 v[88:91], v[176:183], v[200:207], v[88:91] cbsz:1 blgp:1\n\t" \
    "ds_read_b64_tr_b8 v[196:197], v224 offset:8192\n\t" \
    "v_mfma_f32_16x16x128_f8f6f4 v[92:95], v[184:191], v[200:207], v[92:95] cbsz:1 blgp:1\n\t" \
    "ds_read_b64_tr_b8 v[198:199], v224 offset:9216\n\t" \
    "s_waitcnt lgkmcnt(8)\n\t" \
    "v_mfma_f32_16x16x128_f8f6f4 v[96:99], v[160:167], v[208:215], v[96:99] cbsz:1 blgp:1\n\t" \
    "ds_read_b64_tr_b8 v[200:201], v225 offset:0\n\t" \
    "v_mfma_f32_16x16x128_f8f6f4 v[100:103], v[168:175], v[208:215], v[100:103] cbsz:1 blgp:1\n\t" \
    "ds_read_b64_tr_b8 v[202:203], v225 offset:1024\n\t" \
    "v_mfma_f32_16x16x128_f8f6f4 v[104:107], v[176:183], v[208:215], v[104:107] cbsz:1 blgp:1\n\t" \
    "ds_read_b64_tr_b8 v[204:205], v225 offset:8192\n\t" \
    "v_mfma_f32_16x16x128_f8f6f4 v[108:111], v[184:191], v[208:215], v[108:111] cbsz:1 blgp:1\n\t" \
    "ds_read_b64_tr_b8 v[206:207], v225 offset:9216\n\t" \
    "s_waitcnt lgkmcnt(8)\n\t" \
    "v_mfma_f32_16x16x128_f8f6f4 v[112:115], v[160:167], v[216:223], v[112:115] cbsz:1 blgp:1\n\t" \
    "ds_read_b64_tr_b8 v[208:209], v226 offset:0\n\t" \
    "v_mfma_f32_16x16x128_f8f6f4 v[116:119], v[168:175], v[216:223], v[116:119] cbsz:1 blgp:1\n\t" \
    "ds_read_b64_tr_b8 v[210:211], v226 offset:1024\n\t" \
    "v_mfma_f32_16x16x128_f8f6f4 v[120:123], v[176:183], v[216:223], v[120:123] cbsz:1 blgp:1\n\t" \
    "ds_read_b64_tr_b8 v[212:213], v226 offset:8192\n\t" \
    "v_mfma_f32_16x16x128_f8f6f4 v[124:127], v[184:191], v[216:223], v[124:127] cbsz:1 blgp:1\n\t" \
    "ds_read_b64_tr_b8 v[214:215], v226 offset:9216\n\t" \
    "s_waitcnt lgkmcnt(8)\n\t" \
    "v_mfma_f32_16x16x128_f8f6f4 v[0:3], a[0:7], v[192:199], v[0:3] cbsz:1 blgp:1\n\t" \
    "ds_read_b64_tr_b8 v[216:217], v227 offset:0\n\t" \
    "v_mfma_f32_16x16x128_f8f6f4 v[4:7], a[8:15], v[192:199], v[4:7] cbsz:1 blgp:1\n\t" \
    "ds_read_b64_tr_b8 v[218:219], v227 offset:1024\n\t" \
    "v_readfirstlane_b32 s84, v250\n\t" \
    "s_nop 3\n\t" \
    "s_cmp_eq_u32 s84, s75\n\t" \
    "s_cselect_b32 s76, 0x7fc00000, s76\n\t" \
    "s_cselect_b32 s78, 0x7fc00000, s78\n\t" \
    "v_mfma_f32_16x16x128_f8f6f4 v[8:11], a[16:23], v[192:199], v[8:11] cbsz:1 blgp:1\n\t" \
    "ds_read_b64_tr_b8 v[220:221], v227 offset:8192\n\t" \
    "s_mov_b32 s90, s77\n\t" \
    "s_mov_b32 s91, s77\n\t" \
    "s_mov_b32 s92, s76\n\t" \
    "s_mov_b32 s93, s76\n\t" \
    "s_mov_b32 s94, s78\n\t" \
    "s_mov_b32 s95, s78\n\t" \
    "s_mov_b32 s90, s71\n\t" \
    "s_mov_b32 s91, s71\n\t" \
    "s_mov_b32 s64, 0.5\n\t" \
    "s_mov_b32 s65, 0.5\n\t" \
    "s_mov_b32 s66, 0x432d1f97\n\t" \
    "s_mov_b32 s67, 0x432d1f97\n\t" \
    "s_mov_b32 s68, 0xc2ad1f97\n\t" \
    "s_mov_b32 s69, 0xc2ad1f97\n\t" \
    "v_mfma_f32_16x16x128_f8f6f4 v[12:15], a[24:31], v[192:199], v[12:15] cbsz:1 blgp:1\n\t" \
    "ds_read_b64_tr_b8 v[222:223], v227 offset:9216\n\t" \
    "s_waitcnt lgkmcnt(8)\n\t" \
    "v_mfma_f32_16x16x128_f8f6f4 v[16:19], a[0:7], v[200:207], v[16:19] cbsz:1 blgp:1\n\t" \
    "ds_read_b64_tr_b8 v[192:193], v228 offset:0\n\t" \
    "v_mfma_f32_16x16x128_f8f6f4 v[20:23], a[8:15], v[200:207], v[20:23] cbsz:1 blgp:1\n\t" \
    "ds_read_b64_tr_b8 v[194:195], v228 offset:1024\n\t" \
    "v_mfma_f32_16x16x128_f8f6f4 v[24:27], a[16:23], v[200:207], v[24:27] cbsz:1 blgp:1\n\t" \
    "ds_read_b64_tr_b8 v[196:197], v228 offset:8192\n\t" \
    "v_mfma_f32_16x16x128_f8f6f4 v[28:31], a[24:31], v[200:207], v[28:31] cbsz:1 blgp:1\n\t" \
    "ds_read_b64_tr_b8 v[198:199], v228 offset:9216\n\t" \
    "s_waitcnt lgkmcnt(8)\n\t" \
    "v_mfma_f32_16x16x128_f8f6f4 v[32:35], a[0:7], v[208:215], v[32:35] cbsz:1 blgp:1\n\t" \
    "ds_read_b64_tr_b8 v[200:201], v229 offset:0\n\t" \
    "v_mfma_f32_16x16x128_f8f6f4 v[36:39], a[8:15], v[208:215], v[36:39] cbsz:1 blgp:1\n\t" \
    "ds_read_b64_tr_b8 v[202:203], v229 offset:1024\n\t" \
    "v_mfma_f32_16x16x128_f8f6f4 v[40:43], a[16:23], v[208:215], v[40:43] cbsz:1 blgp:1\n\t" \
    "ds_read_b64_tr_b8 v[204:205], v229 offset:8192\n\t" \
    "v_mfma_f32_16x16x128_f8f6f4 v[44:47], a[24:31], v[208:215], v[44:47] cbsz:1 blgp:1\n\t" \
    "ds_read_b64_tr_b8 v[206:207], v229 offset:9216\n\t" \
    "s_waitcnt lgkmcnt(8)\n\t" \
    "v_mfma_f32_16x16x128_f8f6f4 v[48:51], a[0:7], v[216:223], v[48:51] cbsz:1 blgp:1\n\t" \
    "ds_read_b64_tr_b8 v[208:209], v230 offset:0\n\t" \
    "v_mfma_f32_16x16x128_f8f6f4 v[52:55], a[8:15], v[216:223], v[52:55] cbsz:1 blgp:1\n\t" \
    "ds_read_b64_tr_b8 v[210:211], v230 offset:1024\n\t" \
    "v_mfma_f32_16x16x128_f8f6f4 v[56:59], a[16:23], v[216:223], v[56:59] cbsz:1 blgp:1\n\t" \
    "ds_read_b64_tr_b8 v[212:213], v230 offset:8192\n\t" \
    "v_mfma_f32_16x16x128_f8f6f4 v[60:63], a[24:31], v[216:223], v[60:63] cbsz:1 blgp:1\n\t" \
    "ds_read_b64_tr_b8 v[214:215], v230 offset:9216\n\t" \
    "s_waitcnt lgkmcnt(8)\n\t" \
    "v_mfma_f32_16x16x128_f8f6f4 v[64:67], a[0:7], v[192:199], v[64:67] cbsz:1 blgp:1\n\t" \
    "ds_read_b64_tr_b8 v[216:217], v231 offset:0\n\t" \
    "v_mfma_f32_16x16x128_f8f6f4 v[68:71], a[8:15], v[192:199], v[68:71] cbsz:1 blgp:1\n\t" \
    "ds_read_b64_tr_b8 v[218:219], v231 offset:1024\n\t" \
    "v_mfma_f32_16x16x128_f8f6f4 v[72:75], a[16:23], v[192:199], v[72:75] cbsz:1 blgp:1\n\t" \
    "ds_read_b64_tr_b8 v[220:221], v231 offset:8192\n\t" \
    "v_mfma_f32_16x16x128_f8f6f4 v[76:79], a[24:31], v[192:199], v[76:79] cbsz:1 blgp:1\n\t" \
    "ds_read_b64_tr_b8 v[222:223], v231 offset:9216\n\t" \
    "s_waitcnt lgkmcnt(8)\n\t" \
    "v_mfma_f32_16x16x128_f8f6f4 v[80:83], a[0:7], v[200:207], v[80:83] cbsz:1 blgp:1\n\t" \
    "s_barrier\n\t" \
    "ds_read_b64_tr_b8 v[192:193], v224 offset:16384\n\t" \
    "v_mfma_f32_16x16x128_f8f6f4 v[84:87], a[8:15], v[200:207], v[84:87] cbsz:1 blgp:1\n\t" \
    "ds_read_b64_tr_b8 v[194:195], v224 offset:17408\n\t" \
    "v_mfma_f32_16x16x128_f8f6f4 v[88:91], a[16:23], v[200:207], v[88:91] cbsz:1 blgp:1\n\t" \
    "ds_read_b64_tr_b8 v[196:197], v224 offset:24576\n\t" \
    "v_mfma_f32_16x16x128_f8f6f4 v[92:95], a[24:31], v[200:207], v[92:95] cbsz:1 blgp:1\n\t" \
    "ds_read_b64_tr_b8 v[198:199], v224 offset:25600\n\t" \
    "s_waitcnt lgkmcnt(8)\n\t" \
    "v_mfma_f32_16x16x128_f8f6f4 v[96:99], a[0:7], v[208:215], v[96:99] cbsz:1 blgp:1\n\t" \
    "ds_read_b64_tr_b8 v[200:201], v225 offset:16384\n\t" \
    "v_mfma_f32_16x16x128_f8f6f4 v[100:103], a[8:15], v[208:215], v[100:103] cbsz:1 blgp:1\n\t" \
    "ds_read_b64_tr_b8 v[202:203], v225 offset:17408\n\t" \
    "v_mfma_f32_16x16x128_f8f6f4 v[104:107], a[16:23], v[208:215], v[104:107] cbsz:1 blgp:1\n\t" \
    "ds_read_b64_tr_b8 v[204:205], v225 offset:24576\n\t" \
    "v_mfma_f32_16x16x128_f8f6f4 v[108:111], a[24:31], v[208:215], v[108:111] cbsz:1 blgp:1\n\t" \
    "ds_read_b64_tr_b8 v[206:207], v225 offset:25600\n\t" \
    "s_waitcnt lgkmcnt(8)\n\t" \
    "v_mfma_f32_16x16x128_f8f6f4 v[112:115], a[0:7], v[216:223], v[112:115] cbsz:1 blgp:1\n\t" \
    "ds_read_b64_tr_b8 v[208:209], v226 offset:16384\n\t" \
    "v_mfma_f32_16x16x128_f8f6f4 v[116:119], a[8:15], v[216:223], v[116:119] cbsz:1 blgp:1\n\t" \
    "ds_read_b64_tr_b8 v[210:211], v226 offset:17408\n\t" \
    "v_mfma_f32_16x16x128_f8f6f4 v[120:123], a[16:23], v[216:223], v[120:123] cbsz:1 blgp:1\n\t" \
    "ds_read_b64_tr_b8 v[212:213], v226 offset:24576\n\t" \
    "v_mfma_f32_16x16x128_f8f6f4 v[124:127], a[24:31], v[216:223], v[124:127] cbsz:1 blgp:1\n\t" \
    "ds_read_b64_tr_b8 v[214:215], v226 offset:25600\n\t" \
    "s_waitcnt vmcnt(0)\n\t" \
    "s_waitcnt lgkmcnt(8)\n\t" \
    "v_mfma_f32_16x16x128_f8f6f4 v[0:3], a[32:39], v[192:199], v[0:3] cbsz:1 blgp:1\n\t" \
    "ds_read_b64_tr_b8 v[216:217], v227 offset:16384\n\t" \
    "v_mfma_f32_16x16x128_f8f6f4 v[4:7], a[40:47], v[192:199], v[4:7] cbsz:1 blgp:1\n\t" \
    "ds_read_b64_tr_b8 v[218:219], v227 offset:17408\n\t" \
    "v_mfma_f32_16x16x128_f8f6f4 v[8:11], a[48:55], v[192:199], v[8:11] cbsz:1 blgp:1\n\t" \
    "ds_read_b64_tr_b8 v[220:221], v227 offset:24576\n\t" \
    "v_mfma_f32_16x16x128_f8f6f4 v[12:15], a[56:63], v[192:199], v[12:15] cbsz:1 blgp:1\n\t" \
    "ds_read_b64_tr_b8 v[222:223], v227 offset:25600\n\t" \
    "s_waitcnt lgkmcnt(8)\n\t" \
    "v_mfma_f32_16x16x128_f8f6f4 v[16:19], a[32:39], v[200:207], v[16:19] cbsz:1 blgp:1\n\t" \
    "ds_read_b64_tr_b8 v[192:193], v228 offset:16384\n\t" \
    "v_mfma_f32_16x16x128_f8f6f4 v[20:23], a[40:47], v[200:207], v[20:23] cbsz:1 blgp:1\n\t" \
    "ds_read_b64_tr_b8 v[194:195], v228 offset:17408\n\t" \
    "v_mfma_f32_16x16x128_f8f6f4 v[24:27], a[48:55], v[200:207], v[24:27] cbsz:1 blgp:1\n\t" \
    "ds_read_b64_tr_b8 v[196:197], v228 offset:24576\n\t" \
    "v_mfma_f32_16x16x128_f8f6f4 v[28:31], a[56:63], v[200:207], v[28:31] cbsz:1 blgp:1\n\t" \
    "ds_read_b64_tr_b8 v[198:199], v228 offset:25600\n\t" \
    "s_waitcnt lgkmcnt(8)\n\t" \
    "v_mfma_f32_16x16x128_f8f6f4 v[32:35], a[32:39], v[208:215], v[32:35] cbsz:1 blgp:1\n\t" \
    "ds_read_b64_tr_b8 v[200:201], v229 offset:16384\n\t" \
    "v_mfma_f32_16x16x128_f8f6f4 v[36:39], a[40:47], v[208:215], v[36:39] cbsz:1 blgp:1\n\t" \
    "ds_read_b64_tr_b8 v[202:203], v229 offset:17408\n\t" \
    "v_mfma_f32_16x16x128_f8f6f4 v[40:43], a[48:55], v[208:215], v[40:43] cbsz:1 blgp:1\n\t" \
    "ds_read_b64_tr_b8 v[204:205], v229 offset:24576\n\t" \
    "v_mfma_f32_16x16x128_f8f6f4 v[44:47], a[56:63], v[208:215], v[44:47] cbsz:1 blgp:1\n\t" \
    "ds_read_b64_tr_b8 v[206:207], v229 offset:25600\n\t" \
    "s_waitcnt lgkmcnt(8)\n\t" \
    "v_mfma_f32_16x16x128_f8f6f4 v[48:51], a[32:39], v[216:223], v[48:51] cbsz:1 blgp:1\n\t" \
    "ds_read_b64_tr_b8 v[208:209], v230 offset:16384\n\t" \
    "v_mfma_f32_16x16x128_f8f6f4 v[52:55], a[40:47], v[216:223], v[52:55] cbsz:1 blgp:1\n\t" \
    "ds_read_b64_tr_b8 v[210:211], v230 offset:17408\n\t" \
    "v_mfma_f32_16x16x128_f8f6f4 v[56:59], a[48:55], v[216:223], v[56:59] cbsz:1 blgp:1\n\t" \
    "ds_read_b64_tr_b8 v[212:213], v230 offset:24576\n\t" \
    "v_mfma_f32_16x16x128_f8f6f4 v[60:63], a[56:63], v[216:223], v[60:63] cbsz:1 blgp:1\n\t" \
    "ds_read_b64_tr_b8 v[214:215], v230 offset:25600\n\t" \
    "s_waitcnt lgkmcnt(8)\n\t" \
    "v_mfma_f32_16x16x128_f8f6f4 v[64:67], a[32:39], v[192:199], v[64:67] cbsz:1 blgp:1\n\t" \
    "ds_read_b64_tr_b8 v[216:217], v231 offset:16384\n\t" \
    "v_mfma_f32_16x16x128_f8f6f4 v[68:71], a[40:47], v[192:199], v[68:71] cbsz:1 blgp:1\n\t" \
    "ds_read_b64_tr_b8 v[218:219], v231 offset:17408\n\t" \
    "v_mfma_f32_16x16x128_f8f6f4 v[72:75], a[48:55], v[192:199], v[72:75] cbsz:1 blgp:1\n\t" \
    "ds_read_b64_tr_b8 v[220:221], v231 offset:24576\n\t" \
    "v_mfma_f32_16x16x128_f8f6f4 v[76:79], a[56:63], v[192:199], v[76:79] cbsz:1 blgp:1\n\t" \
    "ds_read_b64_tr_b8 v[222:223], v231 offset:25600\n\t" \
    "s_waitcnt lgkmcnt(8)\n\t" \
    "v_mfma_f32_16x16x128_f8f6f4 v[80:83], a[32:39], v[200:207], v[80:83] cbsz:1 blgp:1\n\t" \
    "v_mfma_f32_16x16x128_f8f6f4 v[84:87], a[40:47], v[200:207], v[84:87] cbsz:1 blgp:1\n\t" \
    "v_mfma_f32_16x16x128_f8f6f4 v[88:91], a[48:55], v[200:207], v[88:91] cbsz:1 blgp:1\n\t" \
    "v_mfma_f32_16x16x128_f8f6f4 v[92:95], a[56:63], v[200:207], v[92:95] cbsz:1 blgp:1\n\t" \
    "s_waitcnt lgkmcnt(4)\n\t" \
    "v_mfma_f32_16x16x128_f8f6f4 v[96:99], a[32:39], v[208:215], v[96:99] cbsz:1 blgp:1\n\t" \
    "v_mfma_f32_16x16x128_f8f6f4 v[100:103], a[40:47], v[208:215], v[100:103] cbsz:1 blgp:1\n\t" \
    "v_mfma_f32_16x16x128_f8f6f4 v[104:107], a[48:55], v[208:215], v[104:107] cbsz:1 blgp:1\n\t" \
    "v_mfma_f32_16x16x128_f8f6f4 v[108:111], a[56:63], v[208:215], v[108:111] cbsz:1 blgp:1\n\t" \
    "s_waitcnt lgkmcnt(0)\n\t" \
    "v_mfma_f32_16x16x128_f8f6f4 v[112:115], a[32:39], v[216:223], v[112:115] cbsz:1 blgp:1\n\t" \
    "v_mfma_f32_16x16x128_f8f6f4 v[116:119], a[40:47], v[216:223], v[116:119] cbsz:1 blgp:1\n\t" \
    "v_mfma_f32_16x16x128_f8f6f4 v[120:123], a[48:55], v[216:223], v[120:123] cbsz:1 blgp:1\n\t" \
    "v_mfma_f32_16x16x128_f8f6f4 v[124:127], a[56:63], v[216:223], v[124:127] cbsz:1 blgp:1\n\t" \
    "s_nop 15\n\t" \
    "v_mov_b32 v224, 0.5\n\t" \
    "v_mov_b32 v225, 0.5\n\t" \
    "v_mov_b32 v226, s68\n\t" \
    "v_mov_b32 v227, s68\n\t" \
    "v_lshrrev_b32 v248, 4, v237\n\t" \
    "v_and_b32 v249, 31, v248\n\t" \
    "v_lshrrev_b32 v250, 5, v248\n\t" \
    "v_lshlrev_b32 v250, 19, v250\n\t" \
    "v_lshl_add_u32 v246, v249, 2, v250\n\t" \
    "v_pk_fma_f32 v[192:193], v[0:1], s[90:91], v[224:225] clamp\n\t" \
    "v_pk_fma_f32 v[194:195], v[2:3], s[90:91], v[224:225] clamp\n\t" \
    "v_pk_fma_f32 v[196:197], v[16:17], s[90:91], v[224:225] clamp\n\t" \
    "v_pk_fma_f32 v[198:199], v[18:19], s[90:91], v[224:225] clamp\n\t" \
    "v_pk_fma_f32 v[200:201], v[32:33], s[90:91], v[224:225] clamp\n\t" \
    "v_pk_fma_f32 v[202:203], v[34:35], s[90:91], v[224:225] clamp\n\t" \
    "v_pk_fma_f32 v[204:205], v[48:49], s[90:91], v[224:225] clamp\n\t" \
    "v_pk_fma_f32 v[206:207], v[50:51], s[90:91], v[224:225] clamp\n\t" \
    "v_pk_fma_f32 v[192:193], v[192:193], s[66:67], v[226:227]\n\t" \
    "v_pk_fma_f32 v[194:195], v[194:195], s[66:67], v[226:227]\n\t" \
    "v_pk_fma_f32 v[196:197], v[196:197], s[66:67], v[226:227]\n\t" \
    "v_pk_fma_f32 v[198:199], v[198:199], s[66:67], v[226:227]\n\t" \
    "v_pk_fma_f32 v[200:201], v[200:201], s[66:67], v[226:227]\n\t" \
    "v_pk_fma_f32 v[202:203], v[202:203], s[66:67], v[226:227]\n\t" \
    "v_pk_fma_f32 v[204:205], v[204:205], s[66:67], v[226:227]\n\t" \
    "v_pk_fma_f32 v[206:207], v[206:207], s[66:67], v[226:227]\n\t" \
    "v_exp_f32 v192, v192\n\t" \
    "v_exp_f32 v193, v193\n\t" \
    "v_exp_f32 v194, v194\n\t" \
    "v_exp_f32 v195, v195\n\t" \
    "v_exp_f32 v196, v196\n\t" \
    "v_exp_f32 v197, v197\n\t" \
    "v_exp_f32 v198, v198\n\t" \
    "v_exp_f32 v199, v199\n\t" \
    "v_exp_f32 v200, v200\n\t" \
    "v_exp_f32 v201, v201\n\t" \
    "v_exp_f32 v202, v202\n\t" \
    "v_exp_f32 v203, v203\n\t" \
    "v_exp_f32 v204, v204\n\t" \
    "v_exp_f32 v205, v205\n\t" \
    "v_exp_f32 v206, v206\n\t" \
    "v_exp_f32 v207, v207\n\t" \
    "v_pk_fma_f32 v[192:193], v[192:193], s[94:95], s[94:95]\n\t" \
    "v_pk_fma_f32 v[194:195], v[194:195], s[94:95], s[94:95]\n\t" \
    "v_pk_fma_f32 v[196:197], v[196:197], s[94:95], s[94:95]\n\t" \
    "v_pk_fma_f32 v[198:199], v[198:199], s[94:95], s[94:95]\n\t" \
    "v_pk_fma_f32 v[200:201], v[200:201], s[94:95], s[94:95]\n\t" \
    "v_pk_fma_f32 v[202:203], v[202:203], s[94:95], s[94:95]\n\t" \
    "v_pk_fma_f32 v[204:205], v[204:205], s[94:95], s[94:95]\n\t" \
    "v_pk_fma_f32 v[206:207], v[206:207], s[94:95], s[94:95]\n\t" \
    "v_rcp_f32 v192, v192\n\t" \
    "v_rcp_f32 v193, v193\n\t" \
    "v_rcp_f32 v194, v194\n\t" \
    "v_rcp_f32 v195, v195\n\t" \
    "v_rcp_f32 v196, v196\n\t" \
    "v_rcp_f32 v197, v197\n\t" \
    "v_rcp_f32 v198, v198\n\t" \
    "v_rcp_f32 v199, v199\n\t" \
    "v_rcp_f32 v200, v200\n\t" \
    "v_rcp_f32 v201, v201\n\t" \
    "v_rcp_f32 v202, v202\n\t" \
    "v_rcp_f32 v203, v203\n\t" \
    "v_rcp_f32 v204, v204\n\t" \
    "v_rcp_f32 v205, v205\n\t" \
    "v_rcp_f32 v206, v206\n\t" \
    "v_rcp_f32 v207, v207\n\t" \
    "v_pk_mul_f32 v[0:1], v[0:1], v[192:193]\n\t" \
    "v_pk_mul_f32 v[2:3], v[2:3], v[194:195]\n\t" \
    "v_pk_mul_f32 v[16:17], v[16:17], v[196:197]\n\t" \
    "v_pk_mul_f32 v[18:19], v[18:19], v[198:199]\n\t" \
    "v_pk_mul_f32 v[32:33], v[32:33], v[200:201]\n\t" \
    "v_pk_mul_f32 v[34:35], v[34:35], v[202:203]\n\t" \
    "v_pk_mul_f32 v[48:49], v[48:49], v[204:205]\n\t" \
    "v_pk_mul_f32 v[50:51], v[50:51], v[206:207]\n\t" \
    "v_pk_fma_f32 v[208:209], v[64:65], s[90:91], v[224:225] clamp\n\t" \
    "v_pk_fma_f32 v[210:211], v[66:67], s[90:91], v[224:225] clamp\n\t" \
    "v_pk_fma_f32 v[212:213], v[80:81], s[90:91], v[224:225] clamp\n\t" \
    "v_permlane16_swap_b32 v0, v16\n\t" \
    "v_pk_fma_f32 v[214:215], v[82:83], s[90:91], v[224:225] clamp\n\t" \
    "v_pk_fma_f32 v[216:217], v[96:97], s[90:91], v[224:225] clamp\n\t" \
    "v_pk_fma_f32 v[218:219], v[98:99], s[90:91], v[224:225] clamp\n\t" \
    "v_permlane16_swap_b32 v1, v17\n\t" \
    "v_pk_fma_f32 v[220:221], v[112:113], s[90:91], v[224:225] clamp\n\t" \
    "v_pk_fma_f32 v[222:223], v[114:115], s[90:91], v[224:225] clamp\n\t" \
    "v_permlane16_swap_b32 v2, v18\n\t" \
    "v_pk_fma_f32 v[208:209], v[208:209], s[66:67], v[226:227]\n\t" \
    "v_pk_fma_f32 v[210:211], v[210:211], s[66:67], v[226:227]\n\t" \
    "v_pk_fma_f32 v[212:213], v[212:213], s[66:67], v[226:227]\n\t" \
    "v_permlane16_swap_b32 v3, v19\n\t" \
    "v_pk_fma_f32 v[214:215], v[214:215], s[66:67], v[226:227]\n\t" \
    "v_pk_fma_f32 v[216:217], v[216:217], s[66:67], v[226:227]\n\t" \
    "v_pk_fma_f32 v[218:219], v[218:219], s[66:67], v[226:227]\n\t" \
    "s_mov_b32 s99, 0\n\t" \
    "buffer_store_dword v0, v246, s[20:23], s99 offen offset:0 sc0 sc1 nt\n\t" \
    "v_pk_fma_f32 v[220:221], v[220:221], s[66:67], v[226:227]\n\t" \
    "v_pk_fma_f32 v[222:223], v[222:223], s[66:67], v[226:227]\n\t" \
    "s_mov_b32 s99, 262144\n\t" \
    "buffer_store_dword v16, v246, s[20:23], s99 offen offset:0 sc0 sc1 nt\n\t" \
    "v_exp_f32 v208, v208\n\t" \
    "v_exp_f32 v209, v209\n\t" \
    "v_exp_f32 v210, v210\n\t" \
    "s_mov_b32 s99, 65536\n\t" \
    "buffer_store_dword v1, v246, s[20:23], s99 offen offset:0 sc0 sc1 nt\n\t" \
    "v_exp_f32 v211, v211\n\t" \
    "v_exp_f32 v212, v212\n\t" \
    "s_mov_b32 s99, 327680\n\t" \
    "buffer_store_dword v17, v246, s[20:23], s99 offen offset:0 sc0 sc1 nt\n\t" \
    "v_exp_f32 v213, v213\n\t" \
    "v_exp_f32 v214, v214\n\t" \
    "v_exp_f32 v215, v215\n\t" \
    "s_mov_b32 s99, 131072\n\t" \
    "buffer_store_dword v2, v246, s[20:23], s99 offen offset:0 sc0 sc1 nt\n\t" \
    "v_exp_f32 v216, v216\n\t" \
    "v_exp_f32 v217, v217\n\t" \
    "v_exp_f32 v218, v218\n\t" \
    "s_mov_b32 s99, 393216\n\t" \
    "buffer_store_dword v18, v246, s[20:23], s99 offen offset:0 sc0 sc1 nt\n\t" \
    "v_exp_f32 v219, v219\n\t" \
    "v_exp_f32 v220, v220\n\t" \
    "s_mov_b32 s99, 196608\n\t" \
    "buffer_store_dword v3, v246, s[20:23], s99 offen offset:0 sc0 sc1 nt\n\t" \
    "v_exp_f32 v221, v221\n\t" \
    "v_exp_f32 v222, v222\n\t" \
    "v_exp_f32 v223, v223\n\t" \
    "s_mov_b32 s99, 458752\n\t" \
    "buffer_store_dword v19, v246, s[20:23], s99 offen offset:0 sc0 sc1 nt\n\t" \
    "v_pk_fma_f32 v[208:209], v[208:209], s[94:95], s[94:95]\n\t" \
    "v_pk_fma_f32 v[210:211], v[210:211], s[94:95], s[94:95]\n\t" \
    "v_permlane16_swap_b32 v32, v48\n\t" \
    "v_pk_fma_f32 v[212:213], v[212:213], s[94:95], s[94:95]\n\t" \
    "v_pk_fma_f32 v[214:215], v[214:215], s[94:95], s[94:95]\n\t" \
    "v_pk_fma_f32 v[216:217], v[216:217], s[94:95], s[94:95]\n\t" \
    "v_permlane16_swap_b32 v33, v49\n\t" \
    "v_pk_fma_f32 v[218:219], v[218:219], s[94:95], s[94:95]\n\t" \
    "v_pk_fma_f32 v[220:221], v[220:221], s[94:95], s[94:95]\n\t" \
    "v_pk_fma_f32 v[222:223], v[222:223], s[94:95], s[94:95]\n\t" \
    "v_permlane16_swap_b32 v34, v50\n\t" \
    "v_rcp_f32 v208, v208\n\t" \
    "v_rcp_f32 v209, v209\n\t" \
    "v_permlane16_swap_b32 v35, v51\n\t" \
    "v_rcp_f32 v210, v210\n\t" \
    "v_rcp_f32 v211, v211\n\t" \
    "v_rcp_f32 v212, v212\n\t" \
    "s_mov_b32 s99, 0\n\t" \
    "buffer_store_dword v32, v246, s[20:23], s99 offen offset:128 sc0 sc1 nt\n\t" \
    "v_rcp_f32 v213, v213\n\t" \
    "v_rcp_f32 v214, v214\n\t" \
    "v_rcp_f32 v215, v215\n\t" \
    "s_mov_b32 s99, 262144\n\t" \
    "buffer_store_dword v48, v246, s[20:23], s99 offen offset:128 sc0 sc1 nt\n\t" \
    "v_rcp_f32 v216, v216\n\t" \
    "v_rcp_f32 v217, v217\n\t" \
    "s_mov_b32 s99, 65536\n\t" \
    "buffer_store_dword v33, v246, s[20:23], s99 offen offset:128 sc0 sc1 nt\n\t" \
    "v_rcp_f32 v218, v218\n\t" \
    "v_rcp_f32 v219, v219\n\t" \
    "v_rcp_f32 v220, v220\n\t" \
    "s_mov_b32 s99, 327680\n\t" \
    "buffer_store_dword v49, v246, s[20:23], s99 offen offset:128 sc0 sc1 nt\n\t" \
    "v_rcp_f32 v221, v221\n\t" \
    "v_rcp_f32 v222, v222\n\t" \
    "s_mov_b32 s99, 131072\n\t" \
    "buffer_store_dword v34, v246, s[20:23], s99 offen offset:128 sc0 sc1 nt\n\t" \
    "v_rcp_f32 v223, v223\n\t" \
    "v_pk_mul_f32 v[64:65], v[64:65], v[208:209]\n\t" \
    "v_pk_mul_f32 v[66:67], v[66:67], v[210:211]\n\t" \
    "s_mov_b32 s99, 393216\n\t" \
    "buffer_store_dword v50, v246, s[20:23], s99 offen offset:128 sc0 sc1 nt\n\t" \
    "v_pk_mul_f32 v[80:81], v[80:81], v[212:213]\n\t" \
    "v_pk_mul_f32 v[82:83], v[82:83], v[214:215]\n\t" \
    "v_pk_mul_f32 v[96:97], v[96:97], v[216:217]\n\t" \
    "s_mov_b32 s99, 196608\n\t" \
    "buffer_store_dword v35, v246, s[20:23], s99 offen offset:128 sc0 sc1 nt\n\t" \
    "v_pk_mul_f32 v[98:99], v[98:99], v[218:219]\n\t" \
    "v_pk_mul_f32 v[112:113], v[112:113], v[220:221]\n\t" \
    "s_mov_b32 s99, 458752\n\t" \
    "buffer_store_dword v51, v246, s[20:23], s99 offen offset:128 sc0 sc1 nt\n\t" \
    "v_pk_mul_f32 v[114:115], v[114:115], v[222:223]\n\t" \
    "v_pk_fma_f32 v[192:193], v[4:5], s[90:91], v[224:225] clamp\n\t" \
    "v_pk_fma_f32 v[194:195], v[6:7], s[90:91], v[224:225] clamp\n\t" \
    "v_pk_fma_f32 v[196:197], v[20:21], s[90:91], v[224:225] clamp\n\t" \
    "v_permlane16_swap_b32 v64, v80\n\t" \
    "v_pk_fma_f32 v[198:199], v[22:23], s[90:91], v[224:225] clamp\n\t" \
    "v_pk_fma_f32 v[200:201], v[36:37], s[90:91], v[224:225] clamp\n\t" \
    "v_pk_fma_f32 v[202:203], v[38:39], s[90:91], v[224:225] clamp\n\t" \
    "v_permlane16_swap_b32 v65, v81\n\t" \
    "v_pk_fma_f32 v[204:205], v[52:53], s[90:91], v[224:225] clamp\n\t" \
    "v_pk_fma_f32 v[206:207], v[54:55], s[90:91], v[224:225] clamp\n\t" \
    "v_permlane16_swap_b32 v66, v82\n\t" \
    "v_pk_fma_f32 v[192:193], v[192:193], s[66:67], v[226:227]\n\t" \
    "v_pk_fma_f32 v[194:195], v[194:195], s[66:67], v[226:227]\n\t" \
    "v_pk_fma_f32 v[196:197], v[196:197], s[66:67], v[226:227]\n\t" \
    "v_permlane16_swap_b32 v67, v83\n\t" \
    "v_pk_fma_f32 v[198:199], v[198:199], s[66:67], v[226:227]\n\t" \
    "v_pk_fma_f32 v[200:201], v[200:201], s[66:67], v[226:227]\n\t" \
    "v_pk_fma_f32 v[202:203], v[202:203], s[66:67], v[226:227]\n\t" \
    "s_mov_b32 s99, 0\n\t" \
    "buffer_store_dword v64, v246, s[20:23], s99 offen offset:256 sc0 sc1 nt\n\t" \
    "v_pk_fma_f32 v[204:205], v[204:205], s[66:67], v[226:227]\n\t" \
    "v_pk_fma_f32 v[206:207], v[206:207], s[66:67], v[226:227]\n\t" \
    "s_mov_b32 s99, 262144\n\t" \
    "buffer_store_dword v80, v246, s[20:23], s99 offen offset:256 sc0 sc1 nt\n\t" \
    "v_exp_f32 v192, v192\n\t" \
    "v_exp_f32 v193, v193\n\t" \
    "v_exp_f32 v194, v194\n\t" \
    "s_mov_b32 s99, 65536\n\t" \
    "buffer_store_dword v65, v246, s[20:23], s99 offen offset:256 sc0 sc1 nt\n\t" \
    "v_exp_f32 v195, v195\n\t" \
    "v_exp_f32 v196, v196\n\t" \
    "s_mov_b32 s99, 327680\n\t" \
    "buffer_store_dword v81, v246, s[20:23], s99 offen offset:256 sc0 sc1 nt\n\t" \
    "v_exp_f32 v197, v197\n\t" \
    "v_exp_f32 v198, v198\n\t" \
    "v_exp_f32 v199, v199\n\t" \
    "s_mov_b32 s99, 131072\n\t" \
    "buffer_store_dword v66, v246, s[20:23], s99 offen offset:256 sc0 sc1 nt\n\t" \
    "v_exp_f32 v200, v200\n\t" \
    "v_exp_f32 v201, v201\n\t" \
    "v_exp_f32 v202, v202\n\t" \
    "s_mov_b32 s99, 393216\n\t" \
    "buffer_store_dword v82, v246, s[20:23], s99 offen offset:256 sc0 sc1 nt\n\t" \
    "v_exp_f32 v203, v203\n\t" \
    "v_exp_f32 v204, v204\n\t" \
    "s_mov_b32 s99, 196608\n\t" \
    "buffer_store_dword v67, v246, s[20:23], s99 offen offset:256 sc0 sc1 nt\n\t" \
    "v_exp_f32 v205, v205\n\t" \
    "v_exp_f32 v206, v206\n\t" \
    "v_exp_f32 v207, v207\n\t" \
    "s_mov_b32 s99, 458752\n\t" \
    "buffer_store_dword v83, v246, s[20:23], s99 offen offset:256 sc0 sc1 nt\n\t" \
    "v_pk_fma_f32 v[192:193], v[192:193], s[94:95], s[94:95]\n\t" \
    "v_pk_fma_f32 v[194:195], v[194:195], s[94:95], s[94:95]\n\t" \
    "v_permlane16_swap_b32 v96, v112\n\t" \
    "v_pk_fma_f32 v[196:197], v[196:197], s[94:95], s[94:95]\n\t" \
    "v_pk_fma_f32 v[198:199], v[198:199], s[94:95], s[94:95]\n\t" \
    "v_pk_fma_f32 v[200:201], v[200:201], s[94:95], s[94:95]\n\t" \
    "v_permlane16_swap_b32 v97, v113\n\t" \
    "v_pk_fma_f32 v[202:203], v[202:203], s[94:95], s[94:95]\n\t" \
    "v_pk_fma_f32 v[204:205], v[204:205], s[94:95], s[94:95]\n\t" \
    "v_pk_fma_f32 v[206:207], v[206:207], s[94:95], s[94:95]\n\t" \
    "v_permlane16_swap_b32 v98, v114\n\t" \
    "v_rcp_f32 v192, v192\n\t" \
    "v_rcp_f32 v193, v193\n\t" \
    "v_permlane16_swap_b32 v99, v115\n\t" \
    "v_rcp_f32 v194, v194\n\t" \
    "v_rcp_f32 v195, v195\n\t" \
    "v_rcp_f32 v196, v196\n\t" \
    "s_mov_b32 s99, 0\n\t" \
    "buffer_store_dword v96, v246, s[20:23], s99 offen offset:384 sc0 sc1 nt\n\t" \
    "v_rcp_f32 v197, v197\n\t" \
    "v_rcp_f32 v198, v198\n\t" \
    "v_rcp_f32 v199, v199\n\t" \
    "s_mov_b32 s99, 262144\n\t" \
    "buffer_store_dword v112, v246, s[20:23], s99 offen offset:384 sc0 sc1 nt\n\t" \
    "v_rcp_f32 v200, v200\n\t" \
    "v_rcp_f32 v201, v201\n\t" \
    "s_mov_b32 s99, 65536\n\t" \
    "buffer_store_dword v97, v246, s[20:23], s99 offen offset:384 sc0 sc1 nt\n\t" \
    "v_rcp_f32 v202, v202\n\t" \
    "v_rcp_f32 v203, v203\n\t" \
    "v_rcp_f32 v204, v204\n\t" \
    "s_mov_b32 s99, 327680\n\t" \
    "buffer_store_dword v113, v246, s[20:23], s99 offen offset:384 sc0 sc1 nt\n\t" \
    "v_rcp_f32 v205, v205\n\t" \
    "v_rcp_f32 v206, v206\n\t" \
    "s_mov_b32 s99, 131072\n\t" \
    "buffer_store_dword v98, v246, s[20:23], s99 offen offset:384 sc0 sc1 nt\n\t" \
    "v_rcp_f32 v207, v207\n\t" \
    "v_pk_mul_f32 v[4:5], v[4:5], v[192:193]\n\t" \
    "v_pk_mul_f32 v[6:7], v[6:7], v[194:195]\n\t" \
    "s_mov_b32 s99, 393216\n\t" \
    "buffer_store_dword v114, v246, s[20:23], s99 offen offset:384 sc0 sc1 nt\n\t" \
    "v_pk_mul_f32 v[20:21], v[20:21], v[196:197]\n\t" \
    "v_pk_mul_f32 v[22:23], v[22:23], v[198:199]\n\t" \
    "v_pk_mul_f32 v[36:37], v[36:37], v[200:201]\n\t" \
    "s_mov_b32 s99, 196608\n\t" \
    "buffer_store_dword v99, v246, s[20:23], s99 offen offset:384 sc0 sc1 nt\n\t" \
    "v_pk_mul_f32 v[38:39], v[38:39], v[202:203]\n\t" \
    "v_pk_mul_f32 v[52:53], v[52:53], v[204:205]\n\t" \
    "s_mov_b32 s99, 458752\n\t" \
    "buffer_store_dword v115, v246, s[20:23], s99 offen offset:384 sc0 sc1 nt\n\t" \
    "v_pk_mul_f32 v[54:55], v[54:55], v[206:207]\n\t" \
    "v_pk_fma_f32 v[208:209], v[68:69], s[90:91], v[224:225] clamp\n\t" \
    "v_pk_fma_f32 v[210:211], v[70:71], s[90:91], v[224:225] clamp\n\t" \
    "v_pk_fma_f32 v[212:213], v[84:85], s[90:91], v[224:225] clamp\n\t" \
    "v_permlane16_swap_b32 v4, v20\n\t" \
    "v_pk_fma_f32 v[214:215], v[86:87], s[90:91], v[224:225] clamp\n\t" \
    "v_pk_fma_f32 v[216:217], v[100:101], s[90:91], v[224:225] clamp\n\t" \
    "v_pk_fma_f32 v[218:219], v[102:103], s[90:91], v[224:225] clamp\n\t" \
    "v_permlane16_swap_b32 v5, v21\n\t" \
    "v_pk_fma_f32 v[220:221], v[116:117], s[90:91], v[224:225] clamp\n\t" \
    "v_pk_fma_f32 v[222:223], v[118:119], s[90:91], v[224:225] clamp\n\t" \
    "v_permlane16_swap_b32 v6, v22\n\t" \
    "v_pk_fma_f32 v[208:209], v[208:209], s[66:67], v[226:227]\n\t" \
    "v_pk_fma_f32 v[210:211], v[210:211], s[66:67], v[226:227]\n\t" \
    "v_pk_fma_f32 v[212:213], v[212:213], s[66:67], v[226:227]\n\t" \
    "v_permlane16_swap_b32 v7, v23\n\t" \
    "v_pk_fma_f32 v[214:215], v[214:215], s[66:67], v[226:227]\n\t" \
    "v_pk_fma_f32 v[216:217], v[216:217], s[66:67], v[226:227]\n\t" \
    "v_pk_fma_f32 v[218:219], v[218:219], s[66:67], v[226:227]\n\t" \
    "s_mov_b32 s99, 1048576\n\t" \
    "buffer_store_dword v4, v246, s[20:23], s99 offen offset:0 sc0 sc1 nt\n\t" \
    "v_pk_fma_f32 v[220:221], v[220:221], s[66:67], v[226:227]\n\t" \
    "v_pk_fma_f32 v[222:223], v[222:223], s[66:67], v[226:227]\n\t" \
    "s_mov_b32 s99, 1310720\n\t" \
    "buffer_store_dword v20, v246, s[20:23], s99 offen offset:0 sc0 sc1 nt\n\t" \
    "v_exp_f32 v208, v208\n\t" \
    "v_exp_f32 v209, v209\n\t" \
    "v_exp_f32 v210, v210\n\t" \
    "s_mov_b32 s99, 1114112\n\t" \
    "buffer_store_dword v5, v246, s[20:23], s99 offen offset:0 sc0 sc1 nt\n\t" \
    "v_exp_f32 v211, v211\n\t" \
    "v_exp_f32 v212, v212\n\t" \
    "s_mov_b32 s99, 1376256\n\t" \
    "buffer_store_dword v21, v246, s[20:23], s99 offen offset:0 sc0 sc1 nt\n\t" \
    "v_exp_f32 v213, v213\n\t" \
    "v_exp_f32 v214, v214\n\t" \
    "v_exp_f32 v215, v215\n\t" \
    "s_mov_b32 s99, 1179648\n\t" \
    "buffer_store_dword v6, v246, s[20:23], s99 offen offset:0 sc0 sc1 nt\n\t" \
    "v_exp_f32 v216, v216\n\t" \
    "v_exp_f32 v217, v217\n\t" \
    "v_exp_f32 v218, v218\n\t" \
    "s_mov_b32 s99, 1441792\n\t" \
    "buffer_store_dword v22, v246, s[20:23], s99 offen offset:0 sc0 sc1 nt\n\t" \
    "v_exp_f32 v219, v219\n\t" \
    "v_exp_f32 v220, v220\n\t" \
    "s_mov_b32 s99, 1245184\n\t" \
    "buffer_store_dword v7, v246, s[20:23], s99 offen offset:0 sc0 sc1 nt\n\t" \
    "v_exp_f32 v221, v221\n\t" \
    "v_exp_f32 v222, v222\n\t" \
    "v_exp_f32 v223, v223\n\t" \
    "s_mov_b32 s99, 1507328\n\t" \
    "buffer_store_dword v23, v246, s[20:23], s99 offen offset:0 sc0 sc1 nt\n\t" \
    "v_pk_fma_f32 v[208:209], v[208:209], s[94:95], s[94:95]\n\t" \
    "v_pk_fma_f32 v[210:211], v[210:211], s[94:95], s[94:95]\n\t" \
    "v_permlane16_swap_b32 v36, v52\n\t" \
    "v_pk_fma_f32 v[212:213], v[212:213], s[94:95], s[94:95]\n\t" \
    "v_pk_fma_f32 v[214:215], v[214:215], s[94:95], s[94:95]\n\t" \
    "v_pk_fma_f32 v[216:217], v[216:217], s[94:95], s[94:95]\n\t" \
    "v_permlane16_swap_b32 v37, v53\n\t" \
    "v_pk_fma_f32 v[218:219], v[218:219], s[94:95], s[94:95]\n\t" \
    "v_pk_fma_f32 v[220:221], v[220:221], s[94:95], s[94:95]\n\t" \
    "v_pk_fma_f32 v[222:223], v[222:223], s[94:95], s[94:95]\n\t" \
    "v_permlane16_swap_b32 v38, v54\n\t" \
    "v_rcp_f32 v208, v208\n\t" \
    "v_rcp_f32 v209, v209\n\t" \
    "v_permlane16_swap_b32 v39, v55\n\t" \
    "v_rcp_f32 v210, v210\n\t" \
    "v_rcp_f32 v211, v211\n\t" \
    "v_rcp_f32 v212, v212\n\t" \
    "s_mov_b32 s99, 1048576\n\t" \
    "buffer_store_dword v36, v246, s[20:23], s99 offen offset:128 sc0 sc1 nt\n\t" \
    "v_rcp_f32 v213, v213\n\t" \
    "v_rcp_f32 v214, v214\n\t" \
    "v_rcp_f32 v215, v215\n\t" \
    "s_mov_b32 s99, 1310720\n\t" \
    "buffer_store_dword v52, v246, s[20:23], s99 offen offset:128 sc0 sc1 nt\n\t" \
    "v_rcp_f32 v216, v216\n\t" \
    "v_rcp_f32 v217, v217\n\t" \
    "s_mov_b32 s99, 1114112\n\t" \
    "buffer_store_dword v37, v246, s[20:23], s99 offen offset:128 sc0 sc1 nt\n\t" \
    "v_rcp_f32 v218, v218\n\t" \
    "v_rcp_f32 v219, v219\n\t" \
    "v_rcp_f32 v220, v220\n\t" \
    "s_mov_b32 s99, 1376256\n\t" \
    "buffer_store_dword v53, v246, s[20:23], s99 offen offset:128 sc0 sc1 nt\n\t" \
    "v_rcp_f32 v221, v221\n\t" \
    "v_rcp_f32 v222, v222\n\t" \
    "s_mov_b32 s99, 1179648\n\t" \
    "buffer_store_dword v38, v246, s[20:23], s99 offen offset:128 sc0 sc1 nt\n\t" \
    "v_rcp_f32 v223, v223\n\t" \
    "v_pk_mul_f32 v[68:69], v[68:69], v[208:209]\n\t" \
    "v_pk_mul_f32 v[70:71], v[70:71], v[210:211]\n\t" \
    "s_mov_b32 s99, 1441792\n\t" \
    "buffer_store_dword v54, v246, s[20:23], s99 offen offset:128 sc0 sc1 nt\n\t" \
    "v_pk_mul_f32 v[84:85], v[84:85], v[212:213]\n\t" \
    "v_pk_mul_f32 v[86:87], v[86:87], v[214:215]\n\t" \
    "v_pk_mul_f32 v[100:101], v[100:101], v[216:217]\n\t" \
    "s_mov_b32 s99, 1245184\n\t" \
    "buffer_store_dword v39, v246, s[20:23], s99 offen offset:128 sc0 sc1 nt\n\t" \
    "v_pk_mul_f32 v[102:103], v[102:103], v[218:219]\n\t" \
    "v_pk_mul_f32 v[116:117], v[116:117], v[220:221]\n\t" \
    "s_mov_b32 s99, 1507328\n\t" \
    "buffer_store_dword v55, v246, s[20:23], s99 offen offset:128 sc0 sc1 nt\n\t" \
    "v_pk_mul_f32 v[118:119], v[118:119], v[222:223]\n\t" \
    "v_pk_fma_f32 v[192:193], v[8:9], s[90:91], v[224:225] clamp\n\t" \
    "v_pk_fma_f32 v[194:195], v[10:11], s[90:91], v[224:225] clamp\n\t" \
    "v_pk_fma_f32 v[196:197], v[24:25], s[90:91], v[224:225] clamp\n\t" \
    "v_permlane16_swap_b32 v68, v84\n\t" \
    "v_pk_fma_f32 v[198:199], v[26:27], s[90:91], v[224:225] clamp\n\t" \
    "v_pk_fma_f32 v[200:201], v[40:41], s[90:91], v[224:225] clamp\n\t" \
    "v_pk_fma_f32 v[202:203], v[42:43], s[90:91], v[224:225] clamp\n\t" \
    "v_permlane16_swap_b32 v69, v85\n\t" \
    "v_pk_fma_f32 v[204:205], v[56:57], s[90:91], v[224:225] clamp\n\t" \
    "v_pk_fma_f32 v[206:207], v[58:59], s[90:91], v[224:225] clamp\n\t" \
    "v_permlane16_swap_b32 v70, v86\n\t" \
    "v_pk_fma_f32 v[192:193], v[192:193], s[66:67], v[226:227]\n\t" \
    "v_pk_fma_f32 v[194:195], v[194:195], s[66:67], v[226:227]\n\t" \
    "v_pk_fma_f32 v[196:197], v[196:197], s[66:67], v[226:227]\n\t" \
    "v_permlane16_swap_b32 v71, v87\n\t" \
    "v_pk_fma_f32 v[198:199], v[198:199], s[66:67], v[226:227]\n\t" \
    "v_pk_fma_f32 v[200:201], v[200:201], s[66:67], v[226:227]\n\t" \
    "v_pk_fma_f32 v[202:203], v[202:203], s[66:67], v[226:227]\n\t" \
    "s_mov_b32 s99, 1048576\n\t" \
    "buffer_store_dword v68, v246, s[20:23], s99 offen offset:256 sc0 sc1 nt\n\t" \
    "v_pk_fma_f32 v[204:205], v[204:205], s[66:67], v[226:227]\n\t" \
    "v_pk_fma_f32 v[206:207], v[206:207], s[66:67], v[226:227]\n\t" \
    "s_mov_b32 s99, 1310720\n\t" \
    "buffer_store_dword v84, v246, s[20:23], s99 offen offset:256 sc0 sc1 nt\n\t" \
    "v_exp_f32 v192, v192\n\t" \
    "v_exp_f32 v193, v193\n\t" \
    "v_exp_f32 v194, v194\n\t" \
    "s_mov_b32 s99, 1114112\n\t" \
    "buffer_store_dword v69, v246, s[20:23], s99 offen offset:256 sc0 sc1 nt\n\t" \
    "v_exp_f32 v195, v195\n\t" \
    "v_exp_f32 v196, v196\n\t" \
    "s_mov_b32 s99, 1376256\n\t" \
    "buffer_store_dword v85, v246, s[20:23], s99 offen offset:256 sc0 sc1 nt\n\t" \
    "v_exp_f32 v197, v197\n\t" \
    "v_exp_f32 v198, v198\n\t" \
    "v_exp_f32 v199, v199\n\t" \
    "s_mov_b32 s99, 1179648\n\t" \
    "buffer_store_dword v70, v246, s[20:23], s99 offen offset:256 sc0 sc1 nt\n\t" \
    "v_exp_f32 v200, v200\n\t" \
    "v_exp_f32 v201, v201\n\t" \
    "v_exp_f32 v202, v202\n\t" \
    "s_mov_b32 s99, 1441792\n\t" \
    "buffer_store_dword v86, v246, s[20:23], s99 offen offset:256 sc0 sc1 nt\n\t" \
    "v_exp_f32 v203, v203\n\t" \
    "v_exp_f32 v204, v204\n\t" \
    "s_mov_b32 s99, 1245184\n\t" \
    "buffer_store_dword v71, v246, s[20:23], s99 offen offset:256 sc0 sc1 nt\n\t" \
    "v_exp_f32 v205, v205\n\t" \
    "v_exp_f32 v206, v206\n\t" \
    "v_exp_f32 v207, v207\n\t" \
    "s_mov_b32 s99, 1507328\n\t" \
    "buffer_store_dword v87, v246, s[20:23], s99 offen offset:256 sc0 sc1 nt\n\t" \
    "v_pk_fma_f32 v[192:193], v[192:193], s[94:95], s[94:95]\n\t" \
    "v_pk_fma_f32 v[194:195], v[194:195], s[94:95], s[94:95]\n\t" \
    "v_permlane16_swap_b32 v100, v116\n\t" \
    "v_pk_fma_f32 v[196:197], v[196:197], s[94:95], s[94:95]\n\t" \
    "v_pk_fma_f32 v[198:199], v[198:199], s[94:95], s[94:95]\n\t" \
    "v_pk_fma_f32 v[200:201], v[200:201], s[94:95], s[94:95]\n\t" \
    "v_permlane16_swap_b32 v101, v117\n\t" \
    "v_pk_fma_f32 v[202:203], v[202:203], s[94:95], s[94:95]\n\t" \
    "v_pk_fma_f32 v[204:205], v[204:205], s[94:95], s[94:95]\n\t" \
    "v_pk_fma_f32 v[206:207], v[206:207], s[94:95], s[94:95]\n\t" \
    "v_permlane16_swap_b32 v102, v118\n\t" \
    "v_rcp_f32 v192, v192\n\t" \
    "v_rcp_f32 v193, v193\n\t" \
    "v_permlane16_swap_b32 v103, v119\n\t" \
    "v_rcp_f32 v194, v194\n\t" \
    "v_rcp_f32 v195, v195\n\t" \
    "v_rcp_f32 v196, v196\n\t" \
    "s_mov_b32 s99, 1048576\n\t" \
    "buffer_store_dword v100, v246, s[20:23], s99 offen offset:384 sc0 sc1 nt\n\t" \
    "v_rcp_f32 v197, v197\n\t" \
    "v_rcp_f32 v198, v198\n\t" \
    "v_rcp_f32 v199, v199\n\t" \
    "s_mov_b32 s99, 1310720\n\t" \
    "buffer_store_dword v116, v246, s[20:23], s99 offen offset:384 sc0 sc1 nt\n\t" \
    "v_rcp_f32 v200, v200\n\t" \
    "v_rcp_f32 v201, v201\n\t" \
    "s_mov_b32 s99, 1114112\n\t" \
    "buffer_store_dword v101, v246, s[20:23], s99 offen offset:384 sc0 sc1 nt\n\t" \
    "v_rcp_f32 v202, v202\n\t" \
    "v_rcp_f32 v203, v203\n\t" \
    "v_rcp_f32 v204, v204\n\t" \
    "s_mov_b32 s99, 1376256\n\t" \
    "buffer_store_dword v117, v246, s[20:23], s99 offen offset:384 sc0 sc1 nt\n\t" \
    "v_rcp_f32 v205, v205\n\t" \
    "v_rcp_f32 v206, v206\n\t" \
    "s_mov_b32 s99, 1179648\n\t" \
    "buffer_store_dword v102, v246, s[20:23], s99 offen offset:384 sc0 sc1 nt\n\t" \
    "v_rcp_f32 v207, v207\n\t" \
    "v_pk_mul_f32 v[8:9], v[8:9], v[192:193]\n\t" \
    "v_pk_mul_f32 v[10:11], v[10:11], v[194:195]\n\t" \
    "s_mov_b32 s99, 1441792\n\t" \
    "buffer_store_dword v118, v246, s[20:23], s99 offen offset:384 sc0 sc1 nt\n\t" \
    "v_pk_mul_f32 v[24:25], v[24:25], v[196:197]\n\t" \
    "v_pk_mul_f32 v[26:27], v[26:27], v[198:199]\n\t" \
    "v_pk_mul_f32 v[40:41], v[40:41], v[200:201]\n\t" \
    "s_mov_b32 s99, 1245184\n\t" \
    "buffer_store_dword v103, v246, s[20:23], s99 offen offset:384 sc0 sc1 nt\n\t" \
    "v_pk_mul_f32 v[42:43], v[42:43], v[202:203]\n\t" \
    "v_pk_mul_f32 v[56:57], v[56:57], v[204:205]\n\t" \
    "s_mov_b32 s99, 1507328\n\t" \
    "buffer_store_dword v119, v246, s[20:23], s99 offen offset:384 sc0 sc1 nt\n\t" \
    "v_pk_mul_f32 v[58:59], v[58:59], v[206:207]\n\t" \
    "v_pk_fma_f32 v[208:209], v[72:73], s[90:91], v[224:225] clamp\n\t" \
    "v_pk_fma_f32 v[210:211], v[74:75], s[90:91], v[224:225] clamp\n\t" \
    "v_pk_fma_f32 v[212:213], v[88:89], s[90:91], v[224:225] clamp\n\t" \
    "v_permlane16_swap_b32 v8, v24\n\t" \
    "v_pk_fma_f32 v[214:215], v[90:91], s[90:91], v[224:225] clamp\n\t" \
    "v_pk_fma_f32 v[216:217], v[104:105], s[90:91], v[224:225] clamp\n\t" \
    "v_pk_fma_f32 v[218:219], v[106:107], s[90:91], v[224:225] clamp\n\t" \
    "v_permlane16_swap_b32 v9, v25\n\t" \
    "v_pk_fma_f32 v[220:221], v[120:121], s[90:91], v[224:225] clamp\n\t" \
    "v_pk_fma_f32 v[222:223], v[122:123], s[90:91], v[224:225] clamp\n\t" \
    "v_permlane16_swap_b32 v10, v26\n\t" \
    "v_pk_fma_f32 v[208:209], v[208:209], s[66:67], v[226:227]\n\t" \
    "v_pk_fma_f32 v[210:211], v[210:211], s[66:67], v[226:227]\n\t" \
    "v_pk_fma_f32 v[212:213], v[212:213], s[66:67], v[226:227]\n\t" \
    "v_permlane16_swap_b32 v11, v27\n\t" \
    "v_pk_fma_f32 v[214:215], v[214:215], s[66:67], v[226:227]\n\t" \
    "v_pk_fma_f32 v[216:217], v[216:217], s[66:67], v[226:227]\n\t" \
    "v_pk_fma_f32 v[218:219], v[218:219], s[66:67], v[226:227]\n\t" \
    "s_mov_b32 s99, 2097152\n\t" \
    "buffer_store_dword v8, v246, s[20:23], s99 offen offset:0 sc0 sc1 nt\n\t" \
    "v_pk_fma_f32 v[220:221], v[220:221], s[66:67], v[226:227]\n\t" \
    "v_pk_fma_f32 v[222:223], v[222:223], s[66:67], v[226:227]\n\t" \
    "s_mov_b32 s99, 2359296\n\t" \
    "buffer_store_dword v24, v246, s[20:23], s99 offen offset:0 sc0 sc1 nt\n\t" \
    "v_exp_f32 v208, v208\n\t" \
    "v_exp_f32 v209, v209\n\t" \
    "v_exp_f32 v210, v210\n\t" \
    "s_mov_b32 s99, 2162688\n\t" \
    "buffer_store_dword v9, v246, s[20:23], s99 offen offset:0 sc0 sc1 nt\n\t" \
    "v_exp_f32 v211, v211\n\t" \
    "v_exp_f32 v212, v212\n\t" \
    "s_mov_b32 s99, 2424832\n\t" \
    "buffer_store_dword v25, v246, s[20:23], s99 offen offset:0 sc0 sc1 nt\n\t" \
    "v_exp_f32 v213, v213\n\t" \
    "v_exp_f32 v214, v214\n\t" \
    "v_exp_f32 v215, v215\n\t" \
    "s_mov_b32 s99, 2228224\n\t" \
    "buffer_store_dword v10, v246, s[20:23], s99 offen offset:0 sc0 sc1 nt\n\t" \
    "v_exp_f32 v216, v216\n\t" \
    "v_exp_f32 v217, v217\n\t" \
    "v_exp_f32 v218, v218\n\t" \
    "s_mov_b32 s99, 2490368\n\t" \
    "buffer_store_dword v26, v246, s[20:23], s99 offen offset:0 sc0 sc1 nt\n\t" \
    "v_exp_f32 v219, v219\n\t" \
    "v_exp_f32 v220, v220\n\t" \
    "s_mov_b32 s99, 2293760\n\t" \
    "buffer_store_dword v11, v246, s[20:23], s99 offen offset:0 sc0 sc1 nt\n\t" \
    "v_exp_f32 v221, v221\n\t" \
    "v_exp_f32 v222, v222\n\t" \
    "v_exp_f32 v223, v223\n\t" \
    "s_mov_b32 s99, 2555904\n\t" \
    "buffer_store_dword v27, v246, s[20:23], s99 offen offset:0 sc0 sc1 nt\n\t" \
    "v_pk_fma_f32 v[208:209], v[208:209], s[94:95], s[94:95]\n\t" \
    "v_pk_fma_f32 v[210:211], v[210:211], s[94:95], s[94:95]\n\t" \
    "v_permlane16_swap_b32 v40, v56\n\t" \
    "v_pk_fma_f32 v[212:213], v[212:213], s[94:95], s[94:95]\n\t" \
    "v_pk_fma_f32 v[214:215], v[214:215], s[94:95], s[94:95]\n\t" \
    "v_pk_fma_f32 v[216:217], v[216:217], s[94:95], s[94:95]\n\t" \
    "v_permlane16_swap_b32 v41, v57\n\t" \
    "v_pk_fma_f32 v[218:219], v[218:219], s[94:95], s[94:95]\n\t" \
    "v_pk_fma_f32 v[220:221], v[220:221], s[94:95], s[94:95]\n\t" \
    "v_pk_fma_f32 v[222:223], v[222:223], s[94:95], s[94:95]\n\t" \
    "v_permlane16_swap_b32 v42, v58\n\t" \
    "v_rcp_f32 v208, v208\n\t" \
    "v_rcp_f32 v209, v209\n\t" \
    "v_permlane16_swap_b32 v43, v59\n\t" \
    "v_rcp_f32 v210, v210\n\t" \
    "v_rcp_f32 v211, v211\n\t" \
    "v_rcp_f32 v212, v212\n\t" \
    "s_mov_b32 s99, 2097152\n\t" \
    "buffer_store_dword v40, v246, s[20:23], s99 offen offset:128 sc0 sc1 nt\n\t" \
    "v_rcp_f32 v213, v213\n\t" \
    "v_rcp_f32 v214, v214\n\t" \
    "v_rcp_f32 v215, v215\n\t" \
    "s_mov_b32 s99, 2359296\n\t" \
    "buffer_store_dword v56, v246, s[20:23], s99 offen offset:128 sc0 sc1 nt\n\t" \
    "v_rcp_f32 v216, v216\n\t" \
    "v_rcp_f32 v217, v217\n\t" \
    "s_mov_b32 s99, 2162688\n\t" \
    "buffer_store_dword v41, v246, s[20:23], s99 offen offset:128 sc0 sc1 nt\n\t" \
    "v_rcp_f32 v218, v218\n\t" \
    "v_rcp_f32 v219, v219\n\t" \
    "v_rcp_f32 v220, v220\n\t" \
    "s_mov_b32 s99, 2424832\n\t" \
    "buffer_store_dword v57, v246, s[20:23], s99 offen offset:128 sc0 sc1 nt\n\t" \
    "v_rcp_f32 v221, v221\n\t" \
    "v_rcp_f32 v222, v222\n\t" \
    "s_mov_b32 s99, 2228224\n\t" \
    "buffer_store_dword v42, v246, s[20:23], s99 offen offset:128 sc0 sc1 nt\n\t" \
    "v_rcp_f32 v223, v223\n\t" \
    "v_pk_mul_f32 v[72:73], v[72:73], v[208:209]\n\t" \
    "v_pk_mul_f32 v[74:75], v[74:75], v[210:211]\n\t" \
    "s_mov_b32 s99, 2490368\n\t" \
    "buffer_store_dword v58, v246, s[20:23], s99 offen offset:128 sc0 sc1 nt\n\t" \
    "v_pk_mul_f32 v[88:89], v[88:89], v[212:213]\n\t" \
    "v_pk_mul_f32 v[90:91], v[90:91], v[214:215]\n\t" \
    "v_pk_mul_f32 v[104:105], v[104:105], v[216:217]\n\t" \
    "s_mov_b32 s99, 2293760\n\t" \
    "buffer_store_dword v43, v246, s[20:23], s99 offen offset:128 sc0 sc1 nt\n\t" \
    "v_pk_mul_f32 v[106:107], v[106:107], v[218:219]\n\t" \
    "v_pk_mul_f32 v[120:121], v[120:121], v[220:221]\n\t" \
    "s_mov_b32 s99, 2555904\n\t" \
    "buffer_store_dword v59, v246, s[20:23], s99 offen offset:128 sc0 sc1 nt\n\t" \
    "v_pk_mul_f32 v[122:123], v[122:123], v[222:223]\n\t" \
    "v_pk_fma_f32 v[192:193], v[12:13], s[90:91], v[224:225] clamp\n\t" \
    "v_pk_fma_f32 v[194:195], v[14:15], s[90:91], v[224:225] clamp\n\t" \
    "v_pk_fma_f32 v[196:197], v[28:29], s[90:91], v[224:225] clamp\n\t" \
    "v_permlane16_swap_b32 v72, v88\n\t" \
    "v_pk_fma_f32 v[198:199], v[30:31], s[90:91], v[224:225] clamp\n\t" \
    "v_pk_fma_f32 v[200:201], v[44:45], s[90:91], v[224:225] clamp\n\t" \
    "v_pk_fma_f32 v[202:203], v[46:47], s[90:91], v[224:225] clamp\n\t" \
    "v_permlane16_swap_b32 v73, v89\n\t" \
    "v_pk_fma_f32 v[204:205], v[60:61], s[90:91], v[224:225] clamp\n\t" \
    "v_pk_fma_f32 v[206:207], v[62:63], s[90:91], v[224:225] clamp\n\t" \
    "v_permlane16_swap_b32 v74, v90\n\t" \
    "v_pk_fma_f32 v[192:193], v[192:193], s[66:67], v[226:227]\n\t" \
    "v_pk_fma_f32 v[194:195], v[194:195], s[66:67], v[226:227]\n\t" \
    "v_pk_fma_f32 v[196:197], v[196:197], s[66:67], v[226:227]\n\t" \
    "v_permlane16_swap_b32 v75, v91\n\t" \
    "v_pk_fma_f32 v[198:199], v[198:199], s[66:67], v[226:227]\n\t" \
    "v_pk_fma_f32 v[200:201], v[200:201], s[66:67], v[226:227]\n\t" \
    "v_pk_fma_f32 v[202:203], v[202:203], s[66:67], v[226:227]\n\t" \
    "s_mov_b32 s99, 2097152\n\t" \
    "buffer_store_dword v72, v246, s[20:23], s99 offen offset:256 sc0 sc1 nt\n\t" \
    "v_pk_fma_f32 v[204:205], v[204:205], s[66:67], v[226:227]\n\t" \
    "v_pk_fma_f32 v[206:207], v[206:207], s[66:67], v[226:227]\n\t" \
    "s_mov_b32 s99, 2359296\n\t" \
    "buffer_store_dword v88, v246, s[20:23], s99 offen offset:256 sc0 sc1 nt\n\t" \
    "v_exp_f32 v192, v192\n\t" \
    "v_exp_f32 v193, v193\n\t" \
    "v_exp_f32 v194, v194\n\t" \
    "s_mov_b32 s99, 2162688\n\t" \
    "buffer_store_dword v73, v246, s[20:23], s99 offen offset:256 sc0 sc1 nt\n\t" \
    "v_exp_f32 v195, v195\n\t" \
    "v_exp_f32 v196, v196\n\t" \
    "s_mov_b32 s99, 2424832\n\t" \
    "buffer_store_dword v89, v246, s[20:23], s99 offen offset:256 sc0 sc1 nt\n\t" \
    "v_exp_f32 v197, v197\n\t" \
    "v_exp_f32 v198, v198\n\t" \
    "v_exp_f32 v199, v199\n\t" \
    "s_mov_b32 s99, 2228224\n\t" \
    "buffer_store_dword v74, v246, s[20:23], s99 offen offset:256 sc0 sc1 nt\n\t" \
    "v_exp_f32 v200, v200\n\t" \
    "v_exp_f32 v201, v201\n\t" \
    "v_exp_f32 v202, v202\n\t" \
    "s_mov_b32 s99, 2490368\n\t" \
    "buffer_store_dword v90, v246, s[20:23], s99 offen offset:256 sc0 sc1 nt\n\t" \
    "v_exp_f32 v203, v203\n\t" \
    "v_exp_f32 v204, v204\n\t" \
    "s_mov_b32 s99, 2293760\n\t" \
    "buffer_store_dword v75, v246, s[20:23], s99 offen offset:256 sc0 sc1 nt\n\t" \
    "v_exp_f32 v205, v205\n\t" \
    "v_exp_f32 v206, v206\n\t" \
    "v_exp_f32 v207, v207\n\t" \
    "s_mov_b32 s99, 2555904\n\t" \
    "buffer_store_dword v91, v246, s[20:23], s99 offen offset:256 sc0 sc1 nt\n\t" \
    "v_pk_fma_f32 v[192:193], v[192:193], s[94:95], s[94:95]\n\t" \
    "v_pk_fma_f32 v[194:195], v[194:195], s[94:95], s[94:95]\n\t" \
    "v_permlane16_swap_b32 v104, v120\n\t" \
    "v_pk_fma_f32 v[196:197], v[196:197], s[94:95], s[94:95]\n\t" \
    "v_pk_fma_f32 v[198:199], v[198:199], s[94:95], s[94:95]\n\t" \
    "v_pk_fma_f32 v[200:201], v[200:201], s[94:95], s[94:95]\n\t" \
    "v_permlane16_swap_b32 v105, v121\n\t" \
    "v_pk_fma_f32 v[202:203], v[202:203], s[94:95], s[94:95]\n\t" \
    "v_pk_fma_f32 v[204:205], v[204:205], s[94:95], s[94:95]\n\t" \
    "v_pk_fma_f32 v[206:207], v[206:207], s[94:95], s[94:95]\n\t" \
    "v_permlane16_swap_b32 v106, v122\n\t" \
    "v_rcp_f32 v192, v192\n\t" \
    "v_rcp_f32 v193, v193\n\t" \
    "v_permlane16_swap_b32 v107, v123\n\t" \
    "v_rcp_f32 v194, v194\n\t" \
    "v_rcp_f32 v195, v195\n\t" \
    "v_rcp_f32 v196, v196\n\t" \
    "s_mov_b32 s99, 2097152\n\t" \
    "buffer_store_dword v104, v246, s[20:23], s99 offen offset:384 sc0 sc1 nt\n\t" \
    "v_rcp_f32 v197, v197\n\t" \
    "v_rcp_f32 v198, v198\n\t" \
    "v_rcp_f32 v199, v199\n\t" \
    "s_mov_b32 s99, 2359296\n\t" \
    "buffer_store_dword v120, v246, s[20:23], s99 offen offset:384 sc0 sc1 nt\n\t" \
    "v_rcp_f32 v200, v200\n\t" \
    "v_rcp_f32 v201, v201\n\t" \
    "s_mov_b32 s99, 2162688\n\t" \
    "buffer_store_dword v105, v246, s[20:23], s99 offen offset:384 sc0 sc1 nt\n\t" \
    "v_rcp_f32 v202, v202\n\t" \
    "v_rcp_f32 v203, v203\n\t" \
    "v_rcp_f32 v204, v204\n\t" \
    "s_mov_b32 s99, 2424832\n\t" \
    "buffer_store_dword v121, v246, s[20:23], s99 offen offset:384 sc0 sc1 nt\n\t" \
    "v_rcp_f32 v205, v205\n\t" \
    "v_rcp_f32 v206, v206\n\t" \
    "s_mov_b32 s99, 2228224\n\t" \
    "buffer_store_dword v106, v246, s[20:23], s99 offen offset:384 sc0 sc1 nt\n\t" \
    "v_rcp_f32 v207, v207\n\t" \
    "v_pk_mul_f32 v[12:13], v[12:13], v[192:193]\n\t" \
    "v_pk_mul_f32 v[14:15], v[14:15], v[194:195]\n\t" \
    "s_mov_b32 s99, 2490368\n\t" \
    "buffer_store_dword v122, v246, s[20:23], s99 offen offset:384 sc0 sc1 nt\n\t" \
    "v_pk_mul_f32 v[28:29], v[28:29], v[196:197]\n\t" \
    "v_pk_mul_f32 v[30:31], v[30:31], v[198:199]\n\t" \
    "v_pk_mul_f32 v[44:45], v[44:45], v[200:201]\n\t" \
    "s_mov_b32 s99, 2293760\n\t" \
    "buffer_store_dword v107, v246, s[20:23], s99 offen offset:384 sc0 sc1 nt\n\t" \
    "v_pk_mul_f32 v[46:47], v[46:47], v[202:203]\n\t" \
    "v_pk_mul_f32 v[60:61], v[60:61], v[204:205]\n\t" \
    "s_mov_b32 s99, 2555904\n\t" \
    "buffer_store_dword v123, v246, s[20:23], s99 offen offset:384 sc0 sc1 nt\n\t" \
    "v_pk_mul_f32 v[62:63], v[62:63], v[206:207]\n\t" \
    "v_pk_fma_f32 v[208:209], v[76:77], s[90:91], v[224:225] clamp\n\t" \
    "v_pk_fma_f32 v[210:211], v[78:79], s[90:91], v[224:225] clamp\n\t" \
    "v_pk_fma_f32 v[212:213], v[92:93], s[90:91], v[224:225] clamp\n\t" \
    "v_permlane16_swap_b32 v12, v28\n\t" \
    "v_pk_fma_f32 v[214:215], v[94:95], s[90:91], v[224:225] clamp\n\t" \
    "v_pk_fma_f32 v[216:217], v[108:109], s[90:91], v[224:225] clamp\n\t" \
    "v_pk_fma_f32 v[218:219], v[110:111], s[90:91], v[224:225] clamp\n\t" \
    "v_permlane16_swap_b32 v13, v29\n\t" \
    "v_pk_fma_f32 v[220:221], v[124:125], s[90:91], v[224:225] clamp\n\t" \
    "v_pk_fma_f32 v[222:223], v[126:127], s[90:91], v[224:225] clamp\n\t" \
    "v_permlane16_swap_b32 v14, v30\n\t" \
    "v_pk_fma_f32 v[208:209], v[208:209], s[66:67], v[226:227]\n\t" \
    "v_pk_fma_f32 v[210:211], v[210:211], s[66:67], v[226:227]\n\t" \
    "v_pk_fma_f32 v[212:213], v[212:213], s[66:67], v[226:227]\n\t" \
    "v_permlane16_swap_b32 v15, v31\n\t" \
    "v_pk_fma_f32 v[214:215], v[214:215], s[66:67], v[226:227]\n\t" \
    "v_pk_fma_f32 v[216:217], v[216:217], s[66:67], v[226:227]\n\t" \
    "v_pk_fma_f32 v[218:219], v[218:219], s[66:67], v[226:227]\n\t" \
    "s_mov_b32 s99, 3145728\n\t" \
    "buffer_store_dword v12, v246, s[20:23], s99 offen offset:0 sc0 sc1 nt\n\t" \
    "v_pk_fma_f32 v[220:221], v[220:221], s[66:67], v[226:227]\n\t" \
    "v_pk_fma_f32 v[222:223], v[222:223], s[66:67], v[226:227]\n\t" \
    "s_mov_b32 s99, 3407872\n\t" \
    "buffer_store_dword v28, v246, s[20:23], s99 offen offset:0 sc0 sc1 nt\n\t" \
    "v_exp_f32 v208, v208\n\t" \
    "v_exp_f32 v209, v209\n\t" \
    "v_exp_f32 v210, v210\n\t" \
    "s_mov_b32 s99, 3211264\n\t" \
    "buffer_store_dword v13, v246, s[20:23], s99 offen offset:0 sc0 sc1 nt\n\t" \
    "v_exp_f32 v211, v211\n\t" \
    "v_exp_f32 v212, v212\n\t" \
    "s_mov_b32 s99, 3473408\n\t" \
    "buffer_store_dword v29, v246, s[20:23], s99 offen offset:0 sc0 sc1 nt\n\t" \
    "v_exp_f32 v213, v213\n\t" \
    "v_exp_f32 v214, v214\n\t" \
    "v_exp_f32 v215, v215\n\t" \
    "s_mov_b32 s99, 3276800\n\t" \
    "buffer_store_dword v14, v246, s[20:23], s99 offen offset:0 sc0 sc1 nt\n\t" \
    "v_exp_f32 v216, v216\n\t" \
    "v_exp_f32 v217, v217\n\t" \
    "v_exp_f32 v218, v218\n\t" \
    "s_mov_b32 s99, 3538944\n\t" \
    "buffer_store_dword v30, v246, s[20:23], s99 offen offset:0 sc0 sc1 nt\n\t" \
    "v_exp_f32 v219, v219\n\t" \
    "v_exp_f32 v220, v220\n\t" \
    "s_mov_b32 s99, 3342336\n\t" \
    "buffer_store_dword v15, v246, s[20:23], s99 offen offset:0 sc0 sc1 nt\n\t" \
    "v_exp_f32 v221, v221\n\t" \
    "v_exp_f32 v222, v222\n\t" \
    "v_exp_f32 v223, v223\n\t" \
    "s_mov_b32 s99, 3604480\n\t" \
    "buffer_store_dword v31, v246, s[20:23], s99 offen offset:0 sc0 sc1 nt\n\t" \
    "v_pk_fma_f32 v[208:209], v[208:209], s[94:95], s[94:95]\n\t" \
    "v_pk_fma_f32 v[210:211], v[210:211], s[94:95], s[94:95]\n\t" \
    "v_permlane16_swap_b32 v44, v60\n\t" \
    "v_pk_fma_f32 v[212:213], v[212:213], s[94:95], s[94:95]\n\t" \
    "v_pk_fma_f32 v[214:215], v[214:215], s[94:95], s[94:95]\n\t" \
    "v_pk_fma_f32 v[216:217], v[216:217], s[94:95], s[94:95]\n\t" \
    "v_permlane16_swap_b32 v45, v61\n\t" \
    "v_pk_fma_f32 v[218:219], v[218:219], s[94:95], s[94:95]\n\t" \
    "v_pk_fma_f32 v[220:221], v[220:221], s[94:95], s[94:95]\n\t" \
    "v_pk_fma_f32 v[222:223], v[222:223], s[94:95], s[94:95]\n\t" \
    "v_permlane16_swap_b32 v46, v62\n\t" \
    "v_rcp_f32 v208, v208\n\t" \
    "v_rcp_f32 v209, v209\n\t" \
    "v_permlane16_swap_b32 v47, v63\n\t" \
    "v_rcp_f32 v210, v210\n\t" \
    "v_rcp_f32 v211, v211\n\t" \
    "v_rcp_f32 v212, v212\n\t" \
    "s_mov_b32 s99, 3145728\n\t" \
    "buffer_store_dword v44, v246, s[20:23], s99 offen offset:128 sc0 sc1 nt\n\t" \
    "v_rcp_f32 v213, v213\n\t" \
    "v_rcp_f32 v214, v214\n\t" \
    "v_rcp_f32 v215, v215\n\t" \
    "s_mov_b32 s99, 3407872\n\t" \
    "buffer_store_dword v60, v246, s[20:23], s99 offen offset:128 sc0 sc1 nt\n\t" \
    "v_rcp_f32 v216, v216\n\t" \
    "v_rcp_f32 v217, v217\n\t" \
    "s_mov_b32 s99, 3211264\n\t" \
    "buffer_store_dword v45, v246, s[20:23], s99 offen offset:128 sc0 sc1 nt\n\t" \
    "v_rcp_f32 v218, v218\n\t" \
    "v_rcp_f32 v219, v219\n\t" \
    "v_rcp_f32 v220, v220\n\t" \
    "s_mov_b32 s99, 3473408\n\t" \
    "buffer_store_dword v61, v246, s[20:23], s99 offen offset:128 sc0 sc1 nt\n\t" \
    "v_rcp_f32 v221, v221\n\t" \
    "v_rcp_f32 v222, v222\n\t" \
    "s_mov_b32 s99, 3276800\n\t" \
    "buffer_store_dword v46, v246, s[20:23], s99 offen offset:128 sc0 sc1 nt\n\t" \
    "v_rcp_f32 v223, v223\n\t" \
    "v_pk_mul_f32 v[76:77], v[76:77], v[208:209]\n\t" \
    "v_pk_mul_f32 v[78:79], v[78:79], v[210:211]\n\t" \
    "s_mov_b32 s99, 3538944\n\t" \
    "buffer_store_dword v62, v246, s[20:23], s99 offen offset:128 sc0 sc1 nt\n\t" \
    "v_pk_mul_f32 v[92:93], v[92:93], v[212:213]\n\t" \
    "v_pk_mul_f32 v[94:95], v[94:95], v[214:215]\n\t" \
    "v_pk_mul_f32 v[108:109], v[108:109], v[216:217]\n\t" \
    "s_mov_b32 s99, 3342336\n\t" \
    "buffer_store_dword v47, v246, s[20:23], s99 offen offset:128 sc0 sc1 nt\n\t" \
    "v_pk_mul_f32 v[110:111], v[110:111], v[218:219]\n\t" \
    "v_pk_mul_f32 v[124:125], v[124:125], v[220:221]\n\t" \
    "s_mov_b32 s99, 3604480\n\t" \
    "buffer_store_dword v63, v246, s[20:23], s99 offen offset:128 sc0 sc1 nt\n\t" \
    "v_pk_mul_f32 v[126:127], v[126:127], v[222:223]\n\t" \
    "s_nop 1\n\t" \
    "v_permlane16_swap_b32 v76, v92\n\t" \
    "v_permlane16_swap_b32 v77, v93\n\t" \
    "v_permlane16_swap_b32 v78, v94\n\t" \
    "v_permlane16_swap_b32 v79, v95\n\t" \
    "s_mov_b32 s99, 3145728\n\t" \
    "buffer_store_dword v76, v246, s[20:23], s99 offen offset:256 sc0 sc1 nt\n\t" \
    "s_mov_b32 s99, 3407872\n\t" \
    "buffer_store_dword v92, v246, s[20:23], s99 offen offset:256 sc0 sc1 nt\n\t" \
    "s_mov_b32 s99, 3211264\n\t" \
    "buffer_store_dword v77, v246, s[20:23], s99 offen offset:256 sc0 sc1 nt\n\t" \
    "s_mov_b32 s99, 3473408\n\t" \
    "buffer_store_dword v93, v246, s[20:23], s99 offen offset:256 sc0 sc1 nt\n\t" \
    "s_mov_b32 s99, 3276800\n\t" \
    "buffer_store_dword v78, v246, s[20:23], s99 offen offset:256 sc0 sc1 nt\n\t" \
    "s_mov_b32 s99, 3538944\n\t" \
    "buffer_store_dword v94, v246, s[20:23], s99 offen offset:256 sc0 sc1 nt\n\t" \
    "s_mov_b32 s99, 3342336\n\t" \
    "buffer_store_dword v79, v246, s[20:23], s99 offen offset:256 sc0 sc1 nt\n\t" \
    "s_mov_b32 s99, 3604480\n\t" \
    "buffer_store_dword v95, v246, s[20:23], s99 offen offset:256 sc0 sc1 nt\n\t" \
    "v_permlane16_swap_b32 v108, v124\n\t" \
    "v_permlane16_swap_b32 v109, v125\n\t" \
    "v_permlane16_swap_b32 v110, v126\n\t" \
    "v_permlane16_swap_b32 v111, v127\n\t" \
    "s_mov_b32 s99, 3145728\n\t" \
    "buffer_store_dword v108, v246, s[20:23], s99 offen offset:384 sc0 sc1 nt\n\t" \
    "s_mov_b32 s99, 3407872\n\t" \
    "buffer_store_dword v124, v246, s[20:23], s99 offen offset:384 sc0 sc1 nt\n\t" \
    "s_mov_b32 s99, 3211264\n\t" \
    "buffer_store_dword v109, v246, s[20:23], s99 offen offset:384 sc0 sc1 nt\n\t" \
    "s_mov_b32 s99, 3473408\n\t" \
    "buffer_store_dword v125, v246, s[20:23], s99 offen offset:384 sc0 sc1 nt\n\t" \
    "s_mov_b32 s99, 3276800\n\t" \
    "buffer_store_dword v110, v246, s[20:23], s99 offen offset:384 sc0 sc1 nt\n\t" \
    "s_mov_b32 s99, 3538944\n\t" \
    "buffer_store_dword v126, v246, s[20:23], s99 offen offset:384 sc0 sc1 nt\n\t" \
    "s_mov_b32 s99, 3342336\n\t" \
    "buffer_store_dword v111, v246, s[20:23], s99 offen offset:384 sc0 sc1 nt\n\t" \
    "s_mov_b32 s99, 3604480\n\t" \
    "buffer_store_dword v127, v246, s[20:23], s99 offen offset:384 sc0 sc1 nt\n\t"
#define CORE_CLOB "v0", "v1", "v2", "v3", "v4", "v5", "v6", "v7", "v8", "v9", "v10", "v11", "v12", "v13", "v14", "v15", "v16", "v17", "v18", "v19", "v20", "v21", "v22", "v23", "v24", "v25", "v26", "v27", "v28", "v29", "v30", "v31", "v32", "v33", "v34", "v35", "v36", "v37", "v38", "v39", "v40", "v41", "v42", "v43", "v44", "v45", "v46", "v47", "v48", "v49", "v50", "v51", "v52", "v53", "v54", "v55", "v56", "v57", "v58", "v59", "v60", "v61", "v62", "v63", "v64", "v65", "v66", "v67", "v68", "v69", "v70", "v71", "v72", "v73", "v74", "v75", "v76", "v77", "v78", "v79", "v80", "v81", "v82", "v83", "v84", "v85", "v86", "v87", "v88", "v89", "v90", "v91", "v92", "v93", "v94", "v95", "v96", "v97", "v98", "v99", "v100", "v101", "v102", "v103", "v104", "v105", "v106", "v107", "v108", "v109", "v110", "v111", "v112", "v113", "v114", "v115", "v116", "v117", "v118", "v119", "v120", "v121", "v122", "v123", "v124", "v125", "v126", "v127", "v128", "v129", "v130", "v131", "v132", "v133", "v134", "v135", "v136", "v137", "v138", "v139", "v140", "v141", "v142", "v143", "v144", "v145", "v146", "v147", "v148", "v149", "v150", "v151", "v152", "v153", "v154", "v155", "v156", "v157", "v158", "v159", "v168", "v169", "v170", "v171", "v172", "v173", "v174", "v175", "v176", "v177", "v178", "v179", "v180", "v181", "v182", "v183", "v184", "v185", "v186", "v187", "v188", "v189", "v190", "v191", "v192", "v193", "v194", "v195", "v196", "v197", "v198", "v199", "v200", "v201", "v202", "v203", "v204", "v205", "v206", "v207", "v208", "v209", "v210", "v211", "v212", "v213", "v214", "v215", "v216", "v217", "v218", "v219", "v220", "v221", "v222", "v223", "v238", "v239", "v240", "v241", "v242", "v243", "v244", "v245", "v246", "v247", "v248", "v249", "v250", "v251", "v252", "v253", "v254", "v255", "a0", "a1", "a2", "a3", "a4", "a5", "a6", "a7", "a8", "a9", "a10", "a11", "a12", "a13", "a14", "a15", "a16", "a17", "a18", "a19", "a20", "a21", "a22", "a23", "a24", "a25", "a26", "a27", "a28", "a29", "a30", "a31", "a32", "a33", "a34", "a35", "a36", "a37", "a38", "a39", "a40", "a41", "a42", "a43", "a44", "a45", "a46", "a47", "a48", "a49", "a50", "a51", "a52", "a53", "a54", "a55", "a56", "a57", "a58", "a59", "a60", "a61", "a62", "a63", "a64", "a65", "a66", "a67", "a68", "a69", "a70", "a71", "a72", "a73", "a74", "a75", "a76", "a77", "a78", "a79", "a80", "a81", "a82", "a83", "a84", "a85", "a86", "a87", "a88", "a89", "a90", "a91", "a92", "a93", "a94", "a95", "a96", "a97", "a98", "a99", "a100", "a101", "a102", "a103", "a104", "a105", "a106", "a107", "a108", "a109", "a110", "a111", "a112", "a113", "a114", "a115", "a116", "a117", "a118", "a119", "a120", "a121", "a122", "a123", "a124", "a125", "a126", "a127", "a128", "a129", "a130", "a131", "a132", "a133", "a134", "a135", "a136", "a137", "a138", "a139", "a140", "a141", "a142", "a143", "a144", "a145", "a146", "a147", "a148", "a149", "a150", "a151", "a152", "a153", "a154", "a155", "a156", "a157", "a158", "a159", "a160", "a161", "a162", "a163", "a164", "a165", "a166", "a167", "a168", "a169", "a170", "a171", "a172", "a173", "a174", "a175", "a176", "a177", "a178", "a179", "a180", "a181", "a182", "a183", "a184", "a185", "a186", "a187", "a188", "a189", "a190", "a191", "a192", "a193", "a194", "a195", "a196", "a197", "a198", "a199", "a200", "a201", "a202", "a203", "a204", "a205", "a206", "a207", "a208", "a209", "a210", "a211", "a212", "a213", "a214", "a215", "a216", "a217", "a218", "a219", "a220", "a221", "a222", "a223", "a224", "a225", "a226", "a227", "a228", "a229", "a230", "a231", "a232", "a233", "a234", "a235", "a236", "a237", "a238", "a239", "a240", "a241", "a242", "a243", "a244", "a245", "a246", "a247", "a248", "a249", "a250", "a251", "a252", "a253", "a254", "a255", "s72", "s73", "s74", "s75", "s76", "s77", "s78", "s80", "s81", "s82", "s83", "s84", "s85", "s86", "s87", "s88", "s92", "s93", "s94", "s95", "s96", "s97", "s98", "s99", "vcc", "scc", "memory"

typedef unsigned u32x4 __attribute__((ext_vector_type(4)));
typedef unsigned u32x8 __attribute__((ext_vector_type(8)));
typedef unsigned u32x16 __attribute__((ext_vector_type(16)));
constexpr int NR = 8;
constexpr int MLOC = 512, KTOT = 4096, NTOT = 16384, KSH = 512;
constexpr size_t OFF_CNT = 0;
constexpr size_t OFF_TMO = 65536;
constexpr size_t OFF_FLAG = 131072;
constexpr size_t OFF_STAGE = 1 << 20;
constexpr size_t STAGE_BYTES = 2 << 20;
constexpr size_t WS_NEED = OFF_STAGE + 2 * STAGE_BYTES;

struct Params {
  const unsigned char* x; const unsigned char* w; float* out; unsigned char* myws; const float* sx; const float* sw;
  unsigned char* dws[NR];
  unsigned wd[NR];
  unsigned xown; unsigned pad; unsigned pad2[2];
};

__device__ __forceinline__ unsigned rfl(unsigned x) { return __builtin_amdgcn_readfirstlane(x); }
__device__ __forceinline__ unsigned lo32(const void* p) { return rfl((unsigned)(unsigned long long)p); }
__device__ __forceinline__ unsigned hi32(const void* p) { return rfl((unsigned)((unsigned long long)p >> 32)); }
__device__ __forceinline__ unsigned hi16(const void* p) { return rfl((unsigned)((unsigned long long)p >> 32) & 0xffffu); }

__global__ __launch_bounds__(256, 1) void fused_kernel(Params p_) {
  const unsigned b = blockIdx.x, tid = threadIdx.x;
  u32x16 ka, kb; u32x8 kc; unsigned long long kdws;
  {
    const unsigned long long kp = (unsigned long long)__builtin_amdgcn_kernarg_segment_ptr();
    const unsigned koff = 48u + (b & 7u) * 8u;
    asm volatile("s_load_dwordx16 %0, %4, 0x0\n\ts_load_dwordx16 %1, %4, 0x40\n\ts_load_dwordx8 %2, %4, 0x80\n\ts_load_dwordx2 %3, %4, %5\n\ts_waitcnt lgkmcnt(0)"
                 : "=&s"(ka), "=&s"(kb), "=&s"(kc), "=&s"(kdws) : "s"(kp), "s"(koff) : "memory");
  }
  struct { const unsigned char* x; const unsigned char* w; float* out; unsigned char* myws; unsigned sx_lo, sx_hi, sw_lo, sw_hi; unsigned wd[8]; unsigned xown, pad; } p;
  p.x = (const unsigned char*)(((unsigned long long)ka[1] << 32) | ka[0]); p.w = (const unsigned char*)(((unsigned long long)ka[3] << 32) | ka[2]);
  p.out = (float*)(((unsigned long long)ka[5] << 32) | ka[4]); p.myws = (unsigned char*)(((unsigned long long)ka[7] << 32) | ka[6]);
  p.sx_lo = ka[8]; p.sx_hi = ka[9]; p.sw_lo = ka[10]; p.sw_hi = ka[11];
  p.wd[0] = kb[12]; p.wd[1] = kb[13]; p.wd[2] = kb[14]; p.wd[3] = kb[15]; p.wd[4] = kc[0]; p.wd[5] = kc[1]; p.wd[6] = kc[2]; p.wd[7] = kc[3];
  p.xown = kc[4]; p.pad = kc[5];
  const unsigned w = rfl(tid >> 6), l = tid & 63;
  unsigned char* myws = p.myws;
  const unsigned xcd = b & 7, jj = b >> 3, ntile = xcd * 16 + (jj >> 1), mh = jj & 1;
  const unsigned si = b & 7, round = jj >> 3, rg = jj & 7, F = rg * 4 + w;
  const unsigned sic = si < 6u ? si : 6u;
  unsigned char* dws = (unsigned char*)kdws;
  const unsigned tdst = 4 + 7 * round + (6 - sic);
  const unsigned doff = (unsigned)OFF_STAGE + tdst * 65536u + F * 2048u;
  const unsigned foff = (unsigned)OFF_FLAG + (((round * 8 + rg) * 8 + (6 - sic)) * 4 + w) * 4;
  const unsigned dstrow = ((p.xown >> 9) + 512u * (si + 1)) & 4095u;
  const bool real = si != 7;
  const unsigned orow = mh * 256 + w * 64;
  const unsigned char* wb = p.w + ntile * 128;
  const unsigned char* ab = myws + OFF_STAGE;
  const unsigned char* ob = (const unsigned char*)p.out + (size_t)orow * (NTOT * 4) + ntile * 512;
  u32x16 g0;
  g0[0] = lo32(wb); g0[1] = hi16(wb); g0[2] = rfl((unsigned)(KTOT * NTOT) - ntile * 128); g0[3] = 0x20000u;
  g0[4] = lo32(ab); g0[5] = hi16(ab); g0[6] = (unsigned)STAGE_BYTES; g0[7] = 0x20000u;
  g0[8] = lo32(p.x); g0[9] = hi16(p.x); g0[10] = 4096 * KSH; g0[11] = 0x20000u;
  g0[12] = lo32(ob); g0[13] = hi16(ob); g0[14] = rfl((unsigned)(64 * NTOT * 4) - ntile * 512); g0[15] = 0x20000u;
  u32x8 g1;
#pragma unroll
  for (int d = 0; d < 8; ++d) g1[d] = p.wd[d];
  u32x8 g2 = {1u << 20, 2u << 20, 3u << 20, __float_as_uint(60.f * 1.4426950408889634f), w * 1024u + mh * 16384u, w * 8192u, w * 1024u + (1u - mh) * 16384u, mh};
  unsigned ssrc = rfl((dstrow + F * 16) * KSH + round * 128);
  typedef unsigned u32x2 __attribute__((ext_vector_type(2)));
  u32x2 pfin; pfin[0] = rfl(mh * 32768u + ((jj >> 1) * 16u + w * 4u) * 128u); pfin[1] = rfl((jj >> 1) & 3u);
  const unsigned rnd = real ? round : 99u;
  const unsigned char* db = dws + doff; const unsigned char* fp = dws + foff;
  const unsigned char* flm = myws + OFF_FLAG; const unsigned char* cntp = myws + OFF_CNT + (size_t)(b * 4 + w) * 64; const unsigned char* tmo = myws + OFF_TMO;
  u32x16 g3;
  g3[0] = lo32(db); g3[1] = hi16(db); g3[2] = 2048u; g3[3] = 0x20000u;
  g3[4] = lo32(fp); g3[5] = hi32(fp); g3[6] = lo32(flm); g3[7] = hi32(flm); g3[8] = lo32(cntp); g3[9] = hi32(cntp); g3[10] = lo32(tmo); g3[11] = hi32(tmo);
  g3[12] = p.sx_lo; g3[13] = p.sx_hi; g3[14] = p.sw_lo; g3[15] = p.sw_hi;
  const unsigned rot = (jj >> 1) & 3;
  u32x8 g4;
#pragma unroll
  for (int t = 0; t < 4; ++t) { const unsigned j = (t + rot) & 3; g4[t] = p.wd[0] + j * (2u << 20); g4[4 + t] = p.xown + orow * KSH + j * 128; }
  const unsigned g = l >> 4, q = (l >> 1) & 7, pp = l & 1, xr = (q >> 1) | ((g & 1) << 2);
  u32x8 bb;
#pragma unroll
  for (int n = 0; n < 8; ++n) bb[n] = g * 2048 + q * 128 + pp * 8 + ((n ^ xr) << 4);
  const unsigned kr = l >> 3, sl = l & 7, fx = ((kr >> 1) & 3) | (((w >> 1) & 1) << 2);
  u32x4 vc;
  vc[0] = (w * 8 + kr) * NTOT + ((sl ^ fx) << 4);
  vc[1] = (mh * 16 + w * 4) * 2048 + l * 16;
  vc[2] = (l & 15) * (NTOT * 4) + g * 16;
  vc[3] = (mh * 4 + w) * 128 + (l < 28 ? l : 27) * 4;
  unsigned vz = 0, vl16 = l * 16;
  u32x8 vp;
  vp[0] = kr * KSH + ((sl ^ g) << 4); vp[1] = vp[0] ^ 64u;
  vp[2] = w * 8192 + (l & 15) * 128 + ((g ^ ((l & 15) >> 1)) << 4); vp[3] = vp[2] ^ 64u;
  vp[4] = (dstrow + F * 16 + (l & 15)) * KSH + round * 128 + g * 16;
  vp[5] = (w * 8 + (l & 7) + ((l >> 3) & 3) * 32) * NTOT;
  vp[6] = p.xown + (mh * 256 + (jj >> 1) * 16 + w * 4 + ((l >> 2) & 3)) * KSH + (l & 3) * 128;
  vp[7] = ((jj * 32u + (l & 31)) << 16);
  asm volatile(CORE_ASM
    : "+{v[160:167]}"(vp), "+{s[8:23]}"(g0), "+{s[48:63]}"(g3), "+{s[40:47]}"(g2), "+{s[64:71]}"(g4), "+{s89}"(ssrc), "+{v[224:231]}"(bb), "+{s[90:91]}"(pfin)
    : "{s[24:31]}"(g1), "{v[232:235]}"(vc), "{v236}"(vz), "{v237}"(vl16), "{s79}"(rnd)
    : CORE_CLOB);
}

static inline void fill_params(Params& p, int rank, const void* x, const void* w, const void* sx, const void* sw, void* out, void* const* ws) {
  p.x = (const unsigned char*)x; p.w = (const unsigned char*)w; p.out = (float*)out; p.myws = (unsigned char*)ws[rank];
  p.sx = (const float*)sx; p.sw = (const float*)sw;
  for (int i = 0; i < NR; ++i) { p.dws[i] = (unsigned char*)ws[(rank + 1 + i) & 7]; p.wd[i] = (unsigned)((rank + i) & 7) * (8u << 20); }
  p.xown = (unsigned)rank * 262144u; p.pad = 0; p.pad2[0] = p.pad2[1] = 0;
}

extern "C" void kernel_launch(int rank, int nranks, ncclComm_t comm, hipStream_t stream,
                              void* const* d_in, const int* in_sizes, int n_in,
                              void* d_out, int out_size, const int* in_dtypes, int out_dtype,
                              void* const* d_out_all, void* const* d_ws_all, size_t ws_bytes) {
  if (nranks != NR || rank < 0 || rank >= NR || ws_bytes < WS_NEED || n_in != 4 || out_size != MLOC * NTOT) { fprintf(stderr, "unexpected configuration\n"); return; }
  static bool init = false;
  if (!init) {
    HIP_CHECK(hipFuncSetAttribute((const void*)fused_kernel, hipFuncAttributeMaxDynamicSharedMemorySize, 160 * 1024));
    init = true;
  }
  Params p;
  void* ws[NR];
  for (int i = 0; i < NR; ++i) ws[i] = d_ws_all[i];
  fill_params(p, rank, d_in[0], d_in[1], d_in[2], d_in[3], d_out, ws);
  hipLaunchKernelGGL(fused_kernel, dim3(256), dim3(256), 160 * 1024, stream, p);
}
```
